```python
import numpy as np

kernel_name = "gpu_gemm_a2a_m4096_k4096_n16384_e4m3e5m2_dequant"


ABSMAX_RTOL = 0.05

COMPUTE_DTYPE = "float32"
EPILOGUE = "dequant"
IN_QUANT = None
OUT_QUANT = None
IN_FP8 = "e4m3e5m2"
IN_DTYPES = {"x": "e4m3", "w_mat": "e5m2",
             "scale_x": "float32", "scale_w": "float32"}
WORLD_SIZE = 8

def _snap_fp8(x, mant_bits, max_abs, min_normal):
    a = np.abs(x.astype(np.float64))
    _m, _e = np.frexp(a)
    q = np.where(a >= min_normal, np.ldexp(1.0, _e - 1 - mant_bits),
                 min_normal * 2.0 ** -mant_bits)
    snapped = np.minimum(np.round(a / q) * q, max_abs)
    return (np.sign(x) * snapped).astype(np.float32)

def setup_inputs():

    x = _snap_fp8(np.random.randn(4096, 4096).astype(np.float32), 3, 448.0, 2.0 ** -6)
    w_mat = _snap_fp8(np.random.randn(4096, 16384).astype(np.float32), 2, 57344.0, 2.0 ** -14)
    scale_x = (0.02 * (1.0 + np.random.rand(1))).astype(np.float32)
    scale_w = (0.02 * (1.0 + np.random.rand(1))).astype(np.float32)
    return {"x": x, "w_mat": w_mat, "scale_x": scale_x, "scale_w": scale_w}

def reference(x, w_mat, scale_x, scale_w):
    acc = (x @ w_mat).astype(np.float32)
    y = (acc * (scale_x * scale_w)).astype(np.float32)
    return y
```

```cpp
#include <hip/hip_runtime.h>
#include <rccl/rccl.h>
#include <cstdio>
#include <utility>

#define HIP_CHECK(expr) do { hipError_t _e = (expr); if (_e != hipSuccess) { \
    fprintf(stderr, "HIP error %s at %s:%d\n", hipGetErrorString(_e), __FILE__, __LINE__); \
    return; } } while (0)

namespace {

typedef int v8i __attribute__((ext_vector_type(8)));
typedef int v4i __attribute__((ext_vector_type(4)));
typedef int v2i __attribute__((ext_vector_type(2)));
typedef unsigned v2u __attribute__((ext_vector_type(2)));
typedef float v4f __attribute__((ext_vector_type(4)));
#define LAS __attribute__((address_space(3)))
#define GAS __attribute__((address_space(1)))
typedef __amdgpu_buffer_rsrc_t rsrc_t;

constexpr int NRANKS = 8;
constexpr int KDIM = 4096, NFULL = 16384, NLOC = 2048, MLOC = 512;
constexpr unsigned UNIT_BYTES = 64u * KDIM;
constexpr unsigned SRC_BYTES = (unsigned)MLOC * KDIM;
constexpr unsigned STAGE_BYTES = NRANKS * SRC_BYTES;
constexpr unsigned CTRL = 2u * STAGE_BYTES;
constexpr unsigned CTRL_PROG = CTRL;
constexpr unsigned CTRL_TMO = CTRL + 4096;
constexpr unsigned CTRL_CALL = CTRL + 8192;
constexpr unsigned WS_NEEDED = CTRL_CALL + 256u * 128u;
constexpr int NCOMM = 32;
constexpr int NROUNDS = 16;
constexpr unsigned long long DEADLINE_TICKS = 100000000ull;

#define RING 8
#define COMM_P 1
#define MAPPING 0
#define COMM_W 4
#define SLEEPN 2
#define OSTM 0
#define OST ""
#define XLDM 3
#define XLD "sc0 sc1"
constexpr int R = RING;

constexpr unsigned LDS_MISC = 131072;
constexpr unsigned LDS_BYTES = LDS_MISC + 256;

struct Params {
  const unsigned char* x;
  const unsigned char* w;
  const float* sx;
  const float* sw;
  float* out;
  unsigned char* ws[NRANKS];
  int rank;
};

#define STAMP(ws, idx) do {} while (0)

template <class F, int... I> __device__ __forceinline__ void sfor_impl(F&& f, std::integer_sequence<int, I...>) {
  (f(std::integral_constant<int, I>{}), ...);
}
template <int N, class F> __device__ __forceinline__ void sfor(F&& f) { sfor_impl(f, std::make_integer_sequence<int, N>{}); }

__device__ __forceinline__ rsrc_t mk_rsrc(const void* p, unsigned bytes) {
  return __builtin_amdgcn_make_buffer_rsrc((void*)p, 0, bytes, 0x00020000);
}

struct G {
  v8i Wf[8][4];
  v4i X[R][2];
  v4f acc[4][4];
  v4f acc4[4];
  float own[4][4];
  v4f rd[4];
  unsigned av[4];
  unsigned tmov;
  unsigned badv;
  unsigned vo_loc, vo_stg, vo_out, wa[3], ra, wa5, ra5;
  rsrc_t rs_x, rs_stg, rs_out, rs_tmo;
  unsigned so_loc[5];
  unsigned fo[4];
  unsigned ub_cur, ub_nxt;
  unsigned need_cur, need_nxt;
  unsigned out_prev, out_cur;
  unsigned out5;
  unsigned call;
  float scale;
  const GAS unsigned* prog_ptr;
  GAS unsigned* tmo;
  LAS unsigned* lds_bad;
};

#define MFMA0(acc, wf, xf) asm volatile("v_mfma_f32_16x16x128_f8f6f4 %0, %1, %2, 0 cbsz:1" : "=v"(acc) : "a"(wf), "v"(xf))
#define MFMA1(acc, wf, xf) asm volatile("v_mfma_f32_16x16x128_f8f6f4 %0, %1, %2, %0 cbsz:1" : "+v"(acc) : "a"(wf), "v"(xf))

template <int IMM> __device__ __forceinline__ void ld_loc(v4i& d, unsigned vo, rsrc_t rs, unsigned so) {
  asm volatile("buffer_load_dwordx4 %0, %1, %2, %3 offen offset:%4" : "=v"(d) : "v"(vo), "s"(rs), "s"(so), "i"(IMM) : "memory");
}
template <int IMM> __device__ __forceinline__ void ld_stg(v4i& d, unsigned vo, rsrc_t rs, unsigned so) {
  asm volatile("buffer_load_dwordx4 %0, %1, %2, %3 offen offset:%4 " XLD : "=v"(d) : "v"(vo), "s"(rs), "s"(so), "i"(IMM) : "memory");
}

__device__ __forceinline__ void st_sys(const v4i& d, unsigned vo, rsrc_t rs, unsigned so) {
  asm volatile("buffer_store_dwordx4 %0, %1, %2, %3 offen sc0 sc1" :: "v"(d), "v"(vo), "s"(rs), "s"(so) : "memory");
}

template <int CLS> __device__ __forceinline__ void slow_wait(G& g, unsigned need) {
  unsigned long long t0 = __builtin_amdgcn_s_memrealtime();
  while (true) {
    unsigned v = __hip_atomic_load(g.prog_ptr, __ATOMIC_RELAXED, __HIP_MEMORY_SCOPE_SYSTEM);
    unsigned tvv = __hip_atomic_load(g.tmo, __ATOMIC_RELAXED, __HIP_MEMORY_SCOPE_AGENT);
    asm volatile("" : "+v"(v), "+v"(tvv));
    sfor<4>([&](auto kc) {
      constexpr int k = decltype(kc)::value;
      unsigned lo = __builtin_amdgcn_readlane(v, 2 * k), hi = __builtin_amdgcn_readlane(v, 2 * k + 1);
      g.av[k] = lo < hi ? lo : hi;
    });
    if (g.av[CLS] >= need) break;
    unsigned tv = __builtin_amdgcn_readfirstlane(tvv);
    if (tv == g.call || __builtin_amdgcn_s_memrealtime() - t0 > DEADLINE_TICKS) {
      __hip_atomic_store(g.tmo, g.call, __ATOMIC_RELAXED, __HIP_MEMORY_SCOPE_AGENT);
      *g.lds_bad = 1u;
      break;
    }
    __builtin_amdgcn_s_sleep(SLEEPN);
  }
}

constexpr int ST0 = R + 1;
template <int NF, int PREV, bool HAS_NEXT> struct Sched {
  static constexpr int NG = 8 * NF;
  static constexpr bool has_load(int n) { return HAS_NEXT || (n - 1 + R < NG); }
  static constexpr int n_store(int n) { return PREV == 0 ? 0 : ((n >= ST0 && n < ST0 + 4) ? 1 : 0) + ((PREV == 5 && n == ST0 + 4) ? 1 : 0); }
  static constexpr bool has_tmo(int n) { return PREV != 0 && n == 0; }
  static constexpr int ops(int q) {
    if (q < 0) return 2;
    return (has_load(q) ? 2 : 0) + n_store(q) + (has_tmo(q) ? 1 : 0);
  }
  static constexpr int wait_cnt(int n) {
    int c = 0;
    for (int q = n - R + 2; q <= n - 1; ++q) c += ops(q);
    return c > 63 ? 63 : c;
  }
};

template <int NF, int ROFF, int PREV, bool HAS_NEXT>
__device__ __forceinline__ void unit_body(G& g) {
  using S = Sched<NF, PREV, HAS_NEXT>;
  constexpr int NG = S::NG;
  constexpr bool CUR_LOCAL = (NF == 5);
  sfor<NG>([&](auto nc) {
    constexpr int n = decltype(nc)::value;
    constexpr int i = n / NF, j = n % NF;
    constexpr int slot = (ROFF + n) % R;
    constexpr int pslot = (ROFF + n + R - 1) % R;
    constexpr int t = n - 1 + R;
    constexpr bool t_next = t >= NG;
    constexpr int ti = t_next ? (t - NG) / 4 : t / NF, tj = t_next ? (t - NG) % 4 : t % NF;
    constexpr bool t_remote = t_next || !CUR_LOCAL;

    asm volatile("s_waitcnt vmcnt(%2)" : "+v"(g.X[slot][0]), "+v"(g.X[slot][1]) : "i"(S::wait_cnt(n)));

    if constexpr (PREV != 0 && n == 4) {
      asm volatile("s_waitcnt lgkmcnt(0)\n\ts_barrier" ::: "memory");
    }
    if constexpr (PREV != 0 && n >= 5 && n <= 7) {
      asm volatile("s_waitcnt lgkmcnt(0)" : "+v"(g.rd[0]), "+v"(g.rd[1]), "+v"(g.rd[2]), "+v"(g.rd[3]), "+v"(g.badv));
#pragma unroll
      for (int nf = 0; nf < 4; ++nf)
#pragma unroll
        for (int e = 0; e < 4; ++e) g.own[nf][e] += g.rd[nf][e];
    }
    if constexpr (S::has_load(n) && t_remote && tj == 0) {
      constexpr int cls = ti & 3;
      unsigned need = (t_next ? g.need_nxt : g.need_cur) + (ti >> 2);
      if (g.av[cls] < need) slow_wait<cls>(g, need);
    }
    if constexpr (S::has_tmo(n)) {
      asm volatile("buffer_load_dword %0, off, %1, 0 sc1" : "=v"(g.tmov) : "s"(g.rs_tmo) : "memory");
    }

    v8i xf = {g.X[slot][0][0], g.X[slot][0][1], g.X[slot][0][2], g.X[slot][0][3],
              g.X[slot][1][0], g.X[slot][1][1], g.X[slot][1][2], g.X[slot][1][3]};
    v4f* accp = (j < 4) ? g.acc[j < 4 ? j : 0] : g.acc4;

    auto load_half = [&](auto hc) {
      constexpr int h = decltype(hc)::value;
      if constexpr (S::has_load(n)) {
        if constexpr (!t_remote) ld_loc<h * 64>(g.X[pslot][h], g.vo_loc, g.rs_x, g.so_loc[tj] + ti * 512);
        else ld_stg<h * 1024>(g.X[pslot][h], g.vo_stg, g.rs_stg, (t_next ? g.ub_nxt : g.ub_cur) + ti * 32768 + g.fo[tj]);
      }
    };
    auto side = [&](auto qc) {
      constexpr int q = decltype(qc)::value;
      if constexpr (NF == 4 && n == NG - 2) asm volatile("ds_write_b128 %0, %1 offset:%2" :: "v"(g.wa[0]), "v"(g.acc[0][q]), "i"(q * 1024) : "memory");
      if constexpr (NF == 4 && n == NG - 1) asm volatile("ds_write_b128 %0, %1 offset:%2" :: "v"(g.wa[1]), "v"(g.acc[1][q]), "i"(q * 1024) : "memory");
      if constexpr (NF == 5 && n == NG - 3) asm volatile("ds_write_b128 %0, %1 offset:%2" :: "v"(g.wa[0]), "v"(g.acc[0][q]), "i"(q * 1024) : "memory");
      if constexpr (NF == 5 && n == NG - 2) asm volatile("ds_write_b128 %0, %1 offset:%2" :: "v"(g.wa[1]), "v"(g.acc[1][q]), "i"(q * 1024) : "memory");
      if constexpr (NF == 5 && n == NG - 1) asm volatile("ds_write_b128 %0, %1 offset:%2" :: "v"(g.wa[2]), "v"(g.acc[2][q]), "i"(q * 1024) : "memory");
      if constexpr (PREV == 4 && n == 0) asm volatile("ds_write_b128 %0, %1 offset:%2" :: "v"(g.wa[2]), "v"(g.acc[2][q]), "i"(q * 1024) : "memory");
      if constexpr (PREV == 5 && n == 1) asm volatile("ds_write_b128 %0, %1 offset:%2" :: "v"(g.wa5), "v"(g.acc4[q]), "i"(q * 4096) : "memory");
      if constexpr (PREV != 0 && n >= 4 && n <= 6)
        asm volatile("ds_read_b128 %0, %1 offset:%2" : "=v"(g.rd[q]) : "v"(g.ra), "i"(((n - 4) * 4 + q) * 1024) : "memory");
      if constexpr (PREV != 0 && n == 4 && q == 0)
        asm volatile("ds_read_b32 %0, %1" : "=v"(g.badv) : "v"((unsigned)(LDS_MISC + 4)) : "memory");
      if constexpr (PREV == 5 && n == 7)
        asm volatile("ds_read_b128 %0, %1 offset:%2" : "=v"(g.rd[q]) : "v"(g.ra5), "i"(q * 1024) : "memory");
    };
    if constexpr (PREV != 0 && n == 2) {
#pragma unroll
      for (int nf = 0; nf < 4; ++nf)
#pragma unroll
        for (int e = 0; e < 4; ++e) asm volatile("v_mov_b32 %0, %1" : "=v"(g.own[nf][e]) : "v"(g.acc[3][nf][e]));
    }
    auto store_out = [&]() {
      if constexpr (PREV != 0 && n >= ST0 && n < ST0 + 4) {
        constexpr int nf = n - ST0;
        if constexpr (nf == 0) {
          asm volatile("" : "+v"(g.tmov));
          unsigned bad = __builtin_amdgcn_readfirstlane(g.badv) | (__builtin_amdgcn_readfirstlane(g.tmov) == g.call ? 1u : 0u);
          if (bad) g.scale = __builtin_nanf("");
        }
        v4f v = {g.own[nf][0] * g.scale, g.own[nf][1] * g.scale, g.own[nf][2] * g.scale, g.own[nf][3] * g.scale};
        asm volatile("buffer_store_dwordx4 %0, %1, %2, %3 offen offset:%4 " OST :: "v"(v), "v"(g.vo_out), "s"(g.rs_out), "s"(g.out_prev), "i"(nf * 64) : "memory");
      }
      if constexpr (PREV == 5 && n == ST0 + 4) {
        v4f t = (g.rd[0] + g.rd[1]) + (g.rd[2] + g.rd[3]);
        v4f v = {t[0] * g.scale, t[1] * g.scale, t[2] * g.scale, t[3] * g.scale};
        asm volatile("buffer_store_dwordx4 %0, %1, %2, %3 offen " OST :: "v"(v), "v"(g.vo_out), "s"(g.rs_out), "s"(g.out5) : "memory");
      }
    };
    if constexpr (PREV == 5 && n == 8) {
      asm volatile("s_waitcnt lgkmcnt(0)" : "+v"(g.rd[0]), "+v"(g.rd[1]), "+v"(g.rd[2]), "+v"(g.rd[3]));
    }

    sfor<4>([&](auto fc) {
      constexpr int nf = decltype(fc)::value;
      if constexpr (i == 0) MFMA0(accp[nf], g.Wf[i][nf], xf); else MFMA1(accp[nf], g.Wf[i][nf], xf);
      side(fc);
      if constexpr (nf == 0) load_half(std::integral_constant<int, 0>{});
      if constexpr (nf == 1) store_out();
      if constexpr (nf == 2) load_half(std::integral_constant<int, 1>{});
    });
  });
}

__device__ __forceinline__ void next_unit(G& g, unsigned out_next) {
  g.out_prev = g.out_cur;
  g.out_cur = out_next;
  g.ub_cur = g.ub_nxt;
  g.ub_nxt += UNIT_BYTES;
  g.need_cur = g.need_nxt;
  g.need_nxt += 2;
}
__device__ __forceinline__ void flip_red(G& g) {
  g.wa[0] ^= 0x10000u; g.wa[1] ^= 0x10000u; g.wa[2] ^= 0x10000u;
}

__device__ __forceinline__ void gemm_role(const Params& p, unsigned char* smem_generic) {
  LAS unsigned char* smem = (LAS unsigned char*)smem_generic;
  const int b = blockIdx.x;
  const int xcd = b & 7, slot = (b >> 3) - (NCOMM >> 3);
  const int pi = slot >> 2, cb = xcd * 4 + (slot & 3);
  const int tid = threadIdx.x, lane = tid & 63;
  const int w = __builtin_amdgcn_readfirstlane(tid >> 6);
  const int kg = lane >> 4, li = lane & 15;
  const int rank = p.rank;
  unsigned char* myws = p.ws[rank];
  const int src = (rank + 1 + pi) & 7;
  G g;
  STAMP(myws, 0);

  unsigned ticket = 0;
  if (tid == 0) ticket = __hip_atomic_fetch_add((unsigned*)(myws + CTRL_CALL + b * 128), 1u, __ATOMIC_RELAXED, __HIP_MEMORY_SCOPE_AGENT);

  const int row0 = pi * 80 < 432 ? pi * 80 : 432;
  g.rs_x = mk_rsrc(p.x, SRC_BYTES);
  g.vo_loc = li * KDIM + kg * 16;
  sfor<4>([&](auto jc) { constexpr int j = decltype(jc)::value; g.so_loc[j] = (unsigned)(row0 + ((j + 1 + w) & 3) * 16) * KDIM + w * 128; });
  g.so_loc[4] = (unsigned)(row0 + 64) * KDIM + w * 128;
  sfor<R - 1>([&](auto nc) {
    constexpr int n = decltype(nc)::value;
    ld_loc<0>(g.X[n][0], g.vo_loc, g.rs_x, g.so_loc[n % 5] + (n / 5) * 512);
    ld_loc<64>(g.X[n][1], g.vo_loc, g.rs_x, g.so_loc[n % 5] + (n / 5) * 512);
  });

  {
    const unsigned char* wbase = p.w + (size_t)rank * NLOC + cb * 64 + (size_t)(w * 128 + (lane >> 2)) * NFULL + (lane & 3) * 16;
    LAS unsigned char* lw = smem + w * 32768;
    const unsigned rdaddr = (unsigned)((kg * 16 + (li >> 1)) * 64 + (li & 1) * 8);
    auto issue = [&](int i) {
#pragma unroll
      for (int q = 0; q < 8; ++q)
        __builtin_amdgcn_global_load_lds((const GAS unsigned*)(wbase + (size_t)(i * 512 + q * 16) * NFULL),
                                         (LAS unsigned*)(lw + (i & 3) * 8192 + q * 1024), 16, 0, 0);
    };
#pragma unroll
    for (int i = 0; i < 4; ++i) issue(i);
    sfor<8>([&](auto ic) {
      constexpr int i = decltype(ic)::value;
      asm volatile("s_waitcnt vmcnt(%0)" :: "i"(((i + 3 < 7 ? i + 3 : 7) - i) * 8) : "memory");
#pragma unroll
      for (int nf = 0; nf < 4; ++nf) {
        v8i t;
#pragma unroll
        for (int tt = 0; tt < 4; ++tt) {
          v2i q = __builtin_amdgcn_ds_read_tr8_b64_v2i32((LAS v2i*)(lw + (i & 3) * 8192 + rdaddr + ((tt >> 1) * 64 + (tt & 1) * 8) * 64 + nf * 16));
          t[2 * tt] = q[0]; t[2 * tt + 1] = q[1];
        }
        g.Wf[i][nf] = t;
      }
      asm volatile("s_waitcnt lgkmcnt(0)" ::: "memory");
      if constexpr (i + 4 < 8) issue(i + 4);
    });
  }
  if (tid == 0) {
    *(LAS unsigned*)(smem + LDS_MISC) = ticket + 1u;
    *(LAS unsigned*)(smem + LDS_MISC + 4) = 0u;
  }
  __syncthreads();
  g.call = __builtin_amdgcn_readfirstlane(*(LAS unsigned*)(smem + LDS_MISC));
  g.lds_bad = (LAS unsigned*)(smem + LDS_MISC + 4);

  g.scale = p.sx[0] * p.sw[0];
  g.rs_stg = mk_rsrc(myws + (g.call & 1u) * STAGE_BYTES, STAGE_BYTES);
  g.rs_out = mk_rsrc(p.out, (unsigned)(4096u * NLOC * 4u));
  g.prog_ptr = (const GAS unsigned*)(myws + CTRL_PROG + src * 128 + w * 32 + (lane & 7) * 4);
  g.tmo = (GAS unsigned*)(myws + CTRL_TMO);
  g.rs_tmo = mk_rsrc(myws + CTRL_TMO, 4);
  g.vo_stg = lane * 16;
  g.vo_out = li * (NLOC * 4) + kg * 16;
  sfor<4>([&](auto jc) { constexpr int j = decltype(jc)::value; g.fo[j] = ((j + 1 + w) & 3) * 2048; });
  sfor<3>([&](auto jc) { constexpr int j = decltype(jc)::value; g.wa[j] = lane * 16 + ((j + 1 + w) & 3) * 12288 + j * 4096; });
  g.ra = lane * 16 + w * 12288;
  g.wa5 = 49152 + w * 1024 + lane * 16;
  g.ra5 = 49152 + w * 4096 + lane * 16;
  const unsigned colb = cb * 256;
  auto out_off = [&](int grow) -> unsigned { return (unsigned)(grow + w * 16) * (NLOC * 4) + colb; };
  g.out_cur = out_off(rank * MLOC + row0);
  g.out_prev = g.out_cur;
  g.out5 = (unsigned)(rank * MLOC + row0 + 64) * (NLOC * 4) + colb + w * 64;
  g.ub_cur = 0; g.need_cur = 0;
  g.ub_nxt = (unsigned)src * SRC_BYTES + w * 8192;
  g.need_nxt = (g.call - 1u) * NROUNDS + 1u;
  sfor<4>([&](auto cc) { g.av[decltype(cc)::value] = 0u; });

  STAMP(myws, 1);
  unit_body<5, 0, 0, true>(g);
  STAMP(myws, 2);
  next_unit(g, out_off(src * MLOC));
  flip_red(g);
  unit_body<4, 40 % R, 5, true>(g);
  STAMP(myws, 3);
  g.ra ^= 0x10000u;
  for (int u = 2; u < 8; ++u) {
    next_unit(g, out_off(src * MLOC + (u - 1) * 64));
    g.wa[0] ^= 0x10000u; g.wa[1] ^= 0x10000u;
    unit_body<4, 40 % R, 4, true>(g);
    STAMP(myws, 2 + u);
    g.wa[2] ^= 0x10000u;
    g.ra ^= 0x10000u;
  }
  next_unit(g, out_off(src * MLOC + 7 * 64));
  g.wa[0] ^= 0x10000u; g.wa[1] ^= 0x10000u;
  unit_body<4, 40 % R, 4, false>(g);
  STAMP(myws, 10);
  g.wa[2] ^= 0x10000u;
  g.ra ^= 0x10000u;

  asm volatile("s_nop 7\n\ts_nop 7" : "+v"(g.acc[3][0]), "+v"(g.acc[3][1]), "+v"(g.acc[3][2]), "+v"(g.acc[3][3]));
#pragma unroll
  for (int q = 0; q < 4; ++q) asm volatile("ds_write_b128 %0, %1 offset:%2" :: "v"(g.wa[2]), "v"(g.acc[2][q]), "i"(q * 1024) : "memory");
  unsigned tv = __hip_atomic_load(g.tmo, __ATOMIC_RELAXED, __HIP_MEMORY_SCOPE_AGENT);
  asm volatile("s_waitcnt lgkmcnt(0)\n\ts_barrier" ::: "memory");
  {
    LAS v4f* rp = (LAS v4f*)(smem + g.ra);
    unsigned bad = __builtin_amdgcn_readfirstlane(*g.lds_bad) | (__builtin_amdgcn_readfirstlane(tv) == g.call ? 1u : 0u);
    float sc = bad ? __builtin_nanf("") : g.scale;
#pragma unroll
    for (int nf = 0; nf < 4; ++nf) {
      v4f v = g.acc[3][nf];
      v += rp[(0 * 4 + nf) * 64];
      v += rp[(1 * 4 + nf) * 64];
      v += rp[(2 * 4 + nf) * 64];
      v = (v4f){v[0] * sc, v[1] * sc, v[2] * sc, v[3] * sc};
      asm volatile("buffer_store_dwordx4 %0, %1, %2, %3 offen offset:%4 " OST :: "v"(v), "v"(g.vo_out), "s"(g.rs_out), "s"(g.out_cur), "i"(nf * 64) : "memory");
    }
  }
}

__device__ __forceinline__ void comm_role(const Params& p, unsigned char* smem_generic) {
  LAS unsigned char* smem = (LAS unsigned char*)smem_generic;
  const int j = blockIdx.x;
  const int tid = threadIdx.x, lane = tid & 63;
  const int w = __builtin_amdgcn_readfirstlane(tid >> 6);
  const int cw = j * 4 + w;
  const int rank = p.rank;
  unsigned char* myws = p.ws[rank];

  STAMP(myws, 0);
  unsigned ticket = 0;
  if (tid == 0) ticket = __hip_atomic_fetch_add((unsigned*)(myws + CTRL_CALL + blockIdx.x * 128), 1u, __ATOMIC_RELAXED, __HIP_MEMORY_SCOPE_AGENT);

  const int sl = (cw >> 3) & 3, f = (cw >> 1) & 3, h = cw & 1;
  rsrc_t rs_x = mk_rsrc(p.x, SRC_BYTES);
  const unsigned vo = (unsigned)(f * 16 + (lane & 15)) * KDIM + sl * 128 + h * 64 + (lane >> 4) * 16 + (j >> 3) * 512;
  v4i d[NROUNDS];
#pragma unroll
  for (int r = 0; r < NROUNDS; ++r)
    asm volatile("buffer_load_dwordx4 %0, %1, %2, %3 offen" : "=v"(d[r]) : "v"(vo), "s"(rs_x), "s"((unsigned)((r >> 1) * UNIT_BYTES + (r & 1) * 2048)) : "memory");

  if (tid == 0) *(LAS unsigned*)(smem + LDS_MISC) = ticket + 1u;
  if (tid < NROUNDS) *(LAS unsigned*)(smem + LDS_MISC + 64 + tid * 4) = 0u;
  __syncthreads();
  const unsigned call = __builtin_amdgcn_readfirstlane(*(LAS unsigned*)(smem + LDS_MISC));
  const unsigned dst0 = (call & 1u) * STAGE_BYTES + (unsigned)rank * SRC_BYTES + (unsigned)cw * 1024u;

  rsrc_t rs[7];
#pragma unroll
  for (int q = 0; q < 7; ++q) {
    int pq = (rank + 1 + (q + cw) % 7) & 7;
    rs[q] = mk_rsrc(p.ws[pq], 2u * STAGE_BYTES);
  }
  unsigned* flagp = (unsigned*)(p.ws[(rank + 1 + (lane < 7 ? lane : 0)) & 7] + CTRL_PROG + rank * 128 + (((j >> 1) & 3) * 8 + (j >> 3) * 2 + (j & 1)) * 4);
  const unsigned vs = lane * 16;

  auto publish = [&](int r) {
    unsigned old = 0;
    if (lane == 0) old = __hip_atomic_fetch_add((LAS unsigned*)(smem + LDS_MISC + 64 + r * 4), 1u, __ATOMIC_RELAXED, __HIP_MEMORY_SCOPE_WORKGROUP);
    old = __builtin_amdgcn_readfirstlane(old);
    if (old == 3u) {
      if (lane < 7) __hip_atomic_store(flagp, (call - 1u) * NROUNDS + (unsigned)r + 1u, __ATOMIC_RELAXED, __HIP_MEMORY_SCOPE_SYSTEM);
    }
  };
  sfor<NROUNDS * 7 + COMM_W>([&](auto sc) {
    constexpr int sidx = decltype(sc)::value;
    constexpr int NS = NROUNDS * 7;
    constexpr int r = (sidx < NS ? sidx : NS - 1) / 7, q = sidx % 7;
    if constexpr (sidx >= COMM_W) {
      constexpr int left = sidx < NS ? COMM_W - 1 : NS + COMM_W - 1 - sidx;
      asm volatile("s_waitcnt vmcnt(%1)" : "+v"(d[r]) : "i"(left) : "memory");
      if constexpr ((sidx - COMM_W) % 7 == 6) {
        publish((sidx - COMM_W) / 7);
        STAMP(myws, 1 + (sidx - COMM_W) / 7);
      }
    } else {
      asm volatile("s_waitcnt vmcnt(%1)" : "+v"(d[r]) : "i"(NROUNDS - 1 - r + sidx) : "memory");
    }
    if constexpr (sidx < NS)
      st_sys(d[r], vs, rs[q], dst0 + r * 131072u);
  });
  asm volatile("s_waitcnt vmcnt(0)" ::: "memory");
  STAMP(myws, 20);
}

__global__ void __launch_bounds__(256, 1) fused_ag_gemm(Params p) {
  extern __shared__ __attribute__((aligned(16))) unsigned char smem_dyn[];
  if (blockIdx.x < NCOMM) comm_role(p, smem_dyn);
  else gemm_role(p, smem_dyn);
}

}

extern "C" void kernel_launch(int rank, int nranks, ncclComm_t comm, hipStream_t stream,
                              void* const* d_in, const int* in_sizes, int n_in,
                              void* d_out, int out_size,
                              const int* in_dtypes, int out_dtype,
                              void* const* d_out_all,
                              void* const* d_ws_all, size_t ws_bytes) {
  if (nranks != NRANKS || n_in < 4 || ws_bytes < WS_NEEDED || (unsigned)rank >= (unsigned)NRANKS) {
    fprintf(stderr, "unsupported configuration\n");
    return;
  }
  static bool attr_done = false;
  if (!attr_done) {
    HIP_CHECK(hipFuncSetAttribute((const void*)fused_ag_gemm, hipFuncAttributeMaxDynamicSharedMemorySize, (int)LDS_BYTES));
    attr_done = true;
  }
  Params p;
  p.x = (const unsigned char*)d_in[0];
  p.w = (const unsigned char*)d_in[1];
  p.sx = (const float*)d_in[2];
  p.sw = (const float*)d_in[3];
  p.out = (float*)d_out;
  for (int i = 0; i < NRANKS; ++i) p.ws[i] = (unsigned char*)d_ws_all[i];
  p.rank = rank;
  hipLaunchKernelGGL(fused_ag_gemm, dim3(256), dim3(256), LDS_BYTES, stream, p);
  HIP_CHECK(hipGetLastError());
}
```

```cpp
#include <hip/hip_runtime.h>
#include <rccl/rccl.h>
#include <cstdio>
#include <utility>

#define HIP_CHECK(expr) do { hipError_t _e = (expr); if (_e != hipSuccess) { \
    fprintf(stderr, "HIP error %s at %s:%d\n", hipGetErrorString(_e), __FILE__, __LINE__); \
    return; } } while (0)

namespace {

typedef int v8i __attribute__((ext_vector_type(8)));
typedef int v4i __attribute__((ext_vector_type(4)));
typedef int v2i __attribute__((ext_vector_type(2)));
typedef unsigned v2u __attribute__((ext_vector_type(2)));
typedef float v4f __attribute__((ext_vector_type(4)));
#define LAS __attribute__((address_space(3)))
#define GAS __attribute__((address_space(1)))
typedef __amdgpu_buffer_rsrc_t rsrc_t;

constexpr int NRANKS = 8;
constexpr int KDIM = 4096, NFULL = 16384, NLOC = 2048, MLOC = 512;
constexpr unsigned UNIT_BYTES = 64u * KDIM;
constexpr unsigned SRC_BYTES = (unsigned)MLOC * KDIM;
constexpr unsigned STAGE_BYTES = NRANKS * SRC_BYTES;
constexpr unsigned CTRL = 2u * STAGE_BYTES;
constexpr unsigned CTRL_FLAG = CTRL;
constexpr unsigned CTRL_TMO = CTRL + 16384;
constexpr unsigned CTRL_TICKET = CTRL + 16384 + 128;
constexpr unsigned CTRL_CALL = CTRL + 20480;
constexpr unsigned WS_NEEDED = CTRL_CALL + 256u * 128u;
constexpr int NCOMM = 32;
constexpr unsigned NPIECE = 512;
constexpr unsigned TICKETS_PER_CALL = NPIECE - 2 * NCOMM + NCOMM;
constexpr unsigned long long DEADLINE_TICKS = 100000000ull;

#define RING 8
#define COMM_P 1
#define MAPPING 0
#define COMM_W 4
#define SLEEPN 2
#define POLLW 8
#define OSTM 0
#define OST ""
#define XLDM 3
#define XLD "sc0 sc1"
constexpr int R = RING;

constexpr unsigned LDS_MISC = 131072;
constexpr unsigned LDS_BYTES = LDS_MISC + 256;

struct Params {
  const unsigned char* x;
  const unsigned char* w;
  const float* sx;
  const float* sw;
  float* out;
  unsigned char* ws[NRANKS];
  int rank;
};

#define STAMP(ws, idx) do {} while (0)

template <class F, int... I> __device__ __forceinline__ void sfor_impl(F&& f, std::integer_sequence<int, I...>) {
  (f(std::integral_constant<int, I>{}), ...);
}
template <int N, class F> __device__ __forceinline__ void sfor(F&& f) { sfor_impl(f, std::make_integer_sequence<int, N>{}); }

__device__ __forceinline__ rsrc_t mk_rsrc(const void* p, unsigned bytes) {
  return __builtin_amdgcn_make_buffer_rsrc((void*)p, 0, bytes, 0x00020000);
}

struct G {
  v8i Wf[8][4];
  v4i X[R][2];
  v4f acc[4][4];
  v4f acc4[4];
  float own[4][4];
  v4f rd[4];
  unsigned long long mask;
  unsigned kidx;
  unsigned tmov;
  unsigned badv;
  unsigned vo_loc, vo_stg, vo_out, wa[3], ra, wa5, ra5;
  rsrc_t rs_x, rs_stg, rs_out, rs_tmo;
  unsigned so_loc[5];
  unsigned fo[4];
  unsigned ub_cur, ub_nxt;
  unsigned out_prev, out_cur;
  unsigned out5;
  unsigned call;
  float scale;
  const GAS unsigned long long* flag_ptr;
  GAS unsigned* tmo;
  LAS unsigned* lds_bad;
};

#define MFMA0(acc, wf, xf) asm volatile("v_mfma_f32_16x16x128_f8f6f4 %0, %1, %2, 0 cbsz:1" : "=v"(acc) : "a"(wf), "v"(xf))
#define MFMA1(acc, wf, xf) asm volatile("v_mfma_f32_16x16x128_f8f6f4 %0, %1, %2, %0 cbsz:1" : "+v"(acc) : "a"(wf), "v"(xf))

template <int IMM> __device__ __forceinline__ void ld_loc(v4i& d, unsigned vo, rsrc_t rs, unsigned so) {
  asm volatile("buffer_load_dwordx4 %0, %1, %2, %3 offen offset:%4" : "=v"(d) : "v"(vo), "s"(rs), "s"(so), "i"(IMM) : "memory");
}
template <int IMM> __device__ __forceinline__ void ld_stg(v4i& d, unsigned vo, rsrc_t rs, unsigned so) {
  asm volatile("buffer_load_dwordx4 %0, %1, %2, %3 offen offset:%4 " XLD : "=v"(d) : "v"(vo), "s"(rs), "s"(so), "i"(IMM) : "memory");
}

__device__ __forceinline__ void st_sys(const v4i& d, unsigned vo, rsrc_t rs, unsigned so) {
  asm volatile("buffer_store_dwordx4 %0, %1, %2, %3 offen sc0 sc1" :: "v"(d), "v"(vo), "s"(rs), "s"(so) : "memory");
}

__device__ __forceinline__ void w_prefetch(const Params& p, unsigned& d0, unsigned& d1) {
  const int b = blockIdx.x, lane = threadIdx.x & 63;
  const int wi = ((b >> 3) - (NCOMM >> 3)) * 4 + (threadIdx.x >> 6);
  const unsigned char* a = p.w + (size_t)p.rank * NLOC + (b & 7) * 256 + (lane & 1) * 128;
  const unsigned char* a0 = a + (size_t)(wi * 32 + (lane >> 1)) * NFULL;
  int r1 = 3584 + wi * 5 + (lane >> 1);
  r1 = (lane < 10 && r1 < 4096) ? r1 : 4095;
  const unsigned char* a1 = a + (size_t)r1 * NFULL;
  asm volatile("global_load_dword %0, %1, off" : "=v"(d0) : "v"(a0) : "memory");
  asm volatile("global_load_dword %0, %1, off" : "=v"(d1) : "v"(a1) : "memory");
}

__device__ __forceinline__ void slow_wait(G& g) {
  unsigned long long t0 = __builtin_amdgcn_s_memrealtime();
  const unsigned lane = __lane_id();
  unsigned lo = 0, hi = 64;
  unsigned long long m;
  while (true) {
    unsigned long long v = 0;
    if (lane >= lo && lane < hi) v = __hip_atomic_load(g.flag_ptr, __ATOMIC_RELAXED, __HIP_MEMORY_SCOPE_SYSTEM);
    unsigned tvv = __hip_atomic_load(g.tmo, __ATOMIC_RELAXED, __HIP_MEMORY_SCOPE_AGENT);
    unsigned v0 = (unsigned)v, v1 = (unsigned)(v >> 32);
    asm volatile("" : "+v"(v0), "+v"(v1), "+v"(tvv));
    m = __builtin_amdgcn_ballot_w64((v0 < v1 ? v0 : v1) >= g.call) >> g.kidx;
    if (m & 1ull) break;
    unsigned tv = __builtin_amdgcn_readfirstlane(tvv);
    if (tv == g.call || __builtin_amdgcn_s_memrealtime() - t0 > DEADLINE_TICKS) {
      __hip_atomic_store(g.tmo, g.call, __ATOMIC_RELAXED, __HIP_MEMORY_SCOPE_AGENT);
      *g.lds_bad = 1u;
      m = ~0ull;
      break;
    }
    lo = g.kidx; hi = g.kidx + POLLW;
    __builtin_amdgcn_s_sleep(SLEEPN);
  }
  g.mask = m;
}

constexpr int ST0 = R + 1;
template <int NF, int PREV, bool HAS_NEXT> struct Sched {
  static constexpr int NG = 8 * NF;
  static constexpr bool has_load(int n) { return HAS_NEXT || (n - 1 + R < NG); }
  static constexpr int n_store(int n) { return PREV == 0 ? 0 : ((n >= ST0 && n < ST0 + 4) ? 1 : 0) + ((PREV == 5 && n == ST0 + 4) ? 1 : 0); }
  static constexpr bool has_tmo(int n) { return PREV != 0 && n == 0; }
  static constexpr int ops(int q) {
    if (q < 0) return 2;
    return (has_load(q) ? 2 : 0) + n_store(q) + (has_tmo(q) ? 1 : 0);
  }
  static constexpr int wait_cnt(int n) {
    int c = 0;
    for (int q = n - R + 2; q <= n - 1; ++q) c += ops(q);
    return c > 63 ? 63 : c;
  }
};

template <int NF, int ROFF, int PREV, bool HAS_NEXT>
__device__ __forceinline__ void unit_body(G& g) {
  using S = Sched<NF, PREV, HAS_NEXT>;
  constexpr int NG = S::NG;
  constexpr bool CUR_LOCAL = (NF == 5);
  sfor<NG>([&](auto nc) {
    constexpr int n = decltype(nc)::value;
    constexpr int i = n / NF, j = n % NF;
    constexpr int slot = (ROFF + n) % R;
    constexpr int pslot = (ROFF + n + R - 1) % R;
    constexpr int t = n - 1 + R;
    constexpr bool t_next = t >= NG;
    constexpr int ti = t_next ? (t - NG) / 4 : t / NF, tj = t_next ? (t - NG) % 4 : t % NF;
    constexpr bool t_remote = t_next || !CUR_LOCAL;

    asm volatile("s_waitcnt vmcnt(%2)" : "+v"(g.X[slot][0]), "+v"(g.X[slot][1]) : "i"(S::wait_cnt(n)));

    if constexpr (PREV != 0 && n == 4) {
      asm volatile("s_waitcnt lgkmcnt(0)\n\ts_barrier" ::: "memory");
    }
    if constexpr (PREV != 0 && n >= 5 && n <= 7) {
      asm volatile("s_waitcnt lgkmcnt(0)" : "+v"(g.rd[0]), "+v"(g.rd[1]), "+v"(g.rd[2]), "+v"(g.rd[3]), "+v"(g.badv));
#pragma unroll
      for (int nf = 0; nf < 4; ++nf)
#pragma unroll
        for (int e = 0; e < 4; ++e) g.own[nf][e] += g.rd[nf][e];
    }
    if constexpr (S::has_load(n) && t_remote && tj == 0) {
      if (!(g.mask & 1ull)) slow_wait(g);
      g.mask >>= 1;
      g.kidx += 1;
    }
    if constexpr (S::has_tmo(n)) {
      asm volatile("buffer_load_dword %0, off, %1, 0 sc1" : "=v"(g.tmov) : "s"(g.rs_tmo) : "memory");
    }

    v8i xf = {g.X[slot][0][0], g.X[slot][0][1], g.X[slot][0][2], g.X[slot][0][3],
              g.X[slot][1][0], g.X[slot][1][1], g.X[slot][1][2], g.X[slot][1][3]};
    v4f* accp = (j < 4) ? g.acc[j < 4 ? j : 0] : g.acc4;

    auto load_half = [&](auto hc) {
      constexpr int h = decltype(hc)::value;
      if constexpr (S::has_load(n)) {
        if constexpr (!t_remote) ld_loc<h * 64>(g.X[pslot][h], g.vo_loc, g.rs_x, g.so_loc[tj] + ti * 512);
        else ld_stg<h * 1024>(g.X[pslot][h], g.vo_stg, g.rs_stg, (t_next ? g.ub_nxt : g.ub_cur) + ti * 32768 + g.fo[tj]);
      }
    };
    auto side = [&](auto qc) {
      constexpr int q = decltype(qc)::value;
      if constexpr (NF == 4 && n == NG - 2) asm volatile("ds_write_b128 %0, %1 offset:%2" :: "v"(g.wa[0]), "v"(g.acc[0][q]), "i"(q * 1024) : "memory");
      if constexpr (NF == 4 && n == NG - 1) asm volatile("ds_write_b128 %0, %1 offset:%2" :: "v"(g.wa[1]), "v"(g.acc[1][q]), "i"(q * 1024) : "memory");
      if constexpr (NF == 5 && n == NG - 3) asm volatile("ds_write_b128 %0, %1 offset:%2" :: "v"(g.wa[0]), "v"(g.acc[0][q]), "i"(q * 1024) : "memory");
      if constexpr (NF == 5 && n == NG - 2) asm volatile("ds_write_b128 %0, %1 offset:%2" :: "v"(g.wa[1]), "v"(g.acc[1][q]), "i"(q * 1024) : "memory");
      if constexpr (NF == 5 && n == NG - 1) asm volatile("ds_write_b128 %0, %1 offset:%2" :: "v"(g.wa[2]), "v"(g.acc[2][q]), "i"(q * 1024) : "memory");
      if constexpr (PREV == 4 && n == 0) asm volatile("ds_write_b128 %0, %1 offset:%2" :: "v"(g.wa[2]), "v"(g.acc[2][q]), "i"(q * 1024) : "memory");
      if constexpr (PREV == 5 && n == 1) asm volatile("ds_write_b128 %0, %1 offset:%2" :: "v"(g.wa5), "v"(g.acc4[q]), "i"(q * 4096) : "memory");
      if constexpr (PREV != 0 && n >= 4 && n <= 6)
        asm volatile("ds_read_b128 %0, %1 offset:%2" : "=v"(g.rd[q]) : "v"(g.ra), "i"(((n - 4) * 4 + q) * 1024) : "memory");
      if constexpr (PREV != 0 && n == 4 && q == 0)
        asm volatile("ds_read_b32 %0, %1" : "=v"(g.badv) : "v"((unsigned)(LDS_MISC + 4)) : "memory");
      if constexpr (PREV == 5 && n == 7)
        asm volatile("ds_read_b128 %0, %1 offset:%2" : "=v"(g.rd[q]) : "v"(g.ra5), "i"(q * 1024) : "memory");
    };
    if constexpr (PREV != 0 && n == 2) {
#pragma unroll
      for (int nf = 0; nf < 4; ++nf)
#pragma unroll
        for (int e = 0; e < 4; ++e) asm volatile("v_mov_b32 %0, %1" : "=v"(g.own[nf][e]) : "v"(g.acc[3][nf][e]));
    }
    auto store_out = [&]() {
      if constexpr (PREV != 0 && n >= ST0 && n < ST0 + 4) {
        constexpr int nf = n - ST0;
        if constexpr (nf == 0) {
          asm volatile("" : "+v"(g.tmov));
          unsigned bad = __builtin_amdgcn_readfirstlane(g.badv) | (__builtin_amdgcn_readfirstlane(g.tmov) == g.call ? 1u : 0u);
          if (bad) g.scale = __builtin_nanf("");
        }
        v4f v = {g.own[nf][0] * g.scale, g.own[nf][1] * g.scale, g.own[nf][2] * g.scale, g.own[nf][3] * g.scale};
        asm volatile("buffer_store_dwordx4 %0, %1, %2, %3 offen offset:%4 " OST :: "v"(v), "v"(g.vo_out), "s"(g.rs_out), "s"(g.out_prev), "i"(nf * 64) : "memory");
      }
      if constexpr (PREV == 5 && n == ST0 + 4) {
        v4f t = (g.rd[0] + g.rd[1]) + (g.rd[2] + g.rd[3]);
        v4f v = {t[0] * g.scale, t[1] * g.scale, t[2] * g.scale, t[3] * g.scale};
        asm volatile("buffer_store_dwordx4 %0, %1, %2, %3 offen " OST :: "v"(v), "v"(g.vo_out), "s"(g.rs_out), "s"(g.out5) : "memory");
      }
    };
    if constexpr (PREV == 5 && n == 8) {
      asm volatile("s_waitcnt lgkmcnt(0)" : "+v"(g.rd[0]), "+v"(g.rd[1]), "+v"(g.rd[2]), "+v"(g.rd[3]));
    }

    sfor<4>([&](auto fc) {
      constexpr int nf = decltype(fc)::value;
      if constexpr (i == 0) MFMA0(accp[nf], g.Wf[i][nf], xf); else MFMA1(accp[nf], g.Wf[i][nf], xf);
      side(fc);
      if constexpr (nf == 0) load_half(std::integral_constant<int, 0>{});
      if constexpr (nf == 1) store_out();
      if constexpr (nf == 2) load_half(std::integral_constant<int, 1>{});
    });
  });
}

__device__ __forceinline__ void next_unit(G& g, unsigned out_next) {
  g.out_prev = g.out_cur;
  g.out_cur = out_next;
  g.ub_cur = g.ub_nxt;
  g.ub_nxt += UNIT_BYTES;
}
__device__ __forceinline__ void flip_red(G& g) {
  g.wa[0] ^= 0x10000u; g.wa[1] ^= 0x10000u; g.wa[2] ^= 0x10000u;
}

__device__ __forceinline__ void gemm_role(const Params& p, unsigned char* smem_generic) {
  LAS unsigned char* smem = (LAS unsigned char*)smem_generic;
  const int b = blockIdx.x;
  const int xcd = b & 7, slot = (b >> 3) - (NCOMM >> 3);
  const int pi = slot >> 2, cb = xcd * 4 + (slot & 3);
  const int tid = threadIdx.x, lane = tid & 63;
  const int w = __builtin_amdgcn_readfirstlane(tid >> 6);
  const int kg = lane >> 4, li = lane & 15;
  const int rank = p.rank;
  unsigned char* myws = p.ws[rank];
  const int src = (rank + 1 + pi) & 7;
  G g;
  STAMP(myws, 0);

  unsigned pfd, pfd1;
  w_prefetch(p, pfd, pfd1);
  unsigned ticket = 0;
  if (tid == 0) ticket = __hip_atomic_fetch_add((unsigned*)(myws + CTRL_CALL + b * 128), 1u, __ATOMIC_RELAXED, __HIP_MEMORY_SCOPE_AGENT);

  const int row0 = pi * 80 < 432 ? pi * 80 : 432;
  g.rs_x = mk_rsrc(p.x, SRC_BYTES);
  g.vo_loc = li * KDIM + kg * 16;
  sfor<4>([&](auto jc) { constexpr int j = decltype(jc)::value; g.so_loc[j] = (unsigned)(row0 + ((j + 1 + w) & 3) * 16) * KDIM + w * 128; });
  g.so_loc[4] = (unsigned)(row0 + 64) * KDIM + w * 128;
  sfor<R - 1>([&](auto nc) {
    constexpr int n = decltype(nc)::value;
    ld_loc<0>(g.X[n][0], g.vo_loc, g.rs_x, g.so_loc[n % 5] + (n / 5) * 512);
    ld_loc<64>(g.X[n][1], g.vo_loc, g.rs_x, g.so_loc[n % 5] + (n / 5) * 512);
  });

  {
    const unsigned char* wbase = p.w + (size_t)rank * NLOC + cb * 64 + (size_t)(w * 128 + (lane >> 2)) * NFULL + (lane & 3) * 16;
    LAS unsigned char* lw = smem + w * 32768;
    const unsigned rdaddr = (unsigned)((kg * 16 + (li >> 1)) * 64 + (li & 1) * 8);
    auto issue = [&](int i) {
#pragma unroll
      for (int q = 0; q < 8; ++q)
        __builtin_amdgcn_global_load_lds((const GAS unsigned*)(wbase + (size_t)(i * 512 + q * 16) * NFULL),
                                         (LAS unsigned*)(lw + (i & 3) * 8192 + q * 1024), 16, 0, 0);
    };
#pragma unroll
    for (int i = 0; i < 4; ++i) issue(i);
    sfor<8>([&](auto ic) {
      constexpr int i = decltype(ic)::value;
      asm volatile("s_waitcnt vmcnt(%0)" :: "i"(((i + 3 < 7 ? i + 3 : 7) - i) * 8) : "memory");
      if constexpr (i == 0) asm volatile("" :: "v"(pfd), "v"(pfd1));
#pragma unroll
      for (int nf = 0; nf < 4; ++nf) {
        v8i t;
#pragma unroll
        for (int tt = 0; tt < 4; ++tt) {
          v2i q = __builtin_amdgcn_ds_read_tr8_b64_v2i32((LAS v2i*)(lw + (i & 3) * 8192 + rdaddr + ((tt >> 1) * 64 + (tt & 1) * 8) * 64 + nf * 16));
          t[2 * tt] = q[0]; t[2 * tt + 1] = q[1];
        }
        g.Wf[i][nf] = t;
      }
      asm volatile("s_waitcnt lgkmcnt(0)" ::: "memory");
      if constexpr (i + 4 < 8) issue(i + 4);
    });
  }
  if (tid == 0) {
    *(LAS unsigned*)(smem + LDS_MISC) = ticket + 1u;
    *(LAS unsigned*)(smem + LDS_MISC + 4) = 0u;
  }
  __syncthreads();
  g.call = __builtin_amdgcn_readfirstlane(*(LAS unsigned*)(smem + LDS_MISC));
  g.lds_bad = (LAS unsigned*)(smem + LDS_MISC + 4);

  g.scale = p.sx[0] * p.sw[0];
  g.rs_stg = mk_rsrc(myws + (g.call & 1u) * STAGE_BYTES, STAGE_BYTES);
  g.rs_out = mk_rsrc(p.out, (unsigned)(4096u * NLOC * 4u));
  g.flag_ptr = (const GAS unsigned long long*)(myws + CTRL_FLAG + src * 2048 + w * 512 + lane * 8);
  g.tmo = (GAS unsigned*)(myws + CTRL_TMO);
  g.rs_tmo = mk_rsrc(myws + CTRL_TMO, 4);
  g.vo_stg = lane * 16;
  g.vo_out = li * (NLOC * 4) + kg * 16;
  sfor<4>([&](auto jc) { constexpr int j = decltype(jc)::value; g.fo[j] = ((j + 1 + w) & 3) * 2048; });
  sfor<3>([&](auto jc) { constexpr int j = decltype(jc)::value; g.wa[j] = lane * 16 + ((j + 1 + w) & 3) * 12288 + j * 4096; });
  g.ra = lane * 16 + w * 12288;
  g.wa5 = 49152 + w * 1024 + lane * 16;
  g.ra5 = 49152 + w * 4096 + lane * 16;
  const unsigned colb = cb * 256;
  auto out_off = [&](int grow) -> unsigned { return (unsigned)(grow + w * 16) * (NLOC * 4) + colb; };
  g.out_cur = out_off(rank * MLOC + row0);
  g.out_prev = g.out_cur;
  g.out5 = (unsigned)(rank * MLOC + row0 + 64) * (NLOC * 4) + colb + w * 64;
  g.ub_cur = 0;
  g.ub_nxt = (unsigned)src * SRC_BYTES + w * 8192;
  g.mask = 0ull;
  g.kidx = 0u;

  STAMP(myws, 1);
  unit_body<5, 0, 0, true>(g);
  STAMP(myws, 2);
  next_unit(g, out_off(src * MLOC));
  flip_red(g);
  unit_body<4, 40 % R, 5, true>(g);
  STAMP(myws, 3);
  g.ra ^= 0x10000u;
  for (int u = 2; u < 8; ++u) {
    next_unit(g, out_off(src * MLOC + (u - 1) * 64));
    g.wa[0] ^= 0x10000u; g.wa[1] ^= 0x10000u;
    unit_body<4, 40 % R, 4, true>(g);
    STAMP(myws, 2 + u);
    g.wa[2] ^= 0x10000u;
    g.ra ^= 0x10000u;
  }
  next_unit(g, out_off(src * MLOC + 7 * 64));
  g.wa[0] ^= 0x10000u; g.wa[1] ^= 0x10000u;
  unit_body<4, 40 % R, 4, false>(g);
  STAMP(myws, 10);
  g.wa[2] ^= 0x10000u;
  g.ra ^= 0x10000u;

  asm volatile("s_nop 7\n\ts_nop 7" : "+v"(g.acc[3][0]), "+v"(g.acc[3][1]), "+v"(g.acc[3][2]), "+v"(g.acc[3][3]));
#pragma unroll
  for (int q = 0; q < 4; ++q) asm volatile("ds_write_b128 %0, %1 offset:%2" :: "v"(g.wa[2]), "v"(g.acc[2][q]), "i"(q * 1024) : "memory");
  unsigned tv = __hip_atomic_load(g.tmo, __ATOMIC_RELAXED, __HIP_MEMORY_SCOPE_AGENT);
  asm volatile("s_waitcnt lgkmcnt(0)\n\ts_barrier" ::: "memory");
  {
    LAS v4f* rp = (LAS v4f*)(smem + g.ra);
    unsigned bad = __builtin_amdgcn_readfirstlane(*g.lds_bad) | (__builtin_amdgcn_readfirstlane(tv) == g.call ? 1u : 0u);
    float sc = bad ? __builtin_nanf("") : g.scale;
#pragma unroll
    for (int nf = 0; nf < 4; ++nf) {
      v4f v = g.acc[3][nf];
      v += rp[(0 * 4 + nf) * 64];
      v += rp[(1 * 4 + nf) * 64];
      v += rp[(2 * 4 + nf) * 64];
      v = (v4f){v[0] * sc, v[1] * sc, v[2] * sc, v[3] * sc};
      asm volatile("buffer_store_dwordx4 %0, %1, %2, %3 offen offset:%4 " OST :: "v"(v), "v"(g.vo_out), "s"(g.rs_out), "s"(g.out_cur), "i"(nf * 64) : "memory");
    }
  }
}

#define CSTAMP(n) do {} while (0)
template <bool LEADER>
__device__ __forceinline__ void comm_wave(const Params& p, LAS unsigned char* smem, const unsigned call, const int j, const int w, const int lane, v4i d) {
  const int rank = p.rank;
  unsigned char* myws = p.ws[rank];
  const int cw = j * 4 + w;
  rsrc_t rs_x = mk_rsrc(p.x, SRC_BYTES);
  const unsigned vo = (unsigned)((w >> 1) * 16 + (lane & 15)) * KDIM + (w & 1) * 64 + (lane >> 4) * 16;
  const unsigned dst0 = (call & 1u) * STAGE_BYTES + (unsigned)rank * SRC_BYTES + (unsigned)w * 1024u;
  rsrc_t rs[7];
#pragma unroll
  for (int q = 0; q < 7; ++q) {
    int pq = (rank + 1 + (q + cw) % 7) & 7;
    rs[q] = mk_rsrc(p.ws[pq], 2u * STAGE_BYTES);
  }
  unsigned char* flagb = p.ws[(rank + 1 + (lane < 7 ? lane : 0)) & 7] + CTRL_FLAG + rank * 2048;
  unsigned* ticketp = (unsigned*)(myws + CTRL_TICKET);
  GAS unsigned* tmo = (GAS unsigned*)(myws + CTRL_TMO);
  const unsigned tbase = (call - 1u) * TICKETS_PER_CALL - 2u * NCOMM;
  const unsigned vs = lane * 16;
  LAS unsigned* tk = (LAS unsigned*)(smem + 1024);
  LAS unsigned* cnt = (LAS unsigned*)(smem + LDS_MISC + 64);

  auto publish = [&](unsigned n, unsigned t) {
    unsigned old = 0;
    if (lane == 0) old = __hip_atomic_fetch_add(cnt + (n & 7u), 1u, __ATOMIC_RELAXED, __HIP_MEMORY_SCOPE_WORKGROUP);
    old = __builtin_amdgcn_readfirstlane(old);
    if ((old & 3u) == 3u) {
      if (lane < 7) __hip_atomic_store((unsigned*)(flagb + ((t >> 1) & 3u) * 512u + (t >> 3) * 8u + (t & 1u) * 4u), call, __ATOMIC_RELAXED, __HIP_MEMORY_SCOPE_SYSTEM);
      CSTAMP(n + 1);
    }
  };

  unsigned t_cur = (unsigned)j, t_prev = 0;
  unsigned n = 0;
  while (true) {
    v4i dn;
    unsigned tick = 0, t_nxt = 0;
    const unsigned dst = dst0 + t_cur * 4096u;
    sfor<7>([&](auto qc) {
      constexpr int q = decltype(qc)::value;
      asm volatile("s_waitcnt vmcnt(%0)" :: "i"(COMM_W - 1) : "memory");
      if constexpr (q == COMM_W - 1) { if (n > 0) publish(n - 1, t_prev); }
      if constexpr (LEADER && q == 0) {
        if (n > 0 && lane == 0) asm volatile("global_atomic_add %0, %1, %2, off sc0 sc1" : "+v"(tick) : "v"(ticketp), "v"(1u) : "memory");
      }
      if constexpr (q == 5) {
        if constexpr (LEADER) {
          asm volatile("" : "+v"(tick));
          t_nxt = n == 0 ? (unsigned)(NCOMM + j) : __builtin_amdgcn_readfirstlane(tick) - tbase;
          if (lane == 0) tk[n + 1] = t_nxt + 1u;
        } else {
          unsigned long long t0 = 0;
          while (true) {
            unsigned v = __hip_atomic_load(tk + n + 1, __ATOMIC_RELAXED, __HIP_MEMORY_SCOPE_WORKGROUP);
            v = __builtin_amdgcn_readfirstlane(v);
            if (v) { t_nxt = v - 1u; break; }
            unsigned long long now = __builtin_amdgcn_s_memrealtime();
            if (!t0) t0 = now;
            if (now - t0 > DEADLINE_TICKS) {
              __hip_atomic_store(tmo, call, __ATOMIC_RELAXED, __HIP_MEMORY_SCOPE_AGENT);
              t_nxt = NPIECE; break;
            }
            __builtin_amdgcn_s_sleep(1);
          }
        }
        if (t_nxt < NPIECE)
          asm volatile("buffer_load_dwordx4 %0, %1, %2, %3 offen" : "=v"(dn) : "v"(vo), "s"(rs_x),
                       "s"((t_nxt >> 6) * UNIT_BYTES + (t_nxt & 1u) * (32u * KDIM) + ((t_nxt >> 1) & 31u) * 128u) : "memory");
      }
      st_sys(d, vs, rs[q], dst);
    });
    if (t_nxt >= NPIECE) break;
    asm volatile("s_waitcnt vmcnt(2)" : "+v"(dn) :: "memory");
    t_prev = t_cur; t_cur = t_nxt; d = dn; ++n;
  }
  asm volatile("s_waitcnt vmcnt(0)" ::: "memory");
  publish(n, t_cur);
}

__device__ __forceinline__ void comm_role(const Params& p, unsigned char* smem_generic) {
  LAS unsigned char* smem = (LAS unsigned char*)smem_generic;
  const int j = blockIdx.x;
  const int tid = threadIdx.x, lane = tid & 63;
  const int w = __builtin_amdgcn_readfirstlane(tid >> 6);
  unsigned char* myws = p.ws[p.rank];

  STAMP(myws, 0);
  unsigned ticket = 0;
  if (tid == 0) ticket = __hip_atomic_fetch_add((unsigned*)(myws + CTRL_CALL + blockIdx.x * 128), 1u, __ATOMIC_RELAXED, __HIP_MEMORY_SCOPE_AGENT);
  v4i d;
  {
    rsrc_t rs_x = mk_rsrc(p.x, SRC_BYTES);
    const unsigned vo = (unsigned)((w >> 1) * 16 + (lane & 15)) * KDIM + (w & 1) * 64 + (lane >> 4) * 16;
    asm volatile("buffer_load_dwordx4 %0, %1, %2, %3 offen" : "=v"(d) : "v"(vo), "s"(rs_x), "s"((unsigned)((j & 1) * (32 * KDIM) + (j >> 1) * 128)) : "memory");
  }
  for (int i = tid; i < 520; i += 256) *(LAS unsigned*)(smem + 1024 + i * 4) = 0u;
  if (tid == 0) *(LAS unsigned*)(smem + LDS_MISC) = ticket + 1u;
  if (tid < 8) *(LAS unsigned*)(smem + LDS_MISC + 64 + tid * 4) = 0u;
  __syncthreads();
  const unsigned call = __builtin_amdgcn_readfirstlane(*(LAS unsigned*)(smem + LDS_MISC));
  asm volatile("s_waitcnt vmcnt(0)" : "+v"(d) :: "memory");
  if (w == 0) comm_wave<true>(p, smem, call, j, w, lane, d);
  else comm_wave<false>(p, smem, call, j, w, lane, d);
}

__global__ void __launch_bounds__(256, 1) fused_ag_gemm(Params p) {
  extern __shared__ __attribute__((aligned(16))) unsigned char smem_dyn[];
  if (blockIdx.x < NCOMM) comm_role(p, smem_dyn);
  else gemm_role(p, smem_dyn);
}

}

extern "C" void kernel_launch(int rank, int nranks, ncclComm_t comm, hipStream_t stream,
                              void* const* d_in, const int* in_sizes, int n_in,
                              void* d_out, int out_size,
                              const int* in_dtypes, int out_dtype,
                              void* const* d_out_all,
                              void* const* d_ws_all, size_t ws_bytes) {
  if (nranks != NRANKS || n_in < 4 || ws_bytes < WS_NEEDED || (unsigned)rank >= (unsigned)NRANKS) {
    fprintf(stderr, "unsupported configuration\n");
    return;
  }
  static bool attr_done = false;
  if (!attr_done) {
    HIP_CHECK(hipFuncSetAttribute((const void*)fused_ag_gemm, hipFuncAttributeMaxDynamicSharedMemorySize, (int)LDS_BYTES));
    attr_done = true;
  }
  Params p;
  p.x = (const unsigned char*)d_in[0];
  p.w = (const unsigned char*)d_in[1];
  p.sx = (const float*)d_in[2];
  p.sw = (const float*)d_in[3];
  p.out = (float*)d_out;
  for (int i = 0; i < NRANKS; ++i) p.ws[i] = (unsigned char*)d_ws_all[i];
  p.rank = rank;
  hipLaunchKernelGGL(fused_ag_gemm, dim3(256), dim3(256), LDS_BYTES, stream, p);
  HIP_CHECK(hipGetLastError());
}
```

```cpp
#include <hip/hip_runtime.h>
#include <rccl/rccl.h>
#include <cstdio>
#include <utility>

#define HIP_CHECK(expr) do { hipError_t _e = (expr); if (_e != hipSuccess) { \
    fprintf(stderr, "HIP error %s at %s:%d\n", hipGetErrorString(_e), __FILE__, __LINE__); \
    return; } } while (0)

namespace {

typedef int v8i __attribute__((ext_vector_type(8)));
typedef int v4i __attribute__((ext_vector_type(4)));
typedef int v2i __attribute__((ext_vector_type(2)));
typedef unsigned v2u __attribute__((ext_vector_type(2)));
typedef float v4f __attribute__((ext_vector_type(4)));
#define LAS __attribute__((address_space(3)))
#define GAS __attribute__((address_space(1)))
#define CAS __attribute__((address_space(4)))
typedef __amdgpu_buffer_rsrc_t rsrc_t;

constexpr int NRANKS = 8;
constexpr int KDIM = 4096, NFULL = 16384, NLOC = 2048, MLOC = 512;
constexpr unsigned UNIT_BYTES = 64u * KDIM;
constexpr unsigned SRC_BYTES = (unsigned)MLOC * KDIM;
constexpr unsigned STAGE_BYTES = NRANKS * SRC_BYTES;
constexpr unsigned CTRL = 2u * STAGE_BYTES;
constexpr unsigned CTRL_FLAG = CTRL;
constexpr unsigned CTRL_TMO = CTRL + 16384;
constexpr unsigned CTRL_TICKET = CTRL + 16384 + 128;
constexpr unsigned CTRL_CALL = CTRL + 20480;
constexpr unsigned WS_NEEDED = CTRL_CALL + 1024u * 32u;
constexpr int NCOMM = 32;
constexpr unsigned NPIECE = 512;
constexpr unsigned TICKETS_PER_CALL = NPIECE - 2 * NCOMM + NCOMM;
constexpr unsigned long long DEADLINE_TICKS = 100000000ull;

#define RING 8
#define COMM_P 1
#define MAPPING 0
#define COMM_W 4
#define SLEEPN 2
#define PF_MODE 0
#define POLLW 8
#define OSTM 0
#define OST ""
#define XLDM 3
#define XLD "sc0 sc1"
constexpr int R = RING;

constexpr unsigned LDS_STG = 65536, STG_WAVE = 22528, STG_X = 12288;
constexpr unsigned LDS_MISC = LDS_STG + 4 * STG_WAVE;
constexpr unsigned LDS_BYTES = LDS_MISC + 256;

struct Params {
  const unsigned char* x;
  const unsigned char* w;
  const float* sx;
  const float* sw;
  float* out;
  unsigned char* ws[NRANKS];
  int rank;
};

#define STAMP(ws, idx) do {} while (0)
#define GSTAMP(idx) do {} while (0)

template <class F, int... I> __device__ __forceinline__ void sfor_impl(F&& f, std::integer_sequence<int, I...>) {
  (f(std::integral_constant<int, I>{}), ...);
}
template <int N, class F> __device__ __forceinline__ void sfor(F&& f) { sfor_impl(f, std::make_integer_sequence<int, N>{}); }

__device__ __forceinline__ rsrc_t mk_rsrc(const void* p, unsigned bytes) {
  return __builtin_amdgcn_make_buffer_rsrc((void*)p, 0, bytes, 0x00020000);
}

struct G {
  v8i Wf[8][4];
  v4i X[R][2];
  v4f acc[4][4];
  v4f acc4[4];
  float own[4][4];
  v4f rd[4];
  unsigned long long mask;
  unsigned kidx;
  unsigned tmov;
  unsigned badv;
  unsigned vo_loc, vo_stg, vo_out, wa[3], ra, wa5, ra5;
  rsrc_t rs_x, rs_stg, rs_out, rs_tmo;
  unsigned so_loc[5];
  unsigned fo[4];
  unsigned ub_cur, ub_nxt;
  unsigned out_prev, out_cur;
  unsigned out5;
  unsigned call;
  float scale;
  const GAS unsigned long long* flag_ptr;
  GAS unsigned* tmo;
  LAS unsigned* lds_bad;
};

#define MFMA0(acc, wf, xf) asm volatile("v_mfma_f32_16x16x128_f8f6f4 %0, %1, %2, 0 cbsz:1" : "=v"(acc) : "a"(wf), "v"(xf))
#define MFMA1(acc, wf, xf) asm volatile("v_mfma_f32_16x16x128_f8f6f4 %0, %1, %2, %0 cbsz:1" : "+v"(acc) : "a"(wf), "v"(xf))

template <int IMM> __device__ __forceinline__ void ld_loc(v4i& d, unsigned vo, rsrc_t rs, unsigned so) {
  asm volatile("buffer_load_dwordx4 %0, %1, %2, %3 offen offset:%4" : "=v"(d) : "v"(vo), "s"(rs), "s"(so), "i"(IMM) : "memory");
}
template <int IMM> __device__ __forceinline__ void ld_stg(v4i& d, unsigned vo, rsrc_t rs, unsigned so) {
  asm volatile("buffer_load_dwordx4 %0, %1, %2, %3 offen offset:%4 " XLD : "=v"(d) : "v"(vo), "s"(rs), "s"(so), "i"(IMM) : "memory");
}

__device__ __forceinline__ void st_sys(const v4i& d, unsigned vo, rsrc_t rs, unsigned so) {
  asm volatile("buffer_store_dwordx4 %0, %1, %2, %3 offen sc0 sc1" :: "v"(d), "v"(vo), "s"(rs), "s"(so) : "memory");
}

__device__ __forceinline__ void dma16(unsigned vo, rsrc_t rs, unsigned so, unsigned lds) {
  asm volatile("s_mov_b32 m0, %3\n\tbuffer_load_dwordx4 %0, %1, %2 offen lds" :: "v"(vo), "s"(rs), "s"(so), "s"(lds) : "memory");
}

__device__ __forceinline__ void prefetch_touch(const Params& p, int row0, unsigned (&d)[5]) {
  const int b = blockIdx.x, lane = threadIdx.x & 63;
  const int wi = ((b >> 3) - (NCOMM >> 3)) * 4 + (threadIdx.x >> 6);
  const unsigned char* a = p.w + (size_t)p.rank * NLOC + (b & 7) * 256 + (lane & 1) * 128;
  const unsigned char* a0 = a + (size_t)(wi * 32 + (lane >> 1)) * NFULL;
  int r1   = 3584 + wi * 5 + (lane >> 1);
  r1 = (lane < 10 && r1 < 4096) ? r1 : 4095;
  const unsigned char* a1 = a + (size_t)r1 * NFULL;
  const unsigned char* xb = p.x + (size_t)row0 * KDIM + (size_t)((wi & 15) * 160 + lane) * 128;
  const unsigned char* x2 = xb + (lane < 32 ? 128 * 128 : 0);
  const unsigned char* x1 = xb + 64 * 128;
  asm volatile("global_load_dword %0, %1, off" : "=v"(d[0]) : "v"(xb) : "memory");
  asm volatile("global_load_dword %0, %1, off" : "=v"(d[1]) : "v"(a0) : "memory");
  asm volatile("global_load_dword %0, %1, off" : "=v"(d[2]) : "v"(x1) : "memory");
  asm volatile("global_load_dword %0, %1, off" : "=v"(d[3]) : "v"(a1) : "memory");
  asm volatile("global_load_dword %0, %1, off" : "=v"(d[4]) : "v"(x2) : "memory");
}

__device__ __forceinline__ void slow_wait(G& g) {
  unsigned long long t0 = __builtin_amdgcn_s_memrealtime();
  const unsigned lane = __lane_id();
  unsigned lo = 0, hi = 64;
  unsigned long long m;
  while (true) {
    unsigned long long v = 0;
    if (lane >= lo && lane < hi) v = __hip_atomic_load(g.flag_ptr, __ATOMIC_RELAXED, __HIP_MEMORY_SCOPE_SYSTEM);
    unsigned tvv = __hip_atomic_load(g.tmo, __ATOMIC_RELAXED, __HIP_MEMORY_SCOPE_AGENT);
    unsigned v0 = (unsigned)v, v1 = (unsigned)(v >> 32);
    asm volatile("" : "+v"(v0), "+v"(v1), "+v"(tvv));
    m = __builtin_amdgcn_ballot_w64((v0 < v1 ? v0 : v1) >= g.call) >> g.kidx;
    if (m & 1ull) break;
    unsigned tv = __builtin_amdgcn_readfirstlane(tvv);
    if (tv == g.call || __builtin_amdgcn_s_memrealtime() - t0 > DEADLINE_TICKS) {
      __hip_atomic_store(g.tmo, g.call, __ATOMIC_RELAXED, __HIP_MEMORY_SCOPE_AGENT);
      *g.lds_bad = 1u;
      m = ~0ull;
      break;
    }
    lo = g.kidx; hi = g.kidx + POLLW;
    __builtin_amdgcn_s_sleep(SLEEPN);
  }
  g.mask = m;
}

constexpr int ST0 = R + 1;
template <int NF, int PREV, bool HAS_NEXT> struct Sched {
  static constexpr int NG = 8 * NF;
  static constexpr bool has_load(int n) { return HAS_NEXT || (n - 1 + R < NG); }
  static constexpr int n_store(int n) { return PREV == 0 ? 0 : ((n >= ST0 && n < ST0 + 4) ? 1 : 0) + ((PREV == 5 && n == ST0 + 4) ? 1 : 0); }
  static constexpr bool has_tmo(int n) { return PREV != 0 && n == 0; }
  static constexpr int ops(int q) {
    if (q < 0) return 2;
    return (has_load(q) ? 2 : 0) + n_store(q) + (has_tmo(q) ? 1 : 0);
  }
  static constexpr int wait_cnt(int n) {
    int c = 0;
    for (int q = n - R + 2; q <= n - 1; ++q) c += ops(q);
    return c > 63 ? 63 : c;
  }
};

template <int NF, int ROFF, int PREV, bool HAS_NEXT>
__device__ __forceinline__ void unit_body(G& g) {
  using S = Sched<NF, PREV, HAS_NEXT>;
  constexpr int NG = S::NG;
  constexpr bool CUR_LOCAL = (NF == 5);
  sfor<NG>([&](auto nc) {
    constexpr int n = decltype(nc)::value;
    constexpr int i = n / NF, j = n % NF;
    constexpr int slot = (ROFF + n) % R;
    constexpr int pslot = (ROFF + n + R - 1) % R;
    constexpr int t = n - 1 + R;
    constexpr bool t_next = t >= NG;
    constexpr int ti = t_next ? (t - NG) / 4 : t / NF, tj = t_next ? (t - NG) % 4 : t % NF;
    constexpr bool t_remote = t_next || !CUR_LOCAL;

    asm volatile("s_waitcnt vmcnt(%2)" : "+v"(g.X[slot][0]), "+v"(g.X[slot][1]) : "i"(S::wait_cnt(n)));

    if constexpr (PREV != 0 && n == 4) {
      asm volatile("s_waitcnt lgkmcnt(0)\n\ts_barrier" ::: "memory");
    }
    if constexpr (PREV != 0 && n >= 5 && n <= 7) {
      asm volatile("s_waitcnt lgkmcnt(0)" : "+v"(g.rd[0]), "+v"(g.rd[1]), "+v"(g.rd[2]), "+v"(g.rd[3]), "+v"(g.badv));
#pragma unroll
      for (int nf = 0; nf < 4; ++nf)
#pragma unroll
        for (int e = 0; e < 4; ++e) g.own[nf][e] += g.rd[nf][e];
    }
    if constexpr (S::has_load(n) && t_remote && tj == 0) {
      if (!(g.mask & 1ull)) slow_wait(g);
      g.mask >>= 1;
      g.kidx += 1;
    }
    if constexpr (S::has_tmo(n)) {
      asm volatile("buffer_load_dword %0, off, %1, 0 sc1" : "=v"(g.tmov) : "s"(g.rs_tmo) : "memory");
    }

    v8i xf = {g.X[slot][0][0], g.X[slot][0][1], g.X[slot][0][2], g.X[slot][0][3],
              g.X[slot][1][0], g.X[slot][1][1], g.X[slot][1][2], g.X[slot][1][3]};
    v4f* accp = (j < 4) ? g.acc[j < 4 ? j : 0] : g.acc4;

    auto load_half = [&](auto hc) {
      constexpr int h = decltype(hc)::value;
      if constexpr (S::has_load(n)) {
        if constexpr (!t_remote) ld_loc<h * 64>(g.X[pslot][h], g.vo_loc, g.rs_x, g.so_loc[tj] + ti * 512);
        else ld_stg<h * 1024>(g.X[pslot][h], g.vo_stg, g.rs_stg, (t_next ? g.ub_nxt : g.ub_cur) + ti * 32768 + g.fo[tj]);
      }
    };
    auto side = [&](auto qc) {
      constexpr int q = decltype(qc)::value;
      if constexpr (NF == 4 && n == NG - 2) asm volatile("ds_write_b128 %0, %1 offset:%2" :: "v"(g.wa[0]), "v"(g.acc[0][q]), "i"(q * 1024) : "memory");
      if constexpr (NF == 4 && n == NG - 1) asm volatile("ds_write_b128 %0, %1 offset:%2" :: "v"(g.wa[1]), "v"(g.acc[1][q]), "i"(q * 1024) : "memory");
      if constexpr (NF == 5 && n == NG - 3) asm volatile("ds_write_b128 %0, %1 offset:%2" :: "v"(g.wa[0]), "v"(g.acc[0][q]), "i"(q * 1024) : "memory");
      if constexpr (NF == 5 && n == NG - 2) asm volatile("ds_write_b128 %0, %1 offset:%2" :: "v"(g.wa[1]), "v"(g.acc[1][q]), "i"(q * 1024) : "memory");
      if constexpr (NF == 5 && n == NG - 1) asm volatile("ds_write_b128 %0, %1 offset:%2" :: "v"(g.wa[2]), "v"(g.acc[2][q]), "i"(q * 1024) : "memory");
      if constexpr (PREV == 4 && n == 0) asm volatile("ds_write_b128 %0, %1 offset:%2" :: "v"(g.wa[2]), "v"(g.acc[2][q]), "i"(q * 1024) : "memory");
      if constexpr (PREV == 5 && n == 1) asm volatile("ds_write_b128 %0, %1 offset:%2" :: "v"(g.wa5), "v"(g.acc4[q]), "i"(q * 4096) : "memory");
      if constexpr (PREV != 0 && n >= 4 && n <= 6)
        asm volatile("ds_read_b128 %0, %1 offset:%2" : "=v"(g.rd[q]) : "v"(g.ra), "i"(((n - 4) * 4 + q) * 1024) : "memory");
      if constexpr (PREV != 0 && n == 4 && q == 0)
        asm volatile("ds_read_b32 %0, %1" : "=v"(g.badv) : "v"((unsigned)(LDS_MISC + 4)) : "memory");
      if constexpr (PREV == 5 && n == 7)
        asm volatile("ds_read_b128 %0, %1 offset:%2" : "=v"(g.rd[q]) : "v"(g.ra5), "i"(q * 1024) : "memory");
    };
    if constexpr (PREV != 0 && n == 2) {
#pragma unroll
      for (int nf = 0; nf < 4; ++nf)
#pragma unroll
        for (int e = 0; e < 4; ++e) asm volatile("v_mov_b32 %0, %1" : "=v"(g.own[nf][e]) : "v"(g.acc[3][nf][e]));
    }
    auto store_out = [&]() {
      if constexpr (PREV != 0 && n >= ST0 && n < ST0 + 4) {
        constexpr int nf = n - ST0;
        if constexpr (nf == 0) {
          asm volatile("" : "+v"(g.tmov));
          unsigned bad = __builtin_amdgcn_readfirstlane(g.badv) | (__builtin_amdgcn_readfirstlane(g.tmov) == g.call ? 1u : 0u);
          if (bad) g.scale = __builtin_nanf("");
        }
        v4f v = {g.own[nf][0] * g.scale, g.own[nf][1] * g.scale, g.own[nf][2] * g.scale, g.own[nf][3] * g.scale};
        asm volatile("buffer_store_dwordx4 %0, %1, %2, %3 offen offset:%4 " OST :: "v"(v), "v"(g.vo_out), "s"(g.rs_out), "s"(g.out_prev), "i"(nf * 64) : "memory");
      }
      if constexpr (PREV == 5 && n == ST0 + 4) {
        v4f t = (g.rd[0] + g.rd[1]) + (g.rd[2] + g.rd[3]);
        v4f v = {t[0] * g.scale, t[1] * g.scale, t[2] * g.scale, t[3] * g.scale};
        asm volatile("buffer_store_dwordx4 %0, %1, %2, %3 offen " OST :: "v"(v), "v"(g.vo_out), "s"(g.rs_out), "s"(g.out5) : "memory");
      }
    };
    if constexpr (PREV == 5 && n == 8) {
      asm volatile("s_waitcnt lgkmcnt(0)" : "+v"(g.rd[0]), "+v"(g.rd[1]), "+v"(g.rd[2]), "+v"(g.rd[3]));
    }

    sfor<4>([&](auto fc) {
      constexpr int nf = decltype(fc)::value;
      if constexpr (i == 0) MFMA0(accp[nf], g.Wf[i][nf], xf); else MFMA1(accp[nf], g.Wf[i][nf], xf);
      side(fc);
      if constexpr (nf == 0) load_half(std::integral_constant<int, 0>{});
      if constexpr (nf == 1) store_out();
      if constexpr (nf == 2) load_half(std::integral_constant<int, 1>{});
    });
  });
}

struct U0 {
  unsigned vo_w, vo_x, ra_w, ra_x0, ra_x1;
  unsigned so_w, lds_w, lds_x;
  rsrc_t rs_w;
};
struct U0S {
  static constexpr int NG = 40;
  static constexpr int w_issue(int n, int k) {
    int m = -1;
    if (n == 0) m = 3 + k;
    else if (k == 0 && n % 5 == 3) m = 2 * (n / 5) + 5;
    else if (k == 0 && n % 5 == 0) m = 2 * (n / 5) + 4;
    return m <= 15 ? m : -1;
  }
  static constexpr int w_read(int n) {
    int m = n % 5 == 2 ? 2 * (n / 5 + 1) : (n % 5 == 4 ? 2 * (n / 5 + 1) + 1 : -1);
    return m <= 15 ? m : -1;
  }
  static constexpr bool remote(int n) { return n - 1 + R >= NG; }
  static constexpr int wait_cnt(int n) {
    int pos = 0, pl[NG + 8] = {}, pw[16] = {};
    pos += 1;
    pos += 2; pl[0] = pos; pos += 2; pl[1] = pos;
    pos += 4; pw[0] = pos; pos += 4; pw[1] = pos;
    pos += 5;
    pos += 2; pl[2] = pos; pos += 2; pl[3] = pos;
    pos += 4; pw[2] = pos;
    if (n < 0) return pos - pw[1];
    for (int q = 0; q <= n; ++q) {
      for (int k = 0; k < 2; ++k) if (w_issue(q, k) >= 0) { pos += 4; pw[w_issue(q, k)] = pos; }
      if (q + 4 < NG) { pos += 2; pl[q + 4] = pos; }
      if (remote(q)) pos += 1;
      if (q < n && remote(q)) pos += 1;
    }
    int need = n + 1 < NG ? pl[n + 1] : 0;
    if (w_read(n) >= 0 && pw[w_read(n)] > need) need = pw[w_read(n)];
    int c = pos - need;
    return c > 63 ? 63 : c;
  }
};
template <int M> __device__ __forceinline__ void u0_w_issue(const U0& u) {
  sfor<4>([&](auto qc) {
    constexpr int q = decltype(qc)::value;
    dma16(u.vo_w, u.rs_w, u.so_w + (unsigned)((M >> 1) * 512 + (M & 1) * 64 + q * 16) * NFULL, u.lds_w + (M % 3) * 4096 + q * 1024);
  });
}
template <int N> __device__ __forceinline__ void u0_x_issue(const G& g, const U0& u) {
  dma16(u.vo_x, g.rs_x, g.so_loc[N % 5] + (N / 5) * 512, u.lds_x + (N % 5) * 2048);
  dma16(u.vo_x, g.rs_x, g.so_loc[N % 5] + (N / 5) * 512 + 8 * KDIM, u.lds_x + (N % 5) * 2048 + 1024);
}
template <int M> __device__ __forceinline__ void u0_w_read(G& g, const U0& u) {
#pragma unroll
  for (int tt = 0; tt < 2; ++tt)
#pragma unroll
    for (int nf = 0; nf < 4; ++nf) {
      v2i q = __builtin_amdgcn_ds_read_tr8_b64_v2i32((LAS v2i*)(uintptr_t)(u.ra_w + (M % 3) * 4096 + tt * 512 + nf * 16));
      g.Wf[M >> 1][nf][(M & 1) * 4 + tt * 2] = q[0];
      g.Wf[M >> 1][nf][(M & 1) * 4 + tt * 2 + 1] = q[1];
    }
}
template <int N> __device__ __forceinline__ void u0_x_read(G& g, const U0& u) {
  asm volatile("ds_read_b128 %0, %1 offset:%2" : "=v"(g.X[N % R][0]) : "v"(u.ra_x0), "i"((N % 5) * 2048) : "memory");
  asm volatile("ds_read_b128 %0, %1 offset:%2" : "=v"(g.X[N % R][1]) : "v"(u.ra_x1), "i"((N % 5) * 2048) : "memory");
}

__device__ __forceinline__ void unit0_body(G& g, U0& u) {
  constexpr int NG = U0S::NG;
  sfor<NG>([&](auto nc) {
    constexpr int n = decltype(nc)::value;
    constexpr int i = n / 5, j = n % 5;
    constexpr int slot = n % R, pslot = (n + R - 1) % R;
    constexpr int t = n - 1 + R, ti = U0S::remote(n) ? (t - NG) / 4 : 0, tj = U0S::remote(n) ? (t - NG) % 4 : 0;

    asm volatile("s_waitcnt lgkmcnt(0)" : "+v"(g.X[slot][0]), "+v"(g.X[slot][1]) :: "memory");
    if constexpr (U0S::w_issue(n, 0) >= 0) u0_w_issue<U0S::w_issue(n, 0)>(u);
    if constexpr (U0S::w_issue(n, 1) >= 0) u0_w_issue<U0S::w_issue(n, 1)>(u);
    if constexpr (n + 4 < NG) u0_x_issue<n + 4>(g, u);
    if constexpr (U0S::remote(n) && tj == 0) {
      if (!(g.mask & 1ull)) slow_wait(g);
      g.mask >>= 1;
      g.kidx += 1;
    }
    v8i xf = {g.X[slot][0][0], g.X[slot][0][1], g.X[slot][0][2], g.X[slot][0][3],
              g.X[slot][1][0], g.X[slot][1][1], g.X[slot][1][2], g.X[slot][1][3]};
    v4f* accp = (j < 4) ? g.acc[j < 4 ? j : 0] : g.acc4;
    sfor<4>([&](auto fc) {
      constexpr int nf = decltype(fc)::value;
      if constexpr (i == 0) MFMA0(accp[nf], g.Wf[i][nf], xf); else MFMA1(accp[nf], g.Wf[i][nf], xf);
      if constexpr (n == NG - 3) asm volatile("ds_write_b128 %0, %1 offset:%2" :: "v"(g.wa[0]), "v"(g.acc[0][nf]), "i"(nf * 1024) : "memory");
      if constexpr (n == NG - 2) asm volatile("ds_write_b128 %0, %1 offset:%2" :: "v"(g.wa[1]), "v"(g.acc[1][nf]), "i"(nf * 1024) : "memory");
      if constexpr (n == NG - 1) asm volatile("ds_write_b128 %0, %1 offset:%2" :: "v"(g.wa[2]), "v"(g.acc[2][nf]), "i"(nf * 1024) : "memory");
      if constexpr (U0S::remote(n) && (nf == 0 || nf == 2))
        ld_stg<(nf / 2) * 1024>(g.X[pslot][nf / 2], g.vo_stg, g.rs_stg, g.ub_nxt + ti * 32768 + g.fo[tj]);
      if constexpr (nf == 1 && n + 1 < NG) {
        asm volatile("s_waitcnt vmcnt(%0)" :: "i"(U0S::wait_cnt(n)) : "memory");
        u0_x_read<n + 1>(g, u);
        if constexpr (U0S::w_read(n) >= 0) u0_w_read<U0S::w_read(n)>(g, u);
      }
    });
  });
}

__device__ __forceinline__ void next_unit(G& g, unsigned out_next) {
  g.out_prev = g.out_cur;
  g.out_cur = out_next;
  g.ub_cur = g.ub_nxt;
  g.ub_nxt += UNIT_BYTES;
}
__device__ __forceinline__ void flip_red(G& g) {
  g.wa[0] ^= 0x10000u; g.wa[1] ^= 0x10000u; g.wa[2] ^= 0x10000u;
}

__device__ __forceinline__ void gemm_role(const Params& p, unsigned char* smem_generic) {
  LAS unsigned char* smem = (LAS unsigned char*)smem_generic;
  const int b = blockIdx.x;
  const int xcd = b & 7, slot = (b >> 3) - (NCOMM >> 3);
  const int pi = slot >> 2, cb = xcd * 4 + (slot & 3);
  const int tid = threadIdx.x, lane = tid & 63;
  const int w = __builtin_amdgcn_readfirstlane(tid >> 6);
  const int kg = lane >> 4, li = lane & 15;
  const int rank = p.rank;
  unsigned char* myws = p.ws[rank];
  const int src = (rank + 1 + pi) & 7;
  G g;
  GSTAMP(0);

  unsigned ticket = 0;
  if (lane == 0)
    asm volatile("global_atomic_add %0, %1, %2, off sc0 sc1" : "+v"(ticket) : "v"((unsigned*)(myws + CTRL_CALL + (b * 4 + w) * 32)), "v"(1u) : "memory");
  const int row0 = pi * 80 < 432 ? pi * 80 : 432;
  g.rs_x = mk_rsrc(p.x, SRC_BYTES);
  sfor<4>([&](auto jc) { constexpr int j = decltype(jc)::value; g.so_loc[j] = (unsigned)(row0 + ((j + 1 + w) & 3) * 16) * KDIM + w * 128; });
  g.so_loc[4] = (unsigned)(row0 + 64) * KDIM + w * 128;
  U0 u;
  {
    const unsigned base = (unsigned)(uintptr_t)smem + LDS_STG + w * STG_WAVE;
    u.rs_w = mk_rsrc(p.w, (unsigned)KDIM * NFULL);
    u.vo_w = (lane >> 2) * NFULL + (lane & 3) * 16;
    u.vo_x = (lane >> 3) * KDIM + ((lane & 7) ^ (lane >> 3)) * 16;
    u.so_w = (unsigned)rank * NLOC + cb * 64 + (unsigned)(w * 128) * NFULL;
    u.lds_w = base;
    u.lds_x = base + STG_X;
    u.ra_w = base + (kg * 16 + (li >> 1)) * 64 + (li & 1) * 8;
    u.ra_x0 = base + STG_X + (li >> 3) * 1024 + (li & 7) * 128 + ((kg ^ (li & 7)) * 16);
    u.ra_x1 = u.ra_x0 ^ 64u;
  }
  u0_x_issue<0>(g, u); u0_x_issue<1>(g, u);
  u0_w_issue<0>(u); u0_w_issue<1>(u);
  unsigned pfd[5];
  prefetch_touch(p, row0, pfd);
  u0_x_issue<2>(g, u); u0_x_issue<3>(g, u);
  u0_w_issue<2>(u);
  GSTAMP(12);
  g.lds_bad = (LAS unsigned*)(smem + LDS_MISC + 4);
  if (tid == 0) asm volatile("ds_write_b32 %0, %1" :: "v"((unsigned)(uintptr_t)smem + LDS_MISC + 4), "v"(0u) : "memory");
  asm volatile("s_waitcnt vmcnt(%1) lgkmcnt(0)\n\ts_barrier" : "+v"(ticket) : "i"(U0S::wait_cnt(-1)) : "memory");
  g.call = __builtin_amdgcn_readfirstlane(ticket) + 1u;
  GSTAMP(13);
  u0_x_read<0>(g, u);
  u0_w_read<0>(g, u); u0_w_read<1>(g, u);

  g.scale = *(const CAS float*)p.sx * *(const CAS float*)p.sw;
  g.rs_stg = mk_rsrc(myws + (g.call & 1u) * STAGE_BYTES, STAGE_BYTES);
  g.rs_out = mk_rsrc(p.out, (unsigned)(4096u * NLOC * 4u));
  g.flag_ptr = (const GAS unsigned long long*)(myws + CTRL_FLAG + src * 2048 + w * 512 + lane * 8);
  g.tmo = (GAS unsigned*)(myws + CTRL_TMO);
  g.rs_tmo = mk_rsrc(myws + CTRL_TMO, 4);
  g.vo_stg = lane * 16;
  g.vo_out = li * (NLOC * 4) + kg * 16;
  sfor<4>([&](auto jc) { constexpr int j = decltype(jc)::value; g.fo[j] = ((j + 1 + w) & 3) * 2048; });
  sfor<3>([&](auto jc) { constexpr int j = decltype(jc)::value; g.wa[j] = lane * 16 + ((j + 1 + w) & 3) * 12288 + j * 4096; });
  g.ra = lane * 16 + w * 12288;
  g.wa5 = 49152 + w * 1024 + lane * 16;
  g.ra5 = 49152 + w * 4096 + lane * 16;
  const unsigned colb = cb * 256;
  auto out_off = [&](int grow) -> unsigned { return (unsigned)(grow + w * 16) * (NLOC * 4) + colb; };
  g.out_cur = out_off(rank * MLOC + row0);
  g.out_prev = g.out_cur;
  g.out5 = (unsigned)(rank * MLOC + row0 + 64) * (NLOC * 4) + colb + w * 64;
  g.ub_cur = 0;
  g.ub_nxt = (unsigned)src * SRC_BYTES + w * 8192;
  g.mask = 0ull;
  g.kidx = 0u;

  GSTAMP(1);
  unit0_body(g, u);
  asm volatile("" :: "v"(pfd[0]), "v"(pfd[1]), "v"(pfd[2]), "v"(pfd[3]), "v"(pfd[4]));
  GSTAMP(2);
  next_unit(g, out_off(src * MLOC));
  flip_red(g);
  unit_body<4, 40 % R, 5, true>(g);
  GSTAMP(3);
  g.ra ^= 0x10000u;
  for (int u = 2; u < 8; ++u) {
    next_unit(g, out_off(src * MLOC + (u - 1) * 64));
    g.wa[0] ^= 0x10000u; g.wa[1] ^= 0x10000u;
    unit_body<4, 40 % R, 4, true>(g);
    g.wa[2] ^= 0x10000u;
    g.ra ^= 0x10000u;
  }
  next_unit(g, out_off(src * MLOC + 7 * 64));
  g.wa[0] ^= 0x10000u; g.wa[1] ^= 0x10000u;
  unit_body<4, 40 % R, 4, false>(g);
  GSTAMP(10);
  g.wa[2] ^= 0x10000u;
  g.ra ^= 0x10000u;

  asm volatile("s_nop 7\n\ts_nop 7" : "+v"(g.acc[3][0]), "+v"(g.acc[3][1]), "+v"(g.acc[3][2]), "+v"(g.acc[3][3]));
#pragma unroll
  for (int q = 0; q < 4; ++q) asm volatile("ds_write_b128 %0, %1 offset:%2" :: "v"(g.wa[2]), "v"(g.acc[2][q]), "i"(q * 1024) : "memory");
  unsigned tv = __hip_atomic_load(g.tmo, __ATOMIC_RELAXED, __HIP_MEMORY_SCOPE_AGENT);
  asm volatile("s_waitcnt lgkmcnt(0)\n\ts_barrier" ::: "memory");
  {
    LAS v4f* rp = (LAS v4f*)(smem + g.ra);
    unsigned bad = __builtin_amdgcn_readfirstlane(*g.lds_bad) | (__builtin_amdgcn_readfirstlane(tv) == g.call ? 1u : 0u);
    float sc = bad ? __builtin_nanf("") : g.scale;
#pragma unroll
    for (int nf = 0; nf < 4; ++nf) {
      v4f v = g.acc[3][nf];
      v += rp[(0 * 4 + nf) * 64];
      v += rp[(1 * 4 + nf) * 64];
      v += rp[(2 * 4 + nf) * 64];
      v = (v4f){v[0] * sc, v[1] * sc, v[2] * sc, v[3] * sc};
      asm volatile("buffer_store_dwordx4 %0, %1, %2, %3 offen offset:%4 " OST :: "v"(v), "v"(g.vo_out), "s"(g.rs_out), "s"(g.out_cur), "i"(nf * 64) : "memory");
    }
  }
}

#define CSTAMP(n) do {} while (0)
template <bool LEADER>
__device__ __forceinline__ void comm_wave(const Params& p, LAS unsigned char* smem, const unsigned call, const int j, const int w, const int lane, v4i d) {
  const int rank = p.rank;
  unsigned char* myws = p.ws[rank];
  const int cw = j * 4 + w;
  rsrc_t rs_x = mk_rsrc(p.x, SRC_BYTES);
  const unsigned vo = (unsigned)((w >> 1) * 16 + (lane & 15)) * KDIM + (w & 1) * 64 + (lane >> 4) * 16;
  const unsigned dst0 = (call & 1u) * STAGE_BYTES + (unsigned)rank * SRC_BYTES + (unsigned)w * 1024u;
  rsrc_t rs[7];
#pragma unroll
  for (int q = 0; q < 7; ++q) {
    int pq = (rank + 1 + (q + cw) % 7) & 7;
    rs[q] = mk_rsrc(p.ws[pq], 2u * STAGE_BYTES);
  }
  unsigned char* flagb = p.ws[(rank + 1 + (lane < 7 ? lane : 0)) & 7] + CTRL_FLAG + rank * 2048;
  unsigned* ticketp = (unsigned*)(myws + CTRL_TICKET);
  GAS unsigned* tmo = (GAS unsigned*)(myws + CTRL_TMO);
  const unsigned tbase = (call - 1u) * TICKETS_PER_CALL - 2u * NCOMM;
  const unsigned vs = lane * 16;
  LAS unsigned* tk = (LAS unsigned*)(smem + 1024);
  LAS unsigned* cnt = (LAS unsigned*)(smem + LDS_MISC + 64);

  auto publish = [&](unsigned n, unsigned t) {
    unsigned old = 0;
    if (lane == 0) old = __hip_atomic_fetch_add(cnt + (n & 7u), 1u, __ATOMIC_RELAXED, __HIP_MEMORY_SCOPE_WORKGROUP);
    old = __builtin_amdgcn_readfirstlane(old);
    if ((old & 3u) == 3u) {
      if (lane < 7) __hip_atomic_store((unsigned*)(flagb + ((t >> 1) & 3u) * 512u + (t >> 3) * 8u + (t & 1u) * 4u), call, __ATOMIC_RELAXED, __HIP_MEMORY_SCOPE_SYSTEM);
      CSTAMP(n + 1);
    }
  };

  unsigned t_cur = (unsigned)j, t_prev = 0;
  unsigned n = 0;
  while (true) {
    v4i dn;
    unsigned tick = 0, t_nxt = 0;
    const unsigned dst = dst0 + t_cur * 4096u;
    sfor<7>([&](auto qc) {
      constexpr int q = decltype(qc)::value;
      asm volatile("s_waitcnt vmcnt(%0)" :: "i"(COMM_W - 1) : "memory");
      if constexpr (q == COMM_W - 1) { if (n > 0) publish(n - 1, t_prev); }
      if constexpr (LEADER && q == 0) {
        if (n > 0 && lane == 0) asm volatile("global_atomic_add %0, %1, %2, off sc0 sc1" : "+v"(tick) : "v"(ticketp), "v"(1u) : "memory");
      }
      if constexpr (q == 5) {
        if constexpr (LEADER) {
          asm volatile("" : "+v"(tick));
          t_nxt = n == 0 ? (unsigned)(NCOMM + j) : __builtin_amdgcn_readfirstlane(tick) - tbase;
          if (lane == 0) tk[n + 1] = t_nxt + 1u;
        } else {
          unsigned long long t0 = 0;
          while (true) {
            unsigned v = __hip_atomic_load(tk + n + 1, __ATOMIC_RELAXED, __HIP_MEMORY_SCOPE_WORKGROUP);
            v = __builtin_amdgcn_readfirstlane(v);
            if (v) { t_nxt = v - 1u; break; }
            unsigned long long now = __builtin_amdgcn_s_memrealtime();
            if (!t0) t0 = now;
            if (now - t0 > DEADLINE_TICKS) {
              __hip_atomic_store(tmo, call, __ATOMIC_RELAXED, __HIP_MEMORY_SCOPE_AGENT);
              t_nxt = NPIECE; break;
            }
            __builtin_amdgcn_s_sleep(1);
          }
        }
        if (t_nxt < NPIECE)
          asm volatile("buffer_load_dwordx4 %0, %1, %2, %3 offen" : "=v"(dn) : "v"(vo), "s"(rs_x),
                       "s"((t_nxt >> 6) * UNIT_BYTES + (t_nxt & 1u) * (32u * KDIM) + ((t_nxt >> 1) & 31u) * 128u) : "memory");
      }
      st_sys(d, vs, rs[q], dst);
    });
    if (t_nxt >= NPIECE) break;
    asm volatile("s_waitcnt vmcnt(2)" : "+v"(dn) :: "memory");
    t_prev = t_cur; t_cur = t_nxt; d = dn; ++n;
  }
  asm volatile("s_waitcnt vmcnt(0)" ::: "memory");
  publish(n, t_cur);
}

__device__ __forceinline__ void comm_role(const Params& p, unsigned char* smem_generic) {
  LAS unsigned char* smem = (LAS unsigned char*)smem_generic;
  const int j = blockIdx.x;
  const int tid = threadIdx.x, lane = tid & 63;
  const int w = __builtin_amdgcn_readfirstlane(tid >> 6);
  unsigned char* myws = p.ws[p.rank];

  STAMP(myws, 0);
  unsigned ticket = 0;
  if (tid == 0) ticket = __hip_atomic_fetch_add((unsigned*)(myws + CTRL_CALL + blockIdx.x * 128), 1u, __ATOMIC_RELAXED, __HIP_MEMORY_SCOPE_AGENT);
  v4i d;
  {
    rsrc_t rs_x = mk_rsrc(p.x, SRC_BYTES);
    const unsigned vo = (unsigned)((w >> 1) * 16 + (lane & 15)) * KDIM + (w & 1) * 64 + (lane >> 4) * 16;
    asm volatile("buffer_load_dwordx4 %0, %1, %2, %3 offen" : "=v"(d) : "v"(vo), "s"(rs_x), "s"((unsigned)((j & 1) * (32 * KDIM) + (j >> 1) * 128)) : "memory");
  }
  for (int i = tid; i < 520; i += 256) *(LAS unsigned*)(smem + 1024 + i * 4) = 0u;
  if (tid == 0) *(LAS unsigned*)(smem + LDS_MISC) = ticket + 1u;
  if (tid < 8) *(LAS unsigned*)(smem + LDS_MISC + 64 + tid * 4) = 0u;
  __syncthreads();
  const unsigned call = __builtin_amdgcn_readfirstlane(*(LAS unsigned*)(smem + LDS_MISC));
  asm volatile("s_waitcnt vmcnt(0)" : "+v"(d) :: "memory");
  if (w == 0) comm_wave<true>(p, smem, call, j, w, lane, d);
  else comm_wave<false>(p, smem, call, j, w, lane, d);
}

__global__ void __launch_bounds__(256, 1) fused_ag_gemm(Params p) {
  extern __shared__ __attribute__((aligned(16))) unsigned char smem_dyn[];
  if (blockIdx.x < NCOMM) comm_role(p, smem_dyn);
  else gemm_role(p, smem_dyn);
}

}

extern "C" void kernel_launch(int rank, int nranks, ncclComm_t comm, hipStream_t stream,
                              void* const* d_in, const int* in_sizes, int n_in,
                              void* d_out, int out_size,
                              const int* in_dtypes, int out_dtype,
                              void* const* d_out_all,
                              void* const* d_ws_all, size_t ws_bytes) {
  if (nranks != NRANKS || n_in < 4 || ws_bytes < WS_NEEDED || (unsigned)rank >= (unsigned)NRANKS) {
    fprintf(stderr, "unsupported configuration\n");
    return;
  }
  static bool attr_done = false;
  if (!attr_done) {
    HIP_CHECK(hipFuncSetAttribute((const void*)fused_ag_gemm, hipFuncAttributeMaxDynamicSharedMemorySize, (int)LDS_BYTES));
    attr_done = true;
  }
  Params p;
  p.x = (const unsigned char*)d_in[0];
  p.w = (const unsigned char*)d_in[1];
  p.sx = (const float*)d_in[2];
  p.sw = (const float*)d_in[3];
  p.out = (float*)d_out;
  for (int i = 0; i < NRANKS; ++i) p.ws[i] = (unsigned char*)d_ws_all[i];
  p.rank = rank;
  hipLaunchKernelGGL(fused_ag_gemm, dim3(256), dim3(256), LDS_BYTES, stream, p);
  HIP_CHECK(hipGetLastError());
}
```

```cpp
#include <hip/hip_runtime.h>
#include <rccl/rccl.h>
#include <cstdio>
#include <utility>

#define HIP_CHECK(expr) do { hipError_t _e = (expr); if (_e != hipSuccess) { \
    fprintf(stderr, "HIP error %s at %s:%d\n", hipGetErrorString(_e), __FILE__, __LINE__); \
    return; } } while (0)

namespace {

typedef int v8i __attribute__((ext_vector_type(8)));
typedef int v4i __attribute__((ext_vector_type(4)));
typedef int v2i __attribute__((ext_vector_type(2)));
typedef unsigned v2u __attribute__((ext_vector_type(2)));
typedef float v4f __attribute__((ext_vector_type(4)));
#define LAS __attribute__((address_space(3)))
#define GAS __attribute__((address_space(1)))
#define CAS __attribute__((address_space(4)))
typedef __amdgpu_buffer_rsrc_t rsrc_t;

constexpr int NRANKS = 8;
constexpr int KDIM = 4096, NFULL = 16384, NLOC = 2048, MLOC = 512;
constexpr unsigned UNIT_BYTES = 64u * KDIM;
constexpr unsigned SRC_BYTES = (unsigned)MLOC * KDIM;
constexpr unsigned STAGE_BYTES = NRANKS * SRC_BYTES;
constexpr unsigned NSTAGE = 3;
constexpr unsigned CTRL = NSTAGE * STAGE_BYTES;
constexpr unsigned CTRL_FLAG = CTRL;
constexpr unsigned CTRL_TMO = CTRL + 16384;
constexpr unsigned CTRL_TICKET = CTRL + 16384 + 128;
constexpr unsigned CTRL_CALL = CTRL + 20480;
constexpr unsigned WS_NEEDED = CTRL_CALL + 1024u * 32u;
constexpr int NCOMM = 32;
constexpr unsigned NPIECE = 512;
constexpr unsigned TICKETS_PER_CALL = NPIECE - 2 * NCOMM + NCOMM;
constexpr unsigned long long DEADLINE_TICKS = 100000000ull;

#define RING 8
#define COMM_P 1
#define MAPPING 0
#define XG 2
#define IB_U 6
#define IB_SLEEP 1
#define MK_N 10
#define MKST "sc0 sc1"
#define COMM_W 4
#define SLEEPN 2
#define PF_MODE 0
#define POLLW 8
#define OSTM 0
#define OST ""
#define XLDM 3
#define XLD "sc0 sc1"
constexpr int R = RING;

#define WD 4
#define XD 5
constexpr unsigned RED_ROW = 272, RED_TILE = 16 * RED_ROW, RED_WAVE = 4 * RED_TILE;
constexpr unsigned PART = 39424, LDS_T5 = 2 * RED_WAVE, STG_X = WD * 4096;
constexpr unsigned LDS_MISC = 4 * PART;
constexpr unsigned LDS_BYTES = LDS_MISC + 256;
static_assert(PART % 128 == 0 && LDS_T5 + RED_TILE <= PART && STG_X + XD * 2048 <= PART && LDS_BYTES <= 160 * 1024, "LDS layout");
static_assert(STG_X >= 2 * RED_TILE, "tiles 0 and 1 of unit 0 are written while the last x slots are in use");

struct Params {
  const unsigned char* x;
  const unsigned char* w;
  const float* sx;
  const float* sw;
  float* out;
  unsigned char* ws[NRANKS];
  int rank;
};

#define STAMP(ws, idx) do {} while (0)
#define GSTAMP(idx) do {} while (0)

template <class F, int... I> __device__ __forceinline__ void sfor_impl(F&& f, std::integer_sequence<int, I...>) {
  (f(std::integral_constant<int, I>{}), ...);
}
template <int N, class F> __device__ __forceinline__ void sfor(F&& f) { sfor_impl(f, std::make_integer_sequence<int, N>{}); }

__device__ __forceinline__ rsrc_t mk_rsrc(const void* p, unsigned bytes) {
  return __builtin_amdgcn_make_buffer_rsrc((void*)p, 0, bytes, 0x00020000);
}

struct G {
  v8i Wf[8][4];
  v4f rd[4];
  unsigned long long mask;
  unsigned kidx;
  unsigned tmov;
  unsigned badv;
  unsigned vo_loc, vo_stg, vo_rh, vo_out, wa_cur, wa_prev, ra[4], wa5, ra5;
  rsrc_t rs_x, rs_stg, rs_out, rs_tmo, rs_mk;
  unsigned mk_off, rh_off;
  unsigned so_loc[5];
  unsigned fo[4];
  unsigned ub_cur, ub_nxt;
  unsigned out_prev, out_cur;
  unsigned out5;
  unsigned red_d;
  unsigned call, unit;
  bool ib;
  bool ib_cur, ib_nxt;
  unsigned kbase;
  float scale;
  const GAS unsigned long long* flag_ptr;
  GAS unsigned* tmo;
  LAS unsigned* lds_bad;
};

constexpr int XB = 256 - 8 * R, AB = XB - 80, HD = AB - 1, FD = HD - 3, A4B = FD;
constexpr int acc_reg(int j, int nf) { return AB + 16 * j + 4 * nf; }

template <int J, int NF_, int SLOT, bool FIRST> __device__ __forceinline__ void mfma(const v8i& wf) {
  constexpr int A = acc_reg(J, NF_), X = XB + 8 * SLOT;
  if constexpr (FIRST)
    asm volatile("v_mfma_f32_16x16x128_f8f6f4 v[%c1:%c2], %0, v[%c3:%c4], 0 cbsz:1" :: "a"(wf), "i"(A), "i"(A + 3), "i"(X), "i"(X + 7));
  else
    asm volatile("v_mfma_f32_16x16x128_f8f6f4 v[%c1:%c2], %0, v[%c3:%c4], v[%c1:%c2] cbsz:1" :: "a"(wf), "i"(A), "i"(A + 3), "i"(X), "i"(X + 7));
}
template <int SLOT, int H> __device__ __forceinline__ void ld_stg(unsigned vo, rsrc_t rs, unsigned so) {
  constexpr int X = XB + 8 * SLOT + 4 * H;
  asm volatile("buffer_load_dwordx4 v[%c3:%c4], %0, %1, %2 offen offset:%c5 " XLD :: "v"(vo), "s"(rs), "s"(so), "i"(X), "i"(X + 3), "i"(H * 1024) : "memory");
}
template <int CNT> __device__ __forceinline__ void wait_vm() { asm volatile("s_waitcnt vmcnt(%c0)" :: "i"(CNT) : "memory"); }

__device__ __forceinline__ void st_sys(const v4i& d, unsigned vo, rsrc_t rs, unsigned so) {
  asm volatile("buffer_store_dwordx4 %0, %1, %2, %3 offen sc0 sc1" :: "v"(d), "v"(vo), "s"(rs), "s"(so) : "memory");
}

__device__ __forceinline__ void dma16(unsigned vo, rsrc_t rs, unsigned so, unsigned lds) {
  asm volatile("s_mov_b32 m0, %3\n\tbuffer_load_dwordx4 %0, %1, %2 offen lds" :: "v"(vo), "s"(rs), "s"(so), "s"(lds) : "memory");
}

__device__ __forceinline__ void touch(const unsigned char* a) {
  asm volatile("global_load_dword v%c1, %0, off" :: "v"(a), "i"(HD) : "memory");
}

#define HS0 0
#define XS0 0
constexpr int HW_LINES = (8 - HS0) * 512 * 2 * XG;
constexpr int NHW = (HW_LINES + 112 * 64 - 1) / (112 * 64);
constexpr int HW_LPI = (HW_LINES + 112 * NHW - 1) / (112 * NHW);
constexpr int HX_LINES = (8 - XS0) * 320, HX_WAVES = 16 * XG;
constexpr int NHX = (HX_LINES + HX_WAVES * 64 - 1) / (HX_WAVES * 64);
constexpr int HX_LPI = (HX_LINES + HX_WAVES * NHX - 1) / (HX_WAVES * NHX);
constexpr int NHINT = NHW + NHX;
struct Hint { const unsigned char* wb; const unsigned char* xb; int wi, xi; };
template <int Q> __device__ __forceinline__ void hint(const Hint& h) {
  const int lane = threadIdx.x & 63;
  if constexpr (Q < NHX) {
    int line = (Q * HX_WAVES + h.xi) * HX_LPI + (lane < HX_LPI ? lane : HX_LPI - 1);
    line = line < HX_LINES ? line : HX_LINES - 1;
    const int r = XS0 + line / 320, rem = line % 320;
    touch(h.xb + (rem >> 2) * KDIM + r * 512 + (rem & 3) * 128);
  } else {
    int line = ((Q - NHX) * 112 + h.wi) * HW_LPI + (lane < HW_LPI ? lane : HW_LPI - 1);
    line = line < HW_LINES ? line : HW_LINES - 1;
    touch(h.wb + (size_t)(HS0 * 512 + line / (2 * XG)) * NFULL + (line % (2 * XG)) * 128);
  }
}

__device__ __forceinline__ void slow_wait(G& g) {
  unsigned long long t0 = __builtin_amdgcn_s_memrealtime();
  const unsigned lane = __lane_id();
  unsigned lo = 0, hi = 64;
  unsigned long long m;
  while (true) {
    unsigned long long v = 0;
    if (lane >= lo && lane < hi) v = __hip_atomic_load(g.flag_ptr, __ATOMIC_RELAXED, __HIP_MEMORY_SCOPE_SYSTEM);
    unsigned tvv = __hip_atomic_load(g.tmo, __ATOMIC_RELAXED, __HIP_MEMORY_SCOPE_AGENT);
    unsigned v0 = (unsigned)v, v1 = (unsigned)(v >> 32);
    asm volatile("" : "+v"(v0), "+v"(v1), "+v"(tvv));
    m = __builtin_amdgcn_ballot_w64((v0 < v1 ? v0 : v1) >= g.call) >> g.kidx;
    if (m & 1ull) break;
    unsigned tv = __builtin_amdgcn_readfirstlane(tvv);
    if (tv == g.call || __builtin_amdgcn_s_memrealtime() - t0 > DEADLINE_TICKS) {
      __hip_atomic_store(g.tmo, g.call, __ATOMIC_RELAXED, __HIP_MEMORY_SCOPE_AGENT);
      *g.lds_bad = 1u;
      m = ~0ull;
      break;
    }
    lo = g.kidx; hi = g.kidx + POLLW;
    __builtin_amdgcn_s_sleep(SLEEPN);
  }
  g.mask = m;
}

template <int OFF, int J, int NF_> __device__ __forceinline__ void lds_wr(unsigned a) {
  constexpr int A = acc_reg(J, NF_);
  asm volatile("ds_write_b128 %0, v[%c1:%c2] offset:%c3" :: "v"(a), "i"(A), "i"(A + 3), "i"(OFF) : "memory");
}
template <int OFF> __device__ __forceinline__ void lds_rd(v4f& v, unsigned a) {
  asm volatile("ds_read_b128 %0, %1 offset:%2" : "=v"(v) : "v"(a), "i"(OFF) : "memory");
}

template <int CNT> __device__ __forceinline__ void lds_wait(v4f (&r)[4], unsigned& b) {
  asm volatile("s_waitcnt lgkmcnt(%5)" : "+v"(r[0]), "+v"(r[1]), "+v"(r[2]), "+v"(r[3]), "+v"(b) : "i"(CNT) : "memory");
}
__device__ __forceinline__ void lds_rd32(unsigned& v, unsigned a) {
  asm volatile("ds_read_b32 %0, %1" : "=v"(v) : "v"(a) : "memory");
}
__device__ __forceinline__ void st_out(const v4f& v, unsigned vo, rsrc_t rs, unsigned so) {
  asm volatile("buffer_store_dwordx4 %0, %1, %2, %3 offen " OST :: "v"(v), "v"(vo), "s"(rs), "s"(so) : "memory");
}

__device__ __forceinline__ void st_mark(unsigned vo, rsrc_t rs, unsigned so) {
  const v4i mk = {-1, -1, -1, -1};
  asm volatile("buffer_store_dwordx4 %0, %1, %2, %3 offen " MKST :: "v"(mk), "v"(vo), "s"(rs), "s"(so) : "memory");
}

#define RH_N -1
#define RH0_N -1
constexpr int NRH = 2 / XG > 0 ? 2 / XG : 1;
__device__ __forceinline__ void touch_stg(unsigned vo, rsrc_t rs, unsigned so) {
  asm volatile("buffer_load_dword v%c3, %0, %1, %2 offen " XLD :: "v"(vo), "s"(rs), "s"(so), "i"(HD) : "memory");
}
__device__ __forceinline__ void rhint(G& g) {
  sfor<NRH>([&](auto qc) { touch_stg(g.vo_rh, g.rs_stg, g.ub_nxt + g.rh_off + decltype(qc)::value * 8192u); });
}

#define PK_A 2
#define PK_B 18
#define PK_0 22
__device__ __forceinline__ void peek_issue(G& g) {
  asm volatile("global_load_dwordx2 v[%c1:%c2], %0, off sc0 sc1" :: "v"(g.flag_ptr), "i"(FD), "i"(FD + 1) : "memory");
}
__device__ __forceinline__ void peek_use(G& g) {
  unsigned long long m;
  unsigned t;
  asm volatile("v_min_u32 %1, v%c3, v%c4\n\tv_cmp_le_u32 %0, %2, %1" : "=s"(m), "=&v"(t) : "s"(g.call), "i"(FD), "i"(FD + 1));
  g.mask |= m >> (g.kidx & 63u);
}

template <int NF, int PREV, bool HAS_NEXT> struct Sched {
  static constexpr int NG = 8 * NF;
  static constexpr bool has_load(int n) { return HAS_NEXT || (n - 1 + R < NG); }
  static constexpr int n_out(int n) { return PREV == 0 ? 0 : ((n >= 5 && n <= 8) ? 1 : 0) + ((PREV == 5 && n == 9) ? 1 : 0); }
  static constexpr bool has_mark(int n) { return PREV == 5 && n >= MK_N && n < MK_N + 2 * (9 - IB_U); }
  static constexpr int n_store(int n) { return n_out(n) + (has_mark(n) ? 1 : 0); }
  static constexpr bool has_tmo(int n) { return HAS_NEXT ? n == 30 - R : n == NG - 1; }
  static constexpr bool has_rh(int n) { return HAS_NEXT && n == RH_N; }
  static constexpr bool has_peek(int n) { return HAS_NEXT && (n == PK_A || n == PK_B); }
  static constexpr bool use_peek(int n) { return has_peek(n - R + 1); }
  static constexpr int ops(int q) {
    if (q < 0) return 2;
    return (has_load(q) ? 2 : 0) + n_store(q) + (has_tmo(q) ? 1 : 0) + (has_rh(q) ? NRH : 0) + (has_peek(q) ? 1 : 0);
  }
  static constexpr int wait_cnt(int n) {
    int c = 0;
    for (int q = n - R + 2; q <= n - 1; ++q) c += ops(q);
    return c > 63 ? 63 : c;
  }
};

template <int SLOT> __device__ __forceinline__ bool ib_dirty() {
  constexpr int X = XB + 8 * SLOT;
  unsigned long long m;
  unsigned t0, t1;
  asm volatile("v_max3_u32 %1, v[%c3], v[%c4], v[%c5]\n\t"
               "v_max3_u32 %2, v[%c6], v[%c7], v[%c8]\n\t"
               "v_max3_u32 %1, v[%c9], v[%c10], %1\n\t"
               "v_max_u32 %1, %1, %2\n\t"
               "v_cmp_eq_u32 %0, -1, %1"
               : "=s"(m), "=&v"(t0), "=&v"(t1)
               : "i"(X), "i"(X + 1), "i"(X + 2), "i"(X + 3), "i"(X + 4), "i"(X + 5), "i"(X + 6), "i"(X + 7));
  return m != 0ull;
}
template <int M, int ROFF> __device__ __forceinline__ void ib_load(G& g) {
  constexpr int slot = (ROFF + M) % R;
  const unsigned so = (M < 32 ? g.ub_cur + (M / 4) * 32768 : g.ub_nxt + ((M - 32) / 4) * 32768) + g.fo[M % 4];
  ld_stg<slot, 0>(g.vo_stg, g.rs_stg, so);
  ld_stg<slot, 1>(g.vo_stg, g.rs_stg, so);
}
template <int N, int ROFF, bool HAS_NEXT> __device__ __forceinline__ void ib_retry(G& g) {
  constexpr int KE = N | 3;
  constexpr int LE = (HAS_NEXT || N + R - 2 < 31) ? N + R - 2 : 31;
  constexpr int slot = (ROFF + N) % R;
  const unsigned long long t0 = __builtin_amdgcn_s_memrealtime();
  const unsigned lane = __lane_id();
  bool flagged = false;
  while (true) {
    unsigned long long v = 0;
    if (lane == g.kbase + N / 4) v = __hip_atomic_load(g.flag_ptr, __ATOMIC_RELAXED, __HIP_MEMORY_SCOPE_SYSTEM);
    unsigned tvv = __hip_atomic_load(g.tmo, __ATOMIC_RELAXED, __HIP_MEMORY_SCOPE_AGENT);
    sfor<KE - N + 1>([&](auto mc) { ib_load<N + decltype(mc)::value, ROFF>(g); });
    unsigned v0 = (unsigned)v, v1 = (unsigned)(v >> 32);
    asm volatile("s_waitcnt vmcnt(0)" : "+v"(v0), "+v"(v1), "+v"(tvv) :: "memory");
    if (flagged || !ib_dirty<slot>()) break;
    flagged = __builtin_amdgcn_ballot_w64((v0 < v1 ? v0 : v1) >= g.call) != 0ull;
    unsigned tv = __builtin_amdgcn_readfirstlane(tvv);
    if (tv == g.call || __builtin_amdgcn_s_memrealtime() - t0 > DEADLINE_TICKS) {
      __hip_atomic_store(g.tmo, g.call, __ATOMIC_RELAXED, __HIP_MEMORY_SCOPE_AGENT);
      *g.lds_bad = 1u;
      break;
    }
    if (!flagged) __builtin_amdgcn_s_sleep(IB_SLEEP);
  }
  if constexpr (LE > KE) sfor<LE - KE>([&](auto mc) { ib_load<KE + 1 + decltype(mc)::value, ROFF>(g); });
}

template <int NF, int ROFF, int PREV, bool HAS_NEXT>
__device__ __forceinline__ void unit_body(G& g) {
  using S = Sched<NF, PREV, HAS_NEXT>;
  constexpr int NG = S::NG;
  sfor<NG>([&](auto nc) {
    constexpr int n = decltype(nc)::value;
    constexpr int i = n / NF, j = n % NF;
    constexpr int slot = (ROFF + n) % R;
    constexpr int pslot = (ROFF + n + R - 1) % R;
    constexpr int t = n - 1 + R;
    constexpr bool t_next = t >= NG;
    constexpr int ti = t_next ? (t - NG) / 4 : t / NF, tj = t_next ? (t - NG) % 4 : t % NF;

    wait_vm<S::wait_cnt(n)>();
    if constexpr (IB_U < 9 && PREV == 4) {
      if (__builtin_expect(ib_dirty<slot>(), 0)) ib_retry<n, ROFF, HAS_NEXT>(g);
    }

    if constexpr (PREV != 0 && n == 4) {
      asm volatile("s_waitcnt lgkmcnt(0)\n\ts_barrier" ::: "memory");
    }
    v4f ov;
    if constexpr (S::n_out(n) != 0) {
      asm volatile("s_waitcnt lgkmcnt(0)" : "+v"(g.rd[0]), "+v"(g.rd[1]), "+v"(g.rd[2]), "+v"(g.rd[3]), "+v"(g.badv));
      if constexpr (n == 5) {
        unsigned bad = __builtin_amdgcn_readfirstlane(g.badv) | (__builtin_amdgcn_readfirstlane(g.tmov) == g.call ? 1u : 0u);
        if (bad) g.scale = __builtin_nanf("");
      }
      const v4f sum = (g.rd[0] + g.rd[1]) + (g.rd[2] + g.rd[3]);
      const float sc = g.scale;
      ov = (v4f){sum[0] * sc, sum[1] * sc, sum[2] * sc, sum[3] * sc};
      asm volatile("" : "+v"(ov));
    }
    if constexpr (S::use_peek(n)) peek_use(g);
    if constexpr (S::has_load(n) && tj == 0) {
      if (!(t_next ? g.ib_nxt : g.ib_cur)) {
        if (!(g.mask & 1ull)) slow_wait(g);
        g.mask >>= 1;
        g.kidx += 1;
      }
    }
    if constexpr (S::has_rh(n)) rhint(g);
    if constexpr (S::has_peek(n)) peek_issue(g);
    if constexpr (S::has_tmo(n)) {
      asm volatile("buffer_load_dword %0, off, %1, 0 sc1" : "=v"(g.tmov) : "s"(g.rs_tmo) : "memory");
    }

    auto load_half = [&](auto hc) {
      constexpr int h = decltype(hc)::value;
      if constexpr (S::has_load(n))
        ld_stg<pslot, h>(g.vo_stg, g.rs_stg, (t_next ? g.ub_nxt : g.ub_cur) + ti * 32768 + g.fo[tj]);
    };
    auto side = [&](auto qc) {
      constexpr int q = decltype(qc)::value;
      if constexpr (n == NG - 2) lds_wr<0 * RED_TILE + q * 64, 0, q>(g.wa_cur);
      if constexpr (n == NG - 1) lds_wr<1 * RED_TILE + q * 64, 1, q>(g.wa_cur);
      if constexpr (PREV == 4 && n == 0) lds_wr<2 * RED_TILE + q * 64, 2, q>(g.wa_prev);
      if constexpr (PREV == 4 && n == 1) lds_wr<3 * RED_TILE + q * 64, 3, q>(g.wa_prev);
      if constexpr (PREV == 5 && n == 0) lds_wr<3 * RED_TILE + q * 64, 3, q>(g.wa_prev);
      if constexpr (PREV == 5 && n == 1) lds_wr<q * 64, 4, q>(g.wa5);
      if constexpr (PREV != 0 && n >= 4 && n <= 7) lds_rd<(n - 4) * 4 * RED_ROW>(g.rd[q], g.ra[q]);
      if constexpr (PREV != 0 && n == 4 && q == 0)
        asm volatile("ds_read_b32 %0, %1" : "=v"(g.badv) : "v"((unsigned)(LDS_MISC + 4)) : "memory");
      if constexpr (PREV == 5 && n == 8) lds_rd<(q & 1) * PART>(g.rd[q], q < 2 ? g.ra5 : g.ra5 + 2 * PART);
    };
    auto store_out = [&]() {
      if constexpr (S::n_out(n) != 0) {
        const unsigned so = n < 9 ? g.out_prev + (n - 5) * 4 * (NLOC * 4) : g.out5;
        st_out(ov, g.vo_out, g.rs_out, so);
      }
      if constexpr (S::has_mark(n)) st_mark(g.vo_stg, g.rs_mk, g.mk_off + ((n - MK_N) >> 1) * UNIT_BYTES + ((n - MK_N) & 1) * 1024u);
    };

    sfor<4>([&](auto fc) {
      constexpr int nf = decltype(fc)::value;
      mfma<j, nf, slot, i == 0>(g.Wf[i][nf]);
      side(fc);
      if constexpr (nf == 0) load_half(std::integral_constant<int, 0>{});
      if constexpr (nf == 1) store_out();
      if constexpr (nf == 2) load_half(std::integral_constant<int, 1>{});
    });
  });
}

struct U0 {
  unsigned vo_w, vo_x, ra_w, ra_x0, ra_x1;
  unsigned so_w, lds_w, lds_x;
  rsrc_t rs_w;
};
#define HT_X -2
#define HT_X1 108
#define HT_W0 -2
#define HT_W1 -2
#define HT_W2 108
struct U0S {
  static constexpr int NG = 40;
  static constexpr int w_issue(int n, int k) {
    int m = -1;
    if (n == 0) m = WD + k;
    else if (k == 0 && n % 5 == 3) m = 2 * (n / 5) + 2 + WD;
    else if (k == 0 && n % 5 == 0) m = 2 * (n / 5) + 1 + WD;
    return m <= 15 ? m : -1;
  }
  static constexpr int w_read(int n) {
    int m = n % 5 == 2 ? 2 * (n / 5 + 1) : (n % 5 == 4 ? 2 * (n / 5 + 1) + 1 : -1);
    return m <= 15 ? m : -1;
  }
  static constexpr int w_need(int m) { return m < 2 ? -1 : (m % 2 == 0 ? 5 * (m / 2 - 1) + 2 : 5 * (m / 2 - 1) + 4); }
  static constexpr int hint_t(int q) { return q < NHX ? (q == 0 ? HT_X : HT_X1) : (q == NHX ? HT_W0 : (q == NHX + 1 ? HT_W1 : HT_W2)); }
  static constexpr bool remote(int n) { return n - 1 + R >= NG; }
  static constexpr int NPRO_MAX = WD + XD + NHINT;
  static constexpr int pro_op(int idx) {
    int c = 0;
    for (int t = -2; t < 100; ++t) {
      if (t == -1) { if (c++ == idx) return 100; }
      for (int m = 0; m < WD; ++m) if (w_need(m) == t) { if (c++ == idx) return 200 + m; }
      if (t + 1 < XD && t >= 0) { if (c++ == idx) return 100 + t + 1; }
      for (int q = 0; q < NHINT; ++q) if (hint_t(q) == t) { if (c++ == idx) return 300 + q; }
    }
    return -1;
  }
  static constexpr int op_cnt(int op) { return op < 0 ? 0 : (op < 200 ? 2 : (op < 300 ? 4 : 1)); }
  static constexpr int n_hint(int n) { int c = 0; for (int q = 0; q < NHINT; ++q) if (hint_t(q) == 100 + n) ++c; return c; }
  static constexpr int wait_cnt(int n) {
    int pos = 0, pl[NG + 16] = {}, pw[16] = {};
    pos += 1;
    for (int idx = 0; idx < NPRO_MAX; ++idx) {
      const int op = pro_op(idx);
      pos += op_cnt(op);
      if (op >= 100 && op < 200) pl[op - 100] = pos;
      if (op >= 200 && op < 300) pw[op - 200] = pos;
    }
    if (n < 0) return pos - (pw[1] > pl[0] ? pw[1] : pl[0]);
    for (int q = 0; q <= n; ++q) {
      for (int k = 0; k < 2; ++k) if (w_issue(q, k) >= 0) { pos += 4; pw[w_issue(q, k)] = pos; }
      if (q + XD < NG) { pos += 2; pl[q + XD] = pos; }
      pos += n_hint(q) + (q == RH0_N ? NRH : 0) + (q == PK_0 ? 1 : 0);
      if (remote(q)) pos += 1;
      if (q < n && remote(q)) pos += 1;
    }
    int need = n + 1 < NG ? pl[n + 1] : 0;
    if (w_read(n) >= 0 && pw[w_read(n)] > need) need = pw[w_read(n)];
    int c = pos - need;
    return c > 63 ? 63 : c;
  }
};
template <int M> __device__ __forceinline__ void u0_w_issue(const U0& u) {
  sfor<4>([&](auto qc) {
    constexpr int q = decltype(qc)::value;
    dma16(u.vo_w, u.rs_w, u.so_w + (unsigned)((M >> 1) * 512 + (M & 1) * 64 + q * 16) * NFULL, u.lds_w + (M % WD) * 4096 + q * 1024);
  });
}
template <int N> __device__ __forceinline__ void u0_x_issue(const G& g, const U0& u) {
  dma16(u.vo_x, g.rs_x, g.so_loc[N % 5] + (N / 5) * 512, u.lds_x + (N % XD) * 2048);
  dma16(u.vo_x, g.rs_x, g.so_loc[N % 5] + (N / 5) * 512 + 8 * KDIM, u.lds_x + (N % XD) * 2048 + 1024);
}
template <int M> __device__ __forceinline__ void u0_w_read(G& g, const U0& u) {
#pragma unroll
  for (int tt = 0; tt < 2; ++tt)
#pragma unroll
    for (int nf = 0; nf < 4; ++nf) {
      v2i q = __builtin_amdgcn_ds_read_tr8_b64_v2i32((LAS v2i*)(uintptr_t)(u.ra_w + (M % WD) * 4096 + tt * 512 + nf * 16));
      g.Wf[M >> 1][nf][(M & 1) * 4 + tt * 2] = q[0];
      g.Wf[M >> 1][nf][(M & 1) * 4 + tt * 2 + 1] = q[1];
    }
}
template <int N> __device__ __forceinline__ void u0_x_read(G& g, const U0& u) {
  constexpr int X = XB + 8 * (N % R);
  asm volatile("ds_read_b128 v[%c1:%c2], %0 offset:%c3" :: "v"(u.ra_x0), "i"(X), "i"(X + 3), "i"((N % XD) * 2048) : "memory");
  asm volatile("ds_read_b128 v[%c1:%c2], %0 offset:%c3" :: "v"(u.ra_x1), "i"(X + 4), "i"(X + 7), "i"((N % XD) * 2048) : "memory");
}

__device__ __forceinline__ void unit0_body(G& g, U0& u, const Hint& h) {
  constexpr int NG = U0S::NG;
  sfor<NG>([&](auto nc) {
    constexpr int n = decltype(nc)::value;
    constexpr int i = n / 5, j = n % 5;
    constexpr int slot = n % R, pslot = (n + R - 1) % R;
    constexpr int t = n - 1 + R, ti = U0S::remote(n) ? (t - NG) / 4 : 0, tj = U0S::remote(n) ? (t - NG) % 4 : 0;

    asm volatile("s_waitcnt lgkmcnt(0)" ::: "memory");
    if constexpr (U0S::w_issue(n, 0) >= 0) u0_w_issue<U0S::w_issue(n, 0)>(u);
    if constexpr (U0S::w_issue(n, 1) >= 0) u0_w_issue<U0S::w_issue(n, 1)>(u);
    if constexpr (n + XD < NG) u0_x_issue<n + XD>(g, u);
    sfor<NHINT>([&](auto qc) { constexpr int q = decltype(qc)::value; if constexpr (U0S::hint_t(q) == 100 + n) hint<q>(h); });
    if constexpr (n == RH0_N) rhint(g);
    if constexpr (n == PK_0) peek_issue(g);
    if constexpr (PK_0 >= 0 && n == NG - R) peek_use(g);
    static_assert(PK_0 < 0 || (PK_0 + 1 + XD < NG && PK_0 + XD < NG - R), "peek is complete when it is used");
    if constexpr (U0S::remote(n) && tj == 0) {
      if (!(g.mask & 1ull)) slow_wait(g);
      g.mask >>= 1;
      g.kidx += 1;
    }
    sfor<4>([&](auto fc) {
      constexpr int nf = decltype(fc)::value;
      mfma<j, nf, slot, i == 0>(g.Wf[i][nf]);
      if constexpr (n == NG - 3) lds_wr<0 * RED_TILE + nf * 64, 0, nf>(g.wa_cur);
      if constexpr (n == NG - 2) lds_wr<1 * RED_TILE + nf * 64, 1, nf>(g.wa_cur);
      if constexpr (n == NG - 1) lds_wr<2 * RED_TILE + nf * 64, 2, nf>(g.wa_cur);
      if constexpr (U0S::remote(n) && (nf == 0 || nf == 2))
        ld_stg<pslot, nf / 2>(g.vo_stg, g.rs_stg, g.ub_nxt + ti * 32768 + g.fo[tj]);
      if constexpr (nf == 1 && n + 1 < NG) {
        asm volatile("s_waitcnt vmcnt(%0)" :: "i"(U0S::wait_cnt(n)) : "memory");
        u0_x_read<n + 1>(g, u);
        if constexpr (U0S::w_read(n) >= 0) u0_w_read<U0S::w_read(n)>(g, u);
      }
    });
  });
}

template <bool FIRST> __device__ __forceinline__ void next_unit(G& g, unsigned out_next) {
  g.out_prev = g.out_cur;
  g.out_cur = out_next;
  g.ub_cur = g.ub_nxt;
  g.ub_nxt += UNIT_BYTES;
  g.unit += 1u;
  g.kbase += 8u;
  g.ib_cur = g.ib_nxt;
  g.ib_nxt = g.ib && g.unit + 1u >= (unsigned)IB_U;
  g.wa_prev = g.wa_cur;
  g.wa_cur += g.red_d;
  if (!FIRST) { g.ra[0] -= g.red_d; g.ra[1] -= g.red_d; g.ra[2] -= g.red_d; g.ra[3] -= g.red_d; }
  g.red_d = 0u - g.red_d;
}

__device__ __forceinline__ void gemm_role(const Params& p, unsigned char* smem_generic) {
  LAS unsigned char* smem = (LAS unsigned char*)smem_generic;
  const int b = blockIdx.x;
  const int xcd = b & 7, slot = (b >> 3) - (NCOMM >> 3);
  const int pidx = (xcd % XG) * 28 + slot;
  const int pi = pidx / (4 * XG), cb = (xcd / XG) * (4 * XG) + pidx % (4 * XG);
  const int tid = threadIdx.x, lane = tid & 63;
  const int w = __builtin_amdgcn_readfirstlane(tid >> 6);
  const int kg = lane >> 4, li = lane & 15;
  const int rank = p.rank;
  unsigned char* myws = p.ws[rank];
  const int src = (rank + 1 + pi) & 7;
  G g;
  asm volatile("" ::: "v255");
  GSTAMP(0);

  unsigned ticket = 0;
  if (lane == 0)
    asm volatile("global_atomic_add %0, %1, %2, off sc0 sc1" : "+v"(ticket) : "v"((unsigned*)(myws + CTRL_CALL + (b * 4 + w) * 32)), "v"(1u) : "memory");
  GSTAMP(17);
  const int row0 = pi * 80 < 432 ? pi * 80 : 432;
  g.rs_x = mk_rsrc(p.x, SRC_BYTES);
  sfor<4>([&](auto jc) { constexpr int j = decltype(jc)::value; g.so_loc[j] = (unsigned)(row0 + ((j + 1 + w) & 3) * 16) * KDIM + w * 128; });
  g.so_loc[4] = (unsigned)(row0 + 64) * KDIM + w * 128;
  U0 u;
  {
    const unsigned base = (unsigned)(uintptr_t)smem + w * PART;
    u.rs_w = mk_rsrc(p.w, (unsigned)KDIM * NFULL);
    u.vo_w = (lane >> 2) * NFULL + (lane & 3) * 16;
    u.vo_x = (lane >> 3) * KDIM + ((lane & 7) ^ (lane >> 3)) * 16;
    u.so_w = (unsigned)rank * NLOC + cb * 64 + (unsigned)(w * 128) * NFULL;
    u.lds_w = base;
    u.lds_x = base + STG_X;
    u.ra_w = base + (kg * 16 + (li >> 1)) * 64 + (li & 1) * 8;
    u.ra_x0 = base + STG_X + (li >> 3) * 1024 + (li & 7) * 128 + ((kg ^ (li & 7)) * 16);
    u.ra_x1 = u.ra_x0 ^ 64u;
  }
  Hint h;
  h.wb = p.w + (size_t)rank * NLOC + (xcd / XG) * (256 * XG);
  h.xb = p.x + (size_t)row0 * KDIM;
  h.wi = slot * 4 + w;
  h.xi = (pidx % (4 * XG)) * 4 + w;
  sfor<U0S::NPRO_MAX>([&](auto ic) {
    constexpr int op = U0S::pro_op(decltype(ic)::value);
    if constexpr (op >= 100 && op < 200) u0_x_issue<op - 100>(g, u);
    if constexpr (op >= 200 && op < 300) u0_w_issue<op - 200>(u);
    if constexpr (op >= 300) hint<op - 300>(h);
  });
  GSTAMP(12);
  g.lds_bad = (LAS unsigned*)(smem + LDS_MISC + 4);
  if (tid == 0) asm volatile("ds_write_b32 %0, %1" :: "v"((unsigned)(uintptr_t)smem + LDS_MISC + 4), "v"(0u) : "memory");
  asm volatile("s_waitcnt vmcnt(%1) lgkmcnt(0)\n\ts_barrier" : "+v"(ticket) : "i"(U0S::wait_cnt(-1)) : "memory");
  g.call = __builtin_amdgcn_readfirstlane(ticket) + 1u;
  GSTAMP(13);
  u0_x_read<0>(g, u);
  u0_w_read<0>(g, u); u0_w_read<1>(g, u);

  g.scale = *(const CAS float*)p.sx * *(const CAS float*)p.sw;
  const unsigned cm = __builtin_amdgcn_readfirstlane(g.call % NSTAGE);
  g.rs_stg = mk_rsrc(myws + cm * STAGE_BYTES, STAGE_BYTES);
  {
    const unsigned gw = (unsigned)(b - NCOMM) * 4u + (unsigned)w;
    g.rs_mk = mk_rsrc(myws + (cm == 0u ? 2u : cm - 1u) * STAGE_BYTES, STAGE_BYTES);
    g.mk_off = ((unsigned)(rank + 1 + (gw >> 7)) & 7u) * SRC_BYTES + (IB_U - 1u) * UNIT_BYTES + (gw & 127u) * 2048u;
  }
  g.rs_out = mk_rsrc(p.out, (unsigned)(4096u * NLOC * 4u));
  g.flag_ptr = (const GAS unsigned long long*)(myws + CTRL_FLAG + src * 2048 + w * 512 + lane * 8);
  g.tmo = (GAS unsigned*)(myws + CTRL_TMO);
  g.rs_tmo = mk_rsrc(myws + CTRL_TMO, 4);
  g.vo_stg = lane * 16;
  g.vo_rh = lane * 128;
  g.rh_off = (unsigned)(pidx % (4 * XG)) * (UNIT_BYTES / (4 * XG)) + (unsigned)w * (UNIT_BYTES / (16 * XG) - 8192u);
  g.vo_out = (lane >> 4) * (NLOC * 4) + (lane & 15) * 16;
  sfor<4>([&](auto jc) { constexpr int j = decltype(jc)::value; g.fo[j] = ((j + 1 + w) & 3) * 2048; });
  g.wa_cur = w * PART + li * RED_ROW + kg * 16;
  sfor<4>([&](auto qc) {
    constexpr int q = decltype(qc)::value;
    g.ra[q] = q * PART + ((w - q - 1) & 3) * RED_TILE + (lane >> 4) * RED_ROW + (lane & 15) * 16;
  });
  g.wa5 = w * PART + LDS_T5 + li * RED_ROW + kg * 16;
  g.ra5 = LDS_T5 + (4 * w + (lane >> 4)) * RED_ROW + (lane & 15) * 16;
  g.red_d = RED_WAVE;
  g.tmov = 0u;
  const unsigned colb = cb * 256;
  auto out_off = [&](int grow) -> unsigned { return (unsigned)(grow + w * 16) * (NLOC * 4) + colb; };
  g.out_cur = out_off(rank * MLOC + row0);
  g.out_prev = g.out_cur;
  g.out5 = (unsigned)(rank * MLOC + row0 + 64 + 4 * w) * (NLOC * 4) + colb;
  g.ub_cur = 0;
  g.ub_nxt = (unsigned)src * SRC_BYTES + w * 8192;
  g.mask = 0ull;
  g.ib = g.call > 2u;
  g.ib_cur = false; g.ib_nxt = false;
  g.kbase = 0u - 8u;
  g.unit = 0u;
  static_assert(IB_U >= 3, "the first remote unit is never validated in-band");
  g.kidx = 0u;

  GSTAMP(1);
  unit0_body(g, u, h);
  GSTAMP(2);
  next_unit<true>(g, out_off(src * MLOC));
  unit_body<4, 40 % R, 5, true>(g);
  GSTAMP(3);
  constexpr int PER = (32 % R == 0) ? 1 : 3;
  static_assert((32 * PER) % R == 0 && 6 % PER == 0, "ring size");
  for (int u0 = 2; u0 < 8; u0 += PER) {
    sfor<PER>([&](auto pc) {
      constexpr int pp = decltype(pc)::value;
      const int u = u0 + pp;
      next_unit<false>(g, out_off(src * MLOC + (u - 1) * 64));
      unit_body<4, (40 + 32 * (1 + pp)) % R, 4, true>(g);
    });
  }
  next_unit<false>(g, out_off(src * MLOC + 7 * 64));
  unit_body<4, (40 + 32 * 7) % R, 4, false>(g);
  GSTAMP(10);

  sfor<4>([&](auto qc) { constexpr int q = decltype(qc)::value; lds_wr<2 * RED_TILE + q * 64, 2, q>(g.wa_cur); });
  asm volatile("s_nop 7\n\ts_nop 7\n\ts_nop 7" ::: "memory");
  sfor<4>([&](auto qc) { constexpr int q = decltype(qc)::value; lds_wr<3 * RED_TILE + q * 64, 3, q>(g.wa_cur); });
  asm volatile("s_waitcnt lgkmcnt(0)\n\ts_barrier" ::: "memory");
  {
    v4f r[4][4];
    lds_rd32(g.badv, LDS_MISC + 4);
    sfor<16>([&](auto qc) {
      constexpr int q = decltype(qc)::value;
      lds_rd<(q >> 2) * 4 * RED_ROW>(r[q >> 2][q & 3], g.ra[q & 3] - g.red_d);
    });
    float sc = g.scale;
    sfor<4>([&](auto sc_) {
      constexpr int q = decltype(sc_)::value;
      lds_wait<(3 - q) * 4>(r[q], g.badv);
      if constexpr (q == 0) {
        asm volatile("s_waitcnt vmcnt(0)" : "+v"(g.tmov) :: "memory");
        unsigned bad = __builtin_amdgcn_readfirstlane(g.badv) | (__builtin_amdgcn_readfirstlane(g.tmov) == g.call ? 1u : 0u);
        if (bad) sc = __builtin_nanf("");
      }
      const v4f sum = (r[q][0] + r[q][1]) + (r[q][2] + r[q][3]);
      const v4f v = {sum[0] * sc, sum[1] * sc, sum[2] * sc, sum[3] * sc};
      st_out(v, g.vo_out, g.rs_out, g.out_cur + q * 4 * (NLOC * 4));
    });
  }
}

#define CSTAMP(n) do {} while (0)
template <bool LEADER>
__device__ __forceinline__ void comm_wave(const Params& p, LAS unsigned char* smem, const unsigned call, const int j, const int w, const int lane, v4i d) {
  const int rank = p.rank;
  unsigned char* myws = p.ws[rank];
  const int cw = j * 4 + w;
  rsrc_t rs_x = mk_rsrc(p.x, SRC_BYTES);
  const unsigned vo = (unsigned)((w >> 1) * 16 + (lane & 15)) * KDIM + (w & 1) * 64 + (lane >> 4) * 16;
  const unsigned dst0 = (call % NSTAGE) * STAGE_BYTES + (unsigned)rank * SRC_BYTES + (unsigned)w * 1024u;
  rsrc_t rs[7];
#pragma unroll
  for (int q = 0; q < 7; ++q) {
    int pq = (rank + 1 + (q + cw) % 7) & 7;
    rs[q] = mk_rsrc(p.ws[pq], NSTAGE * STAGE_BYTES);
  }
  unsigned char* flagb = p.ws[(rank + 1 + (lane < 7 ? lane : 0)) & 7] + CTRL_FLAG + rank * 2048;
  unsigned* ticketp = (unsigned*)(myws + CTRL_TICKET);
  GAS unsigned* tmo = (GAS unsigned*)(myws + CTRL_TMO);
  const unsigned tbase = (call - 1u) * TICKETS_PER_CALL - 2u * NCOMM;
  const unsigned vs = lane * 16;
  LAS unsigned* tk = (LAS unsigned*)(smem + 1024);
  LAS unsigned* cnt = (LAS unsigned*)(smem + LDS_MISC + 64);

  auto publish = [&](unsigned n, unsigned t) {
    unsigned old = 0;
    if (lane == 0) old = __hip_atomic_fetch_add(cnt + (n & 7u), 1u, __ATOMIC_RELAXED, __HIP_MEMORY_SCOPE_WORKGROUP);
    old = __builtin_amdgcn_readfirstlane(old);
    if ((old & 3u) == 3u) {
      if (lane < 7) __hip_atomic_store((unsigned*)(flagb + ((t >> 1) & 3u) * 512u + (t >> 3) * 8u + (t & 1u) * 4u), call, __ATOMIC_RELAXED, __HIP_MEMORY_SCOPE_SYSTEM);
      CSTAMP(n + 1);
    }
  };

  unsigned t_cur = (unsigned)j, t_prev = 0;
  unsigned n = 0;
  while (true) {
    v4i dn;
    unsigned tick = 0, t_nxt = 0;
    const unsigned dst = dst0 + t_cur * 4096u;
    sfor<7>([&](auto qc) {
      constexpr int q = decltype(qc)::value;
      asm volatile("s_waitcnt vmcnt(%0)" :: "i"(COMM_W - 1) : "memory");
      if constexpr (q == COMM_W - 1) { if (n > 0) publish(n - 1, t_prev); }
      if constexpr (LEADER && q == 0) {
        if (n > 0 && lane == 0) asm volatile("global_atomic_add %0, %1, %2, off sc0 sc1" : "+v"(tick) : "v"(ticketp), "v"(1u) : "memory");
      }
      if constexpr (q == 5) {
        if constexpr (LEADER) {
          asm volatile("" : "+v"(tick));
          t_nxt = n == 0 ? (unsigned)(NCOMM + j) : __builtin_amdgcn_readfirstlane(tick) - tbase;
          if (lane == 0) tk[n + 1] = t_nxt + 1u;
        } else {
          unsigned long long t0 = 0;
          while (true) {
            unsigned v = __hip_atomic_load(tk + n + 1, __ATOMIC_RELAXED, __HIP_MEMORY_SCOPE_WORKGROUP);
            v = __builtin_amdgcn_readfirstlane(v);
            if (v) { t_nxt = v - 1u; break; }
            unsigned long long now = __builtin_amdgcn_s_memrealtime();
            if (!t0) t0 = now;
            if (now - t0 > DEADLINE_TICKS) {
              __hip_atomic_store(tmo, call, __ATOMIC_RELAXED, __HIP_MEMORY_SCOPE_AGENT);
              t_nxt = NPIECE; break;
            }
            __builtin_amdgcn_s_sleep(1);
          }
        }
        if (t_nxt < NPIECE)
          asm volatile("buffer_load_dwordx4 %0, %1, %2, %3 offen" : "=v"(dn) : "v"(vo), "s"(rs_x),
                       "s"((t_nxt >> 6) * UNIT_BYTES + (t_nxt & 1u) * (32u * KDIM) + ((t_nxt >> 1) & 31u) * 128u) : "memory");
      }
      st_sys(d, vs, rs[q], dst);
    });
    if (t_nxt >= NPIECE) break;
    asm volatile("s_waitcnt vmcnt(2)" : "+v"(dn) :: "memory");
    t_prev = t_cur; t_cur = t_nxt; d = dn; ++n;
  }
  asm volatile("s_waitcnt vmcnt(0)" ::: "memory");
  publish(n, t_cur);
}

__device__ __forceinline__ void comm_role(const Params& p, unsigned char* smem_generic) {
  LAS unsigned char* smem = (LAS unsigned char*)smem_generic;
  const int j = blockIdx.x;
  const int tid = threadIdx.x, lane = tid & 63;
  const int w = __builtin_amdgcn_readfirstlane(tid >> 6);
  unsigned char* myws = p.ws[p.rank];

  STAMP(myws, 0);
  v4i d;
  {
    rsrc_t rs_x = mk_rsrc(p.x, SRC_BYTES);
    const unsigned vo = (unsigned)((w >> 1) * 16 + (lane & 15)) * KDIM + (w & 1) * 64 + (lane >> 4) * 16;
    asm volatile("buffer_load_dwordx4 %0, %1, %2, %3 offen" : "=v"(d) : "v"(vo), "s"(rs_x), "s"((unsigned)((j & 1) * (32 * KDIM) + (j >> 1) * 128)) : "memory");
  }
  unsigned ticket = 0;
  if (lane == 0)
    asm volatile("global_atomic_add %0, %1, %2, off sc0" : "+v"(ticket) : "v"((unsigned*)(myws + CTRL_CALL + (j * 4 + w) * 32)), "v"(1u) : "memory");
  for (int i = tid; i < 520; i += 256) *(LAS unsigned*)(smem + 1024 + i * 4) = 0u;
  if (tid < 8) *(LAS unsigned*)(smem + LDS_MISC + 64 + tid * 4) = 0u;
  asm volatile("s_waitcnt lgkmcnt(0)\n\ts_barrier\n\ts_waitcnt vmcnt(0)" : "+v"(d), "+v"(ticket) :: "memory");
  const unsigned call = __builtin_amdgcn_readfirstlane(ticket) + 1u;
  if (w == 0) comm_wave<true>(p, smem, call, j, w, lane, d);
  else comm_wave<false>(p, smem, call, j, w, lane, d);
}

__global__ void __launch_bounds__(256, 1) fused_ag_gemm(Params p) {
  extern __shared__ __attribute__((aligned(16))) unsigned char smem_dyn[];
  if (blockIdx.x < NCOMM) comm_role(p, smem_dyn);
  else gemm_role(p, smem_dyn);
}

}

extern "C" void kernel_launch(int rank, int nranks, ncclComm_t comm, hipStream_t stream,
                              void* const* d_in, const int* in_sizes, int n_in,
                              void* d_out, int out_size,
                              const int* in_dtypes, int out_dtype,
                              void* const* d_out_all,
                              void* const* d_ws_all, size_t ws_bytes) {
  if (nranks != NRANKS || n_in < 4 || ws_bytes < WS_NEEDED || (unsigned)rank >= (unsigned)NRANKS) {
    fprintf(stderr, "unsupported configuration\n");
    return;
  }
  static bool attr_done = false;
  if (!attr_done) {
    HIP_CHECK(hipFuncSetAttribute((const void*)fused_ag_gemm, hipFuncAttributeMaxDynamicSharedMemorySize, (int)LDS_BYTES));
    attr_done = true;
  }
  Params p;
  p.x = (const unsigned char*)d_in[0];
  p.w = (const unsigned char*)d_in[1];
  p.sx = (const float*)d_in[2];
  p.sw = (const float*)d_in[3];
  p.out = (float*)d_out;
  for (int i = 0; i < NRANKS; ++i) p.ws[i] = (unsigned char*)d_ws_all[i];
  p.rank = rank;
  hipLaunchKernelGGL(fused_ag_gemm, dim3(256), dim3(256), LDS_BYTES, stream, p);
  HIP_CHECK(hipGetLastError());
}
```

```cpp
#include <hip/hip_runtime.h>
#include <rccl/rccl.h>
#include <cstdio>
#include <utility>

#define HIP_CHECK(expr) do { hipError_t _e = (expr); if (_e != hipSuccess) { \
    fprintf(stderr, "HIP error %s at %s:%d\n", hipGetErrorString(_e), __FILE__, __LINE__); \
    return; } } while (0)

namespace {

typedef int v8i __attribute__((ext_vector_type(8)));
typedef int v4i __attribute__((ext_vector_type(4)));
typedef int v2i __attribute__((ext_vector_type(2)));
typedef unsigned v2u __attribute__((ext_vector_type(2)));
typedef float v4f __attribute__((ext_vector_type(4)));
#define LAS __attribute__((address_space(3)))
#define GAS __attribute__((address_space(1)))
#define CAS __attribute__((address_space(4)))
typedef __amdgpu_buffer_rsrc_t rsrc_t;

constexpr int NRANKS = 8;
constexpr int KDIM = 4096, NFULL = 16384, NLOC = 2048, MLOC = 512;
constexpr unsigned UNIT_BYTES = 64u * KDIM;
constexpr unsigned SRC_BYTES = (unsigned)MLOC * KDIM;
constexpr unsigned STAGE_BYTES = NRANKS * SRC_BYTES;
constexpr unsigned NSTAGE = 3;
constexpr unsigned CTRL = NSTAGE * STAGE_BYTES;
constexpr unsigned CTRL_FLAG = CTRL;
constexpr unsigned CTRL_TMO = CTRL + 16384;
constexpr unsigned CTRL_TICKET = CTRL + 16384 + 128;
constexpr unsigned CTRL_CALL = CTRL + 20480;
constexpr unsigned WS_NEEDED = CTRL_CALL + 1024u * 32u;
constexpr int NCOMM = 32;
constexpr unsigned NPIECE = 512;
constexpr unsigned TICKETS_PER_CALL = NPIECE - 2 * NCOMM + NCOMM;
constexpr unsigned long long DEADLINE_TICKS = 100000000ull;

#define RING 8
#define COMM_P 1
#define MAPPING 0
#define XG 2
#define IB_U 6
#define IB_SLEEP 1
#define MK_N 10
#define MK_U 5
#define MKST "sc0 sc1"
#define COMM_W 4
#define SLEEPN 2
#define PF_MODE 0
#define POLLW 8
#define OSTM 0
#define OST ""
#define XLDM 3
#define XLD "sc0 sc1"
constexpr int R = RING;

#define WD 4
#define XD 5
constexpr unsigned RED_ROW = 272, RED_TILE = 16 * RED_ROW, RED_WAVE = 4 * RED_TILE;
constexpr unsigned PART = 39424, LDS_T5 = 2 * RED_WAVE, STG_X = WD * 4096;
constexpr unsigned LDS_MISC = 4 * PART;
constexpr unsigned LDS_BYTES = LDS_MISC + 256;
static_assert(PART % 128 == 0 && LDS_T5 + RED_TILE <= PART && STG_X + XD * 2048 <= PART && LDS_BYTES <= 160 * 1024, "LDS layout");
static_assert(STG_X >= 2 * RED_TILE, "tiles 0 and 1 of unit 0 are written while the last x slots are in use");

struct Params {
  const unsigned char* x;
  const unsigned char* w;
  const float* sx;
  const float* sw;
  float* out;
  unsigned char* ws[NRANKS];
  int rank;
};

#define STAMP(ws, idx) do {} while (0)
#define GSTAMP(idx) do {} while (0)

template <class F, int... I> __device__ __forceinline__ void sfor_impl(F&& f, std::integer_sequence<int, I...>) {
  (f(std::integral_constant<int, I>{}), ...);
}
template <int N, class F> __device__ __forceinline__ void sfor(F&& f) { sfor_impl(f, std::make_integer_sequence<int, N>{}); }

__device__ __forceinline__ rsrc_t mk_rsrc(const void* p, unsigned bytes) {
  return __builtin_amdgcn_make_buffer_rsrc((void*)p, 0, bytes, 0x00020000);
}

struct G {
  v8i Wf[8][4];
  v4f rd[4];
  unsigned long long mask;
  unsigned kidx;
  unsigned tmov;
  unsigned badv;
  unsigned vo_loc, vo_stg, vo_rh, vo_out, wa_cur, wa_prev, ra[4], wa5, ra5;
  rsrc_t rs_x, rs_stg, rs_out, rs_tmo, rs_mk;
  unsigned mk_off, rh_off;
  unsigned so_loc[5];
  unsigned fo[4];
  unsigned ub_cur, ub_nxt;
  unsigned out_prev, out_cur;
  unsigned out5;
  unsigned red_d;
  unsigned call, unit;
  bool ib;
  bool ib_cur, ib_nxt;
  unsigned kbase;
  float scale;
  const GAS unsigned long long* flag_ptr;
  GAS unsigned* tmo;
  LAS unsigned* lds_bad;
};

constexpr int XB = 256 - 8 * R, AB = XB - 80, HD = AB - 1, FD = HD - 3, A4B = FD;
constexpr int acc_reg(int j, int nf) { return AB + 16 * j + 4 * nf; }

template <int J, int NF_, int SLOT, bool FIRST> __device__ __forceinline__ void mfma(const v8i& wf) {
  constexpr int A = acc_reg(J, NF_), X = XB + 8 * SLOT;
  if constexpr (FIRST)
    asm volatile("v_mfma_f32_16x16x128_f8f6f4 v[%c1:%c2], %0, v[%c3:%c4], 0 cbsz:1" :: "a"(wf), "i"(A), "i"(A + 3), "i"(X), "i"(X + 7));
  else
    asm volatile("v_mfma_f32_16x16x128_f8f6f4 v[%c1:%c2], %0, v[%c3:%c4], v[%c1:%c2] cbsz:1" :: "a"(wf), "i"(A), "i"(A + 3), "i"(X), "i"(X + 7));
}
template <int SLOT, int H> __device__ __forceinline__ void ld_stg(unsigned vo, rsrc_t rs, unsigned so) {
  constexpr int X = XB + 8 * SLOT + 4 * H;
  asm volatile("buffer_load_dwordx4 v[%c3:%c4], %0, %1, %2 offen offset:%c5 " XLD :: "v"(vo), "s"(rs), "s"(so), "i"(X), "i"(X + 3), "i"(H * 1024) : "memory");
}
template <int CNT> __device__ __forceinline__ void wait_vm() { asm volatile("s_waitcnt vmcnt(%c0)" :: "i"(CNT) : "memory"); }

__device__ __forceinline__ void st_sys(const v4i& d, unsigned vo, rsrc_t rs, unsigned so) {
  asm volatile("buffer_store_dwordx4 %0, %1, %2, %3 offen sc0 sc1" :: "v"(d), "v"(vo), "s"(rs), "s"(so) : "memory");
}

__device__ __forceinline__ void dma16(unsigned vo, rsrc_t rs, unsigned so, unsigned lds) {
  asm volatile("s_mov_b32 m0, %3\n\tbuffer_load_dwordx4 %0, %1, %2 offen lds" :: "v"(vo), "s"(rs), "s"(so), "s"(lds) : "memory");
}

__device__ __forceinline__ void touch(const unsigned char* a) {
  asm volatile("global_load_dword v%c1, %0, off" :: "v"(a), "i"(HD) : "memory");
}

#define HS0 0
#define XS0 0
constexpr int HW_LINES = (8 - HS0) * 512 * 2 * XG;
constexpr int NHW = (HW_LINES + 112 * 64 - 1) / (112 * 64);
constexpr int HW_LPI = (HW_LINES + 112 * NHW - 1) / (112 * NHW);
constexpr int HX_LINES = (8 - XS0) * 320, HX_WAVES = 16 * XG;
constexpr int NHX = (HX_LINES + HX_WAVES * 64 - 1) / (HX_WAVES * 64);
constexpr int HX_LPI = (HX_LINES + HX_WAVES * NHX - 1) / (HX_WAVES * NHX);
constexpr int NHINT = NHW + NHX;
struct Hint { const unsigned char* wb; const unsigned char* xb; int wi, xi; };
template <int Q> __device__ __forceinline__ void hint(const Hint& h) {
  const int lane = threadIdx.x & 63;
  if constexpr (Q < NHX) {
    int line = (Q * HX_WAVES + h.xi) * HX_LPI + (lane < HX_LPI ? lane : HX_LPI - 1);
    line = line < HX_LINES ? line : HX_LINES - 1;
    const int r = XS0 + line / 320, rem = line % 320;
    touch(h.xb + (rem >> 2) * KDIM + r * 512 + (rem & 3) * 128);
  } else {
    int line = ((Q - NHX) * 112 + h.wi) * HW_LPI + (lane < HW_LPI ? lane : HW_LPI - 1);
    line = line < HW_LINES ? line : HW_LINES - 1;
    touch(h.wb + (size_t)(HS0 * 512 + line / (2 * XG)) * NFULL + (line % (2 * XG)) * 128);
  }
}

__device__ __forceinline__ void slow_wait(G& g) {
  unsigned long long t0 = __builtin_amdgcn_s_memrealtime();
  const unsigned lane = __lane_id();
  unsigned lo = 0, hi = 64;
  unsigned long long m;
  while (true) {
    unsigned long long v = 0;
    if (lane >= lo && lane < hi) v = __hip_atomic_load(g.flag_ptr, __ATOMIC_RELAXED, __HIP_MEMORY_SCOPE_SYSTEM);
    unsigned tvv = __hip_atomic_load(g.tmo, __ATOMIC_RELAXED, __HIP_MEMORY_SCOPE_AGENT);
    unsigned v0 = (unsigned)v, v1 = (unsigned)(v >> 32);
    asm volatile("" : "+v"(v0), "+v"(v1), "+v"(tvv));
    m = __builtin_amdgcn_ballot_w64((v0 < v1 ? v0 : v1) >= g.call) >> g.kidx;
    if (m & 1ull) break;
    unsigned tv = __builtin_amdgcn_readfirstlane(tvv);
    if (tv == g.call || __builtin_amdgcn_s_memrealtime() - t0 > DEADLINE_TICKS) {
      __hip_atomic_store(g.tmo, g.call, __ATOMIC_RELAXED, __HIP_MEMORY_SCOPE_AGENT);
      *g.lds_bad = 1u;
      m = ~0ull;
      break;
    }
    lo = g.kidx; hi = g.kidx + POLLW;
    __builtin_amdgcn_s_sleep(SLEEPN);
  }
  g.mask = m;
}

template <int OFF, int J, int NF_> __device__ __forceinline__ void lds_wr(unsigned a) {
  constexpr int A = acc_reg(J, NF_);
  asm volatile("ds_write_b128 %0, v[%c1:%c2] offset:%c3" :: "v"(a), "i"(A), "i"(A + 3), "i"(OFF) : "memory");
}
template <int OFF> __device__ __forceinline__ void lds_rd(v4f& v, unsigned a) {
  asm volatile("ds_read_b128 %0, %1 offset:%2" : "=v"(v) : "v"(a), "i"(OFF) : "memory");
}

template <int CNT> __device__ __forceinline__ void lds_wait(v4f (&r)[4], unsigned& b) {
  asm volatile("s_waitcnt lgkmcnt(%5)" : "+v"(r[0]), "+v"(r[1]), "+v"(r[2]), "+v"(r[3]), "+v"(b) : "i"(CNT) : "memory");
}
__device__ __forceinline__ void lds_rd32(unsigned& v, unsigned a) {
  asm volatile("ds_read_b32 %0, %1" : "=v"(v) : "v"(a) : "memory");
}
__device__ __forceinline__ void st_out(const v4f& v, unsigned vo, rsrc_t rs, unsigned so) {
  asm volatile("buffer_store_dwordx4 %0, %1, %2, %3 offen " OST :: "v"(v), "v"(vo), "s"(rs), "s"(so) : "memory");
}

__device__ __forceinline__ void st_mark(unsigned vo, rsrc_t rs, unsigned so) {
  const v4i mk = {0, 0, 0, 0};
  asm volatile("buffer_store_dwordx4 %0, %1, %2, %3 offen " MKST :: "v"(mk), "v"(vo), "s"(rs), "s"(so) : "memory");
}

#define RH_N -1
#define RH0_N -1
constexpr int NRH = 2 / XG > 0 ? 2 / XG : 1;
__device__ __forceinline__ void touch_stg(unsigned vo, rsrc_t rs, unsigned so) {
  asm volatile("buffer_load_dword v%c3, %0, %1, %2 offen " XLD :: "v"(vo), "s"(rs), "s"(so), "i"(HD) : "memory");
}
__device__ __forceinline__ void rhint(G& g) {
  sfor<NRH>([&](auto qc) { touch_stg(g.vo_rh, g.rs_stg, g.ub_nxt + g.rh_off + decltype(qc)::value * 8192u); });
}

#define PK_A 2
#define PK_B 18
#define PK_0 22
__device__ __forceinline__ void peek_issue(G& g) {
  asm volatile("global_load_dwordx2 v[%c1:%c2], %0, off sc0 sc1" :: "v"(g.flag_ptr), "i"(FD), "i"(FD + 1) : "memory");
}
__device__ __forceinline__ void peek_use(G& g) {
  unsigned long long m;
  unsigned t;
  asm volatile("v_min_u32 %1, v%c3, v%c4\n\tv_cmp_le_u32 %0, %2, %1" : "=s"(m), "=&v"(t) : "s"(g.call), "i"(FD), "i"(FD + 1));
  g.mask |= m >> (g.kidx & 63u);
}

template <int NF, int PREV, bool HAS_NEXT> struct Sched {
  static constexpr int NG = 8 * NF;
  static constexpr bool has_load(int n) { return HAS_NEXT || (n - 1 + R < NG); }
  static constexpr int n_out(int n) { return PREV == 0 ? 0 : ((n >= 5 && n <= 8) ? 1 : 0) + ((PREV == 5 && n == 9) ? 1 : 0); }
  static constexpr bool mark_slot(int n) { return n >= MK_N && n < MK_N + 2 * (9 - IB_U); }
  static constexpr bool has_mark(int n) { return MK_U == 1 && PREV == 5 && mark_slot(n); }
  static constexpr int n_store(int n) { return n_out(n) + (has_mark(n) ? 1 : 0); }
  static constexpr bool has_tmo(int n) { return HAS_NEXT ? n == 30 - R : n == NG - 1; }
  static constexpr bool has_rh(int n) { return HAS_NEXT && n == RH_N; }
  static constexpr bool has_peek(int n) { return HAS_NEXT && (n == PK_A || n == PK_B); }
  static constexpr bool use_peek(int n) { return has_peek(n - R + 1); }
  static constexpr int ops(int q) {
    if (q < 0) return 2;
    return (has_load(q) ? 2 : 0) + n_store(q) + (has_tmo(q) ? 1 : 0) + (has_rh(q) ? NRH : 0) + (has_peek(q) ? 1 : 0);
  }
  static constexpr int wait_cnt(int n) {
    int c = 0;
    for (int q = n - R + 2; q <= n - 1; ++q) c += ops(q);
    return c > 63 ? 63 : c;
  }
};

template <int SLOT> __device__ __forceinline__ bool ib_dirty() {
  constexpr int X = XB + 8 * SLOT;
  unsigned long long m;
  unsigned t0, t1;
  asm volatile("v_min3_u32 %1, v[%c3], v[%c4], v[%c5]\n\t"
               "v_min3_u32 %2, v[%c6], v[%c7], v[%c8]\n\t"
               "v_min3_u32 %1, v[%c9], v[%c10], %1\n\t"
               "v_min_u32 %1, %1, %2\n\t"
               "v_cmp_eq_u32 %0, 0, %1"
               : "=s"(m), "=&v"(t0), "=&v"(t1)
               : "i"(X), "i"(X + 1), "i"(X + 2), "i"(X + 3), "i"(X + 4), "i"(X + 5), "i"(X + 6), "i"(X + 7));
  return m != 0ull;
}
template <int M, int ROFF> __device__ __forceinline__ void ib_load(G& g) {
  constexpr int slot = (ROFF + M) % R;
  const unsigned so = (M < 32 ? g.ub_cur + (M / 4) * 32768 : g.ub_nxt + ((M - 32) / 4) * 32768) + g.fo[M % 4];
  ld_stg<slot, 0>(g.vo_stg, g.rs_stg, so);
  ld_stg<slot, 1>(g.vo_stg, g.rs_stg, so);
}
template <int N, int ROFF, bool HAS_NEXT> __device__ __forceinline__ void ib_retry(G& g) {
  constexpr int KE = N | 3;
  constexpr int LE = (HAS_NEXT || N + R - 2 < 31) ? N + R - 2 : 31;
  constexpr int slot = (ROFF + N) % R;
  const unsigned long long t0 = __builtin_amdgcn_s_memrealtime();
  const unsigned lane = __lane_id();
  bool flagged = false;
  while (true) {
    unsigned long long v = 0;
    if (lane == g.kbase + N / 4) v = __hip_atomic_load(g.flag_ptr, __ATOMIC_RELAXED, __HIP_MEMORY_SCOPE_SYSTEM);
    unsigned tvv = __hip_atomic_load(g.tmo, __ATOMIC_RELAXED, __HIP_MEMORY_SCOPE_AGENT);
    sfor<KE - N + 1>([&](auto mc) { ib_load<N + decltype(mc)::value, ROFF>(g); });
    unsigned v0 = (unsigned)v, v1 = (unsigned)(v >> 32);
    asm volatile("s_waitcnt vmcnt(0)" : "+v"(v0), "+v"(v1), "+v"(tvv) :: "memory");
    if (flagged || !ib_dirty<slot>()) break;
    flagged = __builtin_amdgcn_ballot_w64((v0 < v1 ? v0 : v1) >= g.call) != 0ull;
    unsigned tv = __builtin_amdgcn_readfirstlane(tvv);
    if (tv == g.call || __builtin_amdgcn_s_memrealtime() - t0 > DEADLINE_TICKS) {
      __hip_atomic_store(g.tmo, g.call, __ATOMIC_RELAXED, __HIP_MEMORY_SCOPE_AGENT);
      *g.lds_bad = 1u;
      break;
    }
    if (!flagged) __builtin_amdgcn_s_sleep(IB_SLEEP);
  }
  if constexpr (LE > KE) sfor<LE - KE>([&](auto mc) { ib_load<KE + 1 + decltype(mc)::value, ROFF>(g); });
}

template <int NF, int ROFF, int PREV, bool HAS_NEXT>
__device__ __forceinline__ void unit_body(G& g) {
  using S = Sched<NF, PREV, HAS_NEXT>;
  constexpr int NG = S::NG;
  sfor<NG>([&](auto nc) {
    constexpr int n = decltype(nc)::value;
    constexpr int i = n / NF, j = n % NF;
    constexpr int slot = (ROFF + n) % R;
    constexpr int pslot = (ROFF + n + R - 1) % R;
    constexpr int t = n - 1 + R;
    constexpr bool t_next = t >= NG;
    constexpr int ti = t_next ? (t - NG) / 4 : t / NF, tj = t_next ? (t - NG) % 4 : t % NF;

    wait_vm<S::wait_cnt(n)>();
    if constexpr (IB_U < 9 && PREV == 4) {
      if (__builtin_expect(ib_dirty<slot>(), 0)) ib_retry<n, ROFF, HAS_NEXT>(g);
    }

    if constexpr (PREV != 0 && n == 4) {
      asm volatile("s_waitcnt lgkmcnt(0)\n\ts_barrier" ::: "memory");
    }
    v4f ov;
    if constexpr (S::n_out(n) != 0) {
      asm volatile("s_waitcnt lgkmcnt(0)" : "+v"(g.rd[0]), "+v"(g.rd[1]), "+v"(g.rd[2]), "+v"(g.rd[3]), "+v"(g.badv));
      if constexpr (n == 5) {
        unsigned bad = __builtin_amdgcn_readfirstlane(g.badv) | (__builtin_amdgcn_readfirstlane(g.tmov) == g.call ? 1u : 0u);
        if (bad) g.scale = __builtin_nanf("");
      }
      const v4f sum = (g.rd[0] + g.rd[1]) + (g.rd[2] + g.rd[3]);
      const float sc = g.scale;
      ov = (v4f){sum[0] * sc, sum[1] * sc, sum[2] * sc, sum[3] * sc};
      asm volatile("" : "+v"(ov));
    }
    if constexpr (S::use_peek(n)) peek_use(g);
    if constexpr (S::has_load(n) && tj == 0) {
      if (!(t_next ? g.ib_nxt : g.ib_cur)) {
        if (!(g.mask & 1ull)) slow_wait(g);
        g.mask >>= 1;
        g.kidx += 1;
      }
    }
    if constexpr (S::has_rh(n)) rhint(g);
    if constexpr (S::has_peek(n)) peek_issue(g);
    if constexpr (S::has_tmo(n)) {
      asm volatile("buffer_load_dword %0, off, %1, 0 sc1" : "=v"(g.tmov) : "s"(g.rs_tmo) : "memory");
    }

    auto load_half = [&](auto hc) {
      constexpr int h = decltype(hc)::value;
      if constexpr (S::has_load(n))
        ld_stg<pslot, h>(g.vo_stg, g.rs_stg, (t_next ? g.ub_nxt : g.ub_cur) + ti * 32768 + g.fo[tj]);
    };
    auto side = [&](auto qc) {
      constexpr int q = decltype(qc)::value;
      if constexpr (n == NG - 2) lds_wr<0 * RED_TILE + q * 64, 0, q>(g.wa_cur);
      if constexpr (n == NG - 1) lds_wr<1 * RED_TILE + q * 64, 1, q>(g.wa_cur);
      if constexpr (PREV == 4 && n == 0) lds_wr<2 * RED_TILE + q * 64, 2, q>(g.wa_prev);
      if constexpr (PREV == 4 && n == 1) lds_wr<3 * RED_TILE + q * 64, 3, q>(g.wa_prev);
      if constexpr (PREV == 5 && n == 0) lds_wr<3 * RED_TILE + q * 64, 3, q>(g.wa_prev);
      if constexpr (PREV == 5 && n == 1) lds_wr<q * 64, 4, q>(g.wa5);
      if constexpr (PREV != 0 && n >= 4 && n <= 7) lds_rd<(n - 4) * 4 * RED_ROW>(g.rd[q], g.ra[q]);
      if constexpr (PREV != 0 && n == 4 && q == 0)
        asm volatile("ds_read_b32 %0, %1" : "=v"(g.badv) : "v"((unsigned)(LDS_MISC + 4)) : "memory");
      if constexpr (PREV == 5 && n == 8) lds_rd<(q & 1) * PART>(g.rd[q], q < 2 ? g.ra5 : g.ra5 + 2 * PART);
    };
    auto store_out = [&]() {
      if constexpr (S::n_out(n) != 0) {
        const unsigned so = n < 9 ? g.out_prev + (n - 5) * 4 * (NLOC * 4) : g.out5;
        st_out(ov, g.vo_out, g.rs_out, so);
      }
      if constexpr (S::has_mark(n)) st_mark(g.vo_stg, g.rs_mk, g.mk_off + ((n - MK_N) >> 1) * UNIT_BYTES + ((n - MK_N) & 1) * 1024u);
      if constexpr (MK_U > 1 && PREV == 4 && HAS_NEXT && S::mark_slot(n)) {
        if (g.unit == (unsigned)MK_U) st_mark(g.vo_stg, g.rs_mk, g.mk_off + ((n - MK_N) >> 1) * UNIT_BYTES + ((n - MK_N) & 1) * 1024u);
      }
    };

    sfor<4>([&](auto fc) {
      constexpr int nf = decltype(fc)::value;
      mfma<j, nf, slot, i == 0>(g.Wf[i][nf]);
      side(fc);
      if constexpr (nf == 0) load_half(std::integral_constant<int, 0>{});
      if constexpr (nf == 1) store_out();
      if constexpr (nf == 2) load_half(std::integral_constant<int, 1>{});
    });
  });
}

struct U0 {
  unsigned vo_w, vo_x, ra_w, ra_x0, ra_x1;
  unsigned so_w, lds_w, lds_x;
  rsrc_t rs_w;
};
#define HT_X -2
#define HT_X1 108
#define HT_W0 -2
#define HT_W1 -2
#define HT_W2 108
struct U0S {
  static constexpr int NG = 40;
  static constexpr int w_issue(int n, int k) {
    int m = -1;
    if (n == 0) m = WD + k;
    else if (k == 0 && n % 5 == 3) m = 2 * (n / 5) + 2 + WD;
    else if (k == 0 && n % 5 == 0) m = 2 * (n / 5) + 1 + WD;
    return m <= 15 ? m : -1;
  }
  static constexpr int w_read(int n) {
    int m = n % 5 == 2 ? 2 * (n / 5 + 1) : (n % 5 == 4 ? 2 * (n / 5 + 1) + 1 : -1);
    return m <= 15 ? m : -1;
  }
  static constexpr int w_need(int m) { return m < 2 ? -1 : (m % 2 == 0 ? 5 * (m / 2 - 1) + 2 : 5 * (m / 2 - 1) + 4); }
  static constexpr int hint_t(int q) { return q < NHX ? (q == 0 ? HT_X : HT_X1) : (q == NHX ? HT_W0 : (q == NHX + 1 ? HT_W1 : HT_W2)); }
  static constexpr bool remote(int n) { return n - 1 + R >= NG; }
  static constexpr int NPRO_MAX = WD + XD + NHINT;
  static constexpr int pro_op(int idx) {
    int c = 0;
    for (int t = -2; t < 100; ++t) {
      if (t == -1) { if (c++ == idx) return 100; }
      for (int m = 0; m < WD; ++m) if (w_need(m) == t) { if (c++ == idx) return 200 + m; }
      if (t + 1 < XD && t >= 0) { if (c++ == idx) return 100 + t + 1; }
      for (int q = 0; q < NHINT; ++q) if (hint_t(q) == t) { if (c++ == idx) return 300 + q; }
    }
    return -1;
  }
  static constexpr int op_cnt(int op) { return op < 0 ? 0 : (op < 200 ? 2 : (op < 300 ? 4 : 1)); }
  static constexpr int n_hint(int n) { int c = 0; for (int q = 0; q < NHINT; ++q) if (hint_t(q) == 100 + n) ++c; return c; }
  static constexpr int wait_cnt(int n) {
    int pos = 0, pl[NG + 16] = {}, pw[16] = {};
    pos += 1;
    for (int idx = 0; idx < NPRO_MAX; ++idx) {
      const int op = pro_op(idx);
      pos += op_cnt(op);
      if (op >= 100 && op < 200) pl[op - 100] = pos;
      if (op >= 200 && op < 300) pw[op - 200] = pos;
    }
    if (n < 0) return pos - (pw[1] > pl[0] ? pw[1] : pl[0]);
    for (int q = 0; q <= n; ++q) {
      for (int k = 0; k < 2; ++k) if (w_issue(q, k) >= 0) { pos += 4; pw[w_issue(q, k)] = pos; }
      if (q + XD < NG) { pos += 2; pl[q + XD] = pos; }
      pos += n_hint(q) + (q == RH0_N ? NRH : 0) + (q == PK_0 ? 1 : 0);
      if (remote(q)) pos += 1;
      if (q < n && remote(q)) pos += 1;
    }
    int need = n + 1 < NG ? pl[n + 1] : 0;
    if (w_read(n) >= 0 && pw[w_read(n)] > need) need = pw[w_read(n)];
    int c = pos - need;
    return c > 63 ? 63 : c;
  }
};
template <int M> __device__ __forceinline__ void u0_w_issue(const U0& u) {
  sfor<4>([&](auto qc) {
    constexpr int q = decltype(qc)::value;
    dma16(u.vo_w, u.rs_w, u.so_w + (unsigned)((M >> 1) * 512 + (M & 1) * 64 + q * 16) * NFULL, u.lds_w + (M % WD) * 4096 + q * 1024);
  });
}
template <int N> __device__ __forceinline__ void u0_x_issue(const G& g, const U0& u) {
  dma16(u.vo_x, g.rs_x, g.so_loc[N % 5] + (N / 5) * 512, u.lds_x + (N % XD) * 2048);
  dma16(u.vo_x, g.rs_x, g.so_loc[N % 5] + (N / 5) * 512 + 8 * KDIM, u.lds_x + (N % XD) * 2048 + 1024);
}
template <int M> __device__ __forceinline__ void u0_w_read(G& g, const U0& u) {
#pragma unroll
  for (int tt = 0; tt < 2; ++tt)
#pragma unroll
    for (int nf = 0; nf < 4; ++nf) {
      v2i q = __builtin_amdgcn_ds_read_tr8_b64_v2i32((LAS v2i*)(uintptr_t)(u.ra_w + (M % WD) * 4096 + tt * 512 + nf * 16));
      g.Wf[M >> 1][nf][(M & 1) * 4 + tt * 2] = q[0];
      g.Wf[M >> 1][nf][(M & 1) * 4 + tt * 2 + 1] = q[1];
    }
}
template <int N> __device__ __forceinline__ void u0_x_read(G& g, const U0& u) {
  constexpr int X = XB + 8 * (N % R);
  asm volatile("ds_read_b128 v[%c1:%c2], %0 offset:%c3" :: "v"(u.ra_x0), "i"(X), "i"(X + 3), "i"((N % XD) * 2048) : "memory");
  asm volatile("ds_read_b128 v[%c1:%c2], %0 offset:%c3" :: "v"(u.ra_x1), "i"(X + 4), "i"(X + 7), "i"((N % XD) * 2048) : "memory");
}

__device__ __forceinline__ void unit0_body(G& g, U0& u, const Hint& h) {
  constexpr int NG = U0S::NG;
  sfor<NG>([&](auto nc) {
    constexpr int n = decltype(nc)::value;
    constexpr int i = n / 5, j = n % 5;
    constexpr int slot = n % R, pslot = (n + R - 1) % R;
    constexpr int t = n - 1 + R, ti = U0S::remote(n) ? (t - NG) / 4 : 0, tj = U0S::remote(n) ? (t - NG) % 4 : 0;

    asm volatile("s_waitcnt lgkmcnt(0)" ::: "memory");
    if constexpr (U0S::w_issue(n, 0) >= 0) u0_w_issue<U0S::w_issue(n, 0)>(u);
    if constexpr (U0S::w_issue(n, 1) >= 0) u0_w_issue<U0S::w_issue(n, 1)>(u);
    if constexpr (n + XD < NG) u0_x_issue<n + XD>(g, u);
    sfor<NHINT>([&](auto qc) { constexpr int q = decltype(qc)::value; if constexpr (U0S::hint_t(q) == 100 + n) hint<q>(h); });
    if constexpr (n == RH0_N) rhint(g);
    if constexpr (n == PK_0) peek_issue(g);
    if constexpr (PK_0 >= 0 && n == NG - R) peek_use(g);
    static_assert(PK_0 < 0 || (PK_0 + 1 + XD < NG && PK_0 + XD < NG - R), "peek is complete when it is used");
    if constexpr (U0S::remote(n) && tj == 0) {
      if (!(g.mask & 1ull)) slow_wait(g);
      g.mask >>= 1;
      g.kidx += 1;
    }
    sfor<4>([&](auto fc) {
      constexpr int nf = decltype(fc)::value;
      mfma<j, nf, slot, i == 0>(g.Wf[i][nf]);
      if constexpr (n == NG - 3) lds_wr<0 * RED_TILE + nf * 64, 0, nf>(g.wa_cur);
      if constexpr (n == NG - 2) lds_wr<1 * RED_TILE + nf * 64, 1, nf>(g.wa_cur);
      if constexpr (n == NG - 1) lds_wr<2 * RED_TILE + nf * 64, 2, nf>(g.wa_cur);
      if constexpr (U0S::remote(n) && (nf == 0 || nf == 2))
        ld_stg<pslot, nf / 2>(g.vo_stg, g.rs_stg, g.ub_nxt + ti * 32768 + g.fo[tj]);
      if constexpr (nf == 1 && n + 1 < NG) {
        asm volatile("s_waitcnt vmcnt(%0)" :: "i"(U0S::wait_cnt(n)) : "memory");
        u0_x_read<n + 1>(g, u);
        if constexpr (U0S::w_read(n) >= 0) u0_w_read<U0S::w_read(n)>(g, u);
      }
    });
  });
}

template <bool FIRST> __device__ __forceinline__ void next_unit(G& g, unsigned out_next) {
  g.out_prev = g.out_cur;
  g.out_cur = out_next;
  g.ub_cur = g.ub_nxt;
  g.ub_nxt += UNIT_BYTES;
  g.unit += 1u;
  g.kbase += 8u;
  g.ib_cur = g.ib_nxt;
  g.ib_nxt = g.ib && g.unit + 1u >= (unsigned)IB_U;
  g.wa_prev = g.wa_cur;
  g.wa_cur += g.red_d;
  if (!FIRST) { g.ra[0] -= g.red_d; g.ra[1] -= g.red_d; g.ra[2] -= g.red_d; g.ra[3] -= g.red_d; }
  g.red_d = 0u - g.red_d;
}

__device__ __forceinline__ void gemm_role(const Params& p, unsigned char* smem_generic) {
  LAS unsigned char* smem = (LAS unsigned char*)smem_generic;
  const int b = blockIdx.x;
  const int xcd = b & 7, slot = (b >> 3) - (NCOMM >> 3);
  const int pidx = (xcd % XG) * 28 + slot;
  const int pi = pidx / (4 * XG), cb = (xcd / XG) * (4 * XG) + pidx % (4 * XG);
  const int tid = threadIdx.x, lane = tid & 63;
  const int w = __builtin_amdgcn_readfirstlane(tid >> 6);
  const int kg = lane >> 4, li = lane & 15;
  const int rank = p.rank;
  unsigned char* myws = p.ws[rank];
  const int src = (rank + 1 + pi) & 7;
  G g;
  asm volatile("" ::: "v255");
  GSTAMP(0);

  unsigned ticket = 0;
  if (lane == 0)
    asm volatile("global_atomic_add %0, %1, %2, off sc0 sc1" : "+v"(ticket) : "v"((unsigned*)(myws + CTRL_CALL + (b * 4 + w) * 32)), "v"(1u) : "memory");
  GSTAMP(17);
  const int row0 = pi * 80 < 432 ? pi * 80 : 432;
  g.rs_x = mk_rsrc(p.x, SRC_BYTES);
  sfor<4>([&](auto jc) { constexpr int j = decltype(jc)::value; g.so_loc[j] = (unsigned)(row0 + ((j + 1 + w) & 3) * 16) * KDIM + w * 128; });
  g.so_loc[4] = (unsigned)(row0 + 64) * KDIM + w * 128;
  U0 u;
  {
    const unsigned base = (unsigned)(uintptr_t)smem + w * PART;
    u.rs_w = mk_rsrc(p.w, (unsigned)KDIM * NFULL);
    u.vo_w = (lane >> 2) * NFULL + (lane & 3) * 16;
    u.vo_x = (lane >> 3) * KDIM + ((lane & 7) ^ (lane >> 3)) * 16;
    u.so_w = (unsigned)rank * NLOC + cb * 64 + (unsigned)(w * 128) * NFULL;
    u.lds_w = base;
    u.lds_x = base + STG_X;
    u.ra_w = base + (kg * 16 + (li >> 1)) * 64 + (li & 1) * 8;
    u.ra_x0 = base + STG_X + (li >> 3) * 1024 + (li & 7) * 128 + ((kg ^ (li & 7)) * 16);
    u.ra_x1 = u.ra_x0 ^ 64u;
  }
  Hint h;
  h.wb = p.w + (size_t)rank * NLOC + (xcd / XG) * (256 * XG);
  h.xb = p.x + (size_t)row0 * KDIM;
  h.wi = slot * 4 + w;
  h.xi = (pidx % (4 * XG)) * 4 + w;
  sfor<U0S::NPRO_MAX>([&](auto ic) {
    constexpr int op = U0S::pro_op(decltype(ic)::value);
    if constexpr (op >= 100 && op < 200) u0_x_issue<op - 100>(g, u);
    if constexpr (op >= 200 && op < 300) u0_w_issue<op - 200>(u);
    if constexpr (op >= 300) hint<op - 300>(h);
  });
  GSTAMP(12);
  g.lds_bad = (LAS unsigned*)(smem + LDS_MISC + 4);
  if (tid == 0) asm volatile("ds_write_b32 %0, %1" :: "v"((unsigned)(uintptr_t)smem + LDS_MISC + 4), "v"(0u) : "memory");
  asm volatile("s_waitcnt vmcnt(%1) lgkmcnt(0)\n\ts_barrier" : "+v"(ticket) : "i"(U0S::wait_cnt(-1)) : "memory");
  g.call = __builtin_amdgcn_readfirstlane(ticket) + 1u;
  GSTAMP(13);
  u0_x_read<0>(g, u);
  u0_w_read<0>(g, u); u0_w_read<1>(g, u);

  g.scale = *(const CAS float*)p.sx * *(const CAS float*)p.sw;
  const unsigned cm = __builtin_amdgcn_readfirstlane(g.call % NSTAGE);
  g.rs_stg = mk_rsrc(myws + cm * STAGE_BYTES, STAGE_BYTES);
  {
    const unsigned gw = (unsigned)(b - NCOMM) * 4u + (unsigned)w;
    g.rs_mk = mk_rsrc(myws + (cm == 0u ? 2u : cm - 1u) * STAGE_BYTES, STAGE_BYTES);
    g.mk_off = ((unsigned)(rank + 1 + (gw >> 7)) & 7u) * SRC_BYTES + (IB_U - 1u) * UNIT_BYTES + (gw & 127u) * 2048u;
  }
  g.rs_out = mk_rsrc(p.out, (unsigned)(4096u * NLOC * 4u));
  g.flag_ptr = (const GAS unsigned long long*)(myws + CTRL_FLAG + src * 2048 + w * 512 + lane * 8);
  g.tmo = (GAS unsigned*)(myws + CTRL_TMO);
  g.rs_tmo = mk_rsrc(myws + CTRL_TMO, 4);
  g.vo_stg = lane * 16;
  g.vo_rh = lane * 128;
  g.rh_off = (unsigned)(pidx % (4 * XG)) * (UNIT_BYTES / (4 * XG)) + (unsigned)w * (UNIT_BYTES / (16 * XG) - 8192u);
  g.vo_out = (lane >> 4) * (NLOC * 4) + (lane & 15) * 16;
  sfor<4>([&](auto jc) { constexpr int j = decltype(jc)::value; g.fo[j] = ((j + 1 + w) & 3) * 2048; });
  g.wa_cur = w * PART + li * RED_ROW + kg * 16;
  sfor<4>([&](auto qc) {
    constexpr int q = decltype(qc)::value;
    g.ra[q] = q * PART + ((w - q - 1) & 3) * RED_TILE + (lane >> 4) * RED_ROW + (lane & 15) * 16;
  });
  g.wa5 = w * PART + LDS_T5 + li * RED_ROW + kg * 16;
  g.ra5 = LDS_T5 + (4 * w + (lane >> 4)) * RED_ROW + (lane & 15) * 16;
  g.red_d = RED_WAVE;
  g.tmov = 0u;
  const unsigned colb = cb * 256;
  auto out_off = [&](int grow) -> unsigned { return (unsigned)(grow + w * 16) * (NLOC * 4) + colb; };
  g.out_cur = out_off(rank * MLOC + row0);
  g.out_prev = g.out_cur;
  g.out5 = (unsigned)(rank * MLOC + row0 + 64 + 4 * w) * (NLOC * 4) + colb;
  g.ub_cur = 0;
  g.ub_nxt = (unsigned)src * SRC_BYTES + w * 8192;
  g.mask = 0ull;
  g.ib = true;
  g.ib_cur = false; g.ib_nxt = false;
  g.kbase = 0u - 8u;
  g.unit = 0u;
  static_assert(IB_U >= 3, "the first remote unit is never validated in-band");
  g.kidx = 0u;

  GSTAMP(1);
  unit0_body(g, u, h);
  GSTAMP(2);
  next_unit<true>(g, out_off(src * MLOC));
  unit_body<4, 40 % R, 5, true>(g);
  GSTAMP(3);
  constexpr int PER = (32 % R == 0) ? 1 : 3;
  static_assert((32 * PER) % R == 0 && 6 % PER == 0, "ring size");
  for (int u0 = 2; u0 < 8; u0 += PER) {
    sfor<PER>([&](auto pc) {
      constexpr int pp = decltype(pc)::value;
      const int u = u0 + pp;
      next_unit<false>(g, out_off(src * MLOC + (u - 1) * 64));
      unit_body<4, (40 + 32 * (1 + pp)) % R, 4, true>(g);
    });
  }
  next_unit<false>(g, out_off(src * MLOC + 7 * 64));
  unit_body<4, (40 + 32 * 7) % R, 4, false>(g);
  GSTAMP(10);

  sfor<4>([&](auto qc) { constexpr int q = decltype(qc)::value; lds_wr<2 * RED_TILE + q * 64, 2, q>(g.wa_cur); });
  asm volatile("s_nop 7\n\ts_nop 7\n\ts_nop 7" ::: "memory");
  sfor<4>([&](auto qc) { constexpr int q = decltype(qc)::value; lds_wr<3 * RED_TILE + q * 64, 3, q>(g.wa_cur); });
  asm volatile("s_waitcnt lgkmcnt(0)\n\ts_barrier" ::: "memory");
  {
    v4f r[4][4];
    lds_rd32(g.badv, LDS_MISC + 4);
    sfor<16>([&](auto qc) {
      constexpr int q = decltype(qc)::value;
      lds_rd<(q >> 2) * 4 * RED_ROW>(r[q >> 2][q & 3], g.ra[q & 3] - g.red_d);
    });
    float sc = g.scale;
    sfor<4>([&](auto sc_) {
      constexpr int q = decltype(sc_)::value;
      lds_wait<(3 - q) * 4>(r[q], g.badv);
      if constexpr (q == 0) {
        asm volatile("s_waitcnt vmcnt(0)" : "+v"(g.tmov) :: "memory");
        unsigned bad = __builtin_amdgcn_readfirstlane(g.badv) | (__builtin_amdgcn_readfirstlane(g.tmov) == g.call ? 1u : 0u);
        if (bad) sc = __builtin_nanf("");
      }
      const v4f sum = (r[q][0] + r[q][1]) + (r[q][2] + r[q][3]);
      const v4f v = {sum[0] * sc, sum[1] * sc, sum[2] * sc, sum[3] * sc};
      st_out(v, g.vo_out, g.rs_out, g.out_cur + q * 4 * (NLOC * 4));
    });
  }
}

#define CSTAMP(n) do {} while (0)
template <bool LEADER>
__device__ __forceinline__ void comm_wave(const Params& p, LAS unsigned char* smem, const unsigned call, const int j, const int w, const int lane, v4i d) {
  const int rank = p.rank;
  unsigned char* myws = p.ws[rank];
  const int cw = j * 4 + w;
  rsrc_t rs_x = mk_rsrc(p.x, SRC_BYTES);
  const unsigned vo = (unsigned)((w >> 1) * 16 + (lane & 15)) * KDIM + (w & 1) * 64 + (lane >> 4) * 16;
  const unsigned dst0 = (call % NSTAGE) * STAGE_BYTES + (unsigned)rank * SRC_BYTES + (unsigned)w * 1024u;
  rsrc_t rs[7];
#pragma unroll
  for (int q = 0; q < 7; ++q) {
    int pq = (rank + 1 + (q + cw) % 7) & 7;
    rs[q] = mk_rsrc(p.ws[pq], NSTAGE * STAGE_BYTES);
  }
  unsigned char* flagb = p.ws[(rank + 1 + (lane < 7 ? lane : 0)) & 7] + CTRL_FLAG + rank * 2048;
  unsigned* ticketp = (unsigned*)(myws + CTRL_TICKET);
  GAS unsigned* tmo = (GAS unsigned*)(myws + CTRL_TMO);
  const unsigned tbase = (call - 1u) * TICKETS_PER_CALL - 2u * NCOMM;
  const unsigned vs = lane * 16;
  LAS unsigned* tk = (LAS unsigned*)(smem + 1024);
  LAS unsigned* cnt = (LAS unsigned*)(smem + LDS_MISC + 64);

  auto publish = [&](unsigned n, unsigned t) {
    unsigned old = 0;
    if (lane == 0) old = __hip_atomic_fetch_add(cnt + (n & 7u), 1u, __ATOMIC_RELAXED, __HIP_MEMORY_SCOPE_WORKGROUP);
    old = __builtin_amdgcn_readfirstlane(old);
    if ((old & 3u) == 3u) {
      if (lane < 7) __hip_atomic_store((unsigned*)(flagb + ((t >> 1) & 3u) * 512u + (t >> 3) * 8u + (t & 1u) * 4u), call, __ATOMIC_RELAXED, __HIP_MEMORY_SCOPE_SYSTEM);
      CSTAMP(n + 1);
    }
  };

  unsigned t_cur = (unsigned)j, t_prev = 0;
  unsigned n = 0;
  while (true) {
    v4i dn;
    unsigned tick = 0, t_nxt = 0;
    const unsigned dst = dst0 + t_cur * 4096u;
    sfor<7>([&](auto qc) {
      constexpr int q = decltype(qc)::value;
      asm volatile("s_waitcnt vmcnt(%0)" :: "i"(COMM_W - 1) : "memory");
      if constexpr (q == COMM_W - 1) { if (n > 0) publish(n - 1, t_prev); }
      if constexpr (LEADER && q == 0) {
        if (n > 0 && lane == 0) asm volatile("global_atomic_add %0, %1, %2, off sc0 sc1" : "+v"(tick) : "v"(ticketp), "v"(1u) : "memory");
      }
      if constexpr (q == 5) {
        if constexpr (LEADER) {
          asm volatile("" : "+v"(tick));
          t_nxt = n == 0 ? (unsigned)(NCOMM + j) : __builtin_amdgcn_readfirstlane(tick) - tbase;
          if (lane == 0) tk[n + 1] = t_nxt + 1u;
        } else {
          unsigned long long t0 = 0;
          while (true) {
            unsigned v = __hip_atomic_load(tk + n + 1, __ATOMIC_RELAXED, __HIP_MEMORY_SCOPE_WORKGROUP);
            v = __builtin_amdgcn_readfirstlane(v);
            if (v) { t_nxt = v - 1u; break; }
            unsigned long long now = __builtin_amdgcn_s_memrealtime();
            if (!t0) t0 = now;
            if (now - t0 > DEADLINE_TICKS) {
              __hip_atomic_store(tmo, call, __ATOMIC_RELAXED, __HIP_MEMORY_SCOPE_AGENT);
              t_nxt = NPIECE; break;
            }
            __builtin_amdgcn_s_sleep(1);
          }
        }
        if (t_nxt < NPIECE)
          asm volatile("buffer_load_dwordx4 %0, %1, %2, %3 offen" : "=v"(dn) : "v"(vo), "s"(rs_x),
                       "s"((t_nxt >> 6) * UNIT_BYTES + (t_nxt & 1u) * (32u * KDIM) + ((t_nxt >> 1) & 31u) * 128u) : "memory");
      }
      st_sys(d, vs, rs[q], dst);
    });
    if (t_nxt >= NPIECE) break;
    asm volatile("s_waitcnt vmcnt(2)" : "+v"(dn) :: "memory");
    t_prev = t_cur; t_cur = t_nxt; d = dn; ++n;
  }
  asm volatile("s_waitcnt vmcnt(0)" ::: "memory");
  publish(n, t_cur);
}

__device__ __forceinline__ void comm_role(const Params& p, unsigned char* smem_generic) {
  LAS unsigned char* smem = (LAS unsigned char*)smem_generic;
  const int j = blockIdx.x;
  const int tid = threadIdx.x, lane = tid & 63;
  const int w = __builtin_amdgcn_readfirstlane(tid >> 6);
  unsigned char* myws = p.ws[p.rank];

  STAMP(myws, 0);
  v4i d;
  {
    rsrc_t rs_x = mk_rsrc(p.x, SRC_BYTES);
    const unsigned vo = (unsigned)((w >> 1) * 16 + (lane & 15)) * KDIM + (w & 1) * 64 + (lane >> 4) * 16;
    asm volatile("buffer_load_dwordx4 %0, %1, %2, %3 offen" : "=v"(d) : "v"(vo), "s"(rs_x), "s"((unsigned)((j & 1) * (32 * KDIM) + (j >> 1) * 128)) : "memory");
  }
  unsigned ticket = 0;
  if (lane == 0)
    asm volatile("global_atomic_add %0, %1, %2, off sc0" : "+v"(ticket) : "v"((unsigned*)(myws + CTRL_CALL + (j * 4 + w) * 32)), "v"(1u) : "memory");
  for (int i = tid; i < 520; i += 256) *(LAS unsigned*)(smem + 1024 + i * 4) = 0u;
  if (tid < 8) *(LAS unsigned*)(smem + LDS_MISC + 64 + tid * 4) = 0u;
  asm volatile("s_waitcnt lgkmcnt(0)\n\ts_barrier\n\ts_waitcnt vmcnt(0)" : "+v"(d), "+v"(ticket) :: "memory");
  const unsigned call = __builtin_amdgcn_readfirstlane(ticket) + 1u;
  if (w == 0) comm_wave<true>(p, smem, call, j, w, lane, d);
  else comm_wave<false>(p, smem, call, j, w, lane, d);
}

__global__ void __launch_bounds__(256, 1) fused_ag_gemm(Params p) {
  extern __shared__ __attribute__((aligned(16))) unsigned char smem_dyn[];
  if (blockIdx.x < NCOMM) comm_role(p, smem_dyn);
  else gemm_role(p, smem_dyn);
}

}

extern "C" void kernel_launch(int rank, int nranks, ncclComm_t comm, hipStream_t stream,
                              void* const* d_in, const int* in_sizes, int n_in,
                              void* d_out, int out_size,
                              const int* in_dtypes, int out_dtype,
                              void* const* d_out_all,
                              void* const* d_ws_all, size_t ws_bytes) {
  if (nranks != NRANKS || n_in < 4 || ws_bytes < WS_NEEDED || (unsigned)rank >= (unsigned)NRANKS) {
    fprintf(stderr, "unsupported configuration\n");
    return;
  }
  static bool attr_done = false;
  if (!attr_done) {
    HIP_CHECK(hipFuncSetAttribute((const void*)fused_ag_gemm, hipFuncAttributeMaxDynamicSharedMemorySize, (int)LDS_BYTES));
    attr_done = true;
  }
  Params p;
  p.x = (const unsigned char*)d_in[0];
  p.w = (const unsigned char*)d_in[1];
  p.sx = (const float*)d_in[2];
  p.sw = (const float*)d_in[3];
  p.out = (float*)d_out;
  for (int i = 0; i < NRANKS; ++i) p.ws[i] = (unsigned char*)d_ws_all[i];
  p.rank = rank;
  hipLaunchKernelGGL(fused_ag_gemm, dim3(256), dim3(256), LDS_BYTES, stream, p);
  HIP_CHECK(hipGetLastError());
}
```

```cpp
#include <hip/hip_runtime.h>
#include <rccl/rccl.h>
#include <cstdio>
#include <utility>

#define HIP_CHECK(expr) do { hipError_t _e = (expr); if (_e != hipSuccess) { \
    fprintf(stderr, "HIP error %s at %s:%d\n", hipGetErrorString(_e), __FILE__, __LINE__); \
    return; } } while (0)

namespace {

typedef int v8i __attribute__((ext_vector_type(8)));
typedef int v4i __attribute__((ext_vector_type(4)));
typedef int v2i __attribute__((ext_vector_type(2)));
typedef unsigned v2u __attribute__((ext_vector_type(2)));
typedef float v4f __attribute__((ext_vector_type(4)));
#define LAS __attribute__((address_space(3)))
#define GAS __attribute__((address_space(1)))
#define CAS __attribute__((address_space(4)))
typedef __amdgpu_buffer_rsrc_t rsrc_t;

constexpr int NRANKS = 8;
constexpr int KDIM = 4096, NFULL = 16384, NLOC = 2048, MLOC = 512;
constexpr unsigned UNIT_BYTES = 64u * KDIM;
constexpr unsigned SRC_BYTES = (unsigned)MLOC * KDIM;
constexpr unsigned STAGE_BYTES = NRANKS * SRC_BYTES;
constexpr unsigned NSTAGE = 3;
constexpr unsigned CTRL = NSTAGE * STAGE_BYTES;
constexpr unsigned CTRL_FLAG = CTRL;
constexpr unsigned CTRL_TMO = CTRL + 16384;
constexpr unsigned CTRL_TICKET = CTRL + 16384 + 128;
constexpr unsigned CTRL_CALL = CTRL + 20480;
constexpr unsigned WS_NEEDED = CTRL_CALL + 1024u * 32u;
constexpr int NCOMM = 32;
constexpr unsigned NPIECE = 512;
constexpr unsigned TICKETS_PER_CALL = NPIECE - 2 * NCOMM + NCOMM;
constexpr unsigned long long DEADLINE_TICKS = 100000000ull;

#define RING 8
#define COMM_P 1
#define MAPPING 0
#define XG 2
#define NROT 1
#define IB_U 6
#define IB_PIPE 2
#define IB_SLEEP 1
#define MK_N 10
#define MK_U 1
#define MKST "sc0 sc1"
#define COMM_MODE 1
#define COMM_W 4
#define SLEEPN 2
#define PF_MODE 0
#define POLLW 8
#define OSTM 0
#define OST ""
#define XLDM 3
#define XLD "sc0 sc1"
constexpr int R = RING;

#define WD 4
#define XD 5
constexpr unsigned RED_ROW = 272, RED_TILE = 16 * RED_ROW, RED_WAVE = 4 * RED_TILE;
constexpr unsigned PART = 39424, LDS_T5 = 2 * RED_WAVE, STG_X = WD * 4096;
constexpr unsigned LDS_MISC = 4 * PART;
constexpr unsigned LDS_BYTES = LDS_MISC + 256;
static_assert(PART % 128 == 0 && LDS_T5 + RED_TILE <= PART && STG_X + XD * 2048 <= PART && LDS_BYTES <= 160 * 1024, "LDS layout");
static_assert(STG_X >= 2 * RED_TILE, "tiles 0 and 1 of unit 0 are written while the last x slots are in use");

constexpr unsigned long long IBM = IB_U <= 8 ? ~0ull << (8 * (IB_U <= 8 ? IB_U - 1 : 0)) : 0ull;

struct Params {
  const unsigned char* x;
  const unsigned char* w;
  const float* sx;
  const float* sw;
  float* out;
  unsigned char* ws[NRANKS];
  int rank;
};

#define STAMP(ws, idx) do {} while (0)
#define GSTAMP(idx) do {} while (0)

template <class F, int... I> __device__ __forceinline__ void sfor_impl(F&& f, std::integer_sequence<int, I...>) {
  (f(std::integral_constant<int, I>{}), ...);
}
template <int N, class F> __device__ __forceinline__ void sfor(F&& f) { sfor_impl(f, std::make_integer_sequence<int, N>{}); }

__device__ __forceinline__ rsrc_t mk_rsrc(const void* p, unsigned bytes) {
  return __builtin_amdgcn_make_buffer_rsrc((void*)p, 0, bytes, 0x00020000);
}

struct G {
  v8i Wf[8][4];
  v4f rd[2][4];
  unsigned sw_n, sw_t;
  unsigned rt_n, rt_it, rt_t, rt_last, rt_lastn;
  unsigned long long mask;
  unsigned kidx;
  unsigned tmov;
  unsigned badv;
  unsigned vo_loc, vo_stg, vo_rh, vo_out, wa_cur, wa_prev, ra[4], wa5, ra5;
  rsrc_t rs_x, rs_stg, rs_out, rs_tmo, rs_mk;
  unsigned mk_off, rh_off;
  unsigned so_loc[5];
  unsigned fo[4];
  unsigned ub_cur, ub_nxt;
  unsigned kd[8];
  unsigned kn[2];
  unsigned rot;
  unsigned out_prev, out_cur;
  unsigned out5;
  unsigned red_d;
  unsigned call, unit;
  unsigned kbase;
  float scale;
  const GAS unsigned long long* flag_ptr;
  GAS unsigned* tmo;
  LAS unsigned* lds_bad;
};

constexpr int XB = 256 - 8 * R, AB = XB - 80, HD = AB - 1, FD = HD - 3, A4B = FD;
constexpr int acc_reg(int j, int nf) { return AB + 16 * j + 4 * nf; }

template <int J, int NF_, int SLOT, bool FIRST> __device__ __forceinline__ void mfma(const v8i& wf) {
  constexpr int A = acc_reg(J, NF_), X = XB + 8 * SLOT;
  if constexpr (FIRST)
    asm volatile("v_mfma_f32_16x16x128_f8f6f4 v[%c1:%c2], %0, v[%c3:%c4], 0 cbsz:1" :: "a"(wf), "i"(A), "i"(A + 3), "i"(X), "i"(X + 7));
  else
    asm volatile("v_mfma_f32_16x16x128_f8f6f4 v[%c1:%c2], %0, v[%c3:%c4], v[%c1:%c2] cbsz:1" :: "a"(wf), "i"(A), "i"(A + 3), "i"(X), "i"(X + 7));
}
template <int SLOT, int H> __device__ __forceinline__ void ld_stg(unsigned vo, rsrc_t rs, unsigned so) {
  constexpr int X = XB + 8 * SLOT + 4 * H;
  asm volatile("buffer_load_dwordx4 v[%c3:%c4], %0, %1, %2 offen offset:%c5 " XLD :: "v"(vo), "s"(rs), "s"(so), "i"(X), "i"(X + 3), "i"(H * 1024) : "memory");
}
template <int CNT> __device__ __forceinline__ void wait_vm() { asm volatile("s_waitcnt vmcnt(%c0)" :: "i"(CNT) : "memory"); }

__device__ __forceinline__ void st_sys(const v4i& d, unsigned vo, rsrc_t rs, unsigned so) {
  asm volatile("buffer_store_dwordx4 %0, %1, %2, %3 offen sc0 sc1\n\ts_nop 1" :: "v"(d), "v"(vo), "s"(rs), "s"(so) : "memory");
}

__device__ __forceinline__ void dma16(unsigned vo, rsrc_t rs, unsigned so, unsigned lds) {
  asm volatile("s_mov_b32 m0, %3\n\tbuffer_load_dwordx4 %0, %1, %2 offen lds" :: "v"(vo), "s"(rs), "s"(so), "s"(lds) : "memory");
}

#define WPOL ""
#define XPOL ""
__device__ __forceinline__ void dma16w(unsigned vo, rsrc_t rs, unsigned so, unsigned lds) {
  asm volatile("s_mov_b32 m0, %3\n\tbuffer_load_dwordx4 %0, %1, %2 offen " WPOL " lds" :: "v"(vo), "s"(rs), "s"(so), "s"(lds) : "memory");
}
__device__ __forceinline__ void dma16x(unsigned vo, rsrc_t rs, unsigned so, unsigned lds) {
  asm volatile("s_mov_b32 m0, %3\n\tbuffer_load_dwordx4 %0, %1, %2 offen " XPOL " lds" :: "v"(vo), "s"(rs), "s"(so), "s"(lds) : "memory");
}
__device__ __forceinline__ void touch(const unsigned char* a) {
  asm volatile("global_load_dword v%c1, %0, off" :: "v"(a), "i"(HD) : "memory");
}

#define HS0 0
#define XS0 0
constexpr int HW_LINES = (8 - HS0) * 512 * 2 * XG;
constexpr int NHW = (HW_LINES + 112 * 64 - 1) / (112 * 64);
constexpr int HW_LPI = (HW_LINES + 112 * NHW - 1) / (112 * NHW);
constexpr int HX_LINES = (8 - XS0) * 320, HX_WAVES = 16 * XG;
constexpr int NHX = (HX_LINES + HX_WAVES * 64 - 1) / (HX_WAVES * 64);
constexpr int HX_LPI = (HX_LINES + HX_WAVES * NHX - 1) / (HX_WAVES * NHX);
constexpr int NHINT = NHW + NHX;
__device__ __forceinline__ int rot_order(int p) { return p / NROT + (p % NROT) * (8 / NROT); }
struct Hint { const unsigned char* wb; const unsigned char* xb; int wi, xi; };
template <int Q> __device__ __forceinline__ void hint(const Hint& h) {
  const int lane = threadIdx.x & 63;
  if constexpr (Q < NHX) {
    int line = (Q * HX_WAVES + h.xi) * HX_LPI + (lane < HX_LPI ? lane : HX_LPI - 1);
    line = line < HX_LINES ? line : HX_LINES - 1;
    const int r = rot_order(XS0 + line / 320), rem = line % 320;
    touch(h.xb + (rem >> 2) * KDIM + r * 512 + (rem & 3) * 128);
  } else {
    int line = ((Q - NHX) * 112 + h.wi) * HW_LPI + (lane < HW_LPI ? lane : HW_LPI - 1);
    line = line < HW_LINES ? line : HW_LINES - 1;
    const int row = HS0 * 512 + line / (2 * XG);
    touch(h.wb + (size_t)(rot_order(row >> 9) * 512 + (row & 511)) * NFULL + (line % (2 * XG)) * 128);
  }
}

__device__ __forceinline__ unsigned long long rt_now() {
  unsigned long long t;
  asm volatile("s_memrealtime %0\n\ts_waitcnt lgkmcnt(0)" : "=s"(t) :: "memory");
  return t;
}

__device__ __forceinline__ void slow_wait(G& g) {
  unsigned long long t0 = rt_now();
  const unsigned lane = __lane_id();
  unsigned lo = 0, hi = 64;
  unsigned long long m;
  while (true) {
    unsigned long long v = 0;
    if (lane >= lo && lane < hi) v = __hip_atomic_load(g.flag_ptr, __ATOMIC_RELAXED, __HIP_MEMORY_SCOPE_SYSTEM);
    unsigned tvv = __hip_atomic_load(g.tmo, __ATOMIC_RELAXED, __HIP_MEMORY_SCOPE_AGENT);
    unsigned v0 = (unsigned)v, v1 = (unsigned)(v >> 32);
    asm volatile("" : "+v"(v0), "+v"(v1), "+v"(tvv));
    m = (__builtin_amdgcn_ballot_w64((v0 < v1 ? v0 : v1) >= g.call) | IBM) >> g.kidx;
    if (m & 1ull) break;
    unsigned tv = __builtin_amdgcn_readfirstlane(tvv);
    if (tv == g.call || rt_now() - t0 > DEADLINE_TICKS) {
      __hip_atomic_store(g.tmo, g.call, __ATOMIC_RELAXED, __HIP_MEMORY_SCOPE_AGENT);
      asm volatile("ds_write_b32 %0, %1" :: "v"((unsigned)(LDS_MISC + 4)), "v"(1u) : "memory");
      m = ~0ull;
      break;
    }
    lo = g.kidx; hi = g.kidx + POLLW;
    __builtin_amdgcn_s_sleep(SLEEPN);
  }
  g.mask = m;
}

template <int OFF, int J, int NF_> __device__ __forceinline__ void lds_wr(unsigned a) {
  constexpr int A = acc_reg(J, NF_);
  asm volatile("ds_write_b128 %0, v[%c1:%c2] offset:%c3" :: "v"(a), "i"(A), "i"(A + 3), "i"(OFF) : "memory");
}
template <int OFF> __device__ __forceinline__ void lds_rd(v4f& v, unsigned a) {
  asm volatile("ds_read_b128 %0, %1 offset:%2" : "=v"(v) : "v"(a), "i"(OFF) : "memory");
}

template <int CNT> __device__ __forceinline__ void lds_wait(v4f (&r)[4], unsigned& b) {
  asm volatile("s_waitcnt lgkmcnt(%5)" : "+v"(r[0]), "+v"(r[1]), "+v"(r[2]), "+v"(r[3]), "+v"(b) : "i"(CNT) : "memory");
}
__device__ __forceinline__ void lds_rd32(unsigned& v, unsigned a) {
  asm volatile("ds_read_b32 %0, %1" : "=v"(v) : "v"(a) : "memory");
}
__device__ __forceinline__ void st_out(const v4f& v, unsigned vo, rsrc_t rs, unsigned so) {
  asm volatile("buffer_store_dwordx4 %0, %1, %2, %3 offen " OST "\n\ts_nop 1" :: "v"(v), "v"(vo), "s"(rs), "s"(so) : "memory");
}

__device__ __forceinline__ void st_mark(unsigned vo, rsrc_t rs, unsigned so) {
  const v4i mk = {0, 0, 0, 0};
  asm volatile("buffer_store_dwordx4 %0, %1, %2, %3 offen " MKST "\n\ts_nop 1" :: "v"(mk), "v"(vo), "s"(rs), "s"(so) : "memory");
}

#define RH_N -1
#define RH0_N -1
constexpr int NRH = 2 / XG > 0 ? 2 / XG : 1;
__device__ __forceinline__ void touch_stg(unsigned vo, rsrc_t rs, unsigned so) {
  asm volatile("buffer_load_dword v%c3, %0, %1, %2 offen " XLD :: "v"(vo), "s"(rs), "s"(so), "i"(HD) : "memory");
}
__device__ __forceinline__ void rhint(G& g) {
  sfor<NRH>([&](auto qc) { touch_stg(g.vo_rh, g.rs_stg, g.ub_nxt + g.rh_off + decltype(qc)::value * 8192u); });
}

#define PK_A 1
#define PK_B 17
#define PK_0 22
__device__ __forceinline__ void peek_issue(G& g) {
  asm volatile("global_load_dwordx2 v[%c1:%c2], %0, off sc0 sc1" :: "v"(g.flag_ptr), "i"(FD), "i"(FD + 1) : "memory");
}
__device__ __forceinline__ void peek_use(G& g) {
  unsigned long long m;
  unsigned t;
  asm volatile("v_min_u32 %1, v%c3, v%c4\n\tv_cmp_le_u32 %0, %2, %1" : "=s"(m), "=&v"(t) : "s"(g.call), "i"(FD), "i"(FD + 1));
  g.mask |= m >> (g.kidx & 63u);
}

template <int NF, int PREV, bool HAS_NEXT> struct Sched {
  static constexpr int NG = 8 * NF;
  static constexpr bool has_load(int n) { return HAS_NEXT || (n - 1 + R < NG); }
  static constexpr int n_out(int n) { return PREV == 0 ? 0 : ((n >= 5 && n <= 8) ? 1 : 0) + ((PREV == 5 && n == 9) ? 1 : 0); }
  static constexpr bool mark_slot(int n) { return n >= MK_N && n < MK_N + 2 * (9 - IB_U); }
  static constexpr bool has_mark(int n) { return MK_U == 1 && PREV == 5 && mark_slot(n); }
  static constexpr int n_store(int n) { return n_out(n) + (has_mark(n) ? 1 : 0); }
  static constexpr bool has_tmo(int n) { return HAS_NEXT ? n == 30 - R : n == NG - 1; }
  static constexpr bool has_rh(int n) { return HAS_NEXT && n == RH_N; }
  static constexpr bool has_peek(int n) { return HAS_NEXT && (n == PK_A || n == PK_B); }
  static constexpr bool use_peek(int n) { return has_peek(n - R + 1); }
  static constexpr int ops(int q) {
    if (q < 0) return 2;
    return (has_load(q) ? 2 : 0) + n_store(q) + (has_tmo(q) ? 1 : 0) + (has_rh(q) ? NRH : 0) + (has_peek(q) ? 1 : 0);
  }
  static constexpr int wait_cnt(int n) {
    int c = 0;
    for (int q = n - R + 2; q <= n - 1; ++q) c += ops(q);
    return c > 63 ? 63 : c;
  }
};

template <int SLOT> __device__ __forceinline__ unsigned ib_min() {
  constexpr int X = XB + 8 * SLOT;
  unsigned t0, t1;
  asm volatile("v_min3_u32 %0, v[%c2], v[%c3], v[%c4]\n\t"
               "v_min3_u32 %1, v[%c5], v[%c6], v[%c7]\n\t"
               "v_min3_u32 %0, v[%c8], v[%c9], %0\n\t"
               "v_min_u32 %0, %0, %1"
               : "=&v"(t0), "=&v"(t1)
               : "i"(X), "i"(X + 1), "i"(X + 2), "i"(X + 3), "i"(X + 4), "i"(X + 5), "i"(X + 6), "i"(X + 7));
  return t0;
}
__device__ __forceinline__ unsigned long long ib_zero(unsigned t) {
  unsigned long long m;
  asm volatile("v_cmp_eq_u32 %0, 0, %1" : "=s"(m) : "v"(t));
  return m;
}
template <int SLOT> __device__ __forceinline__ bool ib_dirty() { return ib_zero(ib_min<SLOT>()) != 0ull; }
template <int M, int ROFF, int ROTL> __device__ __forceinline__ void ib_load(G& g) {
  constexpr int slot = (ROFF + M) % R;
  const unsigned so = (M < 32 ? g.ub_cur + (ROTL < 0 ? g.kd[(M / 4) & 7] : (M / 4) * 32768u) : g.ub_nxt + g.kn[((M - 32) / 4) & 1]) + g.fo[M % 4];
  ld_stg<slot, 0>(g.vo_stg, g.rs_stg, so);
  ld_stg<slot, 1>(g.vo_stg, g.rs_stg, so);
}
template <int N, int ROFF, bool HAS_NEXT, int ROTL> __device__ __forceinline__ void ib_retry(G& g) {
  constexpr int KE = N | 3;
  constexpr int LE = (HAS_NEXT || N + R - 2 < 31) ? N + R - 2 : 31;
  constexpr int slot = (ROFF + N) % R;
  const unsigned long long t0 = rt_now();
  const unsigned lane = __lane_id();
  bool flagged = false;
  while (true) {
    unsigned long long v = 0;
    if (lane == g.kbase + N / 4) v = __hip_atomic_load(g.flag_ptr, __ATOMIC_RELAXED, __HIP_MEMORY_SCOPE_SYSTEM);
    unsigned tvv = __hip_atomic_load(g.tmo, __ATOMIC_RELAXED, __HIP_MEMORY_SCOPE_AGENT);
    sfor<KE - N + 1>([&](auto mc) { ib_load<N + decltype(mc)::value, ROFF, ROTL>(g); });
    unsigned v0 = (unsigned)v, v1 = (unsigned)(v >> 32);
    asm volatile("s_waitcnt vmcnt(0)" : "+v"(v0), "+v"(v1), "+v"(tvv) :: "memory");
    if (flagged || !ib_dirty<slot>()) break;
    flagged = __builtin_amdgcn_ballot_w64((v0 < v1 ? v0 : v1) >= g.call) != 0ull;
    unsigned tv = __builtin_amdgcn_readfirstlane(tvv);
    if (tv == g.call || rt_now() - t0 > DEADLINE_TICKS) {
      __hip_atomic_store(g.tmo, g.call, __ATOMIC_RELAXED, __HIP_MEMORY_SCOPE_AGENT);
      asm volatile("ds_write_b32 %0, %1" :: "v"((unsigned)(LDS_MISC + 4)), "v"(1u) : "memory");
      break;
    }
    if (!flagged) __builtin_amdgcn_s_sleep(IB_SLEEP);
  }
  if constexpr (LE > KE) sfor<LE - KE>([&](auto mc) { ib_load<KE + 1 + decltype(mc)::value, ROFF, ROTL>(g); });
}

template <int NF, int ROFF, int PREV, bool HAS_NEXT, int ROTL, bool IBC>
__device__ __forceinline__ void unit_body(G& g) {
  using S = Sched<NF, PREV, HAS_NEXT>;
  constexpr int NG = S::NG;
  sfor<NG>([&](auto nc) {
    constexpr int n = decltype(nc)::value;
    constexpr int i = n / NF, j = n % NF;
    constexpr int wi = ROTL < 0 ? i : (i + 8 - ROTL) & 7;
    constexpr int slot = (ROFF + n) % R;
    constexpr int pslot = (ROFF + n + R - 1) % R;
    constexpr int t = n - 1 + R;
    constexpr bool t_next = t >= NG;
    constexpr int ti = t_next ? (t - NG) / 4 : t / NF, tj = t_next ? (t - NG) % 4 : t % NF;

    wait_vm<S::wait_cnt(n)>();
    if constexpr (IBC && (n == 0 || !IB_PIPE)) {
      if (__builtin_expect(ib_dirty<slot>(), 0)) ib_retry<n, ROFF, HAS_NEXT, ROTL>(g);
    }

    if constexpr (PREV != 0 && n == 4) {
      asm volatile("s_waitcnt lgkmcnt(0)\n\ts_barrier" ::: "memory");
    }
    v4f ov, ot0, ot1;
    constexpr bool has_rd = PREV != 0 && ((n >= 4 && n <= 7) || (PREV == 5 && n == 8));
    constexpr int roff = (n >= 4 && n <= 7) ? (n - 4) * 4 * (int)RED_ROW : 0;
    constexpr int rb = n & 1;
    if constexpr (S::has_load(n) && tj == 0) {
      if constexpr (!IBC) {
        if (__builtin_expect(!(g.mask & 1ull), 0)) slow_wait(g);
        g.mask >>= 1;
        g.kidx += 1;
      }
    }
    if constexpr (S::has_rh(n)) rhint(g);
    if constexpr (S::has_peek(n)) peek_issue(g);
    if constexpr (S::has_tmo(n)) {
      asm volatile("buffer_load_dword %0, off, %1, 0 sc1" : "=v"(g.tmov) : "s"(g.rs_tmo) : "memory");
    }

    auto load_half = [&](auto hc) {
      constexpr int h = decltype(hc)::value;
      if constexpr (S::has_load(n))
        ld_stg<pslot, h>(g.vo_stg, g.rs_stg, (t_next ? g.ub_nxt + g.kn[ti & 1] : g.ub_cur + (ROTL < 0 ? g.kd[ti & 7] : ti * 32768u)) + g.fo[tj]);
    };
    auto side = [&](auto qc) {
      constexpr int q = decltype(qc)::value;
      if constexpr (n == NG - 2) lds_wr<0 * RED_TILE + q * 64, 0, q>(g.wa_cur);
      if constexpr (n == NG - 1) lds_wr<1 * RED_TILE + q * 64, 1, q>(g.wa_cur);
      if constexpr (PREV == 4 && n == 0) lds_wr<2 * RED_TILE + q * 64, 2, q>(g.wa_prev);
      if constexpr (PREV == 4 && n == 1) lds_wr<3 * RED_TILE + q * 64, 3, q>(g.wa_prev);
      if constexpr (PREV == 5 && n == 0) lds_wr<3 * RED_TILE + q * 64, 3, q>(g.wa_prev);
      if constexpr (PREV == 5 && n == 1) lds_wr<q * 64, 4, q>(g.wa5);
      if constexpr (PREV != 0 && n == 4 && q == 0)
        asm volatile("ds_read_b32 %0, %1" : "=v"(g.badv) : "v"((unsigned)(LDS_MISC + 4)) : "memory");
      if constexpr (has_rd && q < 2) {
        if constexpr (n <= 7) {
          lds_rd<roff>(g.rd[rb][(2 * q) & 3], g.ra[(2 * q) & 3]);
          lds_rd<roff>(g.rd[rb][(2 * q + 1) & 3], g.ra[(2 * q + 1) & 3]);
        } else {
          lds_rd<0>(g.rd[rb][(2 * q) & 3], q == 0 ? g.ra5 : g.ra5 + 2 * PART);
          lds_rd<PART>(g.rd[rb][(2 * q + 1) & 3], q == 0 ? g.ra5 : g.ra5 + 2 * PART);
        }
      }
      if constexpr (S::n_out(n) != 0 && q == 0) {
        constexpr int pb = rb ^ 1;
        asm volatile("s_waitcnt lgkmcnt(%5)" : "+v"(g.rd[pb][0]), "+v"(g.rd[pb][1]), "+v"(g.rd[pb][2]), "+v"(g.rd[pb][3]), "+v"(g.badv) : "i"(has_rd ? 2 : 0));
        ot0 = g.rd[pb][0] + g.rd[pb][1];
        ot1 = g.rd[pb][2] + g.rd[pb][3];
        asm volatile("" : "+v"(ot0), "+v"(ot1));
        if constexpr (n == 5) {
          unsigned bad = __builtin_amdgcn_readfirstlane(g.badv) | (__builtin_amdgcn_readfirstlane(g.tmov) == g.call ? 1u : 0u);
          if (bad) g.scale = __builtin_nanf("");
        }
      }
      if constexpr (S::n_out(n) != 0 && q == 1) {
        const v4f sum = ot0 + ot1;
        const float sc = g.scale;
        ov = (v4f){sum[0] * sc, sum[1] * sc, sum[2] * sc, sum[3] * sc};
        asm volatile("" : "+v"(ov));
      }
    };
    auto store_out = [&]() {
      if constexpr (S::n_out(n) != 0) {
        const unsigned so = n < 9 ? g.out_prev + (n - 5) * 4 * (NLOC * 4) : g.out5;
        st_out(ov, g.vo_out, g.rs_out, so);
      }
      if constexpr (S::has_mark(n)) st_mark(g.vo_stg, g.rs_mk, g.mk_off + ((n - MK_N) >> 1) * UNIT_BYTES + ((n - MK_N) & 1) * 1024u);
      if constexpr (MK_U > 1 && PREV == 4 && HAS_NEXT && S::mark_slot(n)) {
        if (g.unit == (unsigned)MK_U) st_mark(g.vo_stg, g.rs_mk, g.mk_off + ((n - MK_N) >> 1) * UNIT_BYTES + ((n - MK_N) & 1) * 1024u);
      }
    };

    unsigned ibt = 0;
    unsigned long long ibm = 0;
    sfor<4>([&](auto fc) {
      constexpr int nf = decltype(fc)::value;
      mfma<j, nf, slot, i == 0>(g.Wf[wi][nf]);
      side(fc);
      if constexpr (nf == 0) load_half(std::integral_constant<int, 0>{});
      if constexpr (nf == 1) store_out();
      if constexpr (nf == 2) load_half(std::integral_constant<int, 1>{});
      if constexpr (IBC && IB_PIPE == 1 && n + 1 < NG) {
        if constexpr (nf == 2) {
          wait_vm<S::wait_cnt(n + 1)>();
          ibt = ib_min<(ROFF + n + 1) % R>();
        }
        if constexpr (nf == 3) {
          if (__builtin_expect(ib_zero(ibt) != 0ull, 0)) ib_retry<n + 1, ROFF, HAS_NEXT, ROTL>(g);
        }
      }
      if constexpr (IBC && IB_PIPE == 2 && n + 1 < NG) {
        if constexpr (nf == 1) {
          wait_vm<S::wait_cnt(n + 1) - (S::has_load(n) ? 1 : 0)>();
          ibt = ib_min<(ROFF + n + 1) % R>();
        }
        if constexpr (nf == 2) ibm = ib_zero(ibt);
        if constexpr (nf == 3) {
          if (__builtin_expect(ibm != 0ull, 0)) ib_retry<n + 1, ROFF, HAS_NEXT, ROTL>(g);
        }
      }
      if constexpr (S::use_peek(n) && nf == 1) peek_use(g);
    });
  });
}

struct U0 {
  unsigned vo_w, vo_x, ra_w, ra_x0, ra_x1;
  unsigned so_w, lds_w, lds_x;
  rsrc_t rs_w;
};
#define HT_X -1
#define HT_X1 108
#define HT_W0 -1
#define HT_W1 4
#define HT_W2 108
struct U0S {
  static constexpr int NG = 40;
  static constexpr int w_issue(int n, int k) {
    int m = -1;
    if (n == 0) m = WD + k;
    else if (k == 0 && n % 5 == 3) m = 2 * (n / 5) + 2 + WD;
    else if (k == 0 && n % 5 == 0) m = 2 * (n / 5) + 1 + WD;
    return m <= 15 ? m : -1;
  }
  static constexpr int w_read(int n) {
    int m = n % 5 == 2 ? 2 * (n / 5 + 1) : (n % 5 == 4 ? 2 * (n / 5 + 1) + 1 : -1);
    return m <= 15 ? m : -1;
  }
  static constexpr int w_need(int m) { return m < 2 ? -1 : (m % 2 == 0 ? 5 * (m / 2 - 1) + 2 : 5 * (m / 2 - 1) + 4); }
  static constexpr int hint_t(int q) { return q < NHX ? (q == 0 ? HT_X : HT_X1) : (q == NHX ? HT_W0 : (q == NHX + 1 ? HT_W1 : HT_W2)); }
  static constexpr bool remote(int n) { return n - 1 + R >= NG; }
  static constexpr int NPRO_MAX = WD + XD + NHINT;
  static constexpr int pro_op(int idx) {
    int c = 0;
    for (int t = -2; t < 100; ++t) {
      if (t == -1) { if (c++ == idx) return 100; }
      for (int m = 0; m < WD; ++m) if (w_need(m) == t) { if (c++ == idx) return 200 + m; }
      if (t + 1 < XD && t >= 0) { if (c++ == idx) return 100 + t + 1; }
      for (int q = 0; q < NHINT; ++q) if (hint_t(q) == t) { if (c++ == idx) return 300 + q; }
    }
    return -1;
  }
  static constexpr int op_cnt(int op) { return op < 0 ? 0 : (op < 200 ? 2 : (op < 300 ? 4 : 1)); }
  static constexpr int n_hint(int n) { int c = 0; for (int q = 0; q < NHINT; ++q) if (hint_t(q) == 100 + n) ++c; return c; }
  static constexpr int wait_cnt(int n) {
    int pos = 0, pl[NG + 16] = {}, pw[16] = {};
    pos += 1;
    for (int idx = 0; idx < NPRO_MAX; ++idx) {
      const int op = pro_op(idx);
      pos += op_cnt(op);
      if (op >= 100 && op < 200) pl[op - 100] = pos;
      if (op >= 200 && op < 300) pw[op - 200] = pos;
    }
    if (n < 0) return pos - (pw[1] > pl[0] ? pw[1] : pl[0]);
    for (int q = 0; q <= n; ++q) {
      for (int k = 0; k < 2; ++k) if (w_issue(q, k) >= 0) { pos += 4; pw[w_issue(q, k)] = pos; }
      if (q + XD < NG) { pos += 2; pl[q + XD] = pos; }
      pos += n_hint(q) + (q == RH0_N ? NRH : 0) + (q == PK_0 ? 1 : 0);
      if (remote(q)) pos += 1;
      if (q < n && remote(q)) pos += 1;
    }
    int need = n + 1 < NG ? pl[n + 1] : 0;
    if (w_read(n) >= 0 && pw[w_read(n)] > need) need = pw[w_read(n)];
    int c = pos - need;
    return c > 63 ? 63 : c;
  }
};
template <int M> __device__ __forceinline__ void u0_w_issue(const G& g, const U0& u) {
  sfor<4>([&](auto qc) {
    constexpr int q = decltype(qc)::value;
    dma16w(u.vo_w, u.rs_w, u.so_w + (g.kd[M >> 1] << 8) + (unsigned)((M & 1) * 64 + q * 16) * NFULL, u.lds_w + (M % WD) * 4096 + q * 1024);
  });
}
template <int N> __device__ __forceinline__ void u0_x_issue(const G& g, const U0& u) {
  dma16x(u.vo_x, g.rs_x, g.so_loc[N % 5] + (g.kd[N / 5] >> 6), u.lds_x + (N % XD) * 2048);
  dma16x(u.vo_x, g.rs_x, g.so_loc[N % 5] + (g.kd[N / 5] >> 6) + 8 * KDIM, u.lds_x + (N % XD) * 2048 + 1024);
}
template <int M> __device__ __forceinline__ void u0_w_read(G& g, const U0& u) {
#pragma unroll
  for (int tt = 0; tt < 2; ++tt)
#pragma unroll
    for (int nf = 0; nf < 4; ++nf) {
      v2i q = __builtin_amdgcn_ds_read_tr8_b64_v2i32((LAS v2i*)(uintptr_t)(u.ra_w + (M % WD) * 4096 + tt * 512 + nf * 16));
      g.Wf[M >> 1][nf][(M & 1) * 4 + tt * 2] = q[0];
      g.Wf[M >> 1][nf][(M & 1) * 4 + tt * 2 + 1] = q[1];
    }
}
template <int N> __device__ __forceinline__ void u0_x_read(G& g, const U0& u) {
  constexpr int X = XB + 8 * (N % R);
  asm volatile("ds_read_b128 v[%c1:%c2], %0 offset:%c3" :: "v"(u.ra_x0), "i"(X), "i"(X + 3), "i"((N % XD) * 2048) : "memory");
  asm volatile("ds_read_b128 v[%c1:%c2], %0 offset:%c3" :: "v"(u.ra_x1), "i"(X + 4), "i"(X + 7), "i"((N % XD) * 2048) : "memory");
}

__device__ __forceinline__ void unit0_body(G& g, U0& u, const Hint& h) {
  constexpr int NG = U0S::NG;
  sfor<NG>([&](auto nc) {
    constexpr int n = decltype(nc)::value;
    constexpr int i = n / 5, j = n % 5;
    constexpr int slot = n % R, pslot = (n + R - 1) % R;
    constexpr int t = n - 1 + R, ti = U0S::remote(n) ? (t - NG) / 4 : 0, tj = U0S::remote(n) ? (t - NG) % 4 : 0;

    asm volatile("s_waitcnt lgkmcnt(0)" ::: "memory");
    if constexpr (U0S::w_issue(n, 0) >= 0) u0_w_issue<U0S::w_issue(n, 0)>(g, u);
    if constexpr (U0S::w_issue(n, 1) >= 0) u0_w_issue<U0S::w_issue(n, 1)>(g, u);
    if constexpr (n + XD < NG) u0_x_issue<n + XD>(g, u);
    sfor<NHINT>([&](auto qc) { constexpr int q = decltype(qc)::value; if constexpr (U0S::hint_t(q) == 100 + n) hint<q>(h); });
    if constexpr (n == RH0_N) rhint(g);
    if constexpr (n == PK_0) peek_issue(g);
    if constexpr (PK_0 >= 0 && n == NG - R) peek_use(g);
    static_assert(PK_0 < 0 || (PK_0 + 1 + XD < NG && PK_0 + XD < NG - R), "peek is complete when it is used");
    if constexpr (U0S::remote(n) && tj == 0) {
      if (!(g.mask & 1ull)) slow_wait(g);
      g.mask >>= 1;
      g.kidx += 1;
    }
    sfor<4>([&](auto fc) {
      constexpr int nf = decltype(fc)::value;
      mfma<j, nf, slot, i == 0>(g.Wf[i][nf]);
      if constexpr (n == NG - 3) lds_wr<0 * RED_TILE + nf * 64, 0, nf>(g.wa_cur);
      if constexpr (n == NG - 2) lds_wr<1 * RED_TILE + nf * 64, 1, nf>(g.wa_cur);
      if constexpr (n == NG - 1) lds_wr<2 * RED_TILE + nf * 64, 2, nf>(g.wa_cur);
      if constexpr (U0S::remote(n) && (nf == 0 || nf == 2))
        ld_stg<pslot, nf / 2>(g.vo_stg, g.rs_stg, g.ub_nxt + g.kn[ti & 1] + g.fo[tj]);
      if constexpr (nf == 1 && n + 1 < NG) {
        asm volatile("s_waitcnt vmcnt(%0)" :: "i"(U0S::wait_cnt(n)) : "memory");
        u0_x_read<n + 1>(g, u);
        if constexpr (U0S::w_read(n) >= 0) u0_w_read<U0S::w_read(n)>(g, u);
      }
    });
  });
}

template <bool FIRST> __device__ __forceinline__ void next_unit(G& g, unsigned out_next) {
  g.out_prev = g.out_cur;
  g.out_cur = out_next;
  g.ub_cur = g.ub_nxt;
  g.ub_nxt += UNIT_BYTES;
  g.unit += 1u;
  g.kbase += 8u;
  g.kn[0] = g.unit == 7u ? 0u : g.kd[0];
  g.kn[1] = g.unit == 7u ? 32768u : g.kd[1];
  g.wa_prev = g.wa_cur;
  g.wa_cur += g.red_d;
  if (!FIRST) { g.ra[0] -= g.red_d; g.ra[1] -= g.red_d; g.ra[2] -= g.red_d; g.ra[3] -= g.red_d; }
  g.red_d = 0u - g.red_d;
}

__device__ __forceinline__ void gemm_role(const Params& p, unsigned char* smem_generic) {
  LAS unsigned char* smem = (LAS unsigned char*)smem_generic;
  const int b = blockIdx.x;
  const int xcd = b & 7, slot = (b >> 3) - (NCOMM >> 3);
  const int pidx = (xcd % XG) * 28 + slot;
  const int pi = pidx / (4 * XG), cb = (xcd / XG) * (4 * XG) + pidx % (4 * XG);
  const int tid = threadIdx.x, lane = tid & 63;
  const int w = __builtin_amdgcn_readfirstlane(tid >> 6);
  const int kg = lane >> 4, li = lane & 15;
  const int rank = p.rank;
  unsigned char* myws = p.ws[rank];
  const int src = (rank + 1 + pi) & 7;
  G g;
  asm volatile("" ::: "v255");
  GSTAMP(0);

  unsigned ticket = 0;
  if (lane == 0)
    asm volatile("global_atomic_add %0, %1, %2, off sc0 sc1" : "+v"(ticket) : "v"((unsigned*)(myws + CTRL_CALL + (b * 4 + w) * 32)), "v"(1u) : "memory");
  GSTAMP(17);
  const int row0 = pi * 80 < 432 ? pi * 80 : 432;
  g.rs_x = mk_rsrc(p.x, SRC_BYTES);
  g.rot = (unsigned)(pidx % NROT) * (8 / NROT);
  sfor<8>([&](auto ic) { constexpr int i = decltype(ic)::value; g.kd[i] = ((i + g.rot) & 7u) * 32768u; });
  g.kn[0] = g.kd[0]; g.kn[1] = g.kd[1];
  sfor<4>([&](auto jc) { constexpr int j = decltype(jc)::value; g.so_loc[j] = (unsigned)(row0 + ((j + 1 + w) & 3) * 16) * KDIM + w * 128; });
  g.so_loc[4] = (unsigned)(row0 + 64) * KDIM + w * 128;
  U0 u;
  {
    const unsigned base = (unsigned)(uintptr_t)smem + w * PART;
    u.rs_w = mk_rsrc(p.w, (unsigned)KDIM * NFULL);
    u.vo_w = (lane >> 2) * NFULL + (lane & 3) * 16;
    u.vo_x = (lane >> 3) * KDIM + ((lane & 7) ^ (lane >> 3)) * 16;
    u.so_w = (unsigned)rank * NLOC + cb * 64 + (unsigned)(w * 128) * NFULL;
    u.lds_w = base;
    u.lds_x = base + STG_X;
    u.ra_w = base + (kg * 16 + (li >> 1)) * 64 + (li & 1) * 8;
    u.ra_x0 = base + STG_X + (li >> 3) * 1024 + (li & 7) * 128 + ((kg ^ (li & 7)) * 16);
    u.ra_x1 = u.ra_x0 ^ 64u;
  }
  Hint h;
  h.wb = p.w + (size_t)rank * NLOC + (xcd / XG) * (256 * XG);
  h.xb = p.x + (size_t)row0 * KDIM;
  h.wi = slot * 4 + w;
  h.xi = (pidx % (4 * XG)) * 4 + w;
  sfor<U0S::NPRO_MAX>([&](auto ic) {
    constexpr int op = U0S::pro_op(decltype(ic)::value);
    if constexpr (op >= 100 && op < 200) u0_x_issue<op - 100>(g, u);
    if constexpr (op >= 200 && op < 300) u0_w_issue<op - 200>(g, u);
    if constexpr (op >= 300) hint<op - 300>(h);
  });
  GSTAMP(12);
  g.lds_bad = (LAS unsigned*)(smem + LDS_MISC + 4);
  if (tid == 0) asm volatile("ds_write_b32 %0, %1" :: "v"((unsigned)(uintptr_t)smem + LDS_MISC + 4), "v"(0u) : "memory");
  asm volatile("s_waitcnt vmcnt(%1) lgkmcnt(0)\n\ts_barrier" : "+v"(ticket) : "i"(U0S::wait_cnt(-1)) : "memory");
  g.call = __builtin_amdgcn_readfirstlane(ticket) + 1u;
  GSTAMP(13);
  u0_x_read<0>(g, u);
  u0_w_read<0>(g, u); u0_w_read<1>(g, u);

  g.scale = *(const CAS float*)p.sx * *(const CAS float*)p.sw;
  const unsigned cm = __builtin_amdgcn_readfirstlane(g.call % NSTAGE);
  g.rs_stg = mk_rsrc(myws + cm * STAGE_BYTES, STAGE_BYTES);
  {
    const unsigned gw = (unsigned)(b - NCOMM) * 4u + (unsigned)w;
    g.rs_mk = mk_rsrc(myws + (cm == 0u ? 2u : cm - 1u) * STAGE_BYTES, STAGE_BYTES);
    g.mk_off = ((unsigned)(rank + 1 + (gw >> 7)) & 7u) * SRC_BYTES + (IB_U - 1u) * UNIT_BYTES + (gw & 127u) * 2048u;
  }
  g.rs_out = mk_rsrc(p.out, (unsigned)(4096u * NLOC * 4u));
  g.flag_ptr = (const GAS unsigned long long*)(myws + CTRL_FLAG + src * 2048 + w * 512 + (lane < 56 ? (lane & 56) + ((lane + g.rot) & 7) : lane) * 8);
  g.tmo = (GAS unsigned*)(myws + CTRL_TMO);
  g.rs_tmo = mk_rsrc(myws + CTRL_TMO, 4);
  g.vo_stg = lane * 16;
  g.vo_rh = lane * 128;
  g.rh_off = (unsigned)(pidx % (4 * XG)) * (UNIT_BYTES / (4 * XG)) + (unsigned)w * (UNIT_BYTES / (16 * XG) - 8192u);
  g.vo_out = (lane >> 4) * (NLOC * 4) + (lane & 15) * 16;
  sfor<4>([&](auto jc) { constexpr int j = decltype(jc)::value; g.fo[j] = ((j + 1 + w) & 3) * 2048; });
  g.wa_cur = w * PART + li * RED_ROW + kg * 16;
  sfor<4>([&](auto qc) {
    constexpr int q = decltype(qc)::value;
    g.ra[q] = q * PART + ((w - q - 1) & 3) * RED_TILE + (lane >> 4) * RED_ROW + (lane & 15) * 16;
  });
  g.wa5 = w * PART + LDS_T5 + li * RED_ROW + kg * 16;
  g.ra5 = LDS_T5 + (4 * w + (lane >> 4)) * RED_ROW + (lane & 15) * 16;
  g.red_d = RED_WAVE;
  g.tmov = 0u;
  const unsigned colb = cb * 256;
  auto out_off = [&](int grow) -> unsigned { return (unsigned)(grow + w * 16) * (NLOC * 4) + colb; };
  g.out_cur = out_off(rank * MLOC + row0);
  g.out_prev = g.out_cur;
  g.out5 = (unsigned)(rank * MLOC + row0 + 64 + 4 * w) * (NLOC * 4) + colb;
  g.ub_cur = 0;
  g.ub_nxt = (unsigned)src * SRC_BYTES + w * 8192;
  g.mask = IBM;
  g.sw_n = 0; g.sw_t = 0;
  g.rt_n = g.rt_it = g.rt_t = g.rt_last = g.rt_lastn = 0;
  g.kbase = 0u - 8u;
  g.unit = 0u;
  static_assert(IB_U >= 3, "the first remote unit is never validated in-band");
  g.kidx = 0u;

  GSTAMP(1);
  unit0_body(g, u, h);
  GSTAMP(2);
  next_unit<true>(g, out_off(src * MLOC));
  unit_body<4, 40 % R, 5, true, -1, false>(g);
  GSTAMP(3);
  constexpr int PER = (32 % R == 0) ? 1 : 3;
  static_assert((32 * PER) % R == 0 && 6 % PER == 0, "ring size");
  static_assert(PER == 1, "ring size");
  constexpr int ULIM = IB_U < 8 ? IB_U : 8;
#define USTAMP(u) do {} while (0)
  for (int u = 2; u < ULIM; ++u) {
    next_unit<false>(g, out_off(src * MLOC + (u - 1) * 64));
    unit_body<4, (40 + 32) % R, 4, true, -1, false>(g);
    USTAMP(u);
  }
  if constexpr (ULIM < 8) {
    for (int u = ULIM; u < 8; ++u) {
      next_unit<false>(g, out_off(src * MLOC + (u - 1) * 64));
      unit_body<4, (40 + 32) % R, 4, true, -1, true>(g);
      USTAMP(u);
    }
  }
  next_unit<false>(g, out_off(src * MLOC + 7 * 64));
  sfor<NROT>([&](auto rc) {
    constexpr int ro = decltype(rc)::value * (8 / NROT);
    if (NROT == 1 || g.rot == (unsigned)ro) unit_body<4, (40 + 32 * 7) % R, 4, false, ro, (IB_U <= 8)>(g);
  });
  GSTAMP(10);

  sfor<4>([&](auto qc) { constexpr int q = decltype(qc)::value; lds_wr<2 * RED_TILE + q * 64, 2, q>(g.wa_cur); });
  asm volatile("s_nop 7\n\ts_nop 7\n\ts_nop 7" ::: "memory");
  sfor<4>([&](auto qc) { constexpr int q = decltype(qc)::value; lds_wr<3 * RED_TILE + q * 64, 3, q>(g.wa_cur); });
  asm volatile("s_waitcnt lgkmcnt(0)\n\ts_barrier" ::: "memory");
  {
    v4f r[4][4];
    lds_rd32(g.badv, LDS_MISC + 4);
    sfor<16>([&](auto qc) {
      constexpr int q = decltype(qc)::value;
      lds_rd<(q >> 2) * 4 * RED_ROW>(r[q >> 2][q & 3], g.ra[q & 3] - g.red_d);
    });
    float sc = g.scale;
    sfor<4>([&](auto sc_) {
      constexpr int q = decltype(sc_)::value;
      lds_wait<(3 - q) * 4>(r[q], g.badv);
      if constexpr (q == 0) {
        asm volatile("s_waitcnt vmcnt(0)" : "+v"(g.tmov) :: "memory");
        unsigned bad = __builtin_amdgcn_readfirstlane(g.badv) | (__builtin_amdgcn_readfirstlane(g.tmov) == g.call ? 1u : 0u);
        if (bad) sc = __builtin_nanf("");
      }
      const v4f sum = (r[q][0] + r[q][1]) + (r[q][2] + r[q][3]);
      const v4f v = {sum[0] * sc, sum[1] * sc, sum[2] * sc, sum[3] * sc};
      st_out(v, g.vo_out, g.rs_out, g.out_cur + q * 4 * (NLOC * 4));
    });
  }
}

#define CSTAMP(n) do {} while (0)
template <bool LEADER>
__device__ __forceinline__ void comm_wave(const Params& p, LAS unsigned char* smem, const unsigned call, const int j, const int w, const int lane, v4i d) {
  const int rank = p.rank;
  unsigned char* myws = p.ws[rank];
  const int cw = j * 4 + w;
  rsrc_t rs_x = mk_rsrc(p.x, SRC_BYTES);
  const unsigned vo = (unsigned)((w >> 1) * 16 + (lane & 15)) * KDIM + (w & 1) * 64 + (lane >> 4) * 16;
  const unsigned dst0 = (call % NSTAGE) * STAGE_BYTES + (unsigned)rank * SRC_BYTES + (unsigned)w * 1024u;
  rsrc_t rs[7];
#pragma unroll
  for (int q = 0; q < 7; ++q) {
    int pq = (rank + 1 + (q + cw) % 7) & 7;
    rs[q] = mk_rsrc(p.ws[pq], NSTAGE * STAGE_BYTES);
  }
  unsigned char* flagb = p.ws[(rank + 1 + (lane < 7 ? lane : 0)) & 7] + CTRL_FLAG + rank * 2048;
  unsigned* ticketp = (unsigned*)(myws + CTRL_TICKET);
  GAS unsigned* tmo = (GAS unsigned*)(myws + CTRL_TMO);
  const unsigned tbase = (call - 1u) * TICKETS_PER_CALL - 2u * NCOMM;
  const unsigned vs = lane * 16;
  LAS unsigned* tk = (LAS unsigned*)(smem + 1024);
  LAS unsigned* cnt = (LAS unsigned*)(smem + LDS_MISC + 64);

  auto publish = [&](unsigned n, unsigned t) {
    unsigned old = 0;
    if (lane == 0) old = __hip_atomic_fetch_add(cnt + (n & 7u), 1u, __ATOMIC_RELAXED, __HIP_MEMORY_SCOPE_WORKGROUP);
    old = __builtin_amdgcn_readfirstlane(old);
    if ((old & 3u) == 3u) {
      if (lane < 7) __hip_atomic_store((unsigned*)(flagb + ((t >> 1) & 3u) * 512u + (t >> 3) * 8u + (t & 1u) * 4u), call, __ATOMIC_RELAXED, __HIP_MEMORY_SCOPE_SYSTEM);
      CSTAMP(n + 1);
    }
  };

  unsigned t_cur = (unsigned)j, t_prev = 0;
  unsigned n = 0;
  while (true) {
    v4i dn;
    unsigned tick = 0, t_nxt = 0;
    const unsigned dst = dst0 + t_cur * 4096u;
    sfor<7>([&](auto qc) {
      constexpr int q = decltype(qc)::value;
      asm volatile("s_waitcnt vmcnt(%0)" :: "i"(COMM_W - 1) : "memory");
      if constexpr (q == COMM_W - 1) { if (n > 0) publish(n - 1, t_prev); }
      if constexpr (LEADER && q == 0) {
        if (n > 0 && lane == 0) asm volatile("global_atomic_add %0, %1, %2, off sc0 sc1" : "+v"(tick) : "v"(ticketp), "v"(1u) : "memory");
      }
      if constexpr (q == 5) {
        if constexpr (LEADER) {
          asm volatile("" : "+v"(tick));
          t_nxt = n == 0 ? (unsigned)(NCOMM + j) : __builtin_amdgcn_readfirstlane(tick) - tbase;
          if (lane == 0) tk[n + 1] = t_nxt + 1u;
        } else {
          unsigned long long t0 = 0;
          while (true) {
            unsigned v = __hip_atomic_load(tk + n + 1, __ATOMIC_RELAXED, __HIP_MEMORY_SCOPE_WORKGROUP);
            v = __builtin_amdgcn_readfirstlane(v);
            if (v) { t_nxt = v - 1u; break; }
            unsigned long long now = __builtin_amdgcn_s_memrealtime();
            if (!t0) t0 = now;
            if (now - t0 > DEADLINE_TICKS) {
              __hip_atomic_store(tmo, call, __ATOMIC_RELAXED, __HIP_MEMORY_SCOPE_AGENT);
              t_nxt = NPIECE; break;
            }
            __builtin_amdgcn_s_sleep(1);
          }
        }
        if (t_nxt < NPIECE)
          asm volatile("buffer_load_dwordx4 %0, %1, %2, %3 offen" : "=v"(dn) : "v"(vo), "s"(rs_x),
                       "s"((t_nxt >> 6) * UNIT_BYTES + (t_nxt & 1u) * (32u * KDIM) + ((t_nxt >> 1) & 31u) * 128u) : "memory");
      }
      st_sys(d, vs, rs[q], dst);
    });
    if (t_nxt >= NPIECE) break;
    asm volatile("s_waitcnt vmcnt(2)" : "+v"(dn) :: "memory");
    t_prev = t_cur; t_cur = t_nxt; d = dn; ++n;
  }
  asm volatile("s_waitcnt vmcnt(0)" ::: "memory");
  publish(n, t_cur);
}

__device__ __forceinline__ void comm_role(const Params& p, unsigned char* smem_generic) {
  LAS unsigned char* smem = (LAS unsigned char*)smem_generic;
  const int j = blockIdx.x;
  const int tid = threadIdx.x, lane = tid & 63;
  const int w = __builtin_amdgcn_readfirstlane(tid >> 6);
  unsigned char* myws = p.ws[p.rank];

  STAMP(myws, 0);
  v4i d;
  {
    rsrc_t rs_x = mk_rsrc(p.x, SRC_BYTES);
    const unsigned vo = (unsigned)((w >> 1) * 16 + (lane & 15)) * KDIM + (w & 1) * 64 + (lane >> 4) * 16;
    asm volatile("buffer_load_dwordx4 %0, %1, %2, %3 offen" : "=v"(d) : "v"(vo), "s"(rs_x), "s"((unsigned)((j & 1) * (32 * KDIM) + (j >> 1) * 128)) : "memory");
  }
  unsigned ticket = 0;
  if (lane == 0)
    asm volatile("global_atomic_add %0, %1, %2, off sc0" : "+v"(ticket) : "v"((unsigned*)(myws + CTRL_CALL + (j * 4 + w) * 32)), "v"(1u) : "memory");
  for (int i = tid; i < 520; i += 256) *(LAS unsigned*)(smem + 1024 + i * 4) = 0u;
  if (tid < 8) *(LAS unsigned*)(smem + LDS_MISC + 64 + tid * 4) = 0u;
  asm volatile("s_waitcnt lgkmcnt(0)\n\ts_barrier\n\ts_waitcnt vmcnt(0)" : "+v"(d), "+v"(ticket) :: "memory");
  const unsigned call = __builtin_amdgcn_readfirstlane(ticket) + 1u;
  if (w == 0) comm_wave<true>(p, smem, call, j, w, lane, d);
  else comm_wave<false>(p, smem, call, j, w, lane, d);
}

#define CWN 1
constexpr unsigned PGW = 18;
constexpr unsigned PTICKETS = NPIECE - 2 * PGW + PGW;
constexpr int CB = 128, TK = HD - 1;
template <int PAR> __device__ __forceinline__ void ld_piece(const unsigned (&vo)[4], rsrc_t rs_x, unsigned t) {
  const unsigned so = (t >> 6) * UNIT_BYTES + (t & 1u) * (32u * KDIM) + ((t >> 1) & 31u) * 128u;
  asm volatile("buffer_load_dwordx4 v[%c3:%c4], %0, %1, %2 offen" :: "v"(vo[0]), "s"(rs_x), "s"(so), "i"(CB + 16 * PAR), "i"(CB + 16 * PAR + 3) : "memory");
  asm volatile("buffer_load_dwordx4 v[%c3:%c4], %0, %1, %2 offen" :: "v"(vo[1]), "s"(rs_x), "s"(so), "i"(CB + 16 * PAR + 4), "i"(CB + 16 * PAR + 7) : "memory");
  asm volatile("buffer_load_dwordx4 v[%c3:%c4], %0, %1, %2 offen" :: "v"(vo[2]), "s"(rs_x), "s"(so), "i"(CB + 16 * PAR + 8), "i"(CB + 16 * PAR + 11) : "memory");
  asm volatile("buffer_load_dwordx4 v[%c3:%c4], %0, %1, %2 offen" :: "v"(vo[3]), "s"(rs_x), "s"(so), "i"(CB + 16 * PAR + 12), "i"(CB + 16 * PAR + 15) : "memory");
}
template <int PAR, int K> __device__ __forceinline__ void st_piece(unsigned vs, rsrc_t rs_d, unsigned dst) {
  asm volatile("buffer_store_dwordx4 v[%c3:%c4], %0, %1, %2 offen offset:%c5 sc0 sc1"
               :: "v"(vs), "s"(rs_d), "s"(dst), "i"(CB + 16 * PAR + 4 * K), "i"(CB + 16 * PAR + 4 * K + 3), "i"(K * 1024) : "memory");
}
__device__ __forceinline__ void tk_issue(unsigned* ptr) {
  asm volatile("global_atomic_add v%c2, %0, %1, off sc0 sc1" :: "v"(ptr), "v"(1u), "i"(TK) : "memory");
}
__device__ __forceinline__ void dummy_ld(unsigned* ptr) {
  asm volatile("global_load_dword v%c1, %0, off sc0 sc1" :: "v"(ptr), "i"(HD) : "memory");
}
__device__ __forceinline__ unsigned tk_read() {
  unsigned t;
  asm volatile("s_waitcnt vmcnt(5)\n\tv_readfirstlane_b32 %0, v%c1" : "=s"(t) : "i"(TK) : "memory");
  return t;
}
__device__ __forceinline__ void st_flag(unsigned char* a, unsigned v) {
  asm volatile("global_store_dword %0, %1, off sc0 sc1" :: "v"(a), "v"(v) : "memory");
}
struct C2 {
  unsigned vo[4], vs, call, tbase, t0, t1, tp1, tp2;
  rsrc_t rs_x, rs_d;
  unsigned char* flagb;
  unsigned* ticketp;
  unsigned char* flagown;
  int lane;
};
__device__ __forceinline__ void publish2(C2& c, unsigned t) {
  if (c.lane == 0) st_flag(c.flagb + ((t >> 1) & 3u) * 512u + (t >> 3) * 8u + (t & 1u) * 4u, c.call);
}
template <int PAR> __device__ __forceinline__ bool comm_iter(C2& c) {
  constexpr int PER_IT = 10;
  const bool more = c.t1 < NPIECE;
  if (c.lane == 0) {
    if (more) tk_issue(c.ticketp);
    else dummy_ld(c.ticketp);
  }
  const unsigned dst = c.t0 * 4096u;
  const unsigned tpub = CWN == 1 ? c.tp1 : c.tp2;
  wait_vm<PER_IT * CWN - 1>(); st_piece<PAR, 0>(c.vs, c.rs_d, dst);
  wait_vm<PER_IT * CWN - 1>(); st_piece<PAR, 1>(c.vs, c.rs_d, dst);
  wait_vm<PER_IT * CWN - 1>(); st_piece<PAR, 2>(c.vs, c.rs_d, dst);
  wait_vm<PER_IT * CWN - 2>();
  if (tpub < NPIECE) publish2(c, tpub);
  else if (c.lane == 0) dummy_ld(c.ticketp);
  st_piece<PAR, 3>(c.vs, c.rs_d, dst);
  const unsigned tk = tk_read();
  const unsigned t2 = more ? tk - c.tbase : NPIECE;
  ld_piece<PAR>(c.vo, c.rs_x, t2 < NPIECE ? t2 : 0u);
  c.tp2 = c.tp1; c.tp1 = c.t0; c.t0 = c.t1; c.t1 = t2;
  return c.t0 >= NPIECE;
}
__device__ __forceinline__ void comm_role2(const Params& p) {
  const int tid = threadIdx.x, lane = tid & 63;
  const int w = __builtin_amdgcn_readfirstlane(tid >> 6);
  const unsigned cw = blockIdx.x * 4 + w;
  const int rank = p.rank;
  unsigned char* myws = p.ws[rank];
  STAMP(myws, 0);
  if (cw >= 7 * PGW) return;
  const unsigned q = cw % 7u, gi = cw / 7u;
  unsigned char* pws = p.ws[(rank + 1 + q) & 7];
  C2 c;
  c.lane = lane;
  c.rs_x = mk_rsrc(p.x, SRC_BYTES);
#pragma unroll
  for (int k = 0; k < 4; ++k) c.vo[k] = (unsigned)((k >> 1) * 16 + (lane & 15)) * KDIM + (k & 1) * 64 + (lane >> 4) * 16;
  ld_piece<0>(c.vo, c.rs_x, gi);
  ld_piece<1>(c.vo, c.rs_x, PGW + gi);
  unsigned ticket = 0;
  if (lane == 0)
    asm volatile("global_atomic_add %0, %1, %2, off sc0" : "+v"(ticket) : "v"((unsigned*)(myws + CTRL_CALL + cw * 32)), "v"(1u) : "memory");
  asm volatile("s_waitcnt vmcnt(0)" : "+v"(ticket) :: "memory");
  c.call = __builtin_amdgcn_readfirstlane(ticket) + 1u;
  const unsigned cm = __builtin_amdgcn_readfirstlane(c.call % NSTAGE);
  c.rs_d = mk_rsrc(pws + cm * STAGE_BYTES + (unsigned)rank * SRC_BYTES, SRC_BYTES);
  c.flagb = pws + CTRL_FLAG + rank * 2048;
  c.ticketp = (unsigned*)(myws + CTRL_TICKET + q * 128);
  c.tbase = (c.call - 1u) * PTICKETS - 2u * PGW;
  c.flagown = myws + CTRL_CALL + cw * 32;
  c.vs = lane * 16;
  c.t0 = gi; c.t1 = PGW + gi; c.tp1 = NPIECE; c.tp2 = NPIECE;
  while (true) {
    if (comm_iter<0>(c)) break;
    if (comm_iter<1>(c)) break;
  }
  asm volatile("s_waitcnt vmcnt(0)" ::: "memory");
  if (CWN == 2 && c.tp2 < NPIECE) publish2(c, c.tp2);
  publish2(c, c.tp1);
}

__global__ void __launch_bounds__(256, 1) fused_ag_gemm(Params p) {
  extern __shared__ __attribute__((aligned(16))) unsigned char smem_dyn[];
  if (blockIdx.x < NCOMM) comm_role(p, smem_dyn);
  else gemm_role(p, smem_dyn);
}

}

extern "C" void kernel_launch(int rank, int nranks, ncclComm_t comm, hipStream_t stream,
                              void* const* d_in, const int* in_sizes, int n_in,
                              void* d_out, int out_size,
                              const int* in_dtypes, int out_dtype,
                              void* const* d_out_all,
                              void* const* d_ws_all, size_t ws_bytes) {
  if (nranks != NRANKS || n_in < 4 || ws_bytes < WS_NEEDED || (unsigned)rank >= (unsigned)NRANKS) {
    fprintf(stderr, "unsupported configuration\n");
    return;
  }
  static bool attr_done = false;
  if (!attr_done) {
    HIP_CHECK(hipFuncSetAttribute((const void*)fused_ag_gemm, hipFuncAttributeMaxDynamicSharedMemorySize, (int)LDS_BYTES));
    attr_done = true;
  }
  Params p;
  p.x = (const unsigned char*)d_in[0];
  p.w = (const unsigned char*)d_in[1];
  p.sx = (const float*)d_in[2];
  p.sw = (const float*)d_in[3];
  p.out = (float*)d_out;
  for (int i = 0; i < NRANKS; ++i) p.ws[i] = (unsigned char*)d_ws_all[i];
  p.rank = rank;
  hipLaunchKernelGGL(fused_ag_gemm, dim3(256), dim3(256), LDS_BYTES, stream, p);
  HIP_CHECK(hipGetLastError());
}
```

```cpp
#include <hip/hip_runtime.h>
#include <rccl/rccl.h>
#include <cstdio>
#include <utility>

#define HIP_CHECK(expr) do { hipError_t _e = (expr); if (_e != hipSuccess) { \
    fprintf(stderr, "HIP error %s at %s:%d\n", hipGetErrorString(_e), __FILE__, __LINE__); \
    return; } } while (0)

namespace {

typedef int v8i __attribute__((ext_vector_type(8)));
typedef int v4i __attribute__((ext_vector_type(4)));
typedef int v2i __attribute__((ext_vector_type(2)));
typedef unsigned v2u __attribute__((ext_vector_type(2)));
typedef float v4f __attribute__((ext_vector_type(4)));
#define LAS __attribute__((address_space(3)))
#define GAS __attribute__((address_space(1)))
#define CAS __attribute__((address_space(4)))
typedef __amdgpu_buffer_rsrc_t rsrc_t;

constexpr int NRANKS = 8;
constexpr int KDIM = 4096, NFULL = 16384, NLOC = 2048, MLOC = 512;
constexpr unsigned UNIT_BYTES = 64u * KDIM;
constexpr unsigned SRC_BYTES = (unsigned)MLOC * KDIM;
constexpr unsigned STAGE_BYTES = NRANKS * SRC_BYTES;
constexpr unsigned NSTAGE = 3;
constexpr unsigned CTRL = NSTAGE * STAGE_BYTES;
constexpr unsigned CTRL_FLAG = CTRL;
constexpr unsigned CTRL_TMO = CTRL + 16384;
constexpr unsigned CTRL_TICKET = CTRL + 16384 + 128;
constexpr unsigned CTRL_UCNT = CTRL + 16384 + 2048;
constexpr unsigned CTRL_UFLAG = CTRL + 16384 + 3072;
constexpr unsigned CTRL_CALL = CTRL + 20480;
constexpr unsigned WS_NEEDED = CTRL_CALL + 1024u * 32u;
constexpr int NCOMM = 32;
constexpr unsigned NPIECE = 512;
constexpr unsigned TICKETS_PER_CALL = NPIECE - 2 * NCOMM + NCOMM;
constexpr unsigned long long DEADLINE_TICKS = 100000000ull;

#define RING 8
#define COMM_P 1
#define MAPPING 0
#define XG 2
#define NROT 1
#define IB_U 3
#define PF_U 2
#define IB_PIPE 2
#define IB_FAST0 0
#define IB_SLEEP 1
#define MK_N 10
#define MK_U 1
#define MKSM 3
#define MKST "sc0 sc1"
#define COMM_MODE 3
#define COMM_W 4
#define SLEEPN 2
#define PF_MODE 0
#define POLLW 8
#define OSTM 4
#define OST "sc0 sc1 nt"
#define XLDM 3
#define XLD "sc0 sc1"
constexpr int R = RING;

#define WD 4
#define XD 5
constexpr unsigned RED_ROW = 272, RED_TILE = 16 * RED_ROW, RED_WAVE = 4 * RED_TILE;
constexpr unsigned PART = 39424, LDS_T5 = 2 * RED_WAVE, STG_X = WD * 4096;
constexpr unsigned LDS_MISC = 4 * PART;
constexpr unsigned LDS_BYTES = LDS_MISC + 256;
static_assert(PART % 128 == 0 && LDS_T5 + RED_TILE <= PART && STG_X + XD * 2048 <= PART && LDS_BYTES <= 160 * 1024, "LDS layout");
static_assert(STG_X >= 2 * RED_TILE, "tiles 0 and 1 of unit 0 are written while the last x slots are in use");

constexpr unsigned long long IBM = IB_U <= 8 ? ~0ull << (8 * (IB_U <= 8 ? IB_U - 1 : 0)) : 0ull;

struct Params {
  const unsigned char* x;
  const unsigned char* w;
  const float* sx;
  const float* sw;
  float* out;
  unsigned char* ws[NRANKS];
  int rank;
};

#define STAMP(ws, idx) do {} while (0)
#define GSTAMP(idx) do {} while (0)

template <class F, int... I> __device__ __forceinline__ void sfor_impl(F&& f, std::integer_sequence<int, I...>) {
  (f(std::integral_constant<int, I>{}), ...);
}
template <int N, class F> __device__ __forceinline__ void sfor(F&& f) { sfor_impl(f, std::make_integer_sequence<int, N>{}); }

__device__ __forceinline__ rsrc_t mk_rsrc(const void* p, unsigned bytes) {
  return __builtin_amdgcn_make_buffer_rsrc((void*)p, 0, bytes, 0x00020000);
}

struct G {
  v8i Wf[8][4];
  v4f rd[2][4];
  unsigned sw_n, sw_t;
  unsigned rt_n, rt_it, rt_t, rt_last, rt_lastn;
  unsigned long long mask;
  unsigned kidx;
  unsigned tmov;
  unsigned badv;
  unsigned vo_loc, vo_stg, vo_rh, vo_out, wa_cur, wa_prev, ra[4], wa5, ra5;
  rsrc_t rs_x, rs_stg, rs_out, rs_tmo, rs_mk;
  unsigned mk_off, rh_off;
  unsigned so_loc[5];
  unsigned fo[4];
  unsigned ub_cur, ub_nxt;
  unsigned kd[8];
  unsigned kn[2];
  unsigned rot;
  unsigned out_prev, out_cur;
  unsigned out5;
  unsigned red_d;
  unsigned call, unit;
  unsigned kbase;
  float scale;
  const GAS unsigned long long* flag_ptr;
  GAS unsigned* tmo;
  LAS unsigned* lds_bad;
};

constexpr int XB = 256 - 8 * R, AB = XB - 80, HD = AB - 1, FD = HD - 3, A4B = FD;
constexpr int acc_reg(int j, int nf) { return AB + 16 * j + 4 * nf; }

template <int J, int NF_, int SLOT, bool FIRST> __device__ __forceinline__ void mfma(const v8i& wf) {
  constexpr int A = acc_reg(J, NF_), X = XB + 8 * SLOT;
  if constexpr (FIRST)
    asm volatile("v_mfma_f32_16x16x128_f8f6f4 v[%c1:%c2], %0, v[%c3:%c4], 0 cbsz:1" :: "a"(wf), "i"(A), "i"(A + 3), "i"(X), "i"(X + 7));
  else
    asm volatile("v_mfma_f32_16x16x128_f8f6f4 v[%c1:%c2], %0, v[%c3:%c4], v[%c1:%c2] cbsz:1" :: "a"(wf), "i"(A), "i"(A + 3), "i"(X), "i"(X + 7));
}
template <int SLOT, int H> __device__ __forceinline__ void ld_stg(unsigned vo, rsrc_t rs, unsigned so) {
  constexpr int X = XB + 8 * SLOT + 4 * H;
  asm volatile("buffer_load_dwordx4 v[%c3:%c4], %0, %1, %2 offen offset:%c5 " XLD :: "v"(vo), "s"(rs), "s"(so), "i"(X), "i"(X + 3), "i"(H * 1024) : "memory");
}
template <int CNT> __device__ __forceinline__ void wait_vm() { asm volatile("s_waitcnt vmcnt(%c0)" :: "i"(CNT) : "memory"); }

__device__ __forceinline__ void st_sys(const v4i& d, unsigned vo, rsrc_t rs, unsigned so) {
  asm volatile("buffer_store_dwordx4 %0, %1, %2, %3 offen sc0 sc1\n\ts_nop 1" :: "v"(d), "v"(vo), "s"(rs), "s"(so) : "memory");
}

__device__ __forceinline__ void dma16(unsigned vo, rsrc_t rs, unsigned so, unsigned lds) {
  asm volatile("s_mov_b32 m0, %3\n\tbuffer_load_dwordx4 %0, %1, %2 offen lds" :: "v"(vo), "s"(rs), "s"(so), "s"(lds) : "memory");
}

#define WPOL ""
#define XPOL ""
__device__ __forceinline__ void dma16w(unsigned vo, rsrc_t rs, unsigned so, unsigned lds) {
  asm volatile("s_mov_b32 m0, %3\n\tbuffer_load_dwordx4 %0, %1, %2 offen " WPOL " lds" :: "v"(vo), "s"(rs), "s"(so), "s"(lds) : "memory");
}
__device__ __forceinline__ void dma16x(unsigned vo, rsrc_t rs, unsigned so, unsigned lds) {
  asm volatile("s_mov_b32 m0, %3\n\tbuffer_load_dwordx4 %0, %1, %2 offen " XPOL " lds" :: "v"(vo), "s"(rs), "s"(so), "s"(lds) : "memory");
}
__device__ __forceinline__ void touch(const unsigned char* a) {
  asm volatile("global_load_dword v%c1, %0, off" :: "v"(a), "i"(HD) : "memory");
}

#define HS0 0
#define XS0 0
constexpr int HW_LINES = (8 - HS0) * 512 * 2 * XG;
constexpr int NHW = (HW_LINES + 112 * 64 - 1) / (112 * 64);
constexpr int HW_LPI = (HW_LINES + 112 * NHW - 1) / (112 * NHW);
constexpr int HX_LINES = (8 - XS0) * 320, HX_WAVES = 16 * XG;
constexpr int NHX = (HX_LINES + HX_WAVES * 64 - 1) / (HX_WAVES * 64);
constexpr int HX_LPI = (HX_LINES + HX_WAVES * NHX - 1) / (HX_WAVES * NHX);
#define CODE_PF 0
#define CODE_X 0u
constexpr int NHINT = NHW + NHX + CODE_PF;
__device__ __forceinline__ int rot_order(int p) { return p / NROT + (p % NROT) * (8 / NROT); }
struct Hint { const unsigned char* wb; const unsigned char* xb; const unsigned char* cb; unsigned clen; int wi, xi; };
template <int Q> __device__ __forceinline__ void hint(const Hint& h) {
  const int lane = threadIdx.x & 63;
  if constexpr (Q >= NHX + NHW) {
    unsigned off = (unsigned)(h.wi * 8 + (lane & 7)) * 128u;
    off = off < h.clen - 4u ? off : h.clen - 4u;
    touch(h.cb + off);
  } else if constexpr (Q < NHX) {
    int line = (Q * HX_WAVES + h.xi) * HX_LPI + (lane < HX_LPI ? lane : HX_LPI - 1);
    line = line < HX_LINES ? line : HX_LINES - 1;
    const int r = rot_order(XS0 + line / 320), rem = line % 320;
    touch(h.xb + (rem >> 2) * KDIM + r * 512 + (rem & 3) * 128);
  } else {
    int line = ((Q - NHX) * 112 + h.wi) * HW_LPI + (lane < HW_LPI ? lane : HW_LPI - 1);
    line = line < HW_LINES ? line : HW_LINES - 1;
    const int row = HS0 * 512 + line / (2 * XG);
    touch(h.wb + (size_t)(rot_order(row >> 9) * 512 + (row & 511)) * NFULL + (line % (2 * XG)) * 128);
  }
}

__device__ __forceinline__ unsigned long long rt_now() {
  unsigned long long t;
  asm volatile("s_memrealtime %0\n\ts_waitcnt lgkmcnt(0)" : "=s"(t) :: "memory");
  return t;
}

__device__ __forceinline__ void slow_wait(G& g) {
  unsigned long long t0 = rt_now();
  const unsigned lane = __lane_id();
  unsigned lo = 0, hi = 64;
  unsigned long long m;
  while (true) {
    unsigned long long v = 0;
    if (lane >= lo && lane < hi) v = __hip_atomic_load(g.flag_ptr, __ATOMIC_RELAXED, __HIP_MEMORY_SCOPE_SYSTEM);
    unsigned tvv = __hip_atomic_load(g.tmo, __ATOMIC_RELAXED, __HIP_MEMORY_SCOPE_AGENT);
    unsigned v0 = (unsigned)v, v1 = (unsigned)(v >> 32);
    asm volatile("" : "+v"(v0), "+v"(v1), "+v"(tvv));
    m = (__builtin_amdgcn_ballot_w64((v0 < v1 ? v0 : v1) >= g.call) | IBM) >> g.kidx;
    if (m & 1ull) break;
    unsigned tv = __builtin_amdgcn_readfirstlane(tvv);
    if (tv == g.call || rt_now() - t0 > DEADLINE_TICKS) {
      __hip_atomic_store(g.tmo, g.call, __ATOMIC_RELAXED, __HIP_MEMORY_SCOPE_AGENT);
      asm volatile("ds_write_b32 %0, %1" :: "v"((unsigned)(LDS_MISC + 4)), "v"(1u) : "memory");
      m = ~0ull;
      break;
    }
    lo = g.kidx; hi = g.kidx + POLLW;
    __builtin_amdgcn_s_sleep(SLEEPN);
  }
  g.mask = m;
}

template <int OFF, int J, int NF_> __device__ __forceinline__ void lds_wr(unsigned a) {
  constexpr int A = acc_reg(J, NF_);
  asm volatile("ds_write_b128 %0, v[%c1:%c2] offset:%c3" :: "v"(a), "i"(A), "i"(A + 3), "i"(OFF) : "memory");
}
template <int OFF> __device__ __forceinline__ void lds_rd(v4f& v, unsigned a) {
  asm volatile("ds_read_b128 %0, %1 offset:%2" : "=v"(v) : "v"(a), "i"(OFF) : "memory");
}

template <int CNT> __device__ __forceinline__ void lds_wait(v4f (&r)[4], unsigned& b) {
  asm volatile("s_waitcnt lgkmcnt(%5)" : "+v"(r[0]), "+v"(r[1]), "+v"(r[2]), "+v"(r[3]), "+v"(b) : "i"(CNT) : "memory");
}
__device__ __forceinline__ void lds_rd32(unsigned& v, unsigned a) {
  asm volatile("ds_read_b32 %0, %1" : "=v"(v) : "v"(a) : "memory");
}
__device__ __forceinline__ void st_out(const v4f& v, unsigned vo, rsrc_t rs, unsigned so) {
  asm volatile("buffer_store_dwordx4 %0, %1, %2, %3 offen " OST "\n\ts_nop 1" :: "v"(v), "v"(vo), "s"(rs), "s"(so) : "memory");
}

__device__ __forceinline__ void st_mark(unsigned vo, rsrc_t rs, unsigned so) {
  const v4i mk = {0, 0, 0, 0};
  asm volatile("buffer_store_dwordx4 %0, %1, %2, %3 offen " MKST "\n\ts_nop 1" :: "v"(mk), "v"(vo), "s"(rs), "s"(so) : "memory");
}

#define RH_N -1
#define RH0_N -1
constexpr int NRH = 2 / XG > 0 ? 2 / XG : 1;
__device__ __forceinline__ void touch_stg(unsigned vo, rsrc_t rs, unsigned so) {
  asm volatile("buffer_load_dword v%c3, %0, %1, %2 offen " XLD :: "v"(vo), "s"(rs), "s"(so), "i"(HD) : "memory");
}
__device__ __forceinline__ void rhint(G& g) {
  sfor<NRH>([&](auto qc) { touch_stg(g.vo_rh, g.rs_stg, g.ub_nxt + g.rh_off + decltype(qc)::value * 8192u); });
}

#define PK_A 1
#define PK_B 17
#define PK_0 22
__device__ __forceinline__ void peek_issue(G& g) {
  asm volatile("global_load_dwordx2 v[%c1:%c2], %0, off sc0 sc1" :: "v"(g.flag_ptr), "i"(FD), "i"(FD + 1) : "memory");
}
__device__ __forceinline__ void peek_use(G& g) {
  unsigned long long m;
  unsigned t;
  asm volatile("v_min_u32 %1, v%c3, v%c4\n\tv_cmp_le_u32 %0, %2, %1" : "=s"(m), "=&v"(t) : "s"(g.call), "i"(FD), "i"(FD + 1));
  g.mask |= m >> (g.kidx & 63u);
}

template <int NF, int PREV, bool HAS_NEXT> struct Sched {
  static constexpr int NG = 8 * NF;
  static constexpr bool has_load(int n) { return HAS_NEXT || (n - 1 + R < NG); }
  static constexpr int n_out(int n) { return PREV == 0 ? 0 : ((n >= 5 && n <= 8) ? 1 : 0) + ((PREV == 5 && n == 9) ? 1 : 0); }
  static constexpr bool mark_slot(int n) { return n >= MK_N && n < MK_N + 2 * (9 - IB_U); }
  static constexpr bool has_mark(int n) { return MK_U == 1 && PREV == 5 && mark_slot(n); }
  static constexpr int n_store(int n) { return n_out(n) + (has_mark(n) ? 1 : 0); }
  static constexpr bool has_tmo(int n) { return HAS_NEXT ? n == 30 - R : n == NG - 1; }
  static constexpr bool has_rh(int n) { return HAS_NEXT && n == RH_N; }
  static constexpr bool has_peek(int n) { return HAS_NEXT && (n == PK_A || n == PK_B); }
  static constexpr bool use_peek(int n) { return has_peek(n - R + 1); }
  static constexpr int ops(int q) {
    if (q < 0) return 2;
    return (has_load(q) ? 2 : 0) + n_store(q) + (has_tmo(q) ? 1 : 0) + (has_rh(q) ? NRH : 0) + (has_peek(q) ? 1 : 0);
  }
  static constexpr int wait_cnt(int n) {
    int c = 0;
    for (int q = n - R + 2; q <= n - 1; ++q) c += ops(q);
    return c > 63 ? 63 : c;
  }
};

template <int SLOT> __device__ __forceinline__ unsigned ib_min() {
  constexpr int X = XB + 8 * SLOT;
  unsigned t0, t1;
  asm volatile("v_min3_u32 %0, v[%c2], v[%c3], v[%c4]\n\t"
               "v_min3_u32 %1, v[%c5], v[%c6], v[%c7]\n\t"
               "v_min3_u32 %0, v[%c8], v[%c9], %0\n\t"
               "v_min_u32 %0, %0, %1"
               : "=&v"(t0), "=&v"(t1)
               : "i"(X), "i"(X + 1), "i"(X + 2), "i"(X + 3), "i"(X + 4), "i"(X + 5), "i"(X + 6), "i"(X + 7));
  return t0;
}
__device__ __forceinline__ unsigned long long ib_zero(unsigned t) {
  unsigned long long m;
  asm volatile("v_cmp_eq_u32 %0, 0, %1" : "=s"(m) : "v"(t));
  return m;
}
template <int SLOT> __device__ __forceinline__ bool ib_dirty() { return ib_zero(ib_min<SLOT>()) != 0ull; }
template <int M, int ROFF, int ROTL> __device__ __forceinline__ void ib_load(G& g) {
  constexpr int slot = (ROFF + M) % R;
  const unsigned so = (M < 32 ? g.ub_cur + (ROTL < 0 ? g.kd[(M / 4) & 7] : (M / 4) * 32768u) : g.ub_nxt + g.kn[((M - 32) / 4) & 1]) + g.fo[M % 4];
  ld_stg<slot, 0>(g.vo_stg, g.rs_stg, so);
  ld_stg<slot, 1>(g.vo_stg, g.rs_stg, so);
}
template <int N, int ROFF, bool HAS_NEXT, int ROTL> __device__ __forceinline__ void ib_retry(G& g) {
  constexpr int KE = N | 3;
  constexpr int LE = (HAS_NEXT || N + R - 2 < 31) ? N + R - 2 : 31;
  constexpr int slot = (ROFF + N) % R;
  const unsigned long long t0 = rt_now();
  const unsigned lane = __lane_id();
  bool flagged = false;
  while (true) {
    unsigned long long v = 0;
    if (lane == g.kbase + N / 4) v = __hip_atomic_load(g.flag_ptr, __ATOMIC_RELAXED, __HIP_MEMORY_SCOPE_SYSTEM);
    unsigned tvv = __hip_atomic_load(g.tmo, __ATOMIC_RELAXED, __HIP_MEMORY_SCOPE_AGENT);
    sfor<KE - N + 1>([&](auto mc) { ib_load<N + decltype(mc)::value, ROFF, ROTL>(g); });
    unsigned v0 = (unsigned)v, v1 = (unsigned)(v >> 32);
    asm volatile("s_waitcnt vmcnt(0)" : "+v"(v0), "+v"(v1), "+v"(tvv) :: "memory");
    if (flagged || !ib_dirty<slot>()) break;
    flagged = __builtin_amdgcn_ballot_w64((v0 < v1 ? v0 : v1) >= g.call) != 0ull;
    unsigned tv = __builtin_amdgcn_readfirstlane(tvv);
    if (tv == g.call || rt_now() - t0 > DEADLINE_TICKS) {
      __hip_atomic_store(g.tmo, g.call, __ATOMIC_RELAXED, __HIP_MEMORY_SCOPE_AGENT);
      asm volatile("ds_write_b32 %0, %1" :: "v"((unsigned)(LDS_MISC + 4)), "v"(1u) : "memory");
      break;
    }
    if (!flagged) __builtin_amdgcn_s_sleep(IB_SLEEP);
  }
  if constexpr (LE > KE) sfor<LE - KE>([&](auto mc) { ib_load<KE + 1 + decltype(mc)::value, ROFF, ROTL>(g); });
}

template <int NF, int ROFF, int PREV, bool HAS_NEXT, int ROTL, bool IBC>
__device__ __forceinline__ void unit_body(G& g) {
  using S = Sched<NF, PREV, HAS_NEXT>;
  constexpr int NG = S::NG;
  sfor<NG>([&](auto nc) {
    constexpr int n = decltype(nc)::value;
    constexpr int i = n / NF, j = n % NF;
    constexpr int wi = ROTL < 0 ? i : (i + 8 - ROTL) & 7;
    constexpr int slot = (ROFF + n) % R;
    constexpr int pslot = (ROFF + n + R - 1) % R;
    constexpr int t = n - 1 + R;
    constexpr bool t_next = t >= NG;
    constexpr int ti = t_next ? (t - NG) / 4 : t / NF, tj = t_next ? (t - NG) % 4 : t % NF;

    wait_vm<S::wait_cnt(n)>();
    if constexpr (IBC && (n == 0 || !IB_PIPE)) {
      if (__builtin_expect(ib_dirty<slot>(), 0)) ib_retry<n, ROFF, HAS_NEXT, ROTL>(g);
    }

    if constexpr (PREV != 0 && n == 4) {
      asm volatile("s_waitcnt lgkmcnt(0)\n\ts_barrier" ::: "memory");
    }
    v4f ov, ot0, ot1;
    constexpr bool has_rd = PREV != 0 && ((n >= 4 && n <= 7) || (PREV == 5 && n == 8));
    constexpr int roff = (n >= 4 && n <= 7) ? (n - 4) * 4 * (int)RED_ROW : 0;
    constexpr int rb = n & 1;
    if constexpr (S::has_load(n) && tj == 0) {
      if constexpr (!IBC) {
        if (__builtin_expect(!(g.mask & 1ull), 0)) slow_wait(g);
        g.mask >>= 1;
        g.kidx += 1;
      }
    }
    if constexpr (S::has_rh(n)) rhint(g);
    if constexpr (S::has_peek(n)) peek_issue(g);
    if constexpr (S::has_tmo(n)) {
      asm volatile("buffer_load_dword %0, off, %1, 0 sc1" : "=v"(g.tmov) : "s"(g.rs_tmo) : "memory");
    }

    auto load_half = [&](auto hc) {
      constexpr int h = decltype(hc)::value;
      if constexpr (S::has_load(n))
        ld_stg<pslot, h>(g.vo_stg, g.rs_stg, (t_next ? g.ub_nxt + g.kn[ti & 1] : g.ub_cur + (ROTL < 0 ? g.kd[ti & 7] : ti * 32768u)) + g.fo[tj]);
    };
    auto side = [&](auto qc) {
      constexpr int q = decltype(qc)::value;
      if constexpr (n == NG - 2) lds_wr<0 * RED_TILE + q * 64, 0, q>(g.wa_cur);
      if constexpr (n == NG - 1) lds_wr<1 * RED_TILE + q * 64, 1, q>(g.wa_cur);
      if constexpr (PREV == 4 && n == 0) lds_wr<2 * RED_TILE + q * 64, 2, q>(g.wa_prev);
      if constexpr (PREV == 4 && n == 1) lds_wr<3 * RED_TILE + q * 64, 3, q>(g.wa_prev);
      if constexpr (PREV == 5 && n == 0) lds_wr<3 * RED_TILE + q * 64, 3, q>(g.wa_prev);
      if constexpr (PREV == 5 && n == 1) lds_wr<q * 64, 4, q>(g.wa5);
      if constexpr (PREV != 0 && n == 4 && q == 0)
        asm volatile("ds_read_b32 %0, %1" : "=v"(g.badv) : "v"((unsigned)(LDS_MISC + 4)) : "memory");
      if constexpr (has_rd && q < 2) {
        if constexpr (n <= 7) {
          lds_rd<roff>(g.rd[rb][(2 * q) & 3], g.ra[(2 * q) & 3]);
          lds_rd<roff>(g.rd[rb][(2 * q + 1) & 3], g.ra[(2 * q + 1) & 3]);
        } else {
          lds_rd<0>(g.rd[rb][(2 * q) & 3], q == 0 ? g.ra5 : g.ra5 + 2 * PART);
          lds_rd<PART>(g.rd[rb][(2 * q + 1) & 3], q == 0 ? g.ra5 : g.ra5 + 2 * PART);
        }
      }
      if constexpr (S::n_out(n) != 0 && q == 0) {
        constexpr int pb = rb ^ 1;
        asm volatile("s_waitcnt lgkmcnt(%5)" : "+v"(g.rd[pb][0]), "+v"(g.rd[pb][1]), "+v"(g.rd[pb][2]), "+v"(g.rd[pb][3]), "+v"(g.badv) : "i"(has_rd ? 2 : 0));
        ot0 = g.rd[pb][0] + g.rd[pb][1];
        ot1 = g.rd[pb][2] + g.rd[pb][3];
        asm volatile("" : "+v"(ot0), "+v"(ot1));
        if constexpr (n == 5) {
          unsigned bad = __builtin_amdgcn_readfirstlane(g.badv) | (__builtin_amdgcn_readfirstlane(g.tmov) == g.call ? 1u : 0u);
          if (bad) g.scale = __builtin_nanf("");
        }
      }
      if constexpr (S::n_out(n) != 0 && q == 1) {
        const v4f sum = ot0 + ot1;
        const float sc = g.scale;
        ov = (v4f){sum[0] * sc, sum[1] * sc, sum[2] * sc, sum[3] * sc};
        asm volatile("" : "+v"(ov));
      }
    };
    auto store_out = [&]() {
      if constexpr (S::n_out(n) != 0) {
        const unsigned so = n < 9 ? g.out_prev + (n - 5) * 4 * (NLOC * 4) : g.out5;
        st_out(ov, g.vo_out, g.rs_out, so);
      }
      if constexpr (S::has_mark(n)) st_mark(g.vo_stg, g.rs_mk, g.mk_off + ((n - MK_N) >> 1) * UNIT_BYTES + ((n - MK_N) & 1) * 1024u);
      if constexpr (MK_U > 1 && PREV == 4 && HAS_NEXT && S::mark_slot(n)) {
        if (g.unit == (unsigned)MK_U) st_mark(g.vo_stg, g.rs_mk, g.mk_off + ((n - MK_N) >> 1) * UNIT_BYTES + ((n - MK_N) & 1) * 1024u);
      }
    };

    unsigned ibt = 0;
    unsigned long long ibm = 0;
    sfor<4>([&](auto fc) {
      constexpr int nf = decltype(fc)::value;
      mfma<j, nf, slot, i == 0>(g.Wf[wi][nf]);
      side(fc);
      if constexpr (nf == 0) load_half(std::integral_constant<int, 0>{});
      if constexpr (nf == 1) store_out();
      if constexpr (nf == 2) load_half(std::integral_constant<int, 1>{});
      if constexpr (IBC && IB_PIPE == 1 && n + 1 < NG) {
        if constexpr (nf == 2) {
          wait_vm<S::wait_cnt(n + 1)>();
          ibt = ib_min<(ROFF + n + 1) % R>();
        }
        if constexpr (nf == 3) {
          if (__builtin_expect(ib_zero(ibt) != 0ull, 0)) ib_retry<n + 1, ROFF, HAS_NEXT, ROTL>(g);
        }
      }
      if constexpr (IBC && IB_PIPE == 2 && n + 1 < NG) {
        if constexpr (nf == 1) {
          wait_vm<S::wait_cnt(n + 1) - (S::has_load(n) ? 1 : 0)>();
          ibt = ib_min<(ROFF + n + 1) % R>();
        }
        if constexpr (nf == 2) ibm = ib_zero(ibt);
        if constexpr (nf == 3) {
          if (__builtin_expect(ibm != 0ull, 0)) ib_retry<n + 1, ROFF, HAS_NEXT, ROTL>(g);
        }
      }
      if constexpr (S::use_peek(n) && nf == 1) peek_use(g);
    });
  });
}

struct U0 {
  unsigned vo_w, vo_x, ra_w, ra_x0, ra_x1;
  unsigned so_w, lds_w, lds_x;
  rsrc_t rs_w;
};
#define HT_X -1
#define HT_X1 108
#define HT_C 4
#define HT_W0 -1
#define HT_W1 4
#define HT_W2 108
struct U0S {
  static constexpr int NG = 40;
  static constexpr int w_issue(int n, int k) {
    int m = -1;
    if (n == 0) m = WD + k;
    else if (k == 0 && n % 5 == 3) m = 2 * (n / 5) + 2 + WD;
    else if (k == 0 && n % 5 == 0) m = 2 * (n / 5) + 1 + WD;
    return m <= 15 ? m : -1;
  }
  static constexpr int w_read(int n) {
    int m = n % 5 == 2 ? 2 * (n / 5 + 1) : (n % 5 == 4 ? 2 * (n / 5 + 1) + 1 : -1);
    return m <= 15 ? m : -1;
  }
  static constexpr int w_need(int m) { return m < 2 ? -1 : (m % 2 == 0 ? 5 * (m / 2 - 1) + 2 : 5 * (m / 2 - 1) + 4); }
  static constexpr int hint_t(int q) { return q >= NHX + NHW ? HT_C : q < NHX ? (q == 0 ? HT_X : HT_X1) : (q == NHX ? HT_W0 : (q == NHX + 1 ? HT_W1 : HT_W2)); }
  static constexpr bool remote(int n) { return n - 1 + R >= NG; }
  static constexpr int NPRO_MAX = WD + XD + NHINT;
  static constexpr int pro_op(int idx) {
    int c = 0;
    for (int t = -2; t < 100; ++t) {
      if (t == -1) { if (c++ == idx) return 100; }
      for (int m = 0; m < WD; ++m) if (w_need(m) == t) { if (c++ == idx) return 200 + m; }
      if (t + 1 < XD && t >= 0) { if (c++ == idx) return 100 + t + 1; }
      for (int q = 0; q < NHINT; ++q) if (hint_t(q) == t) { if (c++ == idx) return 300 + q; }
    }
    return -1;
  }
  static constexpr int op_cnt(int op) { return op < 0 ? 0 : (op < 200 ? 2 : (op < 300 ? 4 : 1)); }
  static constexpr int n_hint(int n) { int c = 0; for (int q = 0; q < NHINT; ++q) if (hint_t(q) == 100 + n) ++c; return c; }
  static constexpr int wait_cnt(int n) {
    int pos = 0, pl[NG + 16] = {}, pw[16] = {};
    pos += 1;
    for (int idx = 0; idx < NPRO_MAX; ++idx) {
      const int op = pro_op(idx);
      pos += op_cnt(op);
      if (op >= 100 && op < 200) pl[op - 100] = pos;
      if (op >= 200 && op < 300) pw[op - 200] = pos;
    }
    if (n < 0) return pos - (pw[1] > pl[0] ? pw[1] : pl[0]);
    for (int q = 0; q <= n; ++q) {
      for (int k = 0; k < 2; ++k) if (w_issue(q, k) >= 0) { pos += 4; pw[w_issue(q, k)] = pos; }
      if (q + XD < NG) { pos += 2; pl[q + XD] = pos; }
      pos += n_hint(q) + (q == RH0_N ? NRH : 0) + (q == PK_0 ? 1 : 0);
      if (remote(q)) pos += 1;
      if (q < n && remote(q)) pos += 1;
    }
    int need = n + 1 < NG ? pl[n + 1] : 0;
    if (w_read(n) >= 0 && pw[w_read(n)] > need) need = pw[w_read(n)];
    int c = pos - need;
    return c > 63 ? 63 : c;
  }
};
template <int M> __device__ __forceinline__ void u0_w_issue(const G& g, const U0& u) {
  sfor<4>([&](auto qc) {
    constexpr int q = decltype(qc)::value;
    dma16w(u.vo_w, u.rs_w, u.so_w + (g.kd[M >> 1] << 8) + (unsigned)((M & 1) * 64 + q * 16) * NFULL, u.lds_w + (M % WD) * 4096 + q * 1024);
  });
}
template <int N> __device__ __forceinline__ void u0_x_issue(const G& g, const U0& u) {
  dma16x(u.vo_x, g.rs_x, g.so_loc[N % 5] + (g.kd[N / 5] >> 6), u.lds_x + (N % XD) * 2048);
  dma16x(u.vo_x, g.rs_x, g.so_loc[N % 5] + (g.kd[N / 5] >> 6) + 8 * KDIM, u.lds_x + (N % XD) * 2048 + 1024);
}
template <int M> __device__ __forceinline__ void u0_w_read(G& g, const U0& u) {
#pragma unroll
  for (int tt = 0; tt < 2; ++tt)
#pragma unroll
    for (int nf = 0; nf < 4; ++nf) {
      v2i q = __builtin_amdgcn_ds_read_tr8_b64_v2i32((LAS v2i*)(uintptr_t)(u.ra_w + (M % WD) * 4096 + tt * 512 + nf * 16));
      g.Wf[M >> 1][nf][(M & 1) * 4 + tt * 2] = q[0];
      g.Wf[M >> 1][nf][(M & 1) * 4 + tt * 2 + 1] = q[1];
    }
}
template <int N> __device__ __forceinline__ void u0_x_read(G& g, const U0& u) {
  constexpr int X = XB + 8 * (N % R);
  asm volatile("ds_read_b128 v[%c1:%c2], %0 offset:%c3" :: "v"(u.ra_x0), "i"(X), "i"(X + 3), "i"((N % XD) * 2048) : "memory");
  asm volatile("ds_read_b128 v[%c1:%c2], %0 offset:%c3" :: "v"(u.ra_x1), "i"(X + 4), "i"(X + 7), "i"((N % XD) * 2048) : "memory");
}

__device__ __forceinline__ void unit0_body(G& g, U0& u, const Hint& h) {
  constexpr int NG = U0S::NG;
  sfor<NG>([&](auto nc) {
    constexpr int n = decltype(nc)::value;
    constexpr int i = n / 5, j = n % 5;
    constexpr int slot = n % R, pslot = (n + R - 1) % R;
    constexpr int t = n - 1 + R, ti = U0S::remote(n) ? (t - NG) / 4 : 0, tj = U0S::remote(n) ? (t - NG) % 4 : 0;

    asm volatile("s_waitcnt lgkmcnt(0)" ::: "memory");
    if constexpr (U0S::w_issue(n, 0) >= 0) u0_w_issue<U0S::w_issue(n, 0)>(g, u);
    if constexpr (U0S::w_issue(n, 1) >= 0) u0_w_issue<U0S::w_issue(n, 1)>(g, u);
    if constexpr (n + XD < NG) u0_x_issue<n + XD>(g, u);
    sfor<NHINT>([&](auto qc) { constexpr int q = decltype(qc)::value; if constexpr (U0S::hint_t(q) == 100 + n) hint<q>(h); });
    if constexpr (n == RH0_N) rhint(g);
    if constexpr (n == PK_0) peek_issue(g);
    if constexpr (PK_0 >= 0 && n == NG - R) peek_use(g);
    static_assert(PK_0 < 0 || (PK_0 + 1 + XD < NG && PK_0 + XD < NG - R), "peek is complete when it is used");
    if constexpr (U0S::remote(n) && tj == 0) {
      if (!(g.mask & 1ull)) slow_wait(g);
      g.mask >>= 1;
      g.kidx += 1;
    }
    sfor<4>([&](auto fc) {
      constexpr int nf = decltype(fc)::value;
      mfma<j, nf, slot, i == 0>(g.Wf[i][nf]);
      if constexpr (n == NG - 3) lds_wr<0 * RED_TILE + nf * 64, 0, nf>(g.wa_cur);
      if constexpr (n == NG - 2) lds_wr<1 * RED_TILE + nf * 64, 1, nf>(g.wa_cur);
      if constexpr (n == NG - 1) lds_wr<2 * RED_TILE + nf * 64, 2, nf>(g.wa_cur);
      if constexpr (U0S::remote(n) && (nf == 0 || nf == 2))
        ld_stg<pslot, nf / 2>(g.vo_stg, g.rs_stg, g.ub_nxt + g.kn[ti & 1] + g.fo[tj]);
      if constexpr (nf == 1 && n + 1 < NG) {
        asm volatile("s_waitcnt vmcnt(%0)" :: "i"(U0S::wait_cnt(n)) : "memory");
        u0_x_read<n + 1>(g, u);
        if constexpr (U0S::w_read(n) >= 0) u0_w_read<U0S::w_read(n)>(g, u);
      }
    });
  });
}

template <bool FIRST> __device__ __forceinline__ void next_unit(G& g, unsigned out_next) {
  g.out_prev = g.out_cur;
  g.out_cur = out_next;
  g.ub_cur = g.ub_nxt;
  g.ub_nxt += UNIT_BYTES;
  g.unit += 1u;
  g.kbase += 8u;
  g.kn[0] = g.unit == 7u ? 0u : g.kd[0];
  g.kn[1] = g.unit == 7u ? 32768u : g.kd[1];
  g.wa_prev = g.wa_cur;
  g.wa_cur += g.red_d;
  if (!FIRST) { g.ra[0] -= g.red_d; g.ra[1] -= g.red_d; g.ra[2] -= g.red_d; g.ra[3] -= g.red_d; }
  g.red_d = 0u - g.red_d;
}

__device__ __forceinline__ void gemm_role(const Params& p, unsigned char* smem_generic) {
  LAS unsigned char* smem = (LAS unsigned char*)smem_generic;
  const int b = blockIdx.x;
  const int xcd = b & 7, slot = (b >> 3) - (NCOMM >> 3);
  const int pidx = (xcd % XG) * 28 + slot;
  const int pi = pidx / (4 * XG), cb = (xcd / XG) * (4 * XG) + pidx % (4 * XG);
  const int tid = threadIdx.x, lane = tid & 63;
  const int w = __builtin_amdgcn_readfirstlane(tid >> 6);
  const int kg = lane >> 4, li = lane & 15;
  const int rank = p.rank;
  unsigned char* myws = p.ws[rank];
  const int src = (rank + 1 + pi) & 7;
  G g;
  asm volatile("" ::: "v255");
  GSTAMP(0);

  unsigned ticket = 0;
  if (lane == 0)
    asm volatile("global_atomic_add %0, %1, %2, off sc0 sc1" : "+v"(ticket) : "v"((unsigned*)(myws + CTRL_CALL + (b * 4 + w) * 32)), "v"(1u) : "memory");
  GSTAMP(17);
  const int row0 = pi * 80 < 432 ? pi * 80 : 432;
  g.rs_x = mk_rsrc(p.x, SRC_BYTES);
  g.rot = (unsigned)(pidx % NROT) * (8 / NROT);
  sfor<8>([&](auto ic) { constexpr int i = decltype(ic)::value; g.kd[i] = ((i + g.rot) & 7u) * 32768u; });
  g.kn[0] = g.kd[0]; g.kn[1] = g.kd[1];
  sfor<4>([&](auto jc) { constexpr int j = decltype(jc)::value; g.so_loc[j] = (unsigned)(row0 + ((j + 1 + w) & 3) * 16) * KDIM + w * 128; });
  g.so_loc[4] = (unsigned)(row0 + 64) * KDIM + w * 128;
  U0 u;
  {
    const unsigned base = (unsigned)(uintptr_t)smem + w * PART;
    u.rs_w = mk_rsrc(p.w, (unsigned)KDIM * NFULL);
    u.vo_w = (lane >> 2) * NFULL + (lane & 3) * 16;
    u.vo_x = (lane >> 3) * KDIM + ((lane & 7) ^ (lane >> 3)) * 16;
    u.so_w = (unsigned)rank * NLOC + cb * 64 + (unsigned)(w * 128) * NFULL;
    u.lds_w = base;
    u.lds_x = base + STG_X;
    u.ra_w = base + (kg * 16 + (li >> 1)) * 64 + (li & 1) * 8;
    u.ra_x0 = base + STG_X + (li >> 3) * 1024 + (li & 7) * 128 + ((kg ^ (li & 7)) * 16);
    u.ra_x1 = u.ra_x0 ^ 64u;
  }
  Hint h;
  h.wb = p.w + (size_t)rank * NLOC + (xcd / XG) * (256 * XG);
  h.xb = p.x + (size_t)row0 * KDIM;
  h.wi = slot * 4 + w;
  h.xi = (pidx % (4 * XG)) * 4 + w;
  sfor<U0S::NPRO_MAX>([&](auto ic) {
    constexpr int op = U0S::pro_op(decltype(ic)::value);
    if constexpr (op >= 100 && op < 200) u0_x_issue<op - 100>(g, u);
    if constexpr (op >= 200 && op < 300) u0_w_issue<op - 200>(g, u);
    if constexpr (op >= 300) hint<op - 300>(h);
  });
  GSTAMP(12);
  g.lds_bad = (LAS unsigned*)(smem + LDS_MISC + 4);
  if (tid == 0) asm volatile("ds_write_b32 %0, %1" :: "v"((unsigned)(uintptr_t)smem + LDS_MISC + 4), "v"(0u) : "memory");
  asm volatile("s_waitcnt vmcnt(%1) lgkmcnt(0)\n\ts_barrier" : "+v"(ticket) : "i"(U0S::wait_cnt(-1)) : "memory");
  g.call = __builtin_amdgcn_readfirstlane(ticket) + 1u;
  GSTAMP(13);
  u0_x_read<0>(g, u);
  u0_w_read<0>(g, u); u0_w_read<1>(g, u);

  g.scale = *(const CAS float*)p.sx * *(const CAS float*)p.sw;
  const unsigned cm = __builtin_amdgcn_readfirstlane(g.call % NSTAGE);
  g.rs_stg = mk_rsrc(myws + cm * STAGE_BYTES, STAGE_BYTES);
  {
    const unsigned gw = (unsigned)(b - NCOMM) * 4u + (unsigned)w;
    g.rs_mk = mk_rsrc(myws + (cm == 0u ? 2u : cm - 1u) * STAGE_BYTES, STAGE_BYTES);
    g.mk_off = ((unsigned)(rank + 1 + (gw >> 7)) & 7u) * SRC_BYTES + (IB_U - 1u) * UNIT_BYTES + (gw & 127u) * 2048u;
  }
  g.rs_out = mk_rsrc(p.out, (unsigned)(4096u * NLOC * 4u));
  g.flag_ptr = (const GAS unsigned long long*)((lane >> 3) < PF_U
      ? myws + CTRL_FLAG + src * 2048 + w * 512 + (lane < 56 ? (lane & 56) + ((lane + g.rot) & 7) : lane) * 8
      : myws + CTRL_UFLAG + src * 64 + (lane >> 3) * 8);
  g.tmo = (GAS unsigned*)(myws + CTRL_TMO);
  g.rs_tmo = mk_rsrc(myws + CTRL_TMO, 4);
  g.vo_stg = lane * 16;
  g.vo_rh = lane * 128;
  g.rh_off = (unsigned)(pidx % (4 * XG)) * (UNIT_BYTES / (4 * XG)) + (unsigned)w * (UNIT_BYTES / (16 * XG) - 8192u);
  g.vo_out = (lane >> 4) * (NLOC * 4) + (lane & 15) * 16;
  sfor<4>([&](auto jc) { constexpr int j = decltype(jc)::value; g.fo[j] = ((j + 1 + w) & 3) * 2048; });
  g.wa_cur = w * PART + li * RED_ROW + kg * 16;
  sfor<4>([&](auto qc) {
    constexpr int q = decltype(qc)::value;
    g.ra[q] = q * PART + ((w - q - 1) & 3) * RED_TILE + (lane >> 4) * RED_ROW + (lane & 15) * 16;
  });
  g.wa5 = w * PART + LDS_T5 + li * RED_ROW + kg * 16;
  g.ra5 = LDS_T5 + (4 * w + (lane >> 4)) * RED_ROW + (lane & 15) * 16;
  g.red_d = RED_WAVE;
  g.tmov = 0u;
  const unsigned colb = cb * 256;
  auto out_off = [&](int grow) -> unsigned { return (unsigned)(grow + w * 16) * (NLOC * 4) + colb; };
  g.out_cur = out_off(rank * MLOC + row0);
  g.out_prev = g.out_cur;
  g.out5 = (unsigned)(rank * MLOC + row0 + 64 + 4 * w) * (NLOC * 4) + colb;
  g.ub_cur = 0;
  g.ub_nxt = (unsigned)src * SRC_BYTES + w * 8192;
  g.mask = IBM;
  g.sw_n = 0; g.sw_t = 0;
  g.rt_n = g.rt_it = g.rt_t = g.rt_last = g.rt_lastn = 0;
  g.kbase = 0u - 8u;
  g.unit = 0u;
  static_assert(IB_U >= 3, "the first remote unit is never validated in-band");
  g.kidx = 0u;

  GSTAMP(1);
  unit0_body(g, u, h);
  GSTAMP(2);
  next_unit<true>(g, out_off(src * MLOC));
  unit_body<4, 40 % R, 5, true, -1, false>(g);
  GSTAMP(3);
  constexpr int PER = (32 % R == 0) ? 1 : 3;
  static_assert((32 * PER) % R == 0 && 6 % PER == 0, "ring size");
  static_assert(PER == 1, "ring size");
  constexpr int ULIM = IB_U < 8 ? IB_U : 8;
#define USTAMP(u) do {} while (0)
  for (int u = 2; u < ULIM; ++u) {
    next_unit<false>(g, out_off(src * MLOC + (u - 1) * 64));
    unit_body<4, (40 + 32) % R, 4, true, -1, false>(g);
    USTAMP(u);
  }
  if constexpr (ULIM < 8) {
    for (int u = ULIM; u < 8; ++u) {
      next_unit<false>(g, out_off(src * MLOC + (u - 1) * 64));
      unit_body<4, (40 + 32) % R, 4, true, -1, true>(g);
      USTAMP(u);
    }
  }
  next_unit<false>(g, out_off(src * MLOC + 7 * 64));
  sfor<NROT>([&](auto rc) {
    constexpr int ro = decltype(rc)::value * (8 / NROT);
    if (NROT == 1 || g.rot == (unsigned)ro) unit_body<4, (40 + 32 * 7) % R, 4, false, ro, (IB_U <= 8)>(g);
  });
  GSTAMP(10);

  sfor<4>([&](auto qc) { constexpr int q = decltype(qc)::value; lds_wr<2 * RED_TILE + q * 64, 2, q>(g.wa_cur); });
  asm volatile("s_nop 7\n\ts_nop 7\n\ts_nop 7" ::: "memory");
  sfor<4>([&](auto qc) { constexpr int q = decltype(qc)::value; lds_wr<3 * RED_TILE + q * 64, 3, q>(g.wa_cur); });
  asm volatile("s_waitcnt lgkmcnt(0)\n\ts_barrier" ::: "memory");
  {
    v4f r[4][4];
    lds_rd32(g.badv, LDS_MISC + 4);
    sfor<16>([&](auto qc) {
      constexpr int q = decltype(qc)::value;
      lds_rd<(q >> 2) * 4 * RED_ROW>(r[q >> 2][q & 3], g.ra[q & 3] - g.red_d);
    });
    float sc = g.scale;
    sfor<4>([&](auto sc_) {
      constexpr int q = decltype(sc_)::value;
      lds_wait<(3 - q) * 4>(r[q], g.badv);
      if constexpr (q == 0) {
        asm volatile("s_waitcnt vmcnt(0)" : "+v"(g.tmov) :: "memory");
        unsigned bad = __builtin_amdgcn_readfirstlane(g.badv) | (__builtin_amdgcn_readfirstlane(g.tmov) == g.call ? 1u : 0u);
        if (bad) sc = __builtin_nanf("");
      }
      const v4f sum = (r[q][0] + r[q][1]) + (r[q][2] + r[q][3]);
      const v4f v = {sum[0] * sc, sum[1] * sc, sum[2] * sc, sum[3] * sc};
      st_out(v, g.vo_out, g.rs_out, g.out_cur + q * 4 * (NLOC * 4));
    });
  }
}

#define CSTAMP(n) do {} while (0)
template <bool LEADER>
__device__ __forceinline__ void comm_wave(const Params& p, LAS unsigned char* smem, const unsigned call, const int j, const int w, const int lane, v4i d) {
  const int rank = p.rank;
  unsigned char* myws = p.ws[rank];
  const int cw = j * 4 + w;
  rsrc_t rs_x = mk_rsrc(p.x, SRC_BYTES);
  const unsigned vo = (unsigned)((w >> 1) * 16 + (lane & 15)) * KDIM + (w & 1) * 64 + (lane >> 4) * 16;
  const unsigned dst0 = (call % NSTAGE) * STAGE_BYTES + (unsigned)rank * SRC_BYTES + (unsigned)w * 1024u;
  rsrc_t rs[7];
#pragma unroll
  for (int q = 0; q < 7; ++q) {
    int pq = (rank + 1 + (q + cw) % 7) & 7;
    rs[q] = mk_rsrc(p.ws[pq], NSTAGE * STAGE_BYTES);
  }
  unsigned char* flagb = p.ws[(rank + 1 + (lane < 7 ? lane : 0)) & 7] + CTRL_FLAG + rank * 2048;
  unsigned char* uflagb = p.ws[(rank + 1 + (lane < 7 ? lane : 0)) & 7] + CTRL_UFLAG + rank * 64;
  unsigned pend = 0;
  unsigned* ticketp = (unsigned*)(myws + CTRL_TICKET);
  GAS unsigned* tmo = (GAS unsigned*)(myws + CTRL_TMO);
  const unsigned tbase = (call - 1u) * TICKETS_PER_CALL - 2u * NCOMM;
  const unsigned vs = lane * 16;
  LAS unsigned* tk = (LAS unsigned*)(smem + 1024);
  LAS unsigned* cnt = (LAS unsigned*)(smem + LDS_MISC + 64);

  auto publish = [&](unsigned n, unsigned t) {
    unsigned old = 0;
    if (lane == 0) old = __hip_atomic_fetch_add(cnt + (n & 7u), 1u, __ATOMIC_RELAXED, __HIP_MEMORY_SCOPE_WORKGROUP);
    old = __builtin_amdgcn_readfirstlane(old);
    if ((old & 3u) == 3u) {
      if (t < PF_U * 64u) {
        if (lane < 7) __hip_atomic_store((unsigned*)(flagb + ((t >> 1) & 3u) * 512u + (t >> 3) * 8u + (t & 1u) * 4u), call, __ATOMIC_RELAXED, __HIP_MEMORY_SCOPE_SYSTEM);
      } else {
        if (lane == 0)
          asm volatile("global_atomic_add v%c2, %0, %1, off sc0 sc1" :: "v"(myws + CTRL_UCNT + (t >> 6) * 128u), "v"(1u), "i"(FD) : "memory");
        pend = (t >> 6) + 1u;
      }
      CSTAMP(n + 1);
    }
  };
  auto group_done = [&]() {
    if (pend) {
      unsigned o;
      asm volatile("v_readfirstlane_b32 %0, v%c1" : "=s"(o) : "i"(FD) : "memory");
      if (o - (call - 1u) * 64u == 63u && lane < 7)
        asm volatile("global_store_dwordx2 %0, %1, off sc0 sc1" :: "v"(uflagb + (pend - 1u) * 8u), "v"((v2u){call, call}) : "memory");
      pend = 0;
    }
  };

  unsigned t_cur = (unsigned)j, t_prev = 0;
  unsigned n = 0;
  while (true) {
    v4i dn;
    unsigned tick = 0, t_nxt = 0;
    const unsigned dst = dst0 + t_cur * 4096u;
    sfor<7>([&](auto qc) {
      constexpr int q = decltype(qc)::value;
      asm volatile("s_waitcnt vmcnt(%0)" :: "i"(COMM_W - 1) : "memory");
      if constexpr (PF_U < 8 && q == (2 * COMM_W - 2 <= 6 ? 2 * COMM_W - 2 : COMM_W - 1)) group_done();
      if constexpr (q == COMM_W - 1) { if (n > 0) publish(n - 1, t_prev); }
      if constexpr (LEADER && q == 0) {
        if (n > 0 && lane == 0) asm volatile("global_atomic_add %0, %1, %2, off sc0 sc1" : "+v"(tick) : "v"(ticketp), "v"(1u) : "memory");
      }
      if constexpr (q == 5) {
        if constexpr (LEADER) {
          asm volatile("" : "+v"(tick));
          t_nxt = n == 0 ? (unsigned)(NCOMM + j) : __builtin_amdgcn_readfirstlane(tick) - tbase;
          if (lane == 0) tk[n + 1] = t_nxt + 1u;
        } else {
          unsigned long long t0 = 0;
          while (true) {
            unsigned v = __hip_atomic_load(tk + n + 1, __ATOMIC_RELAXED, __HIP_MEMORY_SCOPE_WORKGROUP);
            v = __builtin_amdgcn_readfirstlane(v);
            if (v) { t_nxt = v - 1u; break; }
            unsigned long long now = __builtin_amdgcn_s_memrealtime();
            if (!t0) t0 = now;
            if (now - t0 > DEADLINE_TICKS) {
              __hip_atomic_store(tmo, call, __ATOMIC_RELAXED, __HIP_MEMORY_SCOPE_AGENT);
              t_nxt = NPIECE; break;
            }
            __builtin_amdgcn_s_sleep(1);
          }
        }
        if (t_nxt < NPIECE)
          asm volatile("buffer_load_dwordx4 %0, %1, %2, %3 offen" : "=v"(dn) : "v"(vo), "s"(rs_x),
                       "s"((t_nxt >> 6) * UNIT_BYTES + (t_nxt & 1u) * (32u * KDIM) + ((t_nxt >> 1) & 31u) * 128u) : "memory");
      }
      st_sys(d, vs, rs[q], dst);
    });
    if (t_nxt >= NPIECE) break;
    asm volatile("s_waitcnt vmcnt(2)" : "+v"(dn) :: "memory");
    t_prev = t_cur; t_cur = t_nxt; d = dn; ++n;
  }
  asm volatile("s_waitcnt vmcnt(0)" ::: "memory");
  if constexpr (PF_U < 8) group_done();
  publish(n, t_cur);
  if constexpr (PF_U < 8) {
    asm volatile("s_waitcnt vmcnt(0)" ::: "memory");
    group_done();
  }
}

__device__ __forceinline__ void comm_role(const Params& p, unsigned char* smem_generic) {
  LAS unsigned char* smem = (LAS unsigned char*)smem_generic;
  const int j = blockIdx.x;
  const int tid = threadIdx.x, lane = tid & 63;
  const int w = __builtin_amdgcn_readfirstlane(tid >> 6);
  unsigned char* myws = p.ws[p.rank];

  STAMP(myws, 0);
  v4i d;
  {
    rsrc_t rs_x = mk_rsrc(p.x, SRC_BYTES);
    const unsigned vo = (unsigned)((w >> 1) * 16 + (lane & 15)) * KDIM + (w & 1) * 64 + (lane >> 4) * 16;
    asm volatile("buffer_load_dwordx4 %0, %1, %2, %3 offen" : "=v"(d) : "v"(vo), "s"(rs_x), "s"((unsigned)((j & 1) * (32 * KDIM) + (j >> 1) * 128)) : "memory");
  }
  unsigned ticket = 0;
  if (lane == 0)
    asm volatile("global_atomic_add %0, %1, %2, off sc0" : "+v"(ticket) : "v"((unsigned*)(myws + CTRL_CALL + (j * 4 + w) * 32)), "v"(1u) : "memory");
  for (int i = tid; i < 520; i += 256) *(LAS unsigned*)(smem + 1024 + i * 4) = 0u;
  if (tid < 8) *(LAS unsigned*)(smem + LDS_MISC + 64 + tid * 4) = 0u;
  asm volatile("s_waitcnt lgkmcnt(0)\n\ts_barrier\n\ts_waitcnt vmcnt(0)" : "+v"(d), "+v"(ticket) :: "memory");
  const unsigned call = __builtin_amdgcn_readfirstlane(ticket) + 1u;
  if (w == 0) comm_wave<true>(p, smem, call, j, w, lane, d);
  else comm_wave<false>(p, smem, call, j, w, lane, d);
}

#define CWN 1
constexpr unsigned PGW = 18;
constexpr unsigned PTICKETS = NPIECE - 2 * PGW + PGW;
constexpr int CB = 128, TK = HD - 1;
template <int PAR> __device__ __forceinline__ void ld_piece(const unsigned (&vo)[4], rsrc_t rs_x, unsigned t) {
  const unsigned so = (t >> 6) * UNIT_BYTES + (t & 1u) * (32u * KDIM) + ((t >> 1) & 31u) * 128u;
  asm volatile("buffer_load_dwordx4 v[%c3:%c4], %0, %1, %2 offen" :: "v"(vo[0]), "s"(rs_x), "s"(so), "i"(CB + 16 * PAR), "i"(CB + 16 * PAR + 3) : "memory");
  asm volatile("buffer_load_dwordx4 v[%c3:%c4], %0, %1, %2 offen" :: "v"(vo[1]), "s"(rs_x), "s"(so), "i"(CB + 16 * PAR + 4), "i"(CB + 16 * PAR + 7) : "memory");
  asm volatile("buffer_load_dwordx4 v[%c3:%c4], %0, %1, %2 offen" :: "v"(vo[2]), "s"(rs_x), "s"(so), "i"(CB + 16 * PAR + 8), "i"(CB + 16 * PAR + 11) : "memory");
  asm volatile("buffer_load_dwordx4 v[%c3:%c4], %0, %1, %2 offen" :: "v"(vo[3]), "s"(rs_x), "s"(so), "i"(CB + 16 * PAR + 12), "i"(CB + 16 * PAR + 15) : "memory");
}
template <int PAR, int K> __device__ __forceinline__ void st_piece(unsigned vs, rsrc_t rs_d, unsigned dst) {
  asm volatile("buffer_store_dwordx4 v[%c3:%c4], %0, %1, %2 offen offset:%c5 sc0 sc1"
               :: "v"(vs), "s"(rs_d), "s"(dst), "i"(CB + 16 * PAR + 4 * K), "i"(CB + 16 * PAR + 4 * K + 3), "i"(K * 1024) : "memory");
}
__device__ __forceinline__ void tk_issue(unsigned* ptr) {
  asm volatile("global_atomic_add v%c2, %0, %1, off sc0 sc1" :: "v"(ptr), "v"(1u), "i"(TK) : "memory");
}
__device__ __forceinline__ void dummy_ld(unsigned* ptr) {
  asm volatile("global_load_dword v%c1, %0, off sc0 sc1" :: "v"(ptr), "i"(HD) : "memory");
}
__device__ __forceinline__ unsigned tk_read() {
  unsigned t;
  asm volatile("s_waitcnt vmcnt(5)\n\tv_readfirstlane_b32 %0, v%c1" : "=s"(t) : "i"(TK) : "memory");
  return t;
}
__device__ __forceinline__ void st_flag(unsigned char* a, unsigned v) {
  asm volatile("global_store_dword %0, %1, off sc0 sc1" :: "v"(a), "v"(v) : "memory");
}
struct C2 {
  unsigned vo[4], vs, call, tbase, t0, t1, tp1, tp2;
  rsrc_t rs_x, rs_d;
  unsigned char* flagb;
  unsigned* ticketp;
  unsigned char* flagown;
  int lane;
};
__device__ __forceinline__ void publish2(C2& c, unsigned t) {
  if (c.lane == 0) st_flag(c.flagb + ((t >> 1) & 3u) * 512u + (t >> 3) * 8u + (t & 1u) * 4u, c.call);
}
template <int PAR> __device__ __forceinline__ bool comm_iter(C2& c) {
  constexpr int PER_IT = 10;
  const bool more = c.t1 < NPIECE;
  if (c.lane == 0) {
    if (more) tk_issue(c.ticketp);
    else dummy_ld(c.ticketp);
  }
  const unsigned dst = c.t0 * 4096u;
  const unsigned tpub = CWN == 1 ? c.tp1 : c.tp2;
  wait_vm<PER_IT * CWN - 1>(); st_piece<PAR, 0>(c.vs, c.rs_d, dst);
  wait_vm<PER_IT * CWN - 1>(); st_piece<PAR, 1>(c.vs, c.rs_d, dst);
  wait_vm<PER_IT * CWN - 1>(); st_piece<PAR, 2>(c.vs, c.rs_d, dst);
  wait_vm<PER_IT * CWN - 2>();
  if (tpub < NPIECE) publish2(c, tpub);
  else if (c.lane == 0) dummy_ld(c.ticketp);
  st_piece<PAR, 3>(c.vs, c.rs_d, dst);
  const unsigned tk = tk_read();
  const unsigned t2 = more ? tk - c.tbase : NPIECE;
  ld_piece<PAR>(c.vo, c.rs_x, t2 < NPIECE ? t2 : 0u);
  c.tp2 = c.tp1; c.tp1 = c.t0; c.t0 = c.t1; c.t1 = t2;
  return c.t0 >= NPIECE;
}
__device__ __forceinline__ void comm_role2(const Params& p) {
  const int tid = threadIdx.x, lane = tid & 63;
  const int w = __builtin_amdgcn_readfirstlane(tid >> 6);
  const unsigned cw = blockIdx.x * 4 + w;
  const int rank = p.rank;
  unsigned char* myws = p.ws[rank];
  STAMP(myws, 0);
  if (cw >= 7 * PGW) return;
  const unsigned q = cw % 7u, gi = cw / 7u;
  unsigned char* pws = p.ws[(rank + 1 + q) & 7];
  C2 c;
  c.lane = lane;
  c.rs_x = mk_rsrc(p.x, SRC_BYTES);
#pragma unroll
  for (int k = 0; k < 4; ++k) c.vo[k] = (unsigned)((k >> 1) * 16 + (lane & 15)) * KDIM + (k & 1) * 64 + (lane >> 4) * 16;
  ld_piece<0>(c.vo, c.rs_x, gi);
  ld_piece<1>(c.vo, c.rs_x, PGW + gi);
  unsigned ticket = 0;
  if (lane == 0)
    asm volatile("global_atomic_add %0, %1, %2, off sc0" : "+v"(ticket) : "v"((unsigned*)(myws + CTRL_CALL + cw * 32)), "v"(1u) : "memory");
  asm volatile("s_waitcnt vmcnt(0)" : "+v"(ticket) :: "memory");
  c.call = __builtin_amdgcn_readfirstlane(ticket) + 1u;
  const unsigned cm = __builtin_amdgcn_readfirstlane(c.call % NSTAGE);
  c.rs_d = mk_rsrc(pws + cm * STAGE_BYTES + (unsigned)rank * SRC_BYTES, SRC_BYTES);
  c.flagb = pws + CTRL_FLAG + rank * 2048;
  c.ticketp = (unsigned*)(myws + CTRL_TICKET + q * 128);
  c.tbase = (c.call - 1u) * PTICKETS - 2u * PGW;
  c.flagown = myws + CTRL_CALL + cw * 32;
  c.vs = lane * 16;
  c.t0 = gi; c.t1 = PGW + gi; c.tp1 = NPIECE; c.tp2 = NPIECE;
  while (true) {
    if (comm_iter<0>(c)) break;
    if (comm_iter<1>(c)) break;
  }
  asm volatile("s_waitcnt vmcnt(0)" ::: "memory");
  if (CWN == 2 && c.tp2 < NPIECE) publish2(c, c.tp2);
  publish2(c, c.tp1);
}

#define T_FAST 512
#define C3_ROT 0
#define NACT 32
struct C3 {
  unsigned vo, vs, call, tbase, dst0, t_prev, t_cur, t_nxt, n, pend;
  unsigned char *uflagb, *ucnt;
  rsrc_t rs_x, rs[7];
  unsigned char* flagb;
  unsigned* ticketp;
  GAS unsigned* tmo;
  LAS unsigned *tk, *cnt;
  LAS unsigned char* smem;
  int lane;
};
template <int PAR> __device__ __forceinline__ void ld_chunk(unsigned vo, rsrc_t rs_x, unsigned t) {
  const unsigned so = (t >> 6) * UNIT_BYTES + (t & 1u) * (32u * KDIM) + ((t >> 1) & 31u) * 128u;
  asm volatile("buffer_load_dwordx4 v[%c3:%c4], %0, %1, %2 offen" :: "v"(vo), "s"(rs_x), "s"(so), "i"(CB + 4 * PAR), "i"(CB + 4 * PAR + 3) : "memory");
}
template <int PAR> __device__ __forceinline__ void st_chunk(unsigned vs, rsrc_t rs_d, unsigned dst) {
  asm volatile("buffer_store_dwordx4 v[%c3:%c4], %0, %1, %2 offen sc0 sc1" :: "v"(vs), "s"(rs_d), "s"(dst), "i"(CB + 4 * PAR), "i"(CB + 4 * PAR + 3) : "memory");
}
__device__ __forceinline__ void publish3(C3& c, unsigned n, unsigned t) {
  unsigned old = 0;
  if (c.lane == 0) old = __hip_atomic_fetch_add(c.cnt + n, 1u, __ATOMIC_RELAXED, __HIP_MEMORY_SCOPE_WORKGROUP);
  old = __builtin_amdgcn_readfirstlane(old);
  if (old == 3u) {
    if (t < PF_U * 64u) {
      if (c.lane < 7) st_flag(c.flagb + ((t >> 1) & 3u) * 512u + (t >> 3) * 8u + (t & 1u) * 4u, c.call);
    } else {
      if (c.lane == 0)
        asm volatile("global_atomic_add v%c2, %0, %1, off sc0 sc1" :: "v"(c.ucnt + (t >> 6) * 128u), "v"(1u), "i"(FD) : "memory");
      c.pend = (t >> 6) + 1u;
    }
  }
}
__device__ __forceinline__ void group_done3(C3& c) {
  if (c.pend) {
    unsigned o;
    asm volatile("v_readfirstlane_b32 %0, v%c1" : "=s"(o) : "i"(FD) : "memory");
    if (o - (c.call - 1u) * 64u == 63u && c.lane < 7)
      asm volatile("global_store_dwordx2 %0, %1, off sc0 sc1" :: "v"(c.uflagb + (c.pend - 1u) * 8u), "v"((v2u){c.call, c.call}) : "memory");
    c.pend = 0;
  }
}
template <bool LEADER, int PAR> __device__ __forceinline__ bool comm3_round(C3& c) {
  constexpr int EX = LEADER ? 2 : 1, W = COMM_W;
  const bool more = c.t_nxt < NPIECE;
  unsigned t2 = NPIECE;
  auto ticket_done = [&]() {
    unsigned t;
    asm volatile("v_readfirstlane_b32 %0, v%c1" : "=s"(t) : "i"(TK) : "memory");
    t -= c.tbase;
    t2 = more && t < NPIECE ? t : NPIECE;
    if (c.lane == 0) c.tk[c.n + 2] = t2 + 1u;
  };
  if constexpr (LEADER) {
    if (c.lane == 0) {
      if (more) tk_issue(c.ticketp);
      else dummy_ld(c.ticketp);
    }
  }
  ld_chunk<PAR ^ 1>(c.vo, c.rs_x, more ? c.t_nxt : 0u);
  const unsigned dst = c.dst0 + c.t_cur * 4096u;
  if (c.t_cur < (unsigned)T_FAST) {
    sfor<7>([&](auto qc) {
      constexpr int q = decltype(qc)::value;
      wait_vm<(q < W ? W - 1 + EX : W - 1)>();
      if constexpr (PF_U < 8 && q == (2 * W - 2 <= 6 ? 2 * W - 2 : W - 1)) group_done3(c);
      if constexpr (q == W - 1) { if (c.n > 0) publish3(c, c.n - 1, c.t_prev); }
      if constexpr (LEADER && q == W) ticket_done();
      st_chunk<PAR>(c.vs, c.rs[q], dst);
    });
    if constexpr (LEADER && W == 7) { wait_vm<7>(); ticket_done(); }
  } else {
    sfor<7>([&](auto qc) { st_chunk<PAR>(c.vs, c.rs[decltype(qc)::value], dst); });
    wait_vm<7 + EX>();
    if constexpr (PF_U < 8) group_done3(c);
    if (c.n > 0) publish3(c, c.n - 1, c.t_prev);
    if constexpr (LEADER) { wait_vm<8>(); ticket_done(); }
  }
  if (!more) return true;
  if constexpr (!LEADER) {
    unsigned long long t0 = 0;
    while (true) {
      unsigned v = __hip_atomic_load(c.tk + c.n + 2, __ATOMIC_RELAXED, __HIP_MEMORY_SCOPE_WORKGROUP);
      v = __builtin_amdgcn_readfirstlane(v);
      if (v) { t2 = v - 1u; break; }
      unsigned long long now = __builtin_amdgcn_s_memrealtime();
      if (!t0) t0 = now;
      if (now - t0 > DEADLINE_TICKS) {
        __hip_atomic_store(c.tmo, c.call, __ATOMIC_RELAXED, __HIP_MEMORY_SCOPE_AGENT);
        t2 = NPIECE; break;
      }
      __builtin_amdgcn_s_sleep(1);
    }
  }
  wait_vm<7>();
  c.t_prev = c.t_cur; c.t_cur = c.t_nxt; c.t_nxt = t2; c.n += 1u;
  return false;
}
template <bool LEADER> __device__ __forceinline__ void comm3_wave(C3& c) {
  while (true) {
    if (comm3_round<LEADER, 0>(c)) break;
    if (comm3_round<LEADER, 1>(c)) break;
  }
  wait_vm<0>();
  if constexpr (PF_U < 8) group_done3(c);
  publish3(c, c.n, c.t_cur);
  if constexpr (PF_U < 8) { wait_vm<0>(); group_done3(c); }
}
__device__ __forceinline__ void comm_role3(const Params& p, unsigned char* smem_generic) {
  LAS unsigned char* smem = (LAS unsigned char*)smem_generic;
  const int j = blockIdx.x;
  const int tid = threadIdx.x, lane = tid & 63;
  const int w = __builtin_amdgcn_readfirstlane(tid >> 6);
  const int rank = p.rank;
  unsigned char* myws = p.ws[rank];
  STAMP(myws, 0);
  asm volatile("" ::: "v255");
  if (j >= NACT) return;
  C3 c;
  c.lane = lane;
  c.smem = smem;
  c.rs_x = mk_rsrc(p.x, SRC_BYTES);
  c.vo = (unsigned)((w >> 1) * 16 + (lane & 15)) * KDIM + (w & 1) * 64 + (lane >> 4) * 16;
  ld_chunk<0>(c.vo, c.rs_x, (unsigned)j);
  unsigned ticket = 0;
  if (lane == 0)
    asm volatile("global_atomic_add %0, %1, %2, off sc0" : "+v"(ticket) : "v"((unsigned*)(myws + CTRL_CALL + (j * 4 + w) * 32)), "v"(1u) : "memory");
  for (int i = tid; i < 2 * 520; i += 256) *(LAS unsigned*)(smem + 1024 + i * 4) = 0u;
  c.tk = (LAS unsigned*)(smem + 1024);
  c.cnt = (LAS unsigned*)(smem + 1024 + 520 * 4);
#pragma unroll
  for (int q = 0; q < 7; ++q) {
    const int pq = (rank + 1 + (q + (C3_ROT ? j : j * 4 + w)) % 7) & 7;
    c.rs[q] = mk_rsrc(p.ws[pq], NSTAGE * STAGE_BYTES);
  }
  c.flagb = p.ws[(rank + 1 + (lane < 7 ? lane : 0)) & 7] + CTRL_FLAG + rank * 2048;
  c.ticketp = (unsigned*)(myws + CTRL_TICKET);
  c.tmo = (GAS unsigned*)(myws + CTRL_TMO);
  c.vs = lane * 16;
  asm volatile("s_waitcnt lgkmcnt(0)\n\ts_barrier\n\ts_waitcnt vmcnt(0)" : "+v"(ticket) :: "memory");
  c.call = __builtin_amdgcn_readfirstlane(ticket) + 1u;
  c.tbase = (c.call - 1u) * (NPIECE - NACT) - 2u * NACT;
  c.pend = 0;
  c.uflagb = p.ws[(rank + 1 + (lane < 7 ? lane : 0)) & 7] + CTRL_UFLAG + rank * 64;
  c.ucnt = myws + CTRL_UCNT;
  c.dst0 = __builtin_amdgcn_readfirstlane(c.call % NSTAGE) * STAGE_BYTES + (unsigned)rank * SRC_BYTES + (unsigned)w * 1024u;
  c.t_prev = 0; c.t_cur = (unsigned)j; c.t_nxt = (unsigned)(NACT + j); c.n = 0;
  if (w == 0) comm3_wave<true>(c);
  else comm3_wave<false>(c);
}

__global__ void __launch_bounds__(256, 1) fused_ag_gemm(Params p) {
  extern __shared__ __attribute__((aligned(16))) unsigned char smem_dyn[];
  if (blockIdx.x < NCOMM) comm_role3(p, smem_dyn);
  else gemm_role(p, smem_dyn);
}

}

extern "C" void kernel_launch(int rank, int nranks, ncclComm_t comm, hipStream_t stream,
                              void* const* d_in, const int* in_sizes, int n_in,
                              void* d_out, int out_size,
                              const int* in_dtypes, int out_dtype,
                              void* const* d_out_all,
                              void* const* d_ws_all, size_t ws_bytes) {
  if (nranks != NRANKS || n_in < 4 || ws_bytes < WS_NEEDED || (unsigned)rank >= (unsigned)NRANKS) {
    fprintf(stderr, "unsupported configuration\n");
    return;
  }
  static bool attr_done = false;
  if (!attr_done) {
    HIP_CHECK(hipFuncSetAttribute((const void*)fused_ag_gemm, hipFuncAttributeMaxDynamicSharedMemorySize, (int)LDS_BYTES));
    attr_done = true;
  }
  Params p;
  p.x = (const unsigned char*)d_in[0];
  p.w = (const unsigned char*)d_in[1];
  p.sx = (const float*)d_in[2];
  p.sw = (const float*)d_in[3];
  p.out = (float*)d_out;
  for (int i = 0; i < NRANKS; ++i) p.ws[i] = (unsigned char*)d_ws_all[i];
  p.rank = rank;
  hipLaunchKernelGGL(fused_ag_gemm, dim3(256), dim3(256), LDS_BYTES, stream, p);
  HIP_CHECK(hipGetLastError());
}
```

```cpp
#include <hip/hip_runtime.h>
#include <rccl/rccl.h>
#include <cstdio>
#include <utility>

#define HIP_CHECK(expr) do { hipError_t _e = (expr); if (_e != hipSuccess) { \
    fprintf(stderr, "HIP error %s at %s:%d\n", hipGetErrorString(_e), __FILE__, __LINE__); \
    return; } } while (0)

namespace {

typedef int v8i __attribute__((ext_vector_type(8)));
typedef int v4i __attribute__((ext_vector_type(4)));
typedef int v2i __attribute__((ext_vector_type(2)));
typedef unsigned v2u __attribute__((ext_vector_type(2)));
typedef float v4f __attribute__((ext_vector_type(4)));
#define LAS __attribute__((address_space(3)))
#define GAS __attribute__((address_space(1)))
#define CAS __attribute__((address_space(4)))
typedef __amdgpu_buffer_rsrc_t rsrc_t;

constexpr int NRANKS = 8;
constexpr int KDIM = 4096, NFULL = 16384, NLOC = 2048, MLOC = 512;
constexpr unsigned UNIT_BYTES = 64u * KDIM;
constexpr unsigned SRC_BYTES = (unsigned)MLOC * KDIM;
constexpr unsigned STAGE_BYTES = NRANKS * SRC_BYTES;
constexpr unsigned NSTAGE = 3;
constexpr unsigned CTRL = NSTAGE * STAGE_BYTES;
constexpr unsigned CTRL_FLAG = CTRL;
constexpr unsigned CTRL_TMO = CTRL + 16384;
constexpr unsigned CTRL_TICKET = CTRL + 16384 + 128;
constexpr unsigned CTRL_UCNT = CTRL + 16384 + 2048;
constexpr unsigned CTRL_UFLAG = CTRL + 16384 + 3072;
constexpr unsigned CTRL_CALL = CTRL + 20480;
constexpr unsigned WS_NEEDED = CTRL_CALL + 1024u * 32u;
constexpr int NCOMM = 32;
constexpr unsigned NPIECE = 512;
constexpr unsigned TICKETS_PER_CALL = NPIECE - 2 * NCOMM + NCOMM;
constexpr unsigned long long DEADLINE_TICKS = 100000000ull;

#define RING 8
#define COMM_P 1
#define MAPPING 0
#define XG 2
#define NROT 1
#define IB_U 3
#define PF_U 2
#define IB_PIPE 2
#define IB_FAST0 0
#define IB_SLEEP 1
#define MK_N 10
#define MK_U 1
#define MKSM 3
#define MKST "sc0 sc1"
#define COMM_MODE 3
#define C3_FIX (COMM_MODE == 7)
#define COMM_W 4
#define SLEEPN 2
#define PF_MODE 0
#define POLLW 8
#define OSTM 4
#define OST "sc0 sc1 nt"
#define OSTLM 4
#define OSTL "sc0 sc1 nt"
#define XLDM 3
#define XLD "sc0 sc1"
constexpr int R = RING;

#define WD 4
#define XD 5
constexpr unsigned RED_ROW = 272, RED_TILE = 16 * RED_ROW, RED_WAVE = 4 * RED_TILE;
constexpr unsigned PART = 39424, LDS_T5 = 2 * RED_WAVE, STG_X = WD * 4096;
constexpr unsigned LDS_MISC = 4 * PART;
constexpr unsigned LDS_BYTES = LDS_MISC + 256;
static_assert(PART % 128 == 0 && LDS_T5 + RED_TILE <= PART && STG_X + XD * 2048 <= PART && LDS_BYTES <= 160 * 1024, "LDS layout");
static_assert(STG_X >= 2 * RED_TILE, "tiles 0 and 1 of unit 0 are written while the last x slots are in use");

constexpr unsigned long long IBM = IB_U <= 8 ? ~0ull << (8 * (IB_U <= 8 ? IB_U - 1 : 0)) : 0ull;

struct Params {
  const unsigned char* x;
  const unsigned char* w;
  const float* sx;
  const float* sw;
  float* out;
  unsigned char* ws[NRANKS];
  int rank;
};

#define STAMP(ws, idx) do {} while (0)
#define GSTAMP(idx) do {} while (0)

template <class F, int... I> __device__ __forceinline__ void sfor_impl(F&& f, std::integer_sequence<int, I...>) {
  (f(std::integral_constant<int, I>{}), ...);
}
template <int N, class F> __device__ __forceinline__ void sfor(F&& f) { sfor_impl(f, std::make_integer_sequence<int, N>{}); }

__device__ __forceinline__ rsrc_t mk_rsrc(const void* p, unsigned bytes) {
  return __builtin_amdgcn_make_buffer_rsrc((void*)p, 0, bytes, 0x00020000);
}

struct G {
  v8i Wf[8][4];
  v4f rd[2][4];
  unsigned sw_n, sw_t;
  unsigned rt_n, rt_it, rt_t, rt_last, rt_lastn;
  unsigned long long mask;
  unsigned kidx;
  unsigned tmov;
  unsigned badv;
  unsigned vo_loc, vo_stg, vo_rh, vo_out, wa_cur, wa_prev, ra[4], wa5, ra5;
  rsrc_t rs_x, rs_stg, rs_out, rs_tmo, rs_mk;
  unsigned mk_off, rh_off;
  unsigned so_loc[5];
  unsigned fo[4];
  unsigned ub_cur, ub_nxt;
  unsigned kd[8];
  unsigned kn[2];
  unsigned ke;
  unsigned rot;
  unsigned out_prev, out_cur;
  unsigned out5;
  unsigned red_d;
  unsigned call, unit;
  unsigned kbase;
  float scale;
  const GAS unsigned long long* flag_ptr;
  GAS unsigned* tmo;
  LAS unsigned* lds_bad;
};

constexpr int XB = 256 - 8 * R, AB = XB - 80, HD = AB - 1, FD = HD - 3, A4B = FD;
constexpr int acc_reg(int j, int nf) { return AB + 16 * j + 4 * nf; }

template <int J, int NF_, int SLOT, bool FIRST> __device__ __forceinline__ void mfma(const v8i& wf) {
  constexpr int A = acc_reg(J, NF_), X = XB + 8 * SLOT;
  if constexpr (FIRST)
    asm volatile("v_mfma_f32_16x16x128_f8f6f4 v[%c1:%c2], %0, v[%c3:%c4], 0 cbsz:1" :: "a"(wf), "i"(A), "i"(A + 3), "i"(X), "i"(X + 7));
  else
    asm volatile("v_mfma_f32_16x16x128_f8f6f4 v[%c1:%c2], %0, v[%c3:%c4], v[%c1:%c2] cbsz:1" :: "a"(wf), "i"(A), "i"(A + 3), "i"(X), "i"(X + 7));
}
template <int SLOT, int H> __device__ __forceinline__ void ld_stg(unsigned vo, rsrc_t rs, unsigned so) {
  constexpr int X = XB + 8 * SLOT + 4 * H;
  asm volatile("buffer_load_dwordx4 v[%c3:%c4], %0, %1, %2 offen offset:%c5 " XLD :: "v"(vo), "s"(rs), "s"(so), "i"(X), "i"(X + 3), "i"(H * 1024) : "memory");
}
template <int CNT> __device__ __forceinline__ void wait_vm() { asm volatile("s_waitcnt vmcnt(%c0)" :: "i"(CNT) : "memory"); }

__device__ __forceinline__ void st_sys(const v4i& d, unsigned vo, rsrc_t rs, unsigned so) {
  asm volatile("buffer_store_dwordx4 %0, %1, %2, %3 offen sc0 sc1\n\ts_nop 1" :: "v"(d), "v"(vo), "s"(rs), "s"(so) : "memory");
}

__device__ __forceinline__ void dma16(unsigned vo, rsrc_t rs, unsigned so, unsigned lds) {
  asm volatile("s_mov_b32 m0, %3\n\tbuffer_load_dwordx4 %0, %1, %2 offen lds" :: "v"(vo), "s"(rs), "s"(so), "s"(lds) : "memory");
}

#define WPOL ""
#define XPOL ""
__device__ __forceinline__ void dma16w(unsigned vo, rsrc_t rs, unsigned so, unsigned lds) {
  asm volatile("s_mov_b32 m0, %3\n\tbuffer_load_dwordx4 %0, %1, %2 offen " WPOL " lds" :: "v"(vo), "s"(rs), "s"(so), "s"(lds) : "memory");
}
__device__ __forceinline__ void dma16x(unsigned vo, rsrc_t rs, unsigned so, unsigned lds) {
  asm volatile("s_mov_b32 m0, %3\n\tbuffer_load_dwordx4 %0, %1, %2 offen " XPOL " lds" :: "v"(vo), "s"(rs), "s"(so), "s"(lds) : "memory");
}
__device__ __forceinline__ void touch(const unsigned char* a) {
  asm volatile("global_load_dword v%c1, %0, off" :: "v"(a), "i"(HD) : "memory");
}

#define HS0 0
#define XS0 0
constexpr int HW_LINES = (8 - HS0) * 512 * 2 * XG;
constexpr int NHW = (HW_LINES + 112 * 64 - 1) / (112 * 64);
constexpr int HW_LPI = (HW_LINES + 112 * NHW - 1) / (112 * NHW);
constexpr int HX_LINES = (8 - XS0) * 320, HX_WAVES = 16 * XG;
constexpr int NHX = (HX_LINES + HX_WAVES * 64 - 1) / (HX_WAVES * 64);
constexpr int HX_LPI = (HX_LINES + HX_WAVES * NHX - 1) / (HX_WAVES * NHX);
#define CODE_PF 0
#define CODE_X 0u
constexpr int NHINT = NHW + NHX + CODE_PF;
__device__ __forceinline__ int rot_order(int p) { return p / NROT + (p % NROT) * (8 / NROT); }
struct Hint { const unsigned char* wb; const unsigned char* xb; const unsigned char* cb; unsigned clen; int wi, xi; };
template <int Q> __device__ __forceinline__ void hint(const Hint& h) {
  const int lane = threadIdx.x & 63;
  if constexpr (Q >= NHX + NHW) {
    unsigned off = (unsigned)(h.wi * 8 + (lane & 7)) * 128u;
    off = off < h.clen - 4u ? off : h.clen - 4u;
    touch(h.cb + off);
  } else if constexpr (Q < NHX) {
    int line = (Q * HX_WAVES + h.xi) * HX_LPI + (lane < HX_LPI ? lane : HX_LPI - 1);
    line = line < HX_LINES ? line : HX_LINES - 1;
    const int r = rot_order(XS0 + line / 320), rem = line % 320;
    touch(h.xb + (rem >> 2) * KDIM + r * 512 + (rem & 3) * 128);
  } else {
    int line = ((Q - NHX) * 112 + h.wi) * HW_LPI + (lane < HW_LPI ? lane : HW_LPI - 1);
    line = line < HW_LINES ? line : HW_LINES - 1;
    const int row = HS0 * 512 + line / (2 * XG);
    touch(h.wb + (size_t)(rot_order(row >> 9) * 512 + (row & 511)) * NFULL + (line % (2 * XG)) * 128);
  }
}

__device__ __forceinline__ unsigned long long rt_now() {
  unsigned long long t;
  asm volatile("s_memrealtime %0\n\ts_waitcnt lgkmcnt(0)" : "=s"(t) :: "memory");
  return t;
}

__device__ __forceinline__ void slow_wait(G& g) {
  unsigned long long t0 = rt_now();
  const unsigned lane = __lane_id();
  unsigned lo = 0, hi = 64;
  unsigned long long m;
  while (true) {
    unsigned long long v = 0;
    if (lane >= lo && lane < hi) v = __hip_atomic_load(g.flag_ptr, __ATOMIC_RELAXED, __HIP_MEMORY_SCOPE_SYSTEM);
    unsigned tvv = __hip_atomic_load(g.tmo, __ATOMIC_RELAXED, __HIP_MEMORY_SCOPE_AGENT);
    unsigned v0 = (unsigned)v, v1 = (unsigned)(v >> 32);
    asm volatile("" : "+v"(v0), "+v"(v1), "+v"(tvv));
    m = (__builtin_amdgcn_ballot_w64((v0 < v1 ? v0 : v1) >= g.call) | IBM) >> g.kidx;
    if (m & 1ull) break;
    unsigned tv = __builtin_amdgcn_readfirstlane(tvv);
    if (tv == g.call || rt_now() - t0 > DEADLINE_TICKS) {
      __hip_atomic_store(g.tmo, g.call, __ATOMIC_RELAXED, __HIP_MEMORY_SCOPE_AGENT);
      asm volatile("ds_write_b32 %0, %1" :: "v"((unsigned)(LDS_MISC + 4)), "v"(1u) : "memory");
      m = ~0ull;
      break;
    }
    lo = g.kidx; hi = g.kidx + POLLW;
    __builtin_amdgcn_s_sleep(SLEEPN);
  }
  g.mask = m;
}

template <int OFF, int J, int NF_> __device__ __forceinline__ void lds_wr(unsigned a) {
  constexpr int A = acc_reg(J, NF_);
  asm volatile("ds_write_b128 %0, v[%c1:%c2] offset:%c3" :: "v"(a), "i"(A), "i"(A + 3), "i"(OFF) : "memory");
}
template <int OFF> __device__ __forceinline__ void lds_rd(v4f& v, unsigned a) {
  asm volatile("ds_read_b128 %0, %1 offset:%2" : "=v"(v) : "v"(a), "i"(OFF) : "memory");
}

template <int CNT> __device__ __forceinline__ void lds_wait(v4f (&r)[4], unsigned& b) {
  asm volatile("s_waitcnt lgkmcnt(%5)" : "+v"(r[0]), "+v"(r[1]), "+v"(r[2]), "+v"(r[3]), "+v"(b) : "i"(CNT) : "memory");
}
__device__ __forceinline__ void lds_rd32(unsigned& v, unsigned a) {
  asm volatile("ds_read_b32 %0, %1" : "=v"(v) : "v"(a) : "memory");
}
__device__ __forceinline__ void st_out(const v4f& v, unsigned vo, rsrc_t rs, unsigned so) {
  asm volatile("buffer_store_dwordx4 %0, %1, %2, %3 offen " OST "\n\ts_nop 1" :: "v"(v), "v"(vo), "s"(rs), "s"(so) : "memory");
}
__device__ __forceinline__ void st_out_last(const v4f& v, unsigned vo, rsrc_t rs, unsigned so) {
  asm volatile("buffer_store_dwordx4 %0, %1, %2, %3 offen " OSTL "\n\ts_nop 1" :: "v"(v), "v"(vo), "s"(rs), "s"(so) : "memory");
}

__device__ __forceinline__ void st_mark(unsigned vo, rsrc_t rs, unsigned so) {
  const v4i mk = {0, 0, 0, 0};
  asm volatile("buffer_store_dwordx4 %0, %1, %2, %3 offen " MKST "\n\ts_nop 1" :: "v"(mk), "v"(vo), "s"(rs), "s"(so) : "memory");
}

#define RH_N -1
#define RH0_N -1
constexpr int NRH = 2 / XG > 0 ? 2 / XG : 1;
__device__ __forceinline__ void touch_stg(unsigned vo, rsrc_t rs, unsigned so) {
  asm volatile("buffer_load_dword v%c3, %0, %1, %2 offen " XLD :: "v"(vo), "s"(rs), "s"(so), "i"(HD) : "memory");
}
__device__ __forceinline__ void rhint(G& g) {
  sfor<NRH>([&](auto qc) { touch_stg(g.vo_rh, g.rs_stg, g.ub_nxt + g.rh_off + decltype(qc)::value * 8192u); });
}

#define PK_A 1
#define PK_B 17
#define PK_0 22
__device__ __forceinline__ void peek_issue(G& g) {
  asm volatile("global_load_dwordx2 v[%c1:%c2], %0, off sc0 sc1" :: "v"(g.flag_ptr), "i"(FD), "i"(FD + 1) : "memory");
}
__device__ __forceinline__ void peek_use(G& g) {
  unsigned long long m;
  unsigned t;
  asm volatile("v_min_u32 %1, v%c3, v%c4\n\tv_cmp_le_u32 %0, %2, %1" : "=s"(m), "=&v"(t) : "s"(g.call), "i"(FD), "i"(FD + 1));
  g.mask |= m >> (g.kidx & 63u);
}

template <int NF, int PREV, bool HAS_NEXT> struct Sched {
  static constexpr int NG = 8 * NF;
  static constexpr bool has_load(int n) { return HAS_NEXT || (n - 1 + R < NG); }
  static constexpr int n_out(int n) { return PREV == 0 ? 0 : ((n >= 5 && n <= 8) ? 1 : 0) + ((PREV == 5 && n == 9) ? 1 : 0); }
  static constexpr bool mark_slot(int n) { return n >= MK_N && n < MK_N + 2 * (9 - IB_U); }
  static constexpr bool has_mark(int n) { return MK_U == 1 && PREV == 5 && mark_slot(n); }
  static constexpr int n_store(int n) { return n_out(n) + (has_mark(n) ? 1 : 0); }
  static constexpr bool has_tmo(int n) { return HAS_NEXT ? n == 30 - R : n == NG - 1; }
  static constexpr bool has_rh(int n) { return HAS_NEXT && n == RH_N; }
  static constexpr bool has_peek(int n) { return HAS_NEXT && (n == PK_A || n == PK_B); }
  static constexpr bool use_peek(int n) { return has_peek(n - R + 1); }
  static constexpr int ops(int q) {
    if (q < 0) return 2;
    return (has_load(q) ? 2 : 0) + n_store(q) + (has_tmo(q) ? 1 : 0) + (has_rh(q) ? NRH : 0) + (has_peek(q) ? 1 : 0);
  }
  static constexpr int wait_cnt(int n) {
    int c = 0;
    for (int q = n - R + 2; q <= n - 1; ++q) c += ops(q);
    return c > 63 ? 63 : c;
  }
};

template <int SLOT> __device__ __forceinline__ unsigned ib_min() {
  constexpr int X = XB + 8 * SLOT;
  unsigned t0, t1;
  asm volatile("v_min3_u32 %0, v[%c2], v[%c3], v[%c4]\n\t"
               "v_min3_u32 %1, v[%c5], v[%c6], v[%c7]\n\t"
               "v_min3_u32 %0, v[%c8], v[%c9], %0\n\t"
               "v_min_u32 %0, %0, %1"
               : "=&v"(t0), "=&v"(t1)
               : "i"(X), "i"(X + 1), "i"(X + 2), "i"(X + 3), "i"(X + 4), "i"(X + 5), "i"(X + 6), "i"(X + 7));
  return t0;
}
__device__ __forceinline__ unsigned long long ib_zero(unsigned t) {
  unsigned long long m;
  asm volatile("v_cmp_eq_u32 %0, 0, %1" : "=s"(m) : "v"(t));
  return m;
}
template <int SLOT> __device__ __forceinline__ bool ib_dirty() { return ib_zero(ib_min<SLOT>()) != 0ull; }
constexpr int rl_kk(int rotl, int i) { return (i + 8 - (rotl >> 3)) & 7; }
template <int M, int ROFF, int ROTL> __device__ __forceinline__ void ib_load(G& g) {
  constexpr int slot = (ROFF + M) % R;
  const unsigned so = (M < 32 ? g.ub_cur + (ROTL < 0 ? g.kd[(M / 4) & 7] : rl_kk(ROTL, M / 4) * 32768u) : g.ub_nxt + g.kn[((M - 32) / 4) & 1]) + g.fo[M % 4];
  ld_stg<slot, 0>(g.vo_stg, g.rs_stg, so);
  ld_stg<slot, 1>(g.vo_stg, g.rs_stg, so);
}
template <int N, int ROFF, bool HAS_NEXT, int ROTL> __device__ __forceinline__ void ib_retry(G& g) {
  constexpr int KE = N | 3;
  constexpr int LE = (HAS_NEXT || N + R - 2 < 31) ? N + R - 2 : 31;
  constexpr int slot = (ROFF + N) % R;
  const unsigned long long t0 = rt_now();
  const unsigned lane = __lane_id();
  bool flagged = false;
  while (true) {
    unsigned long long v = 0;
    if (lane == g.kbase + N / 4) v = __hip_atomic_load(g.flag_ptr, __ATOMIC_RELAXED, __HIP_MEMORY_SCOPE_SYSTEM);
    unsigned tvv = __hip_atomic_load(g.tmo, __ATOMIC_RELAXED, __HIP_MEMORY_SCOPE_AGENT);
    sfor<KE - N + 1>([&](auto mc) { ib_load<N + decltype(mc)::value, ROFF, ROTL>(g); });
    unsigned v0 = (unsigned)v, v1 = (unsigned)(v >> 32);
    asm volatile("s_waitcnt vmcnt(0)" : "+v"(v0), "+v"(v1), "+v"(tvv) :: "memory");
    if (flagged || !ib_dirty<slot>()) break;
    flagged = __builtin_amdgcn_ballot_w64((v0 < v1 ? v0 : v1) >= g.call) != 0ull;
    unsigned tv = __builtin_amdgcn_readfirstlane(tvv);
    if (tv == g.call || rt_now() - t0 > DEADLINE_TICKS) {
      __hip_atomic_store(g.tmo, g.call, __ATOMIC_RELAXED, __HIP_MEMORY_SCOPE_AGENT);
      asm volatile("ds_write_b32 %0, %1" :: "v"((unsigned)(LDS_MISC + 4)), "v"(1u) : "memory");
      break;
    }
    if (!flagged) __builtin_amdgcn_s_sleep(IB_SLEEP);
  }
  if constexpr (LE > KE) sfor<LE - KE>([&](auto mc) { ib_load<KE + 1 + decltype(mc)::value, ROFF, ROTL>(g); });
}

template <int NF, int ROFF, int PREV, bool HAS_NEXT, int ROTL, bool IBC>
__device__ __forceinline__ void unit_body(G& g) {
  using S = Sched<NF, PREV, HAS_NEXT>;
  constexpr int NG = S::NG;
  sfor<NG>([&](auto nc) {
    constexpr int n = decltype(nc)::value;
    constexpr int i = n / NF, j = n % NF;
    constexpr int wi = ROTL < 0 ? i : (rl_kk(ROTL, i) + 8 - (ROTL & 7)) & 7;
    constexpr int slot = (ROFF + n) % R;
    constexpr int pslot = (ROFF + n + R - 1) % R;
    constexpr int t = n - 1 + R;
    constexpr bool t_next = t >= NG;
    constexpr int ti = t_next ? (t - NG) / 4 : t / NF, tj = t_next ? (t - NG) % 4 : t % NF;

    wait_vm<S::wait_cnt(n)>();
    if constexpr (IBC && (n == 0 || !IB_PIPE)) {
      if (__builtin_expect(ib_dirty<slot>(), 0)) ib_retry<n, ROFF, HAS_NEXT, ROTL>(g);
    }

    if constexpr (PREV != 0 && n == 4) {
      asm volatile("s_waitcnt lgkmcnt(0)\n\ts_barrier" ::: "memory");
    }
    v4f ov, ot0, ot1;
    constexpr bool has_rd = PREV != 0 && ((n >= 4 && n <= 7) || (PREV == 5 && n == 8));
    constexpr int roff = (n >= 4 && n <= 7) ? (n - 4) * 4 * (int)RED_ROW : 0;
    constexpr int rb = n & 1;
    if constexpr (S::has_load(n) && tj == 0) {
      if constexpr (!IBC) {
        if (__builtin_expect(!(g.mask & 1ull), 0)) slow_wait(g);
        g.mask >>= 1;
        g.kidx += 1;
      }
    }
    if constexpr (S::has_rh(n)) rhint(g);
    if constexpr (S::has_peek(n)) peek_issue(g);
    if constexpr (S::has_tmo(n)) {
      asm volatile("buffer_load_dword %0, off, %1, 0 sc1" : "=v"(g.tmov) : "s"(g.rs_tmo) : "memory");
    }

    auto load_half = [&](auto hc) {
      constexpr int h = decltype(hc)::value;
      if constexpr (S::has_load(n))
        ld_stg<pslot, h>(g.vo_stg, g.rs_stg, (t_next ? g.ub_nxt + g.kn[ti & 1] : g.ub_cur + (ROTL < 0 ? g.kd[ti & 7] : rl_kk(ROTL, ti) * 32768u)) + g.fo[tj]);
    };
    auto side = [&](auto qc) {
      constexpr int q = decltype(qc)::value;
      if constexpr (n == NG - 2) lds_wr<0 * RED_TILE + q * 64, 0, q>(g.wa_cur);
      if constexpr (n == NG - 1) lds_wr<1 * RED_TILE + q * 64, 1, q>(g.wa_cur);
      if constexpr (PREV == 4 && n == 0) lds_wr<2 * RED_TILE + q * 64, 2, q>(g.wa_prev);
      if constexpr (PREV == 4 && n == 1) lds_wr<3 * RED_TILE + q * 64, 3, q>(g.wa_prev);
      if constexpr (PREV == 5 && n == 0) lds_wr<3 * RED_TILE + q * 64, 3, q>(g.wa_prev);
      if constexpr (PREV == 5 && n == 1) lds_wr<q * 64, 4, q>(g.wa5);
      if constexpr (PREV != 0 && n == 4 && q == 0)
        asm volatile("ds_read_b32 %0, %1" : "=v"(g.badv) : "v"((unsigned)(LDS_MISC + 4)) : "memory");
      if constexpr (has_rd && q < 2) {
        if constexpr (n <= 7) {
          lds_rd<roff>(g.rd[rb][(2 * q) & 3], g.ra[(2 * q) & 3]);
          lds_rd<roff>(g.rd[rb][(2 * q + 1) & 3], g.ra[(2 * q + 1) & 3]);
        } else {
          lds_rd<0>(g.rd[rb][(2 * q) & 3], q == 0 ? g.ra5 : g.ra5 + 2 * PART);
          lds_rd<PART>(g.rd[rb][(2 * q + 1) & 3], q == 0 ? g.ra5 : g.ra5 + 2 * PART);
        }
      }
      if constexpr (S::n_out(n) != 0 && q == 0) {
        constexpr int pb = rb ^ 1;
        asm volatile("s_waitcnt lgkmcnt(%5)" : "+v"(g.rd[pb][0]), "+v"(g.rd[pb][1]), "+v"(g.rd[pb][2]), "+v"(g.rd[pb][3]), "+v"(g.badv) : "i"(has_rd ? 2 : 0));
        ot0 = g.rd[pb][0] + g.rd[pb][1];
        ot1 = g.rd[pb][2] + g.rd[pb][3];
        asm volatile("" : "+v"(ot0), "+v"(ot1));
        if constexpr (n == 5) {
          unsigned bad = __builtin_amdgcn_readfirstlane(g.badv) | (__builtin_amdgcn_readfirstlane(g.tmov) == g.call ? 1u : 0u);
          if (bad) g.scale = __builtin_nanf("");
        }
      }
      if constexpr (S::n_out(n) != 0 && q == 1) {
        const v4f sum = ot0 + ot1;
        const float sc = g.scale;
        ov = (v4f){sum[0] * sc, sum[1] * sc, sum[2] * sc, sum[3] * sc};
        asm volatile("" : "+v"(ov));
      }
    };
    auto store_out = [&]() {
      if constexpr (S::n_out(n) != 0) {
        const unsigned so = n < 9 ? g.out_prev + (n - 5) * 4 * (NLOC * 4) : g.out5;
        st_out(ov, g.vo_out, g.rs_out, so);
      }
      if constexpr (S::has_mark(n)) st_mark(g.vo_stg, g.rs_mk, g.mk_off + ((n - MK_N) >> 1) * UNIT_BYTES + ((n - MK_N) & 1) * 1024u);
      if constexpr (MK_U > 1 && PREV == 4 && HAS_NEXT && S::mark_slot(n)) {
        if (g.unit == (unsigned)MK_U) st_mark(g.vo_stg, g.rs_mk, g.mk_off + ((n - MK_N) >> 1) * UNIT_BYTES + ((n - MK_N) & 1) * 1024u);
      }
    };

    unsigned ibt = 0;
    unsigned long long ibm = 0;
    sfor<4>([&](auto fc) {
      constexpr int nf = decltype(fc)::value;
      mfma<j, nf, slot, i == 0>(g.Wf[wi][nf]);
      side(fc);
      if constexpr (nf == 0) load_half(std::integral_constant<int, 0>{});
      if constexpr (nf == 1) store_out();
      if constexpr (nf == 2) load_half(std::integral_constant<int, 1>{});
      if constexpr (IBC && IB_PIPE == 1 && n + 1 < NG) {
        if constexpr (nf == 2) {
          wait_vm<S::wait_cnt(n + 1)>();
          ibt = ib_min<(ROFF + n + 1) % R>();
        }
        if constexpr (nf == 3) {
          if (__builtin_expect(ib_zero(ibt) != 0ull, 0)) ib_retry<n + 1, ROFF, HAS_NEXT, ROTL>(g);
        }
      }
      if constexpr (IBC && IB_PIPE == 2 && n + 1 < NG) {
        if constexpr (nf == 1) {
          wait_vm<S::wait_cnt(n + 1) - (S::has_load(n) ? 1 : 0)>();
          ibt = ib_min<(ROFF + n + 1) % R>();
        }
        if constexpr (nf == 2) ibm = ib_zero(ibt);
        if constexpr (nf == 3) {
          if (__builtin_expect(ibm != 0ull, 0)) ib_retry<n + 1, ROFF, HAS_NEXT, ROTL>(g);
        }
      }
      if constexpr (S::use_peek(n) && nf == 1) peek_use(g);
    });
  });
}

struct U0 {
  unsigned vo_w, vo_x, ra_w, ra_x0, ra_x1;
  unsigned so_w, lds_w, lds_x;
  rsrc_t rs_w;
};
#define HT_X -1
#define HT_X1 108
#define HT_C 4
#define HT_W0 -1
#define HT_W1 4
#define HT_W2 108
struct U0S {
  static constexpr int NG = 40;
  static constexpr int w_issue(int n, int k) {
    int m = -1;
    if (n == 0) m = WD + k;
    else if (k == 0 && n % 5 == 3) m = 2 * (n / 5) + 2 + WD;
    else if (k == 0 && n % 5 == 0) m = 2 * (n / 5) + 1 + WD;
    return m <= 15 ? m : -1;
  }
  static constexpr int w_read(int n) {
    int m = n % 5 == 2 ? 2 * (n / 5 + 1) : (n % 5 == 4 ? 2 * (n / 5 + 1) + 1 : -1);
    return m <= 15 ? m : -1;
  }
  static constexpr int w_need(int m) { return m < 2 ? -1 : (m % 2 == 0 ? 5 * (m / 2 - 1) + 2 : 5 * (m / 2 - 1) + 4); }
  static constexpr int hint_t(int q) { return q >= NHX + NHW ? HT_C : q < NHX ? (q == 0 ? HT_X : HT_X1) : (q == NHX ? HT_W0 : (q == NHX + 1 ? HT_W1 : HT_W2)); }
  static constexpr bool remote(int n) { return n - 1 + R >= NG; }
  static constexpr int NPRO_MAX = WD + XD + NHINT;
  static constexpr int pro_op(int idx) {
    int c = 0;
    for (int t = -2; t < 100; ++t) {
      if (t == -1) { if (c++ == idx) return 100; }
      for (int m = 0; m < WD; ++m) if (w_need(m) == t) { if (c++ == idx) return 200 + m; }
      if (t + 1 < XD && t >= 0) { if (c++ == idx) return 100 + t + 1; }
      for (int q = 0; q < NHINT; ++q) if (hint_t(q) == t) { if (c++ == idx) return 300 + q; }
    }
    return -1;
  }
  static constexpr int op_cnt(int op) { return op < 0 ? 0 : (op < 200 ? 2 : (op < 300 ? 4 : 1)); }
  static constexpr int n_hint(int n) { int c = 0; for (int q = 0; q < NHINT; ++q) if (hint_t(q) == 100 + n) ++c; return c; }
  static constexpr int wait_cnt(int n) {
    int pos = 0, pl[NG + 16] = {}, pw[16] = {};
    pos += 1;
    for (int idx = 0; idx < NPRO_MAX; ++idx) {
      const int op = pro_op(idx);
      pos += op_cnt(op);
      if (op >= 100 && op < 200) pl[op - 100] = pos;
      if (op >= 200 && op < 300) pw[op - 200] = pos;
    }
    if (n < 0) return pos - (pw[1] > pl[0] ? pw[1] : pl[0]);
    for (int q = 0; q <= n; ++q) {
      for (int k = 0; k < 2; ++k) if (w_issue(q, k) >= 0) { pos += 4; pw[w_issue(q, k)] = pos; }
      if (q + XD < NG) { pos += 2; pl[q + XD] = pos; }
      pos += n_hint(q) + (q == RH0_N ? NRH : 0) + (q == PK_0 ? 1 : 0);
      if (remote(q)) pos += 1;
      if (q < n && remote(q)) pos += 1;
    }
    int need = n + 1 < NG ? pl[n + 1] : 0;
    if (w_read(n) >= 0 && pw[w_read(n)] > need) need = pw[w_read(n)];
    int c = pos - need;
    return c > 63 ? 63 : c;
  }
};
template <int M> __device__ __forceinline__ void u0_w_issue(const G& g, const U0& u) {
  sfor<4>([&](auto qc) {
    constexpr int q = decltype(qc)::value;
    dma16w(u.vo_w, u.rs_w, u.so_w + (g.kd[M >> 1] << 8) + (unsigned)((M & 1) * 64 + q * 16) * NFULL, u.lds_w + (M % WD) * 4096 + q * 1024);
  });
}
template <int N> __device__ __forceinline__ void u0_x_issue(const G& g, const U0& u) {
  dma16x(u.vo_x, g.rs_x, g.so_loc[N % 5] + (g.kd[N / 5] >> 6), u.lds_x + (N % XD) * 2048);
  dma16x(u.vo_x, g.rs_x, g.so_loc[N % 5] + (g.kd[N / 5] >> 6) + 8 * KDIM, u.lds_x + (N % XD) * 2048 + 1024);
}
template <int M> __device__ __forceinline__ void u0_w_read(G& g, const U0& u) {
#pragma unroll
  for (int tt = 0; tt < 2; ++tt)
#pragma unroll
    for (int nf = 0; nf < 4; ++nf) {
      v2i q = __builtin_amdgcn_ds_read_tr8_b64_v2i32((LAS v2i*)(uintptr_t)(u.ra_w + (M % WD) * 4096 + tt * 512 + nf * 16));
      g.Wf[M >> 1][nf][(M & 1) * 4 + tt * 2] = q[0];
      g.Wf[M >> 1][nf][(M & 1) * 4 + tt * 2 + 1] = q[1];
    }
}
template <int N> __device__ __forceinline__ void u0_x_read(G& g, const U0& u) {
  constexpr int X = XB + 8 * (N % R);
  asm volatile("ds_read_b128 v[%c1:%c2], %0 offset:%c3" :: "v"(u.ra_x0), "i"(X), "i"(X + 3), "i"((N % XD) * 2048) : "memory");
  asm volatile("ds_read_b128 v[%c1:%c2], %0 offset:%c3" :: "v"(u.ra_x1), "i"(X + 4), "i"(X + 7), "i"((N % XD) * 2048) : "memory");
}

__device__ __forceinline__ void unit0_body(G& g, U0& u, const Hint& h) {
  constexpr int NG = U0S::NG;
  sfor<NG>([&](auto nc) {
    constexpr int n = decltype(nc)::value;
    constexpr int i = n / 5, j = n % 5;
    constexpr int slot = n % R, pslot = (n + R - 1) % R;
    constexpr int t = n - 1 + R, ti = U0S::remote(n) ? (t - NG) / 4 : 0, tj = U0S::remote(n) ? (t - NG) % 4 : 0;

    asm volatile("s_waitcnt lgkmcnt(0)" ::: "memory");
    if constexpr (U0S::w_issue(n, 0) >= 0) u0_w_issue<U0S::w_issue(n, 0)>(g, u);
    if constexpr (U0S::w_issue(n, 1) >= 0) u0_w_issue<U0S::w_issue(n, 1)>(g, u);
    if constexpr (n + XD < NG) u0_x_issue<n + XD>(g, u);
    sfor<NHINT>([&](auto qc) { constexpr int q = decltype(qc)::value; if constexpr (U0S::hint_t(q) == 100 + n) hint<q>(h); });
    if constexpr (n == RH0_N) rhint(g);
    if constexpr (n == PK_0) peek_issue(g);
    if constexpr (PK_0 >= 0 && n == NG - R) peek_use(g);
    static_assert(PK_0 < 0 || (PK_0 + 1 + XD < NG && PK_0 + XD < NG - R), "peek is complete when it is used");
    if constexpr (U0S::remote(n) && tj == 0) {
      if (!(g.mask & 1ull)) slow_wait(g);
      g.mask >>= 1;
      g.kidx += 1;
    }
    sfor<4>([&](auto fc) {
      constexpr int nf = decltype(fc)::value;
      mfma<j, nf, slot, i == 0>(g.Wf[i][nf]);
      if constexpr (n == NG - 3) lds_wr<0 * RED_TILE + nf * 64, 0, nf>(g.wa_cur);
      if constexpr (n == NG - 2) lds_wr<1 * RED_TILE + nf * 64, 1, nf>(g.wa_cur);
      if constexpr (n == NG - 1) lds_wr<2 * RED_TILE + nf * 64, 2, nf>(g.wa_cur);
      if constexpr (U0S::remote(n) && (nf == 0 || nf == 2))
        ld_stg<pslot, nf / 2>(g.vo_stg, g.rs_stg, g.ub_nxt + g.kn[ti & 1] + g.fo[tj]);
      if constexpr (nf == 1 && n + 1 < NG) {
        asm volatile("s_waitcnt vmcnt(%0)" :: "i"(U0S::wait_cnt(n)) : "memory");
        u0_x_read<n + 1>(g, u);
        if constexpr (U0S::w_read(n) >= 0) u0_w_read<U0S::w_read(n)>(g, u);
      }
    });
  });
}

template <bool FIRST> __device__ __forceinline__ void next_unit(G& g, unsigned out_next) {
  g.out_prev = g.out_cur;
  g.out_cur = out_next;
  g.ub_cur = g.ub_nxt;
  g.ub_nxt += UNIT_BYTES;
  g.unit += 1u;
  g.kbase += 8u;
  g.kn[0] = g.unit == 7u ? ((8u - g.ke) & 7u) * 32768u : g.kd[0];
  g.kn[1] = g.unit == 7u ? ((9u - g.ke) & 7u) * 32768u : g.kd[1];
  g.wa_prev = g.wa_cur;
  g.wa_cur += g.red_d;
  if (!FIRST) { g.ra[0] -= g.red_d; g.ra[1] -= g.red_d; g.ra[2] -= g.red_d; g.ra[3] -= g.red_d; }
  g.red_d = 0u - g.red_d;
}

__device__ __forceinline__ void gemm_role(const Params& p, unsigned char* smem_generic) {
  LAS unsigned char* smem = (LAS unsigned char*)smem_generic;
  const int b = blockIdx.x;
  const int xcd = b & 7, slot = (b >> 3) - (NCOMM >> 3);
  const int pidx = (xcd % XG) * 28 + slot;
  const int pi = pidx / (4 * XG), cb = (xcd / XG) * (4 * XG) + pidx % (4 * XG);
  const int tid = threadIdx.x, lane = tid & 63;
  const int w = __builtin_amdgcn_readfirstlane(tid >> 6);
  const int kg = lane >> 4, li = lane & 15;
  const int rank = p.rank;
  unsigned char* myws = p.ws[rank];
  const int src = (rank + 1 + pi) & 7;
  G g;
  asm volatile("" ::: "v255");
  GSTAMP(0);

  unsigned ticket = 0;
  if (lane == 0)
    asm volatile("global_atomic_add %0, %1, %2, off sc0 sc1" : "+v"(ticket) : "v"((unsigned*)(myws + CTRL_CALL + (b * 4 + w) * 32)), "v"(1u) : "memory");
  GSTAMP(17);
  const int row0 = pi * 80 < 432 ? pi * 80 : 432;
  g.rs_x = mk_rsrc(p.x, SRC_BYTES);
  g.rot = (unsigned)(pidx % NROT) * (8 / NROT);
  g.ke = C3_FIX ? (unsigned)(6 - pi + 1) >> 1 : 0u;
  sfor<8>([&](auto ic) { constexpr int i = decltype(ic)::value; g.kd[i] = ((i + g.rot) & 7u) * 32768u; });
  g.kn[0] = g.kd[0]; g.kn[1] = g.kd[1];
  sfor<4>([&](auto jc) { constexpr int j = decltype(jc)::value; g.so_loc[j] = (unsigned)(row0 + ((j + 1 + w) & 3) * 16) * KDIM + w * 128; });
  g.so_loc[4] = (unsigned)(row0 + 64) * KDIM + w * 128;
  U0 u;
  {
    const unsigned base = (unsigned)(uintptr_t)smem + w * PART;
    u.rs_w = mk_rsrc(p.w, (unsigned)KDIM * NFULL);
    u.vo_w = (lane >> 2) * NFULL + (lane & 3) * 16;
    u.vo_x = (lane >> 3) * KDIM + ((lane & 7) ^ (lane >> 3)) * 16;
    u.so_w = (unsigned)rank * NLOC + cb * 64 + (unsigned)(w * 128) * NFULL;
    u.lds_w = base;
    u.lds_x = base + STG_X;
    u.ra_w = base + (kg * 16 + (li >> 1)) * 64 + (li & 1) * 8;
    u.ra_x0 = base + STG_X + (li >> 3) * 1024 + (li & 7) * 128 + ((kg ^ (li & 7)) * 16);
    u.ra_x1 = u.ra_x0 ^ 64u;
  }
  Hint h;
  h.wb = p.w + (size_t)rank * NLOC + (xcd / XG) * (256 * XG);
  h.xb = p.x + (size_t)row0 * KDIM;
  h.wi = slot * 4 + w;
  h.xi = (pidx % (4 * XG)) * 4 + w;
  sfor<U0S::NPRO_MAX>([&](auto ic) {
    constexpr int op = U0S::pro_op(decltype(ic)::value);
    if constexpr (op >= 100 && op < 200) u0_x_issue<op - 100>(g, u);
    if constexpr (op >= 200 && op < 300) u0_w_issue<op - 200>(g, u);
    if constexpr (op >= 300) hint<op - 300>(h);
  });
  GSTAMP(12);
  g.lds_bad = (LAS unsigned*)(smem + LDS_MISC + 4);
  if (tid == 0) asm volatile("ds_write_b32 %0, %1" :: "v"((unsigned)(uintptr_t)smem + LDS_MISC + 4), "v"(0u) : "memory");
  asm volatile("s_waitcnt vmcnt(%1) lgkmcnt(0)\n\ts_barrier" : "+v"(ticket) : "i"(U0S::wait_cnt(-1)) : "memory");
  g.call = __builtin_amdgcn_readfirstlane(ticket) + 1u;
  GSTAMP(13);
  u0_x_read<0>(g, u);
  u0_w_read<0>(g, u); u0_w_read<1>(g, u);

  g.scale = *(const CAS float*)p.sx * *(const CAS float*)p.sw;
  const unsigned cm = __builtin_amdgcn_readfirstlane(g.call % NSTAGE);
  g.rs_stg = mk_rsrc(myws + cm * STAGE_BYTES, STAGE_BYTES);
  {
    const unsigned gw = (unsigned)(b - NCOMM) * 4u + (unsigned)w;
    g.rs_mk = mk_rsrc(myws + (cm == 0u ? 2u : cm - 1u) * STAGE_BYTES, STAGE_BYTES);
    g.mk_off = ((unsigned)(rank + 1 + (gw >> 7)) & 7u) * SRC_BYTES + (IB_U - 1u) * UNIT_BYTES + (gw & 127u) * 2048u;
  }
  g.rs_out = mk_rsrc(p.out, (unsigned)(4096u * NLOC * 4u));
  g.flag_ptr = (const GAS unsigned long long*)((lane >> 3) < PF_U
      ? myws + CTRL_FLAG + src * 2048 + w * 512 + (lane < 56 ? (lane & 56) + ((lane + g.rot) & 7) : lane) * 8
      : myws + CTRL_UFLAG + src * 64 + (lane >> 3) * 8);
  g.tmo = (GAS unsigned*)(myws + CTRL_TMO);
  g.rs_tmo = mk_rsrc(myws + CTRL_TMO, 4);
  g.vo_stg = lane * 16;
  g.vo_rh = lane * 128;
  g.rh_off = (unsigned)(pidx % (4 * XG)) * (UNIT_BYTES / (4 * XG)) + (unsigned)w * (UNIT_BYTES / (16 * XG) - 8192u);
  g.vo_out = (lane >> 4) * (NLOC * 4) + (lane & 15) * 16;
  sfor<4>([&](auto jc) { constexpr int j = decltype(jc)::value; g.fo[j] = ((j + 1 + w) & 3) * 2048; });
  g.wa_cur = w * PART + li * RED_ROW + kg * 16;
  sfor<4>([&](auto qc) {
    constexpr int q = decltype(qc)::value;
    g.ra[q] = q * PART + ((w - q - 1) & 3) * RED_TILE + (lane >> 4) * RED_ROW + (lane & 15) * 16;
  });
  g.wa5 = w * PART + LDS_T5 + li * RED_ROW + kg * 16;
  g.ra5 = LDS_T5 + (4 * w + (lane >> 4)) * RED_ROW + (lane & 15) * 16;
  g.red_d = RED_WAVE;
  g.tmov = 0u;
  const unsigned colb = cb * 256;
  auto out_off = [&](int grow) -> unsigned { return (unsigned)(grow + w * 16) * (NLOC * 4) + colb; };
  g.out_cur = out_off(rank * MLOC + row0);
  g.out_prev = g.out_cur;
  g.out5 = (unsigned)(rank * MLOC + row0 + 64 + 4 * w) * (NLOC * 4) + colb;
  g.ub_cur = 0;
  g.ub_nxt = (unsigned)src * SRC_BYTES + w * 8192;
  g.mask = IBM;
  g.sw_n = 0; g.sw_t = 0;
  g.rt_n = g.rt_it = g.rt_t = g.rt_last = g.rt_lastn = 0;
  g.kbase = 0u - 8u;
  g.unit = 0u;
  static_assert(IB_U >= 3, "the first remote unit is never validated in-band");
  g.kidx = 0u;

  GSTAMP(1);
  unit0_body(g, u, h);
  GSTAMP(2);
  next_unit<true>(g, out_off(src * MLOC));
  unit_body<4, 40 % R, 5, true, -1, false>(g);
  GSTAMP(3);
  constexpr int PER = (32 % R == 0) ? 1 : 3;
  static_assert((32 * PER) % R == 0 && 6 % PER == 0, "ring size");
  static_assert(PER == 1, "ring size");
  constexpr int ULIM = IB_U < 8 ? IB_U : 8;
#define USTAMP(u) do {} while (0)
  for (int u = 2; u < ULIM; ++u) {
    next_unit<false>(g, out_off(src * MLOC + (u - 1) * 64));
    unit_body<4, (40 + 32) % R, 4, true, -1, false>(g);
    USTAMP(u);
  }
  if constexpr (ULIM < 8) {
    for (int u = ULIM; u < 8; ++u) {
      next_unit<false>(g, out_off(src * MLOC + (u - 1) * 64));
      unit_body<4, (40 + 32) % R, 4, true, -1, true>(g);
      USTAMP(u);
    }
  }
  next_unit<false>(g, out_off(src * MLOC + 7 * 64));
  sfor<NROT>([&](auto rc) {
    constexpr int ro = decltype(rc)::value * (8 / NROT);
    if (NROT == 1 || g.rot == (unsigned)ro) unit_body<4, (40 + 32 * 7) % R, 4, false, ro, (IB_U <= 8)>(g);
  });
  GSTAMP(10);

  sfor<4>([&](auto qc) { constexpr int q = decltype(qc)::value; lds_wr<2 * RED_TILE + q * 64, 2, q>(g.wa_cur); });
  asm volatile("s_nop 7\n\ts_nop 7\n\ts_nop 7" ::: "memory");
  sfor<4>([&](auto qc) { constexpr int q = decltype(qc)::value; lds_wr<3 * RED_TILE + q * 64, 3, q>(g.wa_cur); });
  asm volatile("s_waitcnt lgkmcnt(0)\n\ts_barrier" ::: "memory");
  {
    v4f r[4][4];
    lds_rd32(g.badv, LDS_MISC + 4);
    sfor<16>([&](auto qc) {
      constexpr int q = decltype(qc)::value;
      lds_rd<(q >> 2) * 4 * RED_ROW>(r[q >> 2][q & 3], g.ra[q & 3] - g.red_d);
    });
    float sc = g.scale;
    sfor<4>([&](auto sc_) {
      constexpr int q = decltype(sc_)::value;
      lds_wait<(3 - q) * 4>(r[q], g.badv);
      if constexpr (q == 0) {
        asm volatile("s_waitcnt vmcnt(0)" : "+v"(g.tmov) :: "memory");
        unsigned bad = __builtin_amdgcn_readfirstlane(g.badv) | (__builtin_amdgcn_readfirstlane(g.tmov) == g.call ? 1u : 0u);
        if (bad) sc = __builtin_nanf("");
      }
      const v4f sum = (r[q][0] + r[q][1]) + (r[q][2] + r[q][3]);
      const v4f v = {sum[0] * sc, sum[1] * sc, sum[2] * sc, sum[3] * sc};
      st_out_last(v, g.vo_out, g.rs_out, g.out_cur + q * 4 * (NLOC * 4));
    });
  }
}

#define CSTAMP(n) do {} while (0)
template <bool LEADER>
__device__ __forceinline__ void comm_wave(const Params& p, LAS unsigned char* smem, const unsigned call, const int j, const int w, const int lane, v4i d) {
  const int rank = p.rank;
  unsigned char* myws = p.ws[rank];
  const int cw = j * 4 + w;
  rsrc_t rs_x = mk_rsrc(p.x, SRC_BYTES);
  const unsigned vo = (unsigned)((w >> 1) * 16 + (lane & 15)) * KDIM + (w & 1) * 64 + (lane >> 4) * 16;
  const unsigned dst0 = (call % NSTAGE) * STAGE_BYTES + (unsigned)rank * SRC_BYTES + (unsigned)w * 1024u;
  rsrc_t rs[7];
#pragma unroll
  for (int q = 0; q < 7; ++q) {
    int pq = (rank + 1 + (q + cw) % 7) & 7;
    rs[q] = mk_rsrc(p.ws[pq], NSTAGE * STAGE_BYTES);
  }
  unsigned char* flagb = p.ws[(rank + 1 + (lane < 7 ? lane : 0)) & 7] + CTRL_FLAG + rank * 2048;
  unsigned char* uflagb = p.ws[(rank + 1 + (lane < 7 ? lane : 0)) & 7] + CTRL_UFLAG + rank * 64;
  unsigned pend = 0;
  unsigned* ticketp = (unsigned*)(myws + CTRL_TICKET);
  GAS unsigned* tmo = (GAS unsigned*)(myws + CTRL_TMO);
  const unsigned tbase = (call - 1u) * TICKETS_PER_CALL - 2u * NCOMM;
  const unsigned vs = lane * 16;
  LAS unsigned* tk = (LAS unsigned*)(smem + 1024);
  LAS unsigned* cnt = (LAS unsigned*)(smem + LDS_MISC + 64);

  auto publish = [&](unsigned n, unsigned t) {
    unsigned old = 0;
    if (lane == 0) old = __hip_atomic_fetch_add(cnt + (n & 7u), 1u, __ATOMIC_RELAXED, __HIP_MEMORY_SCOPE_WORKGROUP);
    old = __builtin_amdgcn_readfirstlane(old);
    if ((old & 3u) == 3u) {
      if (t < PF_U * 64u) {
        if (lane < 7) __hip_atomic_store((unsigned*)(flagb + ((t >> 1) & 3u) * 512u + (t >> 3) * 8u + (t & 1u) * 4u), call, __ATOMIC_RELAXED, __HIP_MEMORY_SCOPE_SYSTEM);
      } else {
        if (lane == 0)
          asm volatile("global_atomic_add v%c2, %0, %1, off sc0 sc1" :: "v"(myws + CTRL_UCNT + (t >> 6) * 128u), "v"(1u), "i"(FD) : "memory");
        pend = (t >> 6) + 1u;
      }
      CSTAMP(n + 1);
    }
  };
  auto group_done = [&]() {
    if (pend) {
      unsigned o;
      asm volatile("v_readfirstlane_b32 %0, v%c1" : "=s"(o) : "i"(FD) : "memory");
      if (o - (call - 1u) * 64u == 63u && lane < 7)
        asm volatile("global_store_dwordx2 %0, %1, off sc0 sc1" :: "v"(uflagb + (pend - 1u) * 8u), "v"((v2u){call, call}) : "memory");
      pend = 0;
    }
  };

  unsigned t_cur = (unsigned)j, t_prev = 0;
  unsigned n = 0;
  while (true) {
    v4i dn;
    unsigned tick = 0, t_nxt = 0;
    const unsigned dst = dst0 + t_cur * 4096u;
    sfor<7>([&](auto qc) {
      constexpr int q = decltype(qc)::value;
      asm volatile("s_waitcnt vmcnt(%0)" :: "i"(COMM_W - 1) : "memory");
      if constexpr (PF_U < 8 && q == (2 * COMM_W - 2 <= 6 ? 2 * COMM_W - 2 : COMM_W - 1)) group_done();
      if constexpr (q == COMM_W - 1) { if (n > 0) publish(n - 1, t_prev); }
      if constexpr (LEADER && q == 0) {
        if (n > 0 && lane == 0) asm volatile("global_atomic_add %0, %1, %2, off sc0 sc1" : "+v"(tick) : "v"(ticketp), "v"(1u) : "memory");
      }
      if constexpr (q == 5) {
        if constexpr (LEADER) {
          asm volatile("" : "+v"(tick));
          t_nxt = n == 0 ? (unsigned)(NCOMM + j) : __builtin_amdgcn_readfirstlane(tick) - tbase;
          if (lane == 0) tk[n + 1] = t_nxt + 1u;
        } else {
          unsigned long long t0 = 0;
          while (true) {
            unsigned v = __hip_atomic_load(tk + n + 1, __ATOMIC_RELAXED, __HIP_MEMORY_SCOPE_WORKGROUP);
            v = __builtin_amdgcn_readfirstlane(v);
            if (v) { t_nxt = v - 1u; break; }
            unsigned long long now = __builtin_amdgcn_s_memrealtime();
            if (!t0) t0 = now;
            if (now - t0 > DEADLINE_TICKS) {
              __hip_atomic_store(tmo, call, __ATOMIC_RELAXED, __HIP_MEMORY_SCOPE_AGENT);
              t_nxt = NPIECE; break;
            }
            __builtin_amdgcn_s_sleep(1);
          }
        }
        if (t_nxt < NPIECE)
          asm volatile("buffer_load_dwordx4 %0, %1, %2, %3 offen" : "=v"(dn) : "v"(vo), "s"(rs_x),
                       "s"((t_nxt >> 6) * UNIT_BYTES + (t_nxt & 1u) * (32u * KDIM) + ((t_nxt >> 1) & 31u) * 128u) : "memory");
      }
      st_sys(d, vs, rs[q], dst);
    });
    if (t_nxt >= NPIECE) break;
    asm volatile("s_waitcnt vmcnt(2)" : "+v"(dn) :: "memory");
    t_prev = t_cur; t_cur = t_nxt; d = dn; ++n;
  }
  asm volatile("s_waitcnt vmcnt(0)" ::: "memory");
  if constexpr (PF_U < 8) group_done();
  publish(n, t_cur);
  if constexpr (PF_U < 8) {
    asm volatile("s_waitcnt vmcnt(0)" ::: "memory");
    group_done();
  }
}

__device__ __forceinline__ void comm_role(const Params& p, unsigned char* smem_generic) {
  LAS unsigned char* smem = (LAS unsigned char*)smem_generic;
  const int j = blockIdx.x;
  const int tid = threadIdx.x, lane = tid & 63;
  const int w = __builtin_amdgcn_readfirstlane(tid >> 6);
  unsigned char* myws = p.ws[p.rank];

  STAMP(myws, 0);
  v4i d;
  {
    rsrc_t rs_x = mk_rsrc(p.x, SRC_BYTES);
    const unsigned vo = (unsigned)((w >> 1) * 16 + (lane & 15)) * KDIM + (w & 1) * 64 + (lane >> 4) * 16;
    asm volatile("buffer_load_dwordx4 %0, %1, %2, %3 offen" : "=v"(d) : "v"(vo), "s"(rs_x), "s"((unsigned)((j & 1) * (32 * KDIM) + (j >> 1) * 128)) : "memory");
  }
  unsigned ticket = 0;
  if (lane == 0)
    asm volatile("global_atomic_add %0, %1, %2, off sc0" : "+v"(ticket) : "v"((unsigned*)(myws + CTRL_CALL + (j * 4 + w) * 32)), "v"(1u) : "memory");
  for (int i = tid; i < 520; i += 256) *(LAS unsigned*)(smem + 1024 + i * 4) = 0u;
  if (tid < 8) *(LAS unsigned*)(smem + LDS_MISC + 64 + tid * 4) = 0u;
  asm volatile("s_waitcnt lgkmcnt(0)\n\ts_barrier\n\ts_waitcnt vmcnt(0)" : "+v"(d), "+v"(ticket) :: "memory");
  const unsigned call = __builtin_amdgcn_readfirstlane(ticket) + 1u;
  if (w == 0) comm_wave<true>(p, smem, call, j, w, lane, d);
  else comm_wave<false>(p, smem, call, j, w, lane, d);
}

#define CWN 1
constexpr unsigned PGW = 18;
constexpr unsigned PTICKETS = NPIECE - 2 * PGW + PGW;
constexpr int CB = 128, TK = HD - 1;
template <int PAR> __device__ __forceinline__ void ld_piece(const unsigned (&vo)[4], rsrc_t rs_x, unsigned t) {
  const unsigned so = (t >> 6) * UNIT_BYTES + (t & 1u) * (32u * KDIM) + ((t >> 1) & 31u) * 128u;
  asm volatile("buffer_load_dwordx4 v[%c3:%c4], %0, %1, %2 offen" :: "v"(vo[0]), "s"(rs_x), "s"(so), "i"(CB + 16 * PAR), "i"(CB + 16 * PAR + 3) : "memory");
  asm volatile("buffer_load_dwordx4 v[%c3:%c4], %0, %1, %2 offen" :: "v"(vo[1]), "s"(rs_x), "s"(so), "i"(CB + 16 * PAR + 4), "i"(CB + 16 * PAR + 7) : "memory");
  asm volatile("buffer_load_dwordx4 v[%c3:%c4], %0, %1, %2 offen" :: "v"(vo[2]), "s"(rs_x), "s"(so), "i"(CB + 16 * PAR + 8), "i"(CB + 16 * PAR + 11) : "memory");
  asm volatile("buffer_load_dwordx4 v[%c3:%c4], %0, %1, %2 offen" :: "v"(vo[3]), "s"(rs_x), "s"(so), "i"(CB + 16 * PAR + 12), "i"(CB + 16 * PAR + 15) : "memory");
}
template <int PAR, int K> __device__ __forceinline__ void st_piece(unsigned vs, rsrc_t rs_d, unsigned dst) {
  asm volatile("buffer_store_dwordx4 v[%c3:%c4], %0, %1, %2 offen offset:%c5 sc0 sc1"
               :: "v"(vs), "s"(rs_d), "s"(dst), "i"(CB + 16 * PAR + 4 * K), "i"(CB + 16 * PAR + 4 * K + 3), "i"(K * 1024) : "memory");
}
#define TKPOL "sc0 sc1"
__device__ __forceinline__ void tk_issue(unsigned* ptr) {
  asm volatile("global_atomic_add v%c2, %0, %1, off " TKPOL :: "v"(ptr), "v"(1u), "i"(TK) : "memory");
}
__device__ __forceinline__ void dummy_ld(unsigned* ptr) {
  asm volatile("global_load_dword v%c1, %0, off sc0 sc1" :: "v"(ptr), "i"(HD) : "memory");
}
__device__ __forceinline__ unsigned tk_read() {
  unsigned t;
  asm volatile("s_waitcnt vmcnt(5)\n\tv_readfirstlane_b32 %0, v%c1" : "=s"(t) : "i"(TK) : "memory");
  return t;
}
__device__ __forceinline__ void st_flag(unsigned char* a, unsigned v) {
  asm volatile("global_store_dword %0, %1, off sc0 sc1" :: "v"(a), "v"(v) : "memory");
}
struct C2 {
  unsigned vo[4], vs, call, tbase, t0, t1, tp1, tp2;
  rsrc_t rs_x, rs_d;
  unsigned char* flagb;
  unsigned* ticketp;
  unsigned char* flagown;
  int lane;
};
__device__ __forceinline__ void publish2(C2& c, unsigned t) {
  if (c.lane == 0) st_flag(c.flagb + ((t >> 1) & 3u) * 512u + (t >> 3) * 8u + (t & 1u) * 4u, c.call);
}
template <int PAR> __device__ __forceinline__ bool comm_iter(C2& c) {
  constexpr int PER_IT = 10;
  const bool more = c.t1 < NPIECE;
  if (c.lane == 0) {
    if (more) tk_issue(c.ticketp);
    else dummy_ld(c.ticketp);
  }
  const unsigned dst = c.t0 * 4096u;
  const unsigned tpub = CWN == 1 ? c.tp1 : c.tp2;
  wait_vm<PER_IT * CWN - 1>(); st_piece<PAR, 0>(c.vs, c.rs_d, dst);
  wait_vm<PER_IT * CWN - 1>(); st_piece<PAR, 1>(c.vs, c.rs_d, dst);
  wait_vm<PER_IT * CWN - 1>(); st_piece<PAR, 2>(c.vs, c.rs_d, dst);
  wait_vm<PER_IT * CWN - 2>();
  if (tpub < PF_U * 64u) publish2(c, tpub);
  else if (c.lane == 0) dummy_ld(c.ticketp);
  st_piece<PAR, 3>(c.vs, c.rs_d, dst);
  const unsigned tk = tk_read();
  const unsigned t2 = more ? tk - c.tbase : NPIECE;
  ld_piece<PAR>(c.vo, c.rs_x, t2 < NPIECE ? t2 : 0u);
  c.tp2 = c.tp1; c.tp1 = c.t0; c.t0 = c.t1; c.t1 = t2;
  return c.t0 >= NPIECE;
}
__device__ __forceinline__ void comm_role2(const Params& p) {
  const int tid = threadIdx.x, lane = tid & 63;
  const int w = __builtin_amdgcn_readfirstlane(tid >> 6);
  const unsigned cw = blockIdx.x * 4 + w;
  const int rank = p.rank;
  unsigned char* myws = p.ws[rank];
  STAMP(myws, 0);
  if (cw >= 7 * PGW) return;
  const unsigned q = cw % 7u, gi = cw / 7u;
  unsigned char* pws = p.ws[(rank + 1 + q) & 7];
  C2 c;
  c.lane = lane;
  c.rs_x = mk_rsrc(p.x, SRC_BYTES);
#pragma unroll
  for (int k = 0; k < 4; ++k) c.vo[k] = (unsigned)((k >> 1) * 16 + (lane & 15)) * KDIM + (k & 1) * 64 + (lane >> 4) * 16;
  ld_piece<0>(c.vo, c.rs_x, gi);
  ld_piece<1>(c.vo, c.rs_x, PGW + gi);
  unsigned ticket = 0;
  if (lane == 0)
    asm volatile("global_atomic_add %0, %1, %2, off sc0" : "+v"(ticket) : "v"((unsigned*)(myws + CTRL_CALL + cw * 32)), "v"(1u) : "memory");
  asm volatile("s_waitcnt vmcnt(0)" : "+v"(ticket) :: "memory");
  c.call = __builtin_amdgcn_readfirstlane(ticket) + 1u;
  const unsigned cm = __builtin_amdgcn_readfirstlane(c.call % NSTAGE);
  c.rs_d = mk_rsrc(pws + cm * STAGE_BYTES + (unsigned)rank * SRC_BYTES, SRC_BYTES);
  c.flagb = pws + CTRL_FLAG + rank * 2048;
  c.ticketp = (unsigned*)(myws + CTRL_TICKET + q * 128);
  c.tbase = (c.call - 1u) * PTICKETS - 2u * PGW;
  c.flagown = myws + CTRL_CALL + cw * 32;
  c.vs = lane * 16;
  c.t0 = gi; c.t1 = PGW + gi; c.tp1 = NPIECE; c.tp2 = NPIECE;
  while (true) {
    if (comm_iter<0>(c)) break;
    if (comm_iter<1>(c)) break;
  }
  asm volatile("s_waitcnt vmcnt(0)" ::: "memory");
  if (CWN == 2 && c.tp2 < PF_U * 64u) publish2(c, c.tp2);
  if (c.tp1 < PF_U * 64u) publish2(c, c.tp1);
}

#define UF_END 1
__device__ __forceinline__ void publish_groups(unsigned char* ucnt, unsigned char* uflagb, unsigned call, int lane, unsigned nwg) {
  unsigned old = 0;
  if (lane == 0) asm volatile("global_atomic_add %0, %1, %2, off sc0 sc1\n\ts_waitcnt vmcnt(0)" : "+v"(old) : "v"(ucnt), "v"(1u) : "memory");
  old = __builtin_amdgcn_readfirstlane(old);
  if (old - (call - 1u) * nwg == nwg - 1u && lane < 7) {
#pragma unroll
    for (unsigned u = PF_U; u < 8u; ++u)
      asm volatile("global_store_dwordx2 %0, %1, off sc0 sc1" :: "v"(uflagb + u * 8u), "v"((v2u){call, call}) : "memory");
  }
}

#define T_FAST 512
#define C3_ROT 0
#define NACT 32
__device__ __forceinline__ void wait_vm_dyn(unsigned n) {
  switch (n) {
    case 0: wait_vm<0>(); break;
    case 1: wait_vm<1>(); break;
    case 2: wait_vm<2>(); break;
    case 3: wait_vm<3>(); break;
    case 4: wait_vm<4>(); break;
    case 5: wait_vm<5>(); break;
    case 6: wait_vm<6>(); break;
    default: wait_vm<7>(); break;
  }
}
struct C3 {
  unsigned vo, vs, call, tbase, dst0, t_prev, t_cur, t_nxt, n, pend;
  unsigned char *uflagb, *ucnt;
  rsrc_t rs_x, rs[7];
  unsigned char* flagb;
  unsigned* ticketp;
  GAS unsigned* tmo;
  LAS unsigned *tk, *cnt;
  LAS unsigned char* smem;
  int lane;
};
template <int PAR> __device__ __forceinline__ void ld_chunk(unsigned vo, rsrc_t rs_x, unsigned t) {
  const unsigned so = (t >> 6) * UNIT_BYTES + (t & 1u) * (32u * KDIM) + ((t >> 1) & 31u) * 128u;
  asm volatile("buffer_load_dwordx4 v[%c3:%c4], %0, %1, %2 offen" :: "v"(vo), "s"(rs_x), "s"(so), "i"(CB + 4 * PAR), "i"(CB + 4 * PAR + 3) : "memory");
}
template <int PAR> __device__ __forceinline__ void st_chunk(unsigned vs, rsrc_t rs_d, unsigned dst) {
  asm volatile("buffer_store_dwordx4 v[%c3:%c4], %0, %1, %2 offen sc0 sc1" :: "v"(vs), "s"(rs_d), "s"(dst), "i"(CB + 4 * PAR), "i"(CB + 4 * PAR + 3) : "memory");
}
__device__ __forceinline__ void publish3(C3& c, unsigned n, unsigned t) {
  if (t >= PF_U * 64u) return;
  unsigned old = 0;
  if (c.lane == 0) old = __hip_atomic_fetch_add(c.cnt + n, 1u, __ATOMIC_RELAXED, __HIP_MEMORY_SCOPE_WORKGROUP);
  old = __builtin_amdgcn_readfirstlane(old);
  if (old == 3u) {
    if (t < PF_U * 64u) {
      if (c.lane < 7) st_flag(c.flagb + ((t >> 1) & 3u) * 512u + (t >> 3) * 8u + (t & 1u) * 4u, c.call);
    } else if (!UF_END) {
      if (c.lane == 0)
        asm volatile("global_atomic_add v%c2, %0, %1, off sc0 sc1" :: "v"(c.ucnt + (t >> 6) * 128u), "v"(1u), "i"(FD) : "memory");
      c.pend = (t >> 6) + 1u;
    }
  }
}
__device__ __forceinline__ void group_done3(C3& c) {
  if (c.pend) {
    unsigned o;
    asm volatile("v_readfirstlane_b32 %0, v%c1" : "=s"(o) : "i"(FD) : "memory");
    if (o - (c.call - 1u) * 64u == 63u && c.lane < 7)
      asm volatile("global_store_dwordx2 %0, %1, off sc0 sc1" :: "v"(c.uflagb + (c.pend - 1u) * 8u), "v"((v2u){c.call, c.call}) : "memory");
    c.pend = 0;
  }
}
template <bool LEADER, int PAR> __device__ __forceinline__ bool comm3_round(C3& c) {
  constexpr int EX = LEADER ? 2 : 1, W = COMM_W;
  const bool more = c.t_nxt < NPIECE;
  unsigned t2 = NPIECE;
  auto ticket_done = [&]() {
    unsigned t;
    asm volatile("v_readfirstlane_b32 %0, v%c1" : "=s"(t) : "i"(TK) : "memory");
    t -= c.tbase;
    t2 = more && t < NPIECE ? t : NPIECE;
    if (c.lane == 0) c.tk[c.n + 2] = t2 + 1u;
  };
  if constexpr (LEADER) {
    if (c.lane == 0) {
      if (more) tk_issue(c.ticketp);
      else dummy_ld(c.ticketp);
    }
  }
  ld_chunk<PAR ^ 1>(c.vo, c.rs_x, more ? c.t_nxt : 0u);
  const unsigned dst = c.dst0 + c.t_cur * 4096u;
  if (c.t_cur < (unsigned)T_FAST) {
    sfor<7>([&](auto qc) {
      constexpr int q = decltype(qc)::value;
      wait_vm<(q < W ? W - 1 + EX : W - 1)>();
      if constexpr (PF_U < 8 && !UF_END && q == (2 * W - 2 <= 6 ? 2 * W - 2 : W - 1)) group_done3(c);
      if constexpr (q == W - 1) { if (c.n > 0) publish3(c, c.n - 1, c.t_prev); }
      if constexpr (LEADER && q == W) ticket_done();
      st_chunk<PAR>(c.vs, c.rs[q], dst);
    });
    if constexpr (LEADER && W == 7) { wait_vm<7>(); ticket_done(); }
  } else {
    sfor<7>([&](auto qc) { st_chunk<PAR>(c.vs, c.rs[decltype(qc)::value], dst); });
    wait_vm<7 + EX>();
    if constexpr (PF_U < 8 && !UF_END) group_done3(c);
    if (c.n > 0) publish3(c, c.n - 1, c.t_prev);
    if constexpr (LEADER) { wait_vm<8>(); ticket_done(); }
  }
  if (!more) return true;
  if constexpr (!LEADER) {
    unsigned long long t0 = 0;
    while (true) {
      unsigned v = __hip_atomic_load(c.tk + c.n + 2, __ATOMIC_RELAXED, __HIP_MEMORY_SCOPE_WORKGROUP);
      v = __builtin_amdgcn_readfirstlane(v);
      if (v) { t2 = v - 1u; break; }
      unsigned long long now = __builtin_amdgcn_s_memrealtime();
      if (!t0) t0 = now;
      if (now - t0 > DEADLINE_TICKS) {
        __hip_atomic_store(c.tmo, c.call, __ATOMIC_RELAXED, __HIP_MEMORY_SCOPE_AGENT);
        t2 = NPIECE; break;
      }
      __builtin_amdgcn_s_sleep(1);
    }
  }
  wait_vm<7>();
  c.t_prev = c.t_cur; c.t_cur = c.t_nxt; c.t_nxt = t2; c.n += 1u;
  return false;
}
template <bool LEADER> __device__ __forceinline__ void comm3_wave(C3& c) {
  while (true) {
    if (comm3_round<LEADER, 0>(c)) break;
    if (comm3_round<LEADER, 1>(c)) break;
  }
  wait_vm<0>();
  if constexpr (PF_U < 8 && !UF_END) group_done3(c);
  publish3(c, c.n, c.t_cur);
  if constexpr (PF_U < 8 && !UF_END) { wait_vm<0>(); group_done3(c); }
  if constexpr (PF_U < 8 && UF_END) {
    unsigned old = 0;
    if (c.lane == 0) old = __hip_atomic_fetch_add(c.cnt + 519, 1u, __ATOMIC_RELAXED, __HIP_MEMORY_SCOPE_WORKGROUP);
    if (__builtin_amdgcn_readfirstlane(old) == 3u) publish_groups(c.ucnt, c.uflagb, c.call, c.lane, NACT);
  }
}
__device__ __forceinline__ void comm_role3(const Params& p, unsigned char* smem_generic) {
  LAS unsigned char* smem = (LAS unsigned char*)smem_generic;
  const int j = blockIdx.x;
  const int tid = threadIdx.x, lane = tid & 63;
  const int w = __builtin_amdgcn_readfirstlane(tid >> 6);
  const int rank = p.rank;
  unsigned char* myws = p.ws[rank];
  STAMP(myws, 0);
  asm volatile("" ::: "v255");
  if (j >= NACT) return;
  C3 c;
  c.lane = lane;
  c.smem = smem;
  c.rs_x = mk_rsrc(p.x, SRC_BYTES);
  c.vo = (unsigned)((w >> 1) * 16 + (lane & 15)) * KDIM + (w & 1) * 64 + (lane >> 4) * 16;
  ld_chunk<0>(c.vo, c.rs_x, (unsigned)j);
  unsigned ticket = 0;
  if (lane == 0)
    asm volatile("global_atomic_add %0, %1, %2, off sc0" : "+v"(ticket) : "v"((unsigned*)(myws + CTRL_CALL + (j * 4 + w) * 32)), "v"(1u) : "memory");
  for (int i = tid; i < 2 * 520; i += 256) *(LAS unsigned*)(smem + 1024 + i * 4) = 0u;
  c.tk = (LAS unsigned*)(smem + 1024);
  c.cnt = (LAS unsigned*)(smem + 1024 + 520 * 4);
#pragma unroll
  for (int q = 0; q < 7; ++q) {
    const int pq = (rank + 1 + (C3_FIX ? q : (q + (C3_ROT ? j : j * 4 + w)) % 7)) & 7;
    c.rs[q] = mk_rsrc(p.ws[pq], NSTAGE * STAGE_BYTES);
  }
  c.flagb = p.ws[(rank + 1 + (lane < 7 ? lane : 0)) & 7] + CTRL_FLAG + rank * 2048;
  c.ticketp = (unsigned*)(myws + CTRL_TICKET);
  c.tmo = (GAS unsigned*)(myws + CTRL_TMO);
  c.vs = lane * 16;
  asm volatile("s_waitcnt lgkmcnt(0)\n\ts_barrier\n\ts_waitcnt vmcnt(0)" : "+v"(ticket) :: "memory");
  c.call = __builtin_amdgcn_readfirstlane(ticket) + 1u;
  c.tbase = (c.call - 1u) * (NPIECE - NACT) - 2u * NACT;
  c.pend = 0;
  c.uflagb = p.ws[(rank + 1 + (lane < 7 ? lane : 0)) & 7] + CTRL_UFLAG + rank * 64;
  c.ucnt = myws + CTRL_UCNT;
  c.dst0 = __builtin_amdgcn_readfirstlane(c.call % NSTAGE) * STAGE_BYTES + (unsigned)rank * SRC_BYTES + (unsigned)w * 1024u;
  c.t_prev = 0; c.t_cur = (unsigned)j; c.t_nxt = (unsigned)(NACT + j); c.n = 0;
  if (w == 0) comm3_wave<true>(c);
  else comm3_wave<false>(c);
}

constexpr unsigned NTK = NPIECE * 7u / 4u, NTK_U = NTK / 8u;
struct C4 {
  unsigned vo[4], vs, call, tbase, T_cur, T_nxt, t_prev, q_prev, n, pend, tk_out;
  unsigned blo, bhi;
  unsigned char *uflagb, *ucnt, *flagb;
  rsrc_t rs_x;
  unsigned* ticketp;
  GAS unsigned* tmo;
  LAS unsigned *tk, *cnt;
  LAS unsigned char* smem;
  unsigned long long* dbg;
  int lane, w;
};
__device__ __forceinline__ void group_done4(C4& c) {
  if (c.pend) {
    unsigned o;
    asm volatile("v_readfirstlane_b32 %0, v%c1" : "=s"(o) : "i"(FD) : "memory");
    if (o - (c.call - 1u) * NTK_U == NTK_U - 1u && c.lane < 7)
      asm volatile("global_store_dwordx2 %0, %1, off sc0 sc1" :: "v"(c.uflagb + (c.pend - 1u) * 8u), "v"((v2u){c.call, c.call}) : "memory");
    c.pend = 0;
  }
}
__device__ __forceinline__ void publish4(C4& c, unsigned n, unsigned t, unsigned q) {
  if (t < PF_U * 64u) {
    if (c.lane == (int)q) st_flag(c.flagb + ((t >> 1) & 3u) * 512u + (t >> 3) * 8u + (t & 1u) * 4u, c.call);
  } else {
    unsigned old = 0;
    if (c.lane == 0) old = __hip_atomic_fetch_add(c.cnt + n, 1u, __ATOMIC_RELAXED, __HIP_MEMORY_SCOPE_WORKGROUP);
    old = __builtin_amdgcn_readfirstlane(old);
    if (old == 3u) {
      if (c.lane == 0)
        asm volatile("global_atomic_add v%c2, %0, %1, off sc0 sc1" :: "v"(c.ucnt + (t >> 6) * 128u), "v"(1u), "i"(FD) : "memory");
      c.pend = (t >> 6) + 1u;
    }
  }
}
template <int PAR, int X> __device__ __forceinline__ void comm4_stores(C4& c, rsrc_t rs_d, unsigned dst) {
  constexpr int W = COMM_W;
  st_piece<PAR, 0>(c.vs, rs_d, dst);
  wait_vm<W - 1 + X>();
  st_piece<PAR, 1>(c.vs, rs_d, dst);
  wait_vm<W - 1 + X>();
  st_piece<PAR, 2>(c.vs, rs_d, dst);
  wait_vm<W - 1 + X>();
  group_done4(c);
  if (c.n > 0) publish4(c, c.n - 1, c.t_prev, c.q_prev);
  st_piece<PAR, 3>(c.vs, rs_d, dst);
}
template <bool LEADER, int PAR> __device__ __forceinline__ bool comm4_round(C4& c) {
  static_assert(COMM_MODE != 4 || COMM_W == 4, "window");
  wait_vm<3>();
  unsigned T2 = NTK;
  bool tk_now = false;
  if constexpr (LEADER) {
    if (c.n == 0) T2 = 2u * NACT + blockIdx.x;
    else if (c.tk_out) {
      unsigned t;
      asm volatile("v_readfirstlane_b32 %0, v%c1" : "=s"(t) : "i"(TK) : "memory");
      t -= c.tbase;
      T2 = t < NTK ? t : NTK;
    }
    if (c.lane == 0) c.tk[c.n + 2] = T2 + 1u;
    tk_now = T2 < NTK;
    if (tk_now && c.lane == 0) tk_issue(c.ticketp);
    c.tk_out = tk_now ? 1u : 0u;
  }
  const bool more = c.T_nxt < NTK;
  const unsigned tau = c.T_cur * 4u + (unsigned)c.w, t = (tau * 9363u) >> 16, q = tau - 7u * t;
  const unsigned lo = __builtin_amdgcn_readlane(c.blo, q), hi = __builtin_amdgcn_readlane(c.bhi, q);
  const rsrc_t rs_d = mk_rsrc((void*)(((unsigned long long)hi << 32) | lo), SRC_BYTES);
  if (more) {
    const unsigned taun = c.T_nxt * 4u + (unsigned)c.w;
    ld_piece<PAR ^ 1>(c.vo, c.rs_x, (taun * 9363u) >> 16);
    if (LEADER && tk_now) comm4_stores<PAR, 5>(c, rs_d, t * 4096u);
    else comm4_stores<PAR, 4>(c, rs_d, t * 4096u);
  } else {
    comm4_stores<PAR, 0>(c, rs_d, t * 4096u);
  }
  c.t_prev = t; c.q_prev = q;
  if (!more) return true;
  if constexpr (!LEADER) {
    unsigned long long t0 = 0;
    while (true) {
      unsigned v = __hip_atomic_load(c.tk + c.n + 2, __ATOMIC_RELAXED, __HIP_MEMORY_SCOPE_WORKGROUP);
      v = __builtin_amdgcn_readfirstlane(v);
      if (v) { T2 = v - 1u; break; }
      unsigned long long now = __builtin_amdgcn_s_memrealtime();
      if (!t0) t0 = now;
      if (now - t0 > DEADLINE_TICKS) {
        __hip_atomic_store(c.tmo, c.call, __ATOMIC_RELAXED, __HIP_MEMORY_SCOPE_AGENT);
        T2 = NTK; break;
      }
      __builtin_amdgcn_s_sleep(1);
    }
  }
  c.T_cur = c.T_nxt; c.T_nxt = T2; c.n += 1u;
  return false;
}
template <bool LEADER> __device__ __forceinline__ void comm4_wave(C4& c) {
  while (true) {
    if (comm4_round<LEADER, 0>(c)) break;
    if (comm4_round<LEADER, 1>(c)) break;
  }
  wait_vm<0>();
  group_done4(c);
  publish4(c, c.n, c.t_prev, c.q_prev);
  wait_vm<0>();
  group_done4(c);
}
__device__ __forceinline__ void comm_role4(const Params& p, unsigned char* smem_generic) {
  LAS unsigned char* smem = (LAS unsigned char*)smem_generic;
  const int j = blockIdx.x;
  const int tid = threadIdx.x, lane = tid & 63;
  const int w = __builtin_amdgcn_readfirstlane(tid >> 6);
  const int rank = p.rank;
  unsigned char* myws = p.ws[rank];
  STAMP(myws, 0);
  asm volatile("" ::: "v255");
  C4 c;
  c.lane = lane; c.w = w;
  c.smem = smem;
  c.rs_x = mk_rsrc(p.x, SRC_BYTES);
#pragma unroll
  for (int k = 0; k < 4; ++k) c.vo[k] = (unsigned)((k >> 1) * 16 + (lane & 15)) * KDIM + (k & 1) * 64 + (lane >> 4) * 16;
  ld_piece<0>(c.vo, c.rs_x, (((unsigned)j * 4u + (unsigned)w) * 9363u) >> 16);
  unsigned ticket = 0;
  if (lane == 0)
    asm volatile("global_atomic_add %0, %1, %2, off sc0" : "+v"(ticket) : "v"((unsigned*)(myws + CTRL_CALL + (j * 4 + w) * 32)), "v"(1u) : "memory");
  for (int i = tid; i < 2048; i += 256) *(LAS unsigned*)(smem + 32768 + i * 4) = 0u;
  c.dbg = nullptr;
  c.tk = (LAS unsigned*)(smem + 32768);
  c.cnt = (LAS unsigned*)(smem + 32768 + 4096);
  unsigned char* pw = p.ws[(rank + 1 + (lane < 7 ? lane : 0)) & 7];
  c.flagb = pw + CTRL_FLAG + rank * 2048;
  c.uflagb = pw + CTRL_UFLAG + rank * 64;
  c.ucnt = myws + CTRL_UCNT;
  c.ticketp = (unsigned*)(myws + CTRL_TICKET);
  c.tmo = (GAS unsigned*)(myws + CTRL_TMO);
  c.vs = lane * 16;
  asm volatile("s_waitcnt lgkmcnt(0)\n\ts_barrier\n\ts_waitcnt vmcnt(0)" : "+v"(ticket) :: "memory");
  c.call = __builtin_amdgcn_readfirstlane(ticket) + 1u;
  c.tbase = (c.call - 1u) * (NTK - 2u * NACT) - 3u * NACT;
  c.pend = 0; c.tk_out = 0;
  {
    const unsigned long long a = (unsigned long long)(pw + (c.call % NSTAGE) * STAGE_BYTES + (unsigned)rank * SRC_BYTES);
    c.blo = (unsigned)a; c.bhi = (unsigned)(a >> 32);
  }
  c.t_prev = 0; c.q_prev = 0; c.T_cur = (unsigned)j; c.T_nxt = (unsigned)(NACT + j); c.n = 0;
  if (w == 0) comm4_wave<true>(c);
  else comm4_wave<false>(c);
}

#define C6_W 5
#define C6_LD 7
constexpr unsigned NTASK = NPIECE * 7u, NTK6 = (NTASK + 2u) / 3u, C6_LEAD = C6_LD;
constexpr unsigned TK6_PER_CALL = NTK6 - C6_LEAD * NCOMM + 2u * NCOMM;
struct C6 {
  unsigned vo[4], vs, call, T0, T1, T2, n, sig;
  unsigned blo, bhi;
  rsrc_t rs_x;
  GAS unsigned* tmo;
  LAS unsigned *tk, *cnt;
  int lane, w;
};
template <int B, int K> __device__ __forceinline__ void ld_chunk6(unsigned vo, rsrc_t rs_x, unsigned so) {
  asm volatile("buffer_load_dwordx4 v[%c3:%c4], %0, %1, %2 offen" :: "v"(vo), "s"(rs_x), "s"(so), "i"(CB + 16 * B + 4 * K), "i"(CB + 16 * B + 4 * K + 3) : "memory");
}
template <int B> __device__ __forceinline__ void ld_piece6(const unsigned (&vo)[4], rsrc_t rs_x, unsigned t) {
  const unsigned so = (t >> 6) * UNIT_BYTES + (t & 1u) * (32u * KDIM) + ((t >> 1) & 31u) * 128u;
  ld_chunk6<B, 0>(vo[0], rs_x, so);
  ld_chunk6<B, 1>(vo[1], rs_x, so);
  ld_chunk6<B, 2>(vo[2], rs_x, so);
  ld_chunk6<B, 3>(vo[3], rs_x, so);
}
template <int B, int K> __device__ __forceinline__ void st_chunk6(unsigned vs, rsrc_t rs_d, unsigned dst) {
  asm volatile("buffer_store_dwordx4 v[%c3:%c4], %0, %1, %2 offen offset:%c5 sc0 sc1"
               :: "v"(vs), "s"(rs_d), "s"(dst), "i"(CB + 16 * B + 4 * K), "i"(CB + 16 * B + 4 * K + 3), "i"(K * 1024) : "memory");
}
constexpr int c6_cnt(int k) { return C6_W - 1 + 4 * (1 + (C6_W - k - 1) / 4 - (C6_W <= k ? 1 : 0)); }
__device__ __forceinline__ unsigned lds_wait_nz(LAS unsigned* a, unsigned want, GAS unsigned* tmo, unsigned call) {
  unsigned long long t0 = 0;
  while (true) {
    unsigned v = __hip_atomic_load(a, __ATOMIC_RELAXED, __HIP_MEMORY_SCOPE_WORKGROUP);
    v = __builtin_amdgcn_readfirstlane(v);
    if (v >= want) return v;
    unsigned long long now = __builtin_amdgcn_s_memrealtime();
    if (!t0) t0 = now;
    if (now - t0 > DEADLINE_TICKS) {
      __hip_atomic_store(tmo, call, __ATOMIC_RELAXED, __HIP_MEMORY_SCOPE_AGENT);
      return 0;
    }
    __builtin_amdgcn_s_sleep(1);
  }
}
__device__ __forceinline__ unsigned c6_task(unsigned T, int i) { const unsigned tau = T * 3u + (unsigned)i; return tau < NTASK ? tau : NTASK - 1u; }
template <int B> __device__ __forceinline__ bool comm6_round(C6& c) {
  constexpr int KS = (3 + C6_W) % 4, DR = (3 + C6_W) / 4;
  const unsigned tau = c6_task(c.T0, c.w - 1), t = (tau * 9363u) >> 16, q = tau - 7u * t;
  const unsigned lo = __builtin_amdgcn_readlane(c.blo, q), hi = __builtin_amdgcn_readlane(c.bhi, q);
  const rsrc_t rs_d = mk_rsrc((void*)(((unsigned long long)hi << 32) | lo), SRC_BYTES);
  ld_piece6<(B + 2) % 3>(c.vo, c.rs_x, (c6_task(c.T2 < NTK6 ? c.T2 : c.T0, c.w - 1) * 9363u) >> 16);
  sfor<4>([&](auto kc) {
    constexpr int k = decltype(kc)::value;
    wait_vm<c6_cnt(k)>();
    if constexpr (k == KS) {
      if (c.n >= (unsigned)DR) {
        if (c.lane == 0) __hip_atomic_fetch_add(c.cnt + c.sig, 1u, __ATOMIC_RELAXED, __HIP_MEMORY_SCOPE_WORKGROUP);
        c.sig += 1u;
      }
    }
    st_chunk6<B, k>(c.vs, rs_d, t * 4096u);
  });
  if (c.T1 >= NTK6) return true;
  unsigned T3 = NTK6;
  if (c.T2 < NTK6) {
    const unsigned v = lds_wait_nz(c.tk + c.n + 3, 1u, c.tmo, c.call);
    T3 = v ? v - 1u : NTK6;
  }
  c.T0 = c.T1; c.T1 = c.T2; c.T2 = T3; c.n += 1u;
  return false;
}
__device__ __forceinline__ void comm_role6(const Params& p, unsigned char* smem_generic) {
  LAS unsigned char* smem = (LAS unsigned char*)smem_generic;
  const int j = blockIdx.x;
  const int tid = threadIdx.x, lane = tid & 63;
  const int w = __builtin_amdgcn_readfirstlane(tid >> 6);
  const int rank = p.rank;
  unsigned char* myws = p.ws[rank];
  STAMP(myws, 0);
  asm volatile("" ::: "v255");
  C6 c;
  c.lane = lane; c.w = w;
  c.rs_x = mk_rsrc(p.x, SRC_BYTES);
#pragma unroll
  for (int k = 0; k < 4; ++k) c.vo[k] = (unsigned)((k >> 1) * 16 + (lane & 15)) * KDIM + (k & 1) * 64 + (lane >> 4) * 16;
  if (w > 0) {
    ld_piece6<0>(c.vo, c.rs_x, (c6_task((unsigned)j, w - 1) * 9363u) >> 16);
    ld_piece6<1>(c.vo, c.rs_x, (c6_task((unsigned)(NCOMM + j), w - 1) * 9363u) >> 16);
  }
  unsigned ticket = 0;
  if (lane == 0)
    asm volatile("global_atomic_add %0, %1, %2, off sc0" : "+v"(ticket) : "v"((unsigned*)(myws + CTRL_CALL + (j * 4 + w) * 32)), "v"(1u) : "memory");
  LAS unsigned* tk = (LAS unsigned*)(smem + 32768);
  LAS unsigned* cnt = (LAS unsigned*)(smem + 32768 + 8192);
  for (int i = tid; i < 4096; i += 256) tk[i] = i < (int)C6_LEAD ? (unsigned)(i * NCOMM + j + 1) : 0u;
  unsigned char* pw = p.ws[(rank + 1 + (lane < 7 ? lane : 0)) & 7];
  GAS unsigned* tmo = (GAS unsigned*)(myws + CTRL_TMO);
  asm volatile("s_waitcnt lgkmcnt(0)\n\ts_barrier\n\ts_waitcnt vmcnt(0)" : "+v"(ticket) :: "memory");
  const unsigned call = __builtin_amdgcn_readfirstlane(ticket) + 1u;
  if (w > 0) {
    c.call = call; c.tmo = tmo; c.tk = tk; c.cnt = cnt;
    c.vs = lane * 16;
    const unsigned long long a = (unsigned long long)(pw + (call % NSTAGE) * STAGE_BYTES + (unsigned)rank * SRC_BYTES);
    c.blo = (unsigned)a; c.bhi = (unsigned)(a >> 32);
    c.T0 = (unsigned)j; c.T1 = (unsigned)(NCOMM + j); c.T2 = (unsigned)(2 * NCOMM + j); c.n = 0; c.sig = 0;
    while (true) {
      if (comm6_round<0>(c)) break;
      if (comm6_round<1>(c)) break;
      if (comm6_round<2>(c)) break;
    }
    wait_vm<0>();
    for (; c.sig <= c.n; ++c.sig)
      if (lane == 0) __hip_atomic_fetch_add(cnt + c.sig, 1u, __ATOMIC_RELAXED, __HIP_MEMORY_SCOPE_WORKGROUP);
  } else {
    unsigned char* flagb = pw + CTRL_FLAG + rank * 2048;
    unsigned char* uflagb = pw + CTRL_UFLAG + rank * 64;
    unsigned char* ucnt = myws + CTRL_UCNT;
    unsigned* ticketp = (unsigned*)(myws + CTRL_TICKET);
    const unsigned tbase = (call - 1u) * TK6_PER_CALL - C6_LEAD * NCOMM;
    bool drawing = true;
    unsigned young = 0;
    for (unsigned i = 0;; ++i) {
      const unsigned T = __builtin_amdgcn_readfirstlane(__hip_atomic_load(tk + i, __ATOMIC_RELAXED, __HIP_MEMORY_SCOPE_WORKGROUP)) - 1u;
      if (T >= NTK6) break;
      if (lds_wait_nz(cnt + i, 3u, tmo, call) == 0u) break;
      unsigned nf = 0;
      sfor<3>([&](auto ic) {
        const unsigned tau = T * 3u + decltype(ic)::value;
        const unsigned t = (tau * 9363u) >> 16, q = tau - 7u * t;
        if (tau < NTASK && t < PF_U * 64u) {
          if (lane == (int)q) st_flag(flagb + ((t >> 1) & 3u) * 512u + (t >> 3) * 8u + (t & 1u) * 4u, call);
          nf += 1u;
        }
      });
      if (drawing) {
        if (i >= 2u) {
          wait_vm_dyn(young + nf);
          unsigned v;
          if (i & 1u) asm volatile("v_readfirstlane_b32 %0, v%c1" : "=s"(v) : "i"(HD) : "memory");
          else asm volatile("v_readfirstlane_b32 %0, v%c1" : "=s"(v) : "i"(TK) : "memory");
          v -= tbase;
          drawing = v < NTK6;
          if (lane == 0) tk[i - 2u + C6_LEAD] = (drawing ? v : NTK6) + 1u;
        }
        if (drawing && lane == 0) {
          if (i & 1u) asm volatile("global_atomic_add v%c2, %0, %1, off sc0 sc1" :: "v"(ticketp), "v"(1u), "i"(HD) : "memory");
          else asm volatile("global_atomic_add v%c2, %0, %1, off sc0 sc1" :: "v"(ticketp), "v"(1u), "i"(TK) : "memory");
        }
        young = nf + 1u;
      }
    }
    wait_vm<0>();
    if constexpr (PF_U < 8) publish_groups(ucnt, uflagb, call, lane, NCOMM);
  }
}

constexpr unsigned C7_STATIC = PF_U * 64u / NCOMM < 2u ? 2u : PF_U * 64u / NCOMM;
constexpr unsigned C7_TK = NPIECE - C7_STATIC * NCOMM + NCOMM;
struct C7 {
  unsigned vo, vs, call, tbase, dst0, t_prev, t_cur, t_nxt, n;
  rsrc_t rs_x, rs[7];
  unsigned char* flagb;
  unsigned* ticketp;
  LAS unsigned* cnt;
  LAS unsigned char* smem;
  unsigned long long* dbg;
  int lane, w;
};
__device__ __forceinline__ void publish7(C7& c, unsigned n, unsigned t) {
  if (t < PF_U * 64u) {
    unsigned old = 0;
    if (c.lane == 0) old = __hip_atomic_fetch_add(c.cnt + n, 1u, __ATOMIC_RELAXED, __HIP_MEMORY_SCOPE_WORKGROUP);
    old = __builtin_amdgcn_readfirstlane(old);
    if (old == 3u && c.lane < 7) st_flag(c.flagb + ((t >> 1) & 3u) * 512u + (t >> 3) * 8u + (t & 1u) * 4u, c.call);
  }
}
template <bool TKT, int PAR> __device__ __forceinline__ unsigned comm7_stores(C7& c) {
  constexpr int EX = TKT ? 2 : 1, W = COMM_W;
  unsigned t2 = NPIECE;
  auto ticket_done = [&]() {
    unsigned t;
    asm volatile("v_readfirstlane_b32 %0, v%c1" : "=s"(t) : "i"(TK) : "memory");
    t -= c.tbase;
    t2 = t < NPIECE ? t : NPIECE;
  };
  if constexpr (TKT) { if (c.lane == 0) tk_issue(c.ticketp); }
  ld_chunk<PAR ^ 1>(c.vo, c.rs_x, c.t_nxt < NPIECE ? c.t_nxt : 0u);
  const unsigned dst = c.dst0 + c.t_cur * 4096u;
  const unsigned nq = c.t_cur < NPIECE - 32u ? 7u : 7u - (((c.t_cur - (NPIECE - 32u)) * 7u) >> 5);
  if (nq < 7u) {
    sfor<6>([&](auto qc) {
      constexpr int q = decltype(qc)::value;
      if ((unsigned)q < nq) {
        wait_vm<(q < W ? W - 1 + EX : W - 1)>();
        st_chunk<PAR>(c.vs, c.rs[q], dst);
      }
    });
    wait_vm_dyn(nq);
    if (c.n > 0) publish7(c, c.n - 1, c.t_prev);
    if constexpr (TKT) ticket_done();
  } else {
    sfor<7>([&](auto qc) {
      constexpr int q = decltype(qc)::value;
      wait_vm<(q < W ? W - 1 + EX : W - 1)>();
      if constexpr (q == W - 1) { if (c.n > 0) publish7(c, c.n - 1, c.t_prev); }
      if constexpr (TKT && q == W) ticket_done();
      st_chunk<PAR>(c.vs, c.rs[q], dst);
    });
    wait_vm<7>();
  }
  return t2;
}
template <int PAR> __device__ __forceinline__ bool comm7_round(C7& c) {
  const bool more = c.t_nxt < NPIECE;
  unsigned t2;
  if (more && c.n + 2u >= C7_STATIC) t2 = comm7_stores<true, PAR>(c);
  else {
    comm7_stores<false, PAR>(c);
    t2 = more ? (c.n + 2u) * NCOMM + blockIdx.x : NPIECE;
  }
  if (!more) return true;
  c.t_prev = c.t_cur; c.t_cur = c.t_nxt; c.t_nxt = t2; c.n += 1u;
  return false;
}
__device__ __forceinline__ void comm_role7(const Params& p, unsigned char* smem_generic) {
  static_assert(COMM_MODE != 7 || (COMM_W <= 6 && C3_FIX), "window");
  LAS unsigned char* smem = (LAS unsigned char*)smem_generic;
  const int j = blockIdx.x;
  const int tid = threadIdx.x, lane = tid & 63;
  const int w = __builtin_amdgcn_readfirstlane(tid >> 6);
  const int rank = p.rank;
  unsigned char* myws = p.ws[rank];
  STAMP(myws, 0);
  asm volatile("" ::: "v255");
  C7 c;
  c.lane = lane; c.w = w;
  c.smem = smem;
  c.rs_x = mk_rsrc(p.x, SRC_BYTES);
  c.vo = (unsigned)((w >> 1) * 16 + (lane & 15)) * KDIM + (w & 1) * 64 + (lane >> 4) * 16;
  const unsigned ia = (unsigned)(j + 8 * w) & 31u, ta = NPIECE - 32u + ia, ca = (ia * 7u) >> 5;
  ld_chunk<2>(c.vo, c.rs_x, ta);
  ld_chunk<0>(c.vo, c.rs_x, (unsigned)j);
  unsigned ticket = 0;
  if (lane == 0)
    asm volatile("global_atomic_add %0, %1, %2, off sc0" : "+v"(ticket) : "v"((unsigned*)(myws + CTRL_CALL + (j * 4 + w) * 32)), "v"(1u) : "memory");
  if (tid < 64) *(LAS unsigned*)(smem + 1024 + tid * 4) = 0u;
  c.dbg = nullptr;
  c.cnt = (LAS unsigned*)(smem + 1024);
#pragma unroll
  for (int q = 0; q < 7; ++q) c.rs[q] = mk_rsrc(p.ws[(rank + 1 + q) & 7], NSTAGE * STAGE_BYTES);
  unsigned char* pw = p.ws[(rank + 1 + (lane < 7 ? lane : 0)) & 7];
  c.flagb = pw + CTRL_FLAG + rank * 2048;
  c.ticketp = (unsigned*)(myws + CTRL_TICKET + w * 128);
  c.vs = lane * 16;
  asm volatile("s_waitcnt lgkmcnt(0)\n\ts_barrier\n\ts_waitcnt vmcnt(0)" : "+v"(ticket) :: "memory");
  c.call = __builtin_amdgcn_readfirstlane(ticket) + 1u;
  c.tbase = (c.call - 1u) * C7_TK - C7_STATIC * NCOMM;
  c.dst0 = __builtin_amdgcn_readfirstlane(c.call % NSTAGE) * STAGE_BYTES + (unsigned)rank * SRC_BYTES + (unsigned)w * 1024u;
  c.t_prev = 0; c.t_cur = (unsigned)j; c.t_nxt = (unsigned)(NCOMM + j); c.n = 0;
  sfor<6>([&](auto qc) {
    constexpr int q = decltype(qc)::value + 1;
    if ((unsigned)q >= 7u - ca) {
      wait_vm<COMM_W - 1>();
      st_chunk<2>(c.vs, c.rs[q], c.dst0 + ta * 4096u);
    }
  });
  while (true) {
    if (comm7_round<0>(c)) break;
    if (comm7_round<1>(c)) break;
  }
  wait_vm<0>();
  publish7(c, c.n, c.t_cur);
  if constexpr (PF_U < 8) {
    unsigned old = 0;
    if (lane == 0) old = __hip_atomic_fetch_add(c.cnt + 63, 1u, __ATOMIC_RELAXED, __HIP_MEMORY_SCOPE_WORKGROUP);
    if (__builtin_amdgcn_readfirstlane(old) == 3u) publish_groups(myws + CTRL_UCNT, pw + CTRL_UFLAG + rank * 64, c.call, lane, NCOMM);
  }
}

__global__ void __launch_bounds__(256, 1) fused_ag_gemm(Params p) {
  extern __shared__ __attribute__((aligned(16))) unsigned char smem_dyn[];
  if (blockIdx.x < NCOMM) comm_role3(p, smem_dyn);
  else gemm_role(p, smem_dyn);
}

}

extern "C" void kernel_launch(int rank, int nranks, ncclComm_t comm, hipStream_t stream,
                              void* const* d_in, const int* in_sizes, int n_in,
                              void* d_out, int out_size,
                              const int* in_dtypes, int out_dtype,
                              void* const* d_out_all,
                              void* const* d_ws_all, size_t ws_bytes) {
  if (nranks != NRANKS || n_in < 4 || ws_bytes < WS_NEEDED || (unsigned)rank >= (unsigned)NRANKS) {
    fprintf(stderr, "unsupported configuration\n");
    return;
  }
  static bool attr_done = false;
  if (!attr_done) {
    HIP_CHECK(hipFuncSetAttribute((const void*)fused_ag_gemm, hipFuncAttributeMaxDynamicSharedMemorySize, (int)LDS_BYTES));
    attr_done = true;
  }
  Params p;
  p.x = (const unsigned char*)d_in[0];
  p.w = (const unsigned char*)d_in[1];
  p.sx = (const float*)d_in[2];
  p.sw = (const float*)d_in[3];
  p.out = (float*)d_out;
  for (int i = 0; i < NRANKS; ++i) p.ws[i] = (unsigned char*)d_ws_all[i];
  p.rank = rank;
  hipLaunchKernelGGL(fused_ag_gemm, dim3(256), dim3(256), LDS_BYTES, stream, p);
  HIP_CHECK(hipGetLastError());
}
```

```cpp
#include <hip/hip_runtime.h>
#include <rccl/rccl.h>
#include <cstdio>
#include <utility>

#define HIP_CHECK(expr) do { hipError_t _e = (expr); if (_e != hipSuccess) { \
    fprintf(stderr, "HIP error %s at %s:%d\n", hipGetErrorString(_e), __FILE__, __LINE__); \
    return; } } while (0)

namespace {

typedef int v8i __attribute__((ext_vector_type(8)));
typedef int v4i __attribute__((ext_vector_type(4)));
typedef int v2i __attribute__((ext_vector_type(2)));
typedef unsigned v2u __attribute__((ext_vector_type(2)));
typedef float v4f __attribute__((ext_vector_type(4)));
#define LAS __attribute__((address_space(3)))
#define GAS __attribute__((address_space(1)))
#define CAS __attribute__((address_space(4)))
typedef __amdgpu_buffer_rsrc_t rsrc_t;

constexpr int NRANKS = 8;
constexpr int KDIM = 4096, NFULL = 16384, NLOC = 2048, MLOC = 512;
constexpr unsigned UNIT_BYTES = 64u * KDIM;
constexpr unsigned SRC_BYTES = (unsigned)MLOC * KDIM;
constexpr unsigned STAGE_BYTES = NRANKS * SRC_BYTES;
constexpr unsigned NSTAGE = 3;
constexpr unsigned CTRL = NSTAGE * STAGE_BYTES;
constexpr unsigned CTRL_FLAG = CTRL;
constexpr unsigned CTRL_TMO = CTRL + 16384;
constexpr unsigned CTRL_TICKET = CTRL + 16384 + 128;
constexpr unsigned CTRL_UCNT = CTRL + 16384 + 2048;
constexpr unsigned CTRL_UFLAG = CTRL + 16384 + 3072;
constexpr unsigned CTRL_CALL = CTRL + 20480;
constexpr unsigned WS_NEEDED = CTRL_CALL + 1024u * 32u;
constexpr int NCOMM = 32;
constexpr unsigned NPIECE = 512;
constexpr unsigned TICKETS_PER_CALL = NPIECE - 2 * NCOMM + NCOMM;
constexpr unsigned long long DEADLINE_TICKS = 100000000ull;

#define RING 8
#define COMM_P 1
#define MAPPING 0
#define XG 2
#define NROT 1
#define IB_U 3
#define PF_U 2
#define IB_PIPE 2
#define IB_FAST0 0
#define IB_SLEEP 1
#define MK_N 10
#define MK_U 1
#define MK_D 1
#define MKSM 3
#define MKST "sc0 sc1"
#define COMM_MODE 3
#define C3_FIX (COMM_MODE == 7)
#define COMM_W 4
#define SLEEPN 2
#define PF_MODE 0
#define POLLW 8
#define OSTM 4
#define OST "sc0 sc1 nt"
#define OSTLM 4
#define OSTL "sc0 sc1 nt"
#define XLDM 3
#define XLD "sc0 sc1"
constexpr int R = RING;

#define WD 4
#define XD 5
constexpr unsigned RED_ROW = 272, RED_TILE = 16 * RED_ROW, RED_WAVE = 4 * RED_TILE;
constexpr unsigned PART = 39424, LDS_T5 = 2 * RED_WAVE, STG_X = WD * 4096;
constexpr unsigned LDS_MISC = 4 * PART;
constexpr unsigned LDS_BYTES = LDS_MISC + 256;
static_assert(PART % 128 == 0 && LDS_T5 + RED_TILE <= PART && STG_X + XD * 2048 <= PART && LDS_BYTES <= 160 * 1024, "LDS layout");
static_assert(STG_X >= 2 * RED_TILE, "tiles 0 and 1 of unit 0 are written while the last x slots are in use");

constexpr unsigned long long IBM = IB_U <= 8 ? ~0ull << (8 * (IB_U <= 8 ? IB_U - 1 : 0)) : 0ull;

struct Params {
  const unsigned char* x;
  const unsigned char* w;
  const float* sx;
  const float* sw;
  float* out;
  unsigned char* ws[NRANKS];
  int rank;
};

#define STAMP(ws, idx) do {} while (0)
#define GSTAMP(idx) do {} while (0)

template <class F, int... I> __device__ __forceinline__ void sfor_impl(F&& f, std::integer_sequence<int, I...>) {
  (f(std::integral_constant<int, I>{}), ...);
}
template <int N, class F> __device__ __forceinline__ void sfor(F&& f) { sfor_impl(f, std::make_integer_sequence<int, N>{}); }

__device__ __forceinline__ rsrc_t mk_rsrc(const void* p, unsigned bytes) {
  return __builtin_amdgcn_make_buffer_rsrc((void*)p, 0, bytes, 0x00020000);
}

struct G {
  v8i Wf[8][4];
  v4f rd[2][4];
  unsigned sw_n, sw_t;
  unsigned rt_n, rt_it, rt_t, rt_last, rt_lastn;
  unsigned long long mask;
  unsigned kidx;
  unsigned tmov;
  unsigned badv;
  unsigned vo_loc, vo_stg, vo_rh, vo_out, wa_cur, wa_prev, ra[4], wa5, ra5;
  rsrc_t rs_x, rs_stg, rs_out, rs_tmo, rs_mk;
  unsigned mk_off, rh_off;
  unsigned so_loc[5];
  unsigned fo[4];
  unsigned ub_cur, ub_nxt;
  unsigned kd[8];
  unsigned kn[4];
  unsigned ke;
  unsigned rot;
  unsigned out_prev, out_cur;
  unsigned out5;
  unsigned red_d;
  unsigned call, unit;
  unsigned kbase;
  float scale;
  const GAS unsigned long long* flag_ptr;
  GAS unsigned* tmo;
  LAS unsigned* lds_bad;
};

constexpr int XB = 256 - 8 * R, AB = XB - 80, HD = AB - 1, FD = HD - 3, A4B = FD;
constexpr int acc_reg(int j, int nf) { return AB + 16 * j + 4 * nf; }

template <int J, int NF_, int SLOT, bool FIRST> __device__ __forceinline__ void mfma(const v8i& wf) {
  constexpr int A = acc_reg(J, NF_), X = XB + 8 * SLOT;
  if constexpr (FIRST)
    asm volatile("v_mfma_f32_16x16x128_f8f6f4 v[%c1:%c2], %0, v[%c3:%c4], 0 cbsz:1" :: "a"(wf), "i"(A), "i"(A + 3), "i"(X), "i"(X + 7));
  else
    asm volatile("v_mfma_f32_16x16x128_f8f6f4 v[%c1:%c2], %0, v[%c3:%c4], v[%c1:%c2] cbsz:1" :: "a"(wf), "i"(A), "i"(A + 3), "i"(X), "i"(X + 7));
}
template <int SLOT, int H> __device__ __forceinline__ void ld_stg(unsigned vo, rsrc_t rs, unsigned so) {
  constexpr int X = XB + 8 * SLOT + 4 * H;
  asm volatile("buffer_load_dwordx4 v[%c3:%c4], %0, %1, %2 offen offset:%c5 " XLD :: "v"(vo), "s"(rs), "s"(so), "i"(X), "i"(X + 3), "i"(H * 1024) : "memory");
}
template <int CNT> __device__ __forceinline__ void wait_vm() { asm volatile("s_waitcnt vmcnt(%c0)" :: "i"(CNT) : "memory"); }

__device__ __forceinline__ void st_sys(const v4i& d, unsigned vo, rsrc_t rs, unsigned so) {
  asm volatile("buffer_store_dwordx4 %0, %1, %2, %3 offen sc0 sc1\n\ts_nop 1" :: "v"(d), "v"(vo), "s"(rs), "s"(so) : "memory");
}

__device__ __forceinline__ void dma16(unsigned vo, rsrc_t rs, unsigned so, unsigned lds) {
  asm volatile("s_mov_b32 m0, %3\n\tbuffer_load_dwordx4 %0, %1, %2 offen lds" :: "v"(vo), "s"(rs), "s"(so), "s"(lds) : "memory");
}

#define WPOL ""
#define XPOL ""
__device__ __forceinline__ void dma16w(unsigned vo, rsrc_t rs, unsigned so, unsigned lds) {
  asm volatile("s_mov_b32 m0, %3\n\tbuffer_load_dwordx4 %0, %1, %2 offen " WPOL " lds" :: "v"(vo), "s"(rs), "s"(so), "s"(lds) : "memory");
}
__device__ __forceinline__ void dma16x(unsigned vo, rsrc_t rs, unsigned so, unsigned lds) {
  asm volatile("s_mov_b32 m0, %3\n\tbuffer_load_dwordx4 %0, %1, %2 offen " XPOL " lds" :: "v"(vo), "s"(rs), "s"(so), "s"(lds) : "memory");
}
__device__ __forceinline__ void touch(const unsigned char* a) {
  asm volatile("global_load_dword v%c1, %0, off" :: "v"(a), "i"(HD) : "memory");
}

#define HS0 0
#define XS0 0
constexpr int HW_LINES = (8 - HS0) * 512 * 2 * XG;
constexpr int NHW = (HW_LINES + 112 * 64 - 1) / (112 * 64);
constexpr int HW_LPI = (HW_LINES + 112 * NHW - 1) / (112 * NHW);
constexpr int HX_LINES = (8 - XS0) * 320, HX_WAVES = 16 * XG;
constexpr int NHX = (HX_LINES + HX_WAVES * 64 - 1) / (HX_WAVES * 64);
constexpr int HX_LPI = (HX_LINES + HX_WAVES * NHX - 1) / (HX_WAVES * NHX);
#define CODE_PF 0
#define CODE_X 0u
constexpr int NHINT = NHW + NHX + CODE_PF;
__device__ __forceinline__ int rot_order(int p) { return p / NROT + (p % NROT) * (8 / NROT); }
struct Hint { const unsigned char* wb; const unsigned char* xb; const unsigned char* cb; unsigned clen; int wi, xi; };
template <int Q> __device__ __forceinline__ void hint(const Hint& h) {
  const int lane = threadIdx.x & 63;
  if constexpr (Q >= NHX + NHW) {
    unsigned off = (unsigned)(h.wi * 8 + (lane & 7)) * 128u;
    off = off < h.clen - 4u ? off : h.clen - 4u;
    touch(h.cb + off);
  } else if constexpr (Q < NHX) {
    int line = (Q * HX_WAVES + h.xi) * HX_LPI + (lane < HX_LPI ? lane : HX_LPI - 1);
    line = line < HX_LINES ? line : HX_LINES - 1;
    const int r = rot_order(XS0 + line / 320), rem = line % 320;
    touch(h.xb + (rem >> 2) * KDIM + r * 512 + (rem & 3) * 128);
  } else {
    int line = ((Q - NHX) * 112 + h.wi) * HW_LPI + (lane < HW_LPI ? lane : HW_LPI - 1);
    line = line < HW_LINES ? line : HW_LINES - 1;
    const int row = HS0 * 512 + line / (2 * XG);
    touch(h.wb + (size_t)(rot_order(row >> 9) * 512 + (row & 511)) * NFULL + (line % (2 * XG)) * 128);
  }
}

__device__ __forceinline__ unsigned long long rt_now() {
  unsigned long long t;
  asm volatile("s_memrealtime %0\n\ts_waitcnt lgkmcnt(0)" : "=s"(t) :: "memory");
  return t;
}

__device__ __forceinline__ void slow_wait(G& g) {
  unsigned long long t0 = rt_now();
  const unsigned lane = __lane_id();
  unsigned lo = 0, hi = 64;
  unsigned long long m;
  while (true) {
    unsigned long long v = 0;
    if (lane >= lo && lane < hi) v = __hip_atomic_load(g.flag_ptr, __ATOMIC_RELAXED, __HIP_MEMORY_SCOPE_SYSTEM);
    unsigned tvv = __hip_atomic_load(g.tmo, __ATOMIC_RELAXED, __HIP_MEMORY_SCOPE_AGENT);
    unsigned v0 = (unsigned)v, v1 = (unsigned)(v >> 32);
    asm volatile("" : "+v"(v0), "+v"(v1), "+v"(tvv));
    m = (__builtin_amdgcn_ballot_w64((v0 < v1 ? v0 : v1) >= g.call) | IBM) >> g.kidx;
    if (m & 1ull) break;
    unsigned tv = __builtin_amdgcn_readfirstlane(tvv);
    if (tv == g.call || rt_now() - t0 > DEADLINE_TICKS) {
      __hip_atomic_store(g.tmo, g.call, __ATOMIC_RELAXED, __HIP_MEMORY_SCOPE_AGENT);
      asm volatile("ds_write_b32 %0, %1" :: "v"((unsigned)(LDS_MISC + 4)), "v"(1u) : "memory");
      m = ~0ull;
      break;
    }
    lo = g.kidx; hi = g.kidx + POLLW;
    __builtin_amdgcn_s_sleep(SLEEPN);
  }
  g.mask = m;
}

template <int OFF, int J, int NF_> __device__ __forceinline__ void lds_wr(unsigned a) {
  constexpr int A = acc_reg(J, NF_);
  asm volatile("ds_write_b128 %0, v[%c1:%c2] offset:%c3" :: "v"(a), "i"(A), "i"(A + 3), "i"(OFF) : "memory");
}
template <int OFF> __device__ __forceinline__ void lds_rd(v4f& v, unsigned a) {
  asm volatile("ds_read_b128 %0, %1 offset:%2" : "=v"(v) : "v"(a), "i"(OFF) : "memory");
}

template <int CNT> __device__ __forceinline__ void lds_wait(v4f (&r)[4], unsigned& b) {
  asm volatile("s_waitcnt lgkmcnt(%5)" : "+v"(r[0]), "+v"(r[1]), "+v"(r[2]), "+v"(r[3]), "+v"(b) : "i"(CNT) : "memory");
}
__device__ __forceinline__ void lds_rd32(unsigned& v, unsigned a) {
  asm volatile("ds_read_b32 %0, %1" : "=v"(v) : "v"(a) : "memory");
}
__device__ __forceinline__ void st_out(const v4f& v, unsigned vo, rsrc_t rs, unsigned so) {
  asm volatile("buffer_store_dwordx4 %0, %1, %2, %3 offen " OST "\n\ts_nop 1" :: "v"(v), "v"(vo), "s"(rs), "s"(so) : "memory");
}
__device__ __forceinline__ void st_out_last(const v4f& v, unsigned vo, rsrc_t rs, unsigned so) {
  asm volatile("buffer_store_dwordx4 %0, %1, %2, %3 offen " OSTL "\n\ts_nop 1" :: "v"(v), "v"(vo), "s"(rs), "s"(so) : "memory");
}

__device__ __forceinline__ void st_mark(unsigned vo, rsrc_t rs, unsigned so) {
  const v4i mk = {0, 0, 0, 0};
  asm volatile("buffer_store_dwordx4 %0, %1, %2, %3 offen " MKST "\n\ts_nop 1" :: "v"(mk), "v"(vo), "s"(rs), "s"(so) : "memory");
}

#define RH_N -1
#define RH0_N -1
constexpr int NRH = 2 / XG > 0 ? 2 / XG : 1;
__device__ __forceinline__ void touch_stg(unsigned vo, rsrc_t rs, unsigned so) {
  asm volatile("buffer_load_dword v%c3, %0, %1, %2 offen " XLD :: "v"(vo), "s"(rs), "s"(so), "i"(HD) : "memory");
}
__device__ __forceinline__ void rhint(G& g) {
  sfor<NRH>([&](auto qc) { touch_stg(g.vo_rh, g.rs_stg, g.ub_nxt + g.rh_off + decltype(qc)::value * 8192u); });
}

#define PK_A 1
#define PK_B 17
#define PK_0 22
__device__ __forceinline__ void peek_issue(G& g) {
  asm volatile("global_load_dwordx2 v[%c1:%c2], %0, off sc0 sc1" :: "v"(g.flag_ptr), "i"(FD), "i"(FD + 1) : "memory");
}
__device__ __forceinline__ void peek_use(G& g) {
  unsigned long long m;
  unsigned t;
  asm volatile("v_min_u32 %1, v%c3, v%c4\n\tv_cmp_le_u32 %0, %2, %1" : "=s"(m), "=&v"(t) : "s"(g.call), "i"(FD), "i"(FD + 1));
  g.mask |= m >> (g.kidx & 63u);
}

#define LAST_B 4
template <int NF, int PREV, bool HAS_NEXT> struct Sched {
  static constexpr int NG = 8 * NF;
  static constexpr int B = HAS_NEXT ? 4 : LAST_B;
  static constexpr bool has_load(int n) { return HAS_NEXT || (n - 1 + R < NG); }
  static constexpr int n_out(int n) { return PREV == 0 ? 0 : ((n >= B + 1 && n <= B + 4) ? 1 : 0) + ((PREV == 5 && n == B + 5) ? 1 : 0); }
  static constexpr bool mark_slot(int n) { return n >= MK_N && n < MK_N + 2 * (9 - IB_U); }
  static constexpr bool has_mark(int n) {
    return MK_U == 0 ? HAS_NEXT && PREV != 0 && (n == MK_N || n == MK_N + MK_D) : MK_U == 1 && PREV == 5 && mark_slot(n);
  }
  static constexpr int n_store(int n) { return n_out(n) + (has_mark(n) ? 1 : 0); }
  static constexpr bool has_tmo(int n) { return HAS_NEXT ? n == 30 - R : n == NG - 1; }
  static constexpr bool has_rh(int n) { return HAS_NEXT && n == RH_N; }
  static constexpr bool has_peek(int n) { return HAS_NEXT && (n == PK_A || n == PK_B); }
  static constexpr bool use_peek(int n) { return has_peek(n - R + 1); }
  static constexpr int ops(int q) {
    if (q < 0) return 2;
    return (has_load(q) ? 2 : 0) + n_store(q) + (has_tmo(q) ? 1 : 0) + (has_rh(q) ? NRH : 0) + (has_peek(q) ? 1 : 0);
  }
  static constexpr int wait_cnt(int n) {
    int c = 0;
    for (int q = n - R + 2; q <= n - 1; ++q) c += ops(q);
    return c > 63 ? 63 : c;
  }
};

template <int SLOT> __device__ __forceinline__ unsigned ib_min() {
  constexpr int X = XB + 8 * SLOT;
  unsigned t0, t1;
  asm volatile("v_min3_u32 %0, v[%c2], v[%c3], v[%c4]\n\t"
               "v_min3_u32 %1, v[%c5], v[%c6], v[%c7]\n\t"
               "v_min3_u32 %0, v[%c8], v[%c9], %0\n\t"
               "v_min_u32 %0, %0, %1"
               : "=&v"(t0), "=&v"(t1)
               : "i"(X), "i"(X + 1), "i"(X + 2), "i"(X + 3), "i"(X + 4), "i"(X + 5), "i"(X + 6), "i"(X + 7));
  return t0;
}
__device__ __forceinline__ unsigned long long ib_zero(unsigned t) {
  unsigned long long m;
  asm volatile("v_cmp_eq_u32 %0, 0, %1" : "=s"(m) : "v"(t));
  return m;
}
template <int SLOT> __device__ __forceinline__ bool ib_dirty() { return ib_zero(ib_min<SLOT>()) != 0ull; }
constexpr int rl_kk(int rotl, int i) { return (i + 8 - (rotl >> 3)) & 7; }
template <int M, int ROFF, int ROTL> __device__ __forceinline__ void ib_load(G& g) {
  constexpr int slot = (ROFF + M) % R;
  const unsigned so = (M < 32 ? g.ub_cur + (ROTL < 0 ? g.kd[(M / 4) & 7] : rl_kk(ROTL, M / 4) * 32768u) : g.ub_nxt + g.kn[((M - 32) / 4) & 3]) + g.fo[M % 4];
  ld_stg<slot, 0>(g.vo_stg, g.rs_stg, so);
  ld_stg<slot, 1>(g.vo_stg, g.rs_stg, so);
}
template <int N, int ROFF, bool HAS_NEXT, int ROTL> __device__ __forceinline__ void ib_retry(G& g) {
  constexpr int KE = N | 3;
  constexpr int LE = (HAS_NEXT || N + R - 2 < 31) ? N + R - 2 : 31;
  constexpr int slot = (ROFF + N) % R;
  const unsigned long long t0 = rt_now();
  const unsigned lane = __lane_id();
  bool flagged = false;
  while (true) {
    unsigned long long v = 0;
    if (lane == g.kbase + N / 4) v = __hip_atomic_load(g.flag_ptr, __ATOMIC_RELAXED, __HIP_MEMORY_SCOPE_SYSTEM);
    unsigned tvv = __hip_atomic_load(g.tmo, __ATOMIC_RELAXED, __HIP_MEMORY_SCOPE_AGENT);
    sfor<KE - N + 1>([&](auto mc) { ib_load<N + decltype(mc)::value, ROFF, ROTL>(g); });
    unsigned v0 = (unsigned)v, v1 = (unsigned)(v >> 32);
    asm volatile("s_waitcnt vmcnt(0)" : "+v"(v0), "+v"(v1), "+v"(tvv) :: "memory");
    if (flagged || !ib_dirty<slot>()) break;
    flagged = __builtin_amdgcn_ballot_w64((v0 < v1 ? v0 : v1) >= g.call) != 0ull;
    unsigned tv = __builtin_amdgcn_readfirstlane(tvv);
    if (tv == g.call || rt_now() - t0 > DEADLINE_TICKS) {
      __hip_atomic_store(g.tmo, g.call, __ATOMIC_RELAXED, __HIP_MEMORY_SCOPE_AGENT);
      asm volatile("ds_write_b32 %0, %1" :: "v"((unsigned)(LDS_MISC + 4)), "v"(1u) : "memory");
      break;
    }
    if (!flagged) __builtin_amdgcn_s_sleep(IB_SLEEP);
  }
  if constexpr (LE > KE) sfor<LE - KE>([&](auto mc) { ib_load<KE + 1 + decltype(mc)::value, ROFF, ROTL>(g); });
}

template <int NF, int ROFF, int PREV, bool HAS_NEXT, int ROTL, bool IBC>
__device__ __forceinline__ void unit_body(G& g) {
  using S = Sched<NF, PREV, HAS_NEXT>;
  constexpr int NG = S::NG, B = S::B;
  sfor<NG>([&](auto nc) {
    constexpr int n = decltype(nc)::value;
    constexpr int i = n / NF, j = n % NF;
    constexpr int wi = ROTL < 0 ? i : (rl_kk(ROTL, i) + 8 - (ROTL & 7)) & 7;
    constexpr int slot = (ROFF + n) % R;
    constexpr int pslot = (ROFF + n + R - 1) % R;
    constexpr int t = n - 1 + R;
    constexpr bool t_next = t >= NG;
    constexpr int ti = t_next ? (t - NG) / 4 : t / NF, tj = t_next ? (t - NG) % 4 : t % NF;

    wait_vm<S::wait_cnt(n)>();
    if constexpr (IBC && (n == 0 || !IB_PIPE)) {
      if (__builtin_expect(ib_dirty<slot>(), 0)) ib_retry<n, ROFF, HAS_NEXT, ROTL>(g);
    }

    if constexpr (PREV != 0 && n == B) {
      asm volatile("s_waitcnt lgkmcnt(0)\n\ts_barrier" ::: "memory");
    }
    v4f ov, ot0, ot1;
    constexpr bool has_rd = PREV != 0 && ((n >= B && n <= B + 3) || (PREV == 5 && n == B + 4));
    constexpr int roff = (n >= B && n <= B + 3) ? (n - B) * 4 * (int)RED_ROW : 0;
    constexpr int rb = n & 1;
    if constexpr (S::has_load(n) && tj == 0) {
      if constexpr (!IBC) {
        if (__builtin_expect(!(g.mask & 1ull), 0)) slow_wait(g);
        g.mask >>= 1;
        g.kidx += 1;
      }
    }
    if constexpr (S::has_rh(n)) rhint(g);
    if constexpr (S::has_peek(n)) peek_issue(g);
    if constexpr (S::has_tmo(n)) {
      asm volatile("buffer_load_dword %0, off, %1, 0 sc1" : "=v"(g.tmov) : "s"(g.rs_tmo) : "memory");
    }

    auto load_half = [&](auto hc) {
      constexpr int h = decltype(hc)::value;
      if constexpr (S::has_load(n))
        ld_stg<pslot, h>(g.vo_stg, g.rs_stg, (t_next ? g.ub_nxt + g.kn[ti & 3] : g.ub_cur + (ROTL < 0 ? g.kd[ti & 7] : rl_kk(ROTL, ti) * 32768u)) + g.fo[tj]);
    };
    auto side = [&](auto qc) {
      constexpr int q = decltype(qc)::value;
      if constexpr (n == NG - 2) lds_wr<0 * RED_TILE + q * 64, 0, q>(g.wa_cur);
      if constexpr (n == NG - 1) lds_wr<1 * RED_TILE + q * 64, 1, q>(g.wa_cur);
      if constexpr (PREV == 4 && n == 0) lds_wr<2 * RED_TILE + q * 64, 2, q>(g.wa_prev);
      if constexpr (PREV == 4 && n == 1) lds_wr<3 * RED_TILE + q * 64, 3, q>(g.wa_prev);
      if constexpr (PREV == 5 && n == 0) lds_wr<3 * RED_TILE + q * 64, 3, q>(g.wa_prev);
      if constexpr (PREV == 5 && n == 1) lds_wr<q * 64, 4, q>(g.wa5);
      if constexpr (PREV != 0 && n == B && q == 0)
        asm volatile("ds_read_b32 %0, %1" : "=v"(g.badv) : "v"((unsigned)(LDS_MISC + 4)) : "memory");
      if constexpr (has_rd && q < 2) {
        if constexpr (n <= B + 3) {
          lds_rd<roff>(g.rd[rb][(2 * q) & 3], g.ra[(2 * q) & 3]);
          lds_rd<roff>(g.rd[rb][(2 * q + 1) & 3], g.ra[(2 * q + 1) & 3]);
        } else {
          lds_rd<0>(g.rd[rb][(2 * q) & 3], q == 0 ? g.ra5 : g.ra5 + 2 * PART);
          lds_rd<PART>(g.rd[rb][(2 * q + 1) & 3], q == 0 ? g.ra5 : g.ra5 + 2 * PART);
        }
      }
      if constexpr (S::n_out(n) != 0 && q == 0) {
        constexpr int pb = rb ^ 1;
        asm volatile("s_waitcnt lgkmcnt(%5)" : "+v"(g.rd[pb][0]), "+v"(g.rd[pb][1]), "+v"(g.rd[pb][2]), "+v"(g.rd[pb][3]), "+v"(g.badv) : "i"(has_rd ? 2 : 0));
        ot0 = g.rd[pb][0] + g.rd[pb][1];
        ot1 = g.rd[pb][2] + g.rd[pb][3];
        asm volatile("" : "+v"(ot0), "+v"(ot1));
        if constexpr (n == B + 1) {
          unsigned bad = __builtin_amdgcn_readfirstlane(g.badv) | (__builtin_amdgcn_readfirstlane(g.tmov) == g.call ? 1u : 0u);
          if (bad) g.scale = __builtin_nanf("");
        }
      }
      if constexpr (S::n_out(n) != 0 && q == 1) {
        const v4f sum = ot0 + ot1;
        const float sc = g.scale;
        ov = (v4f){sum[0] * sc, sum[1] * sc, sum[2] * sc, sum[3] * sc};
        asm volatile("" : "+v"(ov));
      }
    };
    auto store_out = [&]() {
      if constexpr (S::n_out(n) != 0) {
        const unsigned so = n < B + 5 ? g.out_prev + (n - B - 1) * 4 * (NLOC * 4) : g.out5;
        st_out(ov, g.vo_out, g.rs_out, so);
      }
      if constexpr (S::has_mark(n) && MK_U == 0)
        st_mark(g.vo_stg, g.rs_mk, g.mk_off + (g.unit < 9u - IB_U ? g.unit - 1u : 8u - IB_U) * UNIT_BYTES + (n == MK_N ? 0u : 1024u));
      if constexpr (S::has_mark(n) && MK_U != 0) st_mark(g.vo_stg, g.rs_mk, g.mk_off + ((n - MK_N) >> 1) * UNIT_BYTES + ((n - MK_N) & 1) * 1024u);
      if constexpr (MK_U > 1 && PREV == 4 && HAS_NEXT && S::mark_slot(n)) {
        if (g.unit == (unsigned)MK_U) st_mark(g.vo_stg, g.rs_mk, g.mk_off + ((n - MK_N) >> 1) * UNIT_BYTES + ((n - MK_N) & 1) * 1024u);
      }
    };

    unsigned ibt = 0;
    unsigned long long ibm = 0;
    sfor<4>([&](auto fc) {
      constexpr int nf = decltype(fc)::value;
      mfma<j, nf, slot, i == 0>(g.Wf[wi][nf]);
      side(fc);
      if constexpr (nf == 0) load_half(std::integral_constant<int, 0>{});
      if constexpr (nf == 1) store_out();
      if constexpr (nf == 2) load_half(std::integral_constant<int, 1>{});
      if constexpr (IBC && IB_PIPE == 1 && n + 1 < NG) {
        if constexpr (nf == 2) {
          wait_vm<S::wait_cnt(n + 1)>();
          ibt = ib_min<(ROFF + n + 1) % R>();
        }
        if constexpr (nf == 3) {
          if (__builtin_expect(ib_zero(ibt) != 0ull, 0)) ib_retry<n + 1, ROFF, HAS_NEXT, ROTL>(g);
        }
      }
      if constexpr (IBC && IB_PIPE == 2 && n + 1 < NG) {
        if constexpr (nf == 1) {
          wait_vm<S::wait_cnt(n + 1) - (S::has_load(n) ? 1 : 0)>();
          ibt = ib_min<(ROFF + n + 1) % R>();
        }
        if constexpr (nf == 2) ibm = ib_zero(ibt);
        if constexpr (nf == 3) {
          if (__builtin_expect(ibm != 0ull, 0)) ib_retry<n + 1, ROFF, HAS_NEXT, ROTL>(g);
        }
      }
      if constexpr (S::use_peek(n) && nf == 1) peek_use(g);
    });
  });
}

struct U0 {
  unsigned vo_w, vo_x, ra_w, ra_x0, ra_x1;
  unsigned so_w, lds_w, lds_x;
  rsrc_t rs_w;
};
#define HT_X -1
#define HT_X1 108
#define HT_C 4
#define HT_W0 -1
#define HT_W1 4
#define HT_W2 108
struct U0S {
  static constexpr int NG = 40;
  static constexpr int w_issue(int n, int k) {
    int m = -1;
    if (n == 0) m = WD + k;
    else if (k == 0 && n % 5 == 3) m = 2 * (n / 5) + 2 + WD;
    else if (k == 0 && n % 5 == 0) m = 2 * (n / 5) + 1 + WD;
    return m <= 15 ? m : -1;
  }
  static constexpr int w_read(int n) {
    int m = n % 5 == 2 ? 2 * (n / 5 + 1) : (n % 5 == 4 ? 2 * (n / 5 + 1) + 1 : -1);
    return m <= 15 ? m : -1;
  }
  static constexpr int w_need(int m) { return m < 2 ? -1 : (m % 2 == 0 ? 5 * (m / 2 - 1) + 2 : 5 * (m / 2 - 1) + 4); }
  static constexpr int hint_t(int q) { return q >= NHX + NHW ? HT_C : q < NHX ? (q == 0 ? HT_X : HT_X1) : (q == NHX ? HT_W0 : (q == NHX + 1 ? HT_W1 : HT_W2)); }
  static constexpr bool remote(int n) { return n - 1 + R >= NG; }
  static constexpr int NPRO_MAX = WD + XD + NHINT;
  static constexpr int pro_op(int idx) {
    int c = 0;
    for (int t = -2; t < 100; ++t) {
      if (t == -1) { if (c++ == idx) return 100; }
      for (int m = 0; m < WD; ++m) if (w_need(m) == t) { if (c++ == idx) return 200 + m; }
      if (t + 1 < XD && t >= 0) { if (c++ == idx) return 100 + t + 1; }
      for (int q = 0; q < NHINT; ++q) if (hint_t(q) == t) { if (c++ == idx) return 300 + q; }
    }
    return -1;
  }
  static constexpr int op_cnt(int op) { return op < 0 ? 0 : (op < 200 ? 2 : (op < 300 ? 4 : 1)); }
  static constexpr int n_hint(int n) { int c = 0; for (int q = 0; q < NHINT; ++q) if (hint_t(q) == 100 + n) ++c; return c; }
  static constexpr int wait_cnt(int n) {
    int pos = 0, pl[NG + 16] = {}, pw[16] = {};
    pos += 1;
    for (int idx = 0; idx < NPRO_MAX; ++idx) {
      const int op = pro_op(idx);
      pos += op_cnt(op);
      if (op >= 100 && op < 200) pl[op - 100] = pos;
      if (op >= 200 && op < 300) pw[op - 200] = pos;
    }
    if (n < 0) return pos - (pw[1] > pl[0] ? pw[1] : pl[0]);
    for (int q = 0; q <= n; ++q) {
      for (int k = 0; k < 2; ++k) if (w_issue(q, k) >= 0) { pos += 4; pw[w_issue(q, k)] = pos; }
      if (q + XD < NG) { pos += 2; pl[q + XD] = pos; }
      pos += n_hint(q) + (q == RH0_N ? NRH : 0) + (q == PK_0 ? 1 : 0);
      if (remote(q)) pos += 1;
      if (q < n && remote(q)) pos += 1;
    }
    int need = n + 1 < NG ? pl[n + 1] : 0;
    if (w_read(n) >= 0 && pw[w_read(n)] > need) need = pw[w_read(n)];
    int c = pos - need;
    return c > 63 ? 63 : c;
  }
};
template <int M> __device__ __forceinline__ void u0_w_issue(const G& g, const U0& u) {
  sfor<4>([&](auto qc) {
    constexpr int q = decltype(qc)::value;
    dma16w(u.vo_w, u.rs_w, u.so_w + (g.kd[M >> 1] << 8) + (unsigned)((M & 1) * 64 + q * 16) * NFULL, u.lds_w + (M % WD) * 4096 + q * 1024);
  });
}
template <int N> __device__ __forceinline__ void u0_x_issue(const G& g, const U0& u) {
  dma16x(u.vo_x, g.rs_x, g.so_loc[N % 5] + (g.kd[N / 5] >> 6), u.lds_x + (N % XD) * 2048);
  dma16x(u.vo_x, g.rs_x, g.so_loc[N % 5] + (g.kd[N / 5] >> 6) + 8 * KDIM, u.lds_x + (N % XD) * 2048 + 1024);
}
template <int M> __device__ __forceinline__ void u0_w_read(G& g, const U0& u) {
#pragma unroll
  for (int tt = 0; tt < 2; ++tt)
#pragma unroll
    for (int nf = 0; nf < 4; ++nf) {
      v2i q = __builtin_amdgcn_ds_read_tr8_b64_v2i32((LAS v2i*)(uintptr_t)(u.ra_w + (M % WD) * 4096 + tt * 512 + nf * 16));
      g.Wf[M >> 1][nf][(M & 1) * 4 + tt * 2] = q[0];
      g.Wf[M >> 1][nf][(M & 1) * 4 + tt * 2 + 1] = q[1];
    }
}
template <int N> __device__ __forceinline__ void u0_x_read(G& g, const U0& u) {
  constexpr int X = XB + 8 * (N % R);
  asm volatile("ds_read_b128 v[%c1:%c2], %0 offset:%c3" :: "v"(u.ra_x0), "i"(X), "i"(X + 3), "i"((N % XD) * 2048) : "memory");
  asm volatile("ds_read_b128 v[%c1:%c2], %0 offset:%c3" :: "v"(u.ra_x1), "i"(X + 4), "i"(X + 7), "i"((N % XD) * 2048) : "memory");
}

__device__ __forceinline__ void unit0_body(G& g, U0& u, const Hint& h) {
  constexpr int NG = U0S::NG;
  sfor<NG>([&](auto nc) {
    constexpr int n = decltype(nc)::value;
    constexpr int i = n / 5, j = n % 5;
    constexpr int slot = n % R, pslot = (n + R - 1) % R;
    constexpr int t = n - 1 + R, ti = U0S::remote(n) ? (t - NG) / 4 : 0, tj = U0S::remote(n) ? (t - NG) % 4 : 0;

    asm volatile("s_waitcnt lgkmcnt(0)" ::: "memory");
    if constexpr (U0S::w_issue(n, 0) >= 0) u0_w_issue<U0S::w_issue(n, 0)>(g, u);
    if constexpr (U0S::w_issue(n, 1) >= 0) u0_w_issue<U0S::w_issue(n, 1)>(g, u);
    if constexpr (n + XD < NG) u0_x_issue<n + XD>(g, u);
    sfor<NHINT>([&](auto qc) { constexpr int q = decltype(qc)::value; if constexpr (U0S::hint_t(q) == 100 + n) hint<q>(h); });
    if constexpr (n == RH0_N) rhint(g);
    if constexpr (n == PK_0) peek_issue(g);
    if constexpr (PK_0 >= 0 && n == NG - R) peek_use(g);
    static_assert(PK_0 < 0 || (PK_0 + 1 + XD < NG && PK_0 + XD < NG - R), "peek is complete when it is used");
    if constexpr (U0S::remote(n) && tj == 0) {
      if (!(g.mask & 1ull)) slow_wait(g);
      g.mask >>= 1;
      g.kidx += 1;
    }
    sfor<4>([&](auto fc) {
      constexpr int nf = decltype(fc)::value;
      mfma<j, nf, slot, i == 0>(g.Wf[i][nf]);
      if constexpr (n == NG - 3) lds_wr<0 * RED_TILE + nf * 64, 0, nf>(g.wa_cur);
      if constexpr (n == NG - 2) lds_wr<1 * RED_TILE + nf * 64, 1, nf>(g.wa_cur);
      if constexpr (n == NG - 1) lds_wr<2 * RED_TILE + nf * 64, 2, nf>(g.wa_cur);
      if constexpr (U0S::remote(n) && (nf == 0 || nf == 2))
        ld_stg<pslot, nf / 2>(g.vo_stg, g.rs_stg, g.ub_nxt + g.kn[ti & 3] + g.fo[tj]);
      if constexpr (nf == 1 && n + 1 < NG) {
        asm volatile("s_waitcnt vmcnt(%0)" :: "i"(U0S::wait_cnt(n)) : "memory");
        u0_x_read<n + 1>(g, u);
        if constexpr (U0S::w_read(n) >= 0) u0_w_read<U0S::w_read(n)>(g, u);
      }
    });
  });
}

template <bool FIRST> __device__ __forceinline__ void next_unit(G& g, unsigned out_next) {
  g.out_prev = g.out_cur;
  g.out_cur = out_next;
  g.ub_cur = g.ub_nxt;
  g.ub_nxt += UNIT_BYTES;
  g.unit += 1u;
  g.kbase += 8u;
  sfor<(R + 2) / 4>([&](auto ic) { constexpr int i = decltype(ic)::value; g.kn[i] = g.unit == 7u ? ((8u + i - g.ke) & 7u) * 32768u : g.kd[i]; });
  g.wa_prev = g.wa_cur;
  g.wa_cur += g.red_d;
  if (!FIRST) { g.ra[0] -= g.red_d; g.ra[1] -= g.red_d; g.ra[2] -= g.red_d; g.ra[3] -= g.red_d; }
  g.red_d = 0u - g.red_d;
}

__device__ __forceinline__ void gemm_role(const Params& p, unsigned char* smem_generic) {
  LAS unsigned char* smem = (LAS unsigned char*)smem_generic;
  const int b = blockIdx.x;
  const int xcd = b & 7, slot = (b >> 3) - (NCOMM >> 3);
  const int pidx = (xcd % XG) * 28 + slot;
  const int pi = pidx / (4 * XG), cb = (xcd / XG) * (4 * XG) + pidx % (4 * XG);
  const int tid = threadIdx.x, lane = tid & 63;
  const int w = __builtin_amdgcn_readfirstlane(tid >> 6);
  const int kg = lane >> 4, li = lane & 15;
  const int rank = p.rank;
  unsigned char* myws = p.ws[rank];
  const int src = (rank + 1 + pi) & 7;
  G g;
  asm volatile("" ::: "v255");
  GSTAMP(0);

  unsigned ticket = 0;
  if (lane == 0)
    asm volatile("global_atomic_add %0, %1, %2, off sc0 sc1" : "+v"(ticket) : "v"((unsigned*)(myws + CTRL_CALL + (b * 4 + w) * 32)), "v"(1u) : "memory");
  GSTAMP(17);
  const int row0 = pi * 80 < 432 ? pi * 80 : 432;
  g.rs_x = mk_rsrc(p.x, SRC_BYTES);
  g.rot = (unsigned)(pidx % NROT) * (8 / NROT);
  g.ke = C3_FIX ? (unsigned)(6 - pi + 1) >> 1 : 0u;
  sfor<8>([&](auto ic) { constexpr int i = decltype(ic)::value; g.kd[i] = ((i + g.rot) & 7u) * 32768u; });
  g.kn[0] = g.kd[0]; g.kn[1] = g.kd[1]; g.kn[2] = g.kd[2]; g.kn[3] = g.kd[3];
  sfor<4>([&](auto jc) { constexpr int j = decltype(jc)::value; g.so_loc[j] = (unsigned)(row0 + ((j + 1 + w) & 3) * 16) * KDIM + w * 128; });
  g.so_loc[4] = (unsigned)(row0 + 64) * KDIM + w * 128;
  U0 u;
  {
    const unsigned base = (unsigned)(uintptr_t)smem + w * PART;
    u.rs_w = mk_rsrc(p.w, (unsigned)KDIM * NFULL);
    u.vo_w = (lane >> 2) * NFULL + (lane & 3) * 16;
    u.vo_x = (lane >> 3) * KDIM + ((lane & 7) ^ (lane >> 3)) * 16;
    u.so_w = (unsigned)rank * NLOC + cb * 64 + (unsigned)(w * 128) * NFULL;
    u.lds_w = base;
    u.lds_x = base + STG_X;
    u.ra_w = base + (kg * 16 + (li >> 1)) * 64 + (li & 1) * 8;
    u.ra_x0 = base + STG_X + (li >> 3) * 1024 + (li & 7) * 128 + ((kg ^ (li & 7)) * 16);
    u.ra_x1 = u.ra_x0 ^ 64u;
  }
  Hint h;
  h.wb = p.w + (size_t)rank * NLOC + (xcd / XG) * (256 * XG);
  h.xb = p.x + (size_t)row0 * KDIM;
  h.wi = slot * 4 + w;
  h.xi = (pidx % (4 * XG)) * 4 + w;
  sfor<U0S::NPRO_MAX>([&](auto ic) {
    constexpr int op = U0S::pro_op(decltype(ic)::value);
    if constexpr (op >= 100 && op < 200) u0_x_issue<op - 100>(g, u);
    if constexpr (op >= 200 && op < 300) u0_w_issue<op - 200>(g, u);
    if constexpr (op >= 300) hint<op - 300>(h);
  });
  GSTAMP(12);
  g.lds_bad = (LAS unsigned*)(smem + LDS_MISC + 4);
  if (tid == 0) asm volatile("ds_write_b32 %0, %1" :: "v"((unsigned)(uintptr_t)smem + LDS_MISC + 4), "v"(0u) : "memory");
  asm volatile("s_waitcnt vmcnt(%1) lgkmcnt(0)\n\ts_barrier" : "+v"(ticket) : "i"(U0S::wait_cnt(-1)) : "memory");
  g.call = __builtin_amdgcn_readfirstlane(ticket) + 1u;
  GSTAMP(13);
  u0_x_read<0>(g, u);
  u0_w_read<0>(g, u); u0_w_read<1>(g, u);

  g.scale = *(const CAS float*)p.sx * *(const CAS float*)p.sw;
  const unsigned cm = __builtin_amdgcn_readfirstlane(g.call % NSTAGE);
  g.rs_stg = mk_rsrc(myws + cm * STAGE_BYTES, STAGE_BYTES);
  {
    const unsigned gw = (unsigned)(b - NCOMM) * 4u + (unsigned)w;
    g.rs_mk = mk_rsrc(myws + (cm == 0u ? 2u : cm - 1u) * STAGE_BYTES, STAGE_BYTES);
    g.mk_off = ((unsigned)(rank + 1 + (gw >> 7)) & 7u) * SRC_BYTES + (IB_U - 1u) * UNIT_BYTES + (gw & 127u) * 2048u;
  }
  g.rs_out = mk_rsrc(p.out, (unsigned)(4096u * NLOC * 4u));
  g.flag_ptr = (const GAS unsigned long long*)((lane >> 3) < PF_U
      ? myws + CTRL_FLAG + src * 2048 + w * 512 + (lane < 56 ? (lane & 56) + ((lane + g.rot) & 7) : lane) * 8
      : myws + CTRL_UFLAG + src * 64 + (lane >> 3) * 8);
  g.tmo = (GAS unsigned*)(myws + CTRL_TMO);
  g.rs_tmo = mk_rsrc(myws + CTRL_TMO, 4);
  g.vo_stg = lane * 16;
  g.vo_rh = lane * 128;
  g.rh_off = (unsigned)(pidx % (4 * XG)) * (UNIT_BYTES / (4 * XG)) + (unsigned)w * (UNIT_BYTES / (16 * XG) - 8192u);
  g.vo_out = (lane >> 4) * (NLOC * 4) + (lane & 15) * 16;
  sfor<4>([&](auto jc) { constexpr int j = decltype(jc)::value; g.fo[j] = ((j + 1 + w) & 3) * 2048; });
  g.wa_cur = w * PART + li * RED_ROW + kg * 16;
  sfor<4>([&](auto qc) {
    constexpr int q = decltype(qc)::value;
    g.ra[q] = q * PART + ((w - q - 1) & 3) * RED_TILE + (lane >> 4) * RED_ROW + (lane & 15) * 16;
  });
  g.wa5 = w * PART + LDS_T5 + li * RED_ROW + kg * 16;
  g.ra5 = LDS_T5 + (4 * w + (lane >> 4)) * RED_ROW + (lane & 15) * 16;
  g.red_d = RED_WAVE;
  g.tmov = 0u;
  const unsigned colb = cb * 256;
  auto out_off = [&](int grow) -> unsigned { return (unsigned)(grow + w * 16) * (NLOC * 4) + colb; };
  g.out_cur = out_off(rank * MLOC + row0);
  g.out_prev = g.out_cur;
  g.out5 = (unsigned)(rank * MLOC + row0 + 64 + 4 * w) * (NLOC * 4) + colb;
  g.ub_cur = 0;
  g.ub_nxt = (unsigned)src * SRC_BYTES + w * 8192;
  g.mask = IBM;
  g.sw_n = 0; g.sw_t = 0;
  g.rt_n = g.rt_it = g.rt_t = g.rt_last = g.rt_lastn = 0;
  g.kbase = 0u - 8u;
  g.unit = 0u;
  static_assert(IB_U >= 1 && PF_U < IB_U, "groups with flags / validated in-band");
  g.kidx = 0u;

  GSTAMP(1);
  unit0_body(g, u, h);
  GSTAMP(2);
  next_unit<true>(g, out_off(src * MLOC));
  unit_body<4, 40 % R, 5, true, -1, (IB_U <= 1)>(g);
  GSTAMP(3);
#define USTAMP(u) do {} while (0)
  constexpr int ULIM = IB_U < 2 ? 2 : (IB_U < 8 ? IB_U : 8);
  if constexpr (32 % R == 0) {
    for (int u = 2; u < ULIM; ++u) {
      next_unit<false>(g, out_off(src * MLOC + (u - 1) * 64));
      unit_body<4, (40 + 32) % R, 4, true, -1, false>(g);
      USTAMP(u);
    }
    if constexpr (ULIM < 8) {
      for (int u = ULIM; u < 8; ++u) {
        next_unit<false>(g, out_off(src * MLOC + (u - 1) * 64));
        unit_body<4, (40 + 32) % R, 4, true, -1, true>(g);
        USTAMP(u);
      }
    }
  } else {
    static_assert(32 % R == 0 || (96 % R == 0 && ULIM == 3), "ring size");
    next_unit<false>(g, out_off(src * MLOC + 64));
    unit_body<4, (40 + 32) % R, 4, true, -1, false>(g);
    USTAMP(2);
    sfor<5>([&](auto uc) {
      constexpr int u = 3 + decltype(uc)::value;
      next_unit<false>(g, out_off(src * MLOC + (u - 1) * 64));
      unit_body<4, (40 + 32 * (u - 1)) % R, 4, true, -1, true>(g);
      USTAMP(u);
    });
  }
  next_unit<false>(g, out_off(src * MLOC + 7 * 64));
  sfor<NROT>([&](auto rc) {
    constexpr int ro = decltype(rc)::value * (8 / NROT);
    if (NROT == 1 || g.rot == (unsigned)ro) unit_body<4, (40 + 32 * 7) % R, 4, false, ro, (IB_U <= 8)>(g);
  });
  GSTAMP(10);

  sfor<4>([&](auto qc) { constexpr int q = decltype(qc)::value; lds_wr<2 * RED_TILE + q * 64, 2, q>(g.wa_cur); });
  asm volatile("s_nop 7\n\ts_nop 7\n\ts_nop 7" ::: "memory");
  sfor<4>([&](auto qc) { constexpr int q = decltype(qc)::value; lds_wr<3 * RED_TILE + q * 64, 3, q>(g.wa_cur); });
  asm volatile("s_waitcnt lgkmcnt(0)\n\ts_barrier" ::: "memory");
  {
    v4f r[4][4];
    lds_rd32(g.badv, LDS_MISC + 4);
    sfor<16>([&](auto qc) {
      constexpr int q = decltype(qc)::value;
      lds_rd<(q >> 2) * 4 * RED_ROW>(r[q >> 2][q & 3], g.ra[q & 3] - g.red_d);
    });
    float sc = g.scale;
    sfor<4>([&](auto sc_) {
      constexpr int q = decltype(sc_)::value;
      lds_wait<(3 - q) * 4>(r[q], g.badv);
      if constexpr (q == 0) {
        asm volatile("s_waitcnt vmcnt(0)" : "+v"(g.tmov) :: "memory");
        unsigned bad = __builtin_amdgcn_readfirstlane(g.badv) | (__builtin_amdgcn_readfirstlane(g.tmov) == g.call ? 1u : 0u);
        if (bad) sc = __builtin_nanf("");
      }
      const v4f sum = (r[q][0] + r[q][1]) + (r[q][2] + r[q][3]);
      const v4f v = {sum[0] * sc, sum[1] * sc, sum[2] * sc, sum[3] * sc};
      st_out_last(v, g.vo_out, g.rs_out, g.out_cur + q * 4 * (NLOC * 4));
    });
  }
}

#define CSTAMP(n) do {} while (0)
template <bool LEADER>
__device__ __forceinline__ void comm_wave(const Params& p, LAS unsigned char* smem, const unsigned call, const int j, const int w, const int lane, v4i d) {
  const int rank = p.rank;
  unsigned char* myws = p.ws[rank];
  const int cw = j * 4 + w;
  rsrc_t rs_x = mk_rsrc(p.x, SRC_BYTES);
  const unsigned vo = (unsigned)((w >> 1) * 16 + (lane & 15)) * KDIM + (w & 1) * 64 + (lane >> 4) * 16;
  const unsigned dst0 = (call % NSTAGE) * STAGE_BYTES + (unsigned)rank * SRC_BYTES + (unsigned)w * 1024u;
  rsrc_t rs[7];
#pragma unroll
  for (int q = 0; q < 7; ++q) {
    int pq = (rank + 1 + (q + cw) % 7) & 7;
    rs[q] = mk_rsrc(p.ws[pq], NSTAGE * STAGE_BYTES);
  }
  unsigned char* flagb = p.ws[(rank + 1 + (lane < 7 ? lane : 0)) & 7] + CTRL_FLAG + rank * 2048;
  unsigned char* uflagb = p.ws[(rank + 1 + (lane < 7 ? lane : 0)) & 7] + CTRL_UFLAG + rank * 64;
  unsigned pend = 0;
  unsigned* ticketp = (unsigned*)(myws + CTRL_TICKET);
  GAS unsigned* tmo = (GAS unsigned*)(myws + CTRL_TMO);
  const unsigned tbase = (call - 1u) * TICKETS_PER_CALL - 2u * NCOMM;
  const unsigned vs = lane * 16;
  LAS unsigned* tk = (LAS unsigned*)(smem + 1024);
  LAS unsigned* cnt = (LAS unsigned*)(smem + LDS_MISC + 64);

  auto publish = [&](unsigned n, unsigned t) {
    unsigned old = 0;
    if (lane == 0) old = __hip_atomic_fetch_add(cnt + (n & 7u), 1u, __ATOMIC_RELAXED, __HIP_MEMORY_SCOPE_WORKGROUP);
    old = __builtin_amdgcn_readfirstlane(old);
    if ((old & 3u) == 3u) {
      if (t < PF_U * 64u) {
        if (lane < 7) __hip_atomic_store((unsigned*)(flagb + ((t >> 1) & 3u) * 512u + (t >> 3) * 8u + (t & 1u) * 4u), call, __ATOMIC_RELAXED, __HIP_MEMORY_SCOPE_SYSTEM);
      } else {
        if (lane == 0)
          asm volatile("global_atomic_add v%c2, %0, %1, off sc0 sc1" :: "v"(myws + CTRL_UCNT + (t >> 6) * 128u), "v"(1u), "i"(FD) : "memory");
        pend = (t >> 6) + 1u;
      }
      CSTAMP(n + 1);
    }
  };
  auto group_done = [&]() {
    if (pend) {
      unsigned o;
      asm volatile("v_readfirstlane_b32 %0, v%c1" : "=s"(o) : "i"(FD) : "memory");
      if (o - (call - 1u) * 64u == 63u && lane < 7)
        asm volatile("global_store_dwordx2 %0, %1, off sc0 sc1" :: "v"(uflagb + (pend - 1u) * 8u), "v"((v2u){call, call}) : "memory");
      pend = 0;
    }
  };

  unsigned t_cur = (unsigned)j, t_prev = 0;
  unsigned n = 0;
  while (true) {
    v4i dn;
    unsigned tick = 0, t_nxt = 0;
    const unsigned dst = dst0 + t_cur * 4096u;
    sfor<7>([&](auto qc) {
      constexpr int q = decltype(qc)::value;
      asm volatile("s_waitcnt vmcnt(%0)" :: "i"(COMM_W - 1) : "memory");
      if constexpr (PF_U < 8 && q == (2 * COMM_W - 2 <= 6 ? 2 * COMM_W - 2 : COMM_W - 1)) group_done();
      if constexpr (q == COMM_W - 1) { if (n > 0) publish(n - 1, t_prev); }
      if constexpr (LEADER && q == 0) {
        if (n > 0 && lane == 0) asm volatile("global_atomic_add %0, %1, %2, off sc0 sc1" : "+v"(tick) : "v"(ticketp), "v"(1u) : "memory");
      }
      if constexpr (q == 5) {
        if constexpr (LEADER) {
          asm volatile("" : "+v"(tick));
          t_nxt = n == 0 ? (unsigned)(NCOMM + j) : __builtin_amdgcn_readfirstlane(tick) - tbase;
          if (lane == 0) tk[n + 1] = t_nxt + 1u;
        } else {
          unsigned long long t0 = 0;
          while (true) {
            unsigned v = __hip_atomic_load(tk + n + 1, __ATOMIC_RELAXED, __HIP_MEMORY_SCOPE_WORKGROUP);
            v = __builtin_amdgcn_readfirstlane(v);
            if (v) { t_nxt = v - 1u; break; }
            unsigned long long now = __builtin_amdgcn_s_memrealtime();
            if (!t0) t0 = now;
            if (now - t0 > DEADLINE_TICKS) {
              __hip_atomic_store(tmo, call, __ATOMIC_RELAXED, __HIP_MEMORY_SCOPE_AGENT);
              t_nxt = NPIECE; break;
            }
            __builtin_amdgcn_s_sleep(1);
          }
        }
        if (t_nxt < NPIECE)
          asm volatile("buffer_load_dwordx4 %0, %1, %2, %3 offen" : "=v"(dn) : "v"(vo), "s"(rs_x),
                       "s"((t_nxt >> 6) * UNIT_BYTES + (t_nxt & 1u) * (32u * KDIM) + ((t_nxt >> 1) & 31u) * 128u) : "memory");
      }
      st_sys(d, vs, rs[q], dst);
    });
    if (t_nxt >= NPIECE) break;
    asm volatile("s_waitcnt vmcnt(2)" : "+v"(dn) :: "memory");
    t_prev = t_cur; t_cur = t_nxt; d = dn; ++n;
  }
  asm volatile("s_waitcnt vmcnt(0)" ::: "memory");
  if constexpr (PF_U < 8) group_done();
  publish(n, t_cur);
  if constexpr (PF_U < 8) {
    asm volatile("s_waitcnt vmcnt(0)" ::: "memory");
    group_done();
  }
}

__device__ __forceinline__ void comm_role(const Params& p, unsigned char* smem_generic) {
  LAS unsigned char* smem = (LAS unsigned char*)smem_generic;
  const int j = blockIdx.x;
  const int tid = threadIdx.x, lane = tid & 63;
  const int w = __builtin_amdgcn_readfirstlane(tid >> 6);
  unsigned char* myws = p.ws[p.rank];

  STAMP(myws, 0);
  v4i d;
  {
    rsrc_t rs_x = mk_rsrc(p.x, SRC_BYTES);
    const unsigned vo = (unsigned)((w >> 1) * 16 + (lane & 15)) * KDIM + (w & 1) * 64 + (lane >> 4) * 16;
    asm volatile("buffer_load_dwordx4 %0, %1, %2, %3 offen" : "=v"(d) : "v"(vo), "s"(rs_x), "s"((unsigned)((j & 1) * (32 * KDIM) + (j >> 1) * 128)) : "memory");
  }
  unsigned ticket = 0;
  if (lane == 0)
    asm volatile("global_atomic_add %0, %1, %2, off sc0" : "+v"(ticket) : "v"((unsigned*)(myws + CTRL_CALL + (j * 4 + w) * 32)), "v"(1u) : "memory");
  for (int i = tid; i < 520; i += 256) *(LAS unsigned*)(smem + 1024 + i * 4) = 0u;
  if (tid < 8) *(LAS unsigned*)(smem + LDS_MISC + 64 + tid * 4) = 0u;
  asm volatile("s_waitcnt lgkmcnt(0)\n\ts_barrier\n\ts_waitcnt vmcnt(0)" : "+v"(d), "+v"(ticket) :: "memory");
  const unsigned call = __builtin_amdgcn_readfirstlane(ticket) + 1u;
  if (w == 0) comm_wave<true>(p, smem, call, j, w, lane, d);
  else comm_wave<false>(p, smem, call, j, w, lane, d);
}

#define CWN 1
constexpr unsigned PGW = 18;
constexpr unsigned PTICKETS = NPIECE - 2 * PGW + PGW;
constexpr int CB = 128, TK = HD - 1;
template <int PAR> __device__ __forceinline__ void ld_piece(const unsigned (&vo)[4], rsrc_t rs_x, unsigned t) {
  const unsigned so = (t >> 6) * UNIT_BYTES + (t & 1u) * (32u * KDIM) + ((t >> 1) & 31u) * 128u;
  asm volatile("buffer_load_dwordx4 v[%c3:%c4], %0, %1, %2 offen" :: "v"(vo[0]), "s"(rs_x), "s"(so), "i"(CB + 16 * PAR), "i"(CB + 16 * PAR + 3) : "memory");
  asm volatile("buffer_load_dwordx4 v[%c3:%c4], %0, %1, %2 offen" :: "v"(vo[1]), "s"(rs_x), "s"(so), "i"(CB + 16 * PAR + 4), "i"(CB + 16 * PAR + 7) : "memory");
  asm volatile("buffer_load_dwordx4 v[%c3:%c4], %0, %1, %2 offen" :: "v"(vo[2]), "s"(rs_x), "s"(so), "i"(CB + 16 * PAR + 8), "i"(CB + 16 * PAR + 11) : "memory");
  asm volatile("buffer_load_dwordx4 v[%c3:%c4], %0, %1, %2 offen" :: "v"(vo[3]), "s"(rs_x), "s"(so), "i"(CB + 16 * PAR + 12), "i"(CB + 16 * PAR + 15) : "memory");
}
template <int PAR, int K> __device__ __forceinline__ void st_piece(unsigned vs, rsrc_t rs_d, unsigned dst) {
  asm volatile("buffer_store_dwordx4 v[%c3:%c4], %0, %1, %2 offen offset:%c5 sc0 sc1"
               :: "v"(vs), "s"(rs_d), "s"(dst), "i"(CB + 16 * PAR + 4 * K), "i"(CB + 16 * PAR + 4 * K + 3), "i"(K * 1024) : "memory");
}
#define TKPOL "sc0 sc1"
__device__ __forceinline__ void tk_issue(unsigned* ptr) {
  asm volatile("global_atomic_add v%c2, %0, %1, off " TKPOL :: "v"(ptr), "v"(1u), "i"(TK) : "memory");
}
__device__ __forceinline__ void dummy_ld(unsigned* ptr) {
  asm volatile("global_load_dword v%c1, %0, off sc0 sc1" :: "v"(ptr), "i"(HD) : "memory");
}
__device__ __forceinline__ unsigned tk_read() {
  unsigned t;
  asm volatile("s_waitcnt vmcnt(5)\n\tv_readfirstlane_b32 %0, v%c1" : "=s"(t) : "i"(TK) : "memory");
  return t;
}
__device__ __forceinline__ void st_flag(unsigned char* a, unsigned v) {
  asm volatile("global_store_dword %0, %1, off sc0 sc1" :: "v"(a), "v"(v) : "memory");
}
struct C2 {
  unsigned vo[4], vs, call, tbase, t0, t1, tp1, tp2;
  rsrc_t rs_x, rs_d;
  unsigned char* flagb;
  unsigned* ticketp;
  unsigned char* flagown;
  int lane;
};
__device__ __forceinline__ void publish2(C2& c, unsigned t) {
  if (c.lane == 0) st_flag(c.flagb + ((t >> 1) & 3u) * 512u + (t >> 3) * 8u + (t & 1u) * 4u, c.call);
}
template <int PAR> __device__ __forceinline__ bool comm_iter(C2& c) {
  constexpr int PER_IT = 10;
  const bool more = c.t1 < NPIECE;
  if (c.lane == 0) {
    if (more) tk_issue(c.ticketp);
    else dummy_ld(c.ticketp);
  }
  const unsigned dst = c.t0 * 4096u;
  const unsigned tpub = CWN == 1 ? c.tp1 : c.tp2;
  wait_vm<PER_IT * CWN - 1>(); st_piece<PAR, 0>(c.vs, c.rs_d, dst);
  wait_vm<PER_IT * CWN - 1>(); st_piece<PAR, 1>(c.vs, c.rs_d, dst);
  wait_vm<PER_IT * CWN - 1>(); st_piece<PAR, 2>(c.vs, c.rs_d, dst);
  wait_vm<PER_IT * CWN - 2>();
  if (tpub < PF_U * 64u) publish2(c, tpub);
  else if (c.lane == 0) dummy_ld(c.ticketp);
  st_piece<PAR, 3>(c.vs, c.rs_d, dst);
  const unsigned tk = tk_read();
  const unsigned t2 = more ? tk - c.tbase : NPIECE;
  ld_piece<PAR>(c.vo, c.rs_x, t2 < NPIECE ? t2 : 0u);
  c.tp2 = c.tp1; c.tp1 = c.t0; c.t0 = c.t1; c.t1 = t2;
  return c.t0 >= NPIECE;
}
__device__ __forceinline__ void comm_role2(const Params& p) {
  const int tid = threadIdx.x, lane = tid & 63;
  const int w = __builtin_amdgcn_readfirstlane(tid >> 6);
  const unsigned cw = blockIdx.x * 4 + w;
  const int rank = p.rank;
  unsigned char* myws = p.ws[rank];
  STAMP(myws, 0);
  if (cw >= 7 * PGW) return;
  const unsigned q = cw % 7u, gi = cw / 7u;
  unsigned char* pws = p.ws[(rank + 1 + q) & 7];
  C2 c;
  c.lane = lane;
  c.rs_x = mk_rsrc(p.x, SRC_BYTES);
#pragma unroll
  for (int k = 0; k < 4; ++k) c.vo[k] = (unsigned)((k >> 1) * 16 + (lane & 15)) * KDIM + (k & 1) * 64 + (lane >> 4) * 16;
  ld_piece<0>(c.vo, c.rs_x, gi);
  ld_piece<1>(c.vo, c.rs_x, PGW + gi);
  unsigned ticket = 0;
  if (lane == 0)
    asm volatile("global_atomic_add %0, %1, %2, off sc0" : "+v"(ticket) : "v"((unsigned*)(myws + CTRL_CALL + cw * 32)), "v"(1u) : "memory");
  asm volatile("s_waitcnt vmcnt(0)" : "+v"(ticket) :: "memory");
  c.call = __builtin_amdgcn_readfirstlane(ticket) + 1u;
  const unsigned cm = __builtin_amdgcn_readfirstlane(c.call % NSTAGE);
  c.rs_d = mk_rsrc(pws + cm * STAGE_BYTES + (unsigned)rank * SRC_BYTES, SRC_BYTES);
  c.flagb = pws + CTRL_FLAG + rank * 2048;
  c.ticketp = (unsigned*)(myws + CTRL_TICKET + q * 128);
  c.tbase = (c.call - 1u) * PTICKETS - 2u * PGW;
  c.flagown = myws + CTRL_CALL + cw * 32;
  c.vs = lane * 16;
  c.t0 = gi; c.t1 = PGW + gi; c.tp1 = NPIECE; c.tp2 = NPIECE;
  while (true) {
    if (comm_iter<0>(c)) break;
    if (comm_iter<1>(c)) break;
  }
  asm volatile("s_waitcnt vmcnt(0)" ::: "memory");
  if (CWN == 2 && c.tp2 < PF_U * 64u) publish2(c, c.tp2);
  if (c.tp1 < PF_U * 64u) publish2(c, c.tp1);
}

#define UF_END 1
__device__ __forceinline__ void publish_groups(unsigned char* ucnt, unsigned char* uflagb, unsigned call, int lane, unsigned nwg) {
  unsigned old = 0;
  if (lane == 0) asm volatile("global_atomic_add %0, %1, %2, off sc0 sc1\n\ts_waitcnt vmcnt(0)" : "+v"(old) : "v"(ucnt), "v"(1u) : "memory");
  old = __builtin_amdgcn_readfirstlane(old);
  if (old - (call - 1u) * nwg == nwg - 1u && lane < 7) {
#pragma unroll
    for (unsigned u = PF_U; u < 8u; ++u)
      asm volatile("global_store_dwordx2 %0, %1, off sc0 sc1" :: "v"(uflagb + u * 8u), "v"((v2u){call, call}) : "memory");
  }
}

#define T_FAST 512
#define C3_ROT 0
#define NACT 32
__device__ __forceinline__ void wait_vm_dyn(unsigned n) {
  switch (n) {
    case 0: wait_vm<0>(); break;
    case 1: wait_vm<1>(); break;
    case 2: wait_vm<2>(); break;
    case 3: wait_vm<3>(); break;
    case 4: wait_vm<4>(); break;
    case 5: wait_vm<5>(); break;
    case 6: wait_vm<6>(); break;
    default: wait_vm<7>(); break;
  }
}
__device__ __forceinline__ unsigned lds_wait_nz(LAS unsigned* a, unsigned want, GAS unsigned* tmo, unsigned call) {
  unsigned long long t0 = 0;
  while (true) {
    unsigned v = __hip_atomic_load(a, __ATOMIC_RELAXED, __HIP_MEMORY_SCOPE_WORKGROUP);
    v = __builtin_amdgcn_readfirstlane(v);
    if (v >= want) return v;
    unsigned long long now = __builtin_amdgcn_s_memrealtime();
    if (!t0) t0 = now;
    if (now - t0 > DEADLINE_TICKS) {
      __hip_atomic_store(tmo, call, __ATOMIC_RELAXED, __HIP_MEMORY_SCOPE_AGENT);
      return 0;
    }
    __builtin_amdgcn_s_sleep(1);
  }
}
#define C3_DIAG 512
constexpr unsigned TPD = C3_DIAG, UD = NPIECE - TPD;
static_assert(UD % 7u == 0u, "range of the permuted tickets");
constexpr int DB = CB + 16;
template <int PAR, int Q> __device__ __forceinline__ void ld_diag(unsigned vo, rsrc_t rs_x, unsigned so) {
  asm volatile("s_nop 4\n\tbuffer_load_dwordx4 v[%c3:%c4], %0, %1, %2 offen" :: "v"(vo), "s"(rs_x), "s"(so), "i"(DB + 28 * PAR + 4 * Q), "i"(DB + 28 * PAR + 4 * Q + 3) : "memory");
}
template <int PAR, int Q> __device__ __forceinline__ void st_diag(unsigned vs, unsigned lo, unsigned hi) {
  const v4i rs = {(int)lo, (int)hi, 1024, 0x00020000};
  asm volatile("s_nop 4\n\tbuffer_store_dwordx4 v[%c2:%c3], %0, %1, 0 offen sc0 sc1" :: "v"(vs), "s"(rs), "i"(DB + 28 * PAR + 4 * Q), "i"(DB + 28 * PAR + 4 * Q + 3) : "memory");
}
struct C3 {
  unsigned vo, vs, call, tbase, dst0, t_prev, t_cur, t_nxt, n, pend;
  unsigned blo, bhi, w;
  unsigned alo, ahi, nlo, nhi;
  unsigned char *uflagb, *ucnt;
  rsrc_t rs_x, rs[7];
  unsigned char* flagb;
  unsigned* ticketp;
  GAS unsigned* tmo;
  LAS unsigned *tk, *cnt;
  LAS unsigned char* smem;
  int lane;
};
template <int PAR> __device__ __forceinline__ void ld_chunk(unsigned vo, rsrc_t rs_x, unsigned t) {
  const unsigned so = (t >> 6) * UNIT_BYTES + (t & 1u) * (32u * KDIM) + ((t >> 1) & 31u) * 128u;
  asm volatile("buffer_load_dwordx4 v[%c3:%c4], %0, %1, %2 offen" :: "v"(vo), "s"(rs_x), "s"(so), "i"(CB + 4 * PAR), "i"(CB + 4 * PAR + 3) : "memory");
}
template <int PAR> __device__ __forceinline__ void st_chunk(unsigned vs, rsrc_t rs_d, unsigned dst) {
  asm volatile("buffer_store_dwordx4 v[%c3:%c4], %0, %1, %2 offen sc0 sc1" :: "v"(vs), "s"(rs_d), "s"(dst), "i"(CB + 4 * PAR), "i"(CB + 4 * PAR + 3) : "memory");
}
__device__ __forceinline__ void publish3(C3& c, unsigned n, unsigned t) {
  if (t >= PF_U * 64u) return;
  unsigned old = 0;
  if (c.lane == 0) old = __hip_atomic_fetch_add(c.cnt + n, 1u, __ATOMIC_RELAXED, __HIP_MEMORY_SCOPE_WORKGROUP);
  old = __builtin_amdgcn_readfirstlane(old);
  if (old == 3u) {
    if (t < PF_U * 64u) {
      if (c.lane < 7) st_flag(c.flagb + ((t >> 1) & 3u) * 512u + (t >> 3) * 8u + (t & 1u) * 4u, c.call);
    } else if (!UF_END) {
      if (c.lane == 0)
        asm volatile("global_atomic_add v%c2, %0, %1, off sc0 sc1" :: "v"(c.ucnt + (t >> 6) * 128u), "v"(1u), "i"(FD) : "memory");
      c.pend = (t >> 6) + 1u;
    }
  }
}
__device__ __forceinline__ void group_done3(C3& c) {
  if (c.pend) {
    unsigned o;
    asm volatile("v_readfirstlane_b32 %0, v%c1" : "=s"(o) : "i"(FD) : "memory");
    if (o - (c.call - 1u) * 64u == 63u && c.lane < 7)
      asm volatile("global_store_dwordx2 %0, %1, off sc0 sc1" :: "v"(c.uflagb + (c.pend - 1u) * 8u), "v"((v2u){c.call, c.call}) : "memory");
    c.pend = 0;
  }
}
template <bool LEADER, int PAR, bool CD, bool ND> __device__ __forceinline__ bool comm3_diag(C3& c) {
  constexpr int W = COMM_W, ET = LEADER && ND ? 1 : 0;
  static_assert(UD == 0 || (W < 7 && UF_END && !C3_FIX && TPD >= 2 * NACT && TPD >= PF_U * 64u), "window");
  unsigned t2 = NPIECE, so = 0;
  if constexpr (ET) { if (c.lane == 0) tk_issue(c.ticketp); }
  if constexpr (ND) {
    const int q = c.lane < 6 ? c.lane : 6;
    const int m = (int)(c.t_nxt - TPD) + q;
    const int a = (m * 9363) >> 16, v = m - 7 * a, sl = 2 * a + q;
    int r = 7 * (sl >> 1) + 4 * (sl & 1) + (q >> 1) - (v == 6 ? 18 : 25);
    r = r < 0 ? r + (int)UD : r;
    const unsigned pc = TPD + (unsigned)r;
    so = (pc >> 6) * UNIT_BYTES + (pc & 1u) * (32u * KDIM) + ((pc >> 1) & 31u) * 128u;
    int d = v + (int)c.w;
    d = d >= 7 ? d - 7 : d;
    const unsigned long long ad = (((unsigned long long)(unsigned)__builtin_amdgcn_ds_bpermute(d * 4, (int)c.bhi) << 32) | (unsigned)__builtin_amdgcn_ds_bpermute(d * 4, (int)c.blo)) + pc * 4096u;
    c.nlo = (unsigned)ad; c.nhi = (unsigned)(ad >> 32) & 0xffffu;
  }
  sfor<7>([&](auto qc) {
    constexpr int q = decltype(qc)::value;
    unsigned lo = 0, hi = 0, sq = 0;
    if constexpr (CD) { lo = __builtin_amdgcn_readlane(c.alo, q); hi = __builtin_amdgcn_readlane(c.ahi, q); }
    if constexpr (ND) sq = __builtin_amdgcn_readlane(so, q);
    constexpr int prev = q < W ? (W - 1 - q) + (CD ? W - q : 0) + ET : 0;
    constexpr int ns = q < W ? q : W - 1, nl = ND ? (q < W ? q : W) : 0;
    wait_vm<prev + ns + nl>();
    if constexpr (q == W - 1) { if (c.n > 0) publish3(c, c.n - 1, c.t_prev); }
    if constexpr (ET && q == W) {
      unsigned t;
      asm volatile("v_readfirstlane_b32 %0, v%c1" : "=s"(t) : "i"(TK) : "memory");
      t -= c.tbase;
      t2 = t < NPIECE ? t : NPIECE;
      if (c.lane == 0) c.tk[c.n + 2] = t2 + 1u;
    }
    if constexpr (CD) st_diag<PAR, q>(c.vs, lo, hi);
    else st_chunk<PAR>(c.vs, c.rs[q], c.dst0 + c.t_cur * 4096u);
    if constexpr (ND) ld_diag<PAR ^ 1, q>(c.vo, c.rs_x, sq);
  });
  if constexpr (!ND) return true;
  if constexpr (!LEADER) {
    const unsigned v = lds_wait_nz(c.tk + c.n + 2, 1u, c.tmo, c.call);
    t2 = v ? v - 1u : NPIECE;
  }
  c.t_prev = c.t_cur; c.t_cur = c.t_nxt; c.t_nxt = t2; c.n += 1u; c.alo = c.nlo; c.ahi = c.nhi;
  return false;
}
template <bool LEADER, int PAR> __device__ __forceinline__ bool comm3_round(C3& c) {
  constexpr int EX = LEADER ? 2 : 1, W = COMM_W;
  const bool more = c.t_nxt < NPIECE;
  if constexpr (UD != 0) {
    if (c.t_cur >= TPD) return more ? comm3_diag<LEADER, PAR, true, true>(c) : comm3_diag<LEADER, PAR, true, false>(c);
    if (more && c.t_nxt >= TPD) return comm3_diag<LEADER, PAR, false, true>(c);
  }
  unsigned t2 = NPIECE;
  auto ticket_done = [&]() {
    unsigned t;
    asm volatile("v_readfirstlane_b32 %0, v%c1" : "=s"(t) : "i"(TK) : "memory");
    t -= c.tbase;
    t2 = more && t < NPIECE ? t : NPIECE;
    if (c.lane == 0) c.tk[c.n + 2] = t2 + 1u;
  };
  if constexpr (LEADER) {
    if (c.lane == 0) {
      if (more) tk_issue(c.ticketp);
      else dummy_ld(c.ticketp);
    }
  }
  ld_chunk<PAR ^ 1>(c.vo, c.rs_x, more ? c.t_nxt : 0u);
  const unsigned dst = c.dst0 + c.t_cur * 4096u;
  if (c.t_cur < (unsigned)T_FAST) {
    sfor<7>([&](auto qc) {
      constexpr int q = decltype(qc)::value;
      wait_vm<(q < W ? W - 1 + EX : W - 1)>();
      if constexpr (PF_U < 8 && !UF_END && q == (2 * W - 2 <= 6 ? 2 * W - 2 : W - 1)) group_done3(c);
      if constexpr (q == W - 1) { if (c.n > 0) publish3(c, c.n - 1, c.t_prev); }
      if constexpr (LEADER && q == W) ticket_done();
      st_chunk<PAR>(c.vs, c.rs[q], dst);
    });
    if constexpr (LEADER && W == 7) { wait_vm<7>(); ticket_done(); }
  } else {
    sfor<7>([&](auto qc) { st_chunk<PAR>(c.vs, c.rs[decltype(qc)::value], dst); });
    wait_vm<7 + EX>();
    if constexpr (PF_U < 8 && !UF_END) group_done3(c);
    if (c.n > 0) publish3(c, c.n - 1, c.t_prev);
    if constexpr (LEADER) { wait_vm<8>(); ticket_done(); }
  }
  if (!more) return true;
  if constexpr (!LEADER) {
    unsigned long long t0 = 0;
    while (true) {
      unsigned v = __hip_atomic_load(c.tk + c.n + 2, __ATOMIC_RELAXED, __HIP_MEMORY_SCOPE_WORKGROUP);
      v = __builtin_amdgcn_readfirstlane(v);
      if (v) { t2 = v - 1u; break; }
      unsigned long long now = __builtin_amdgcn_s_memrealtime();
      if (!t0) t0 = now;
      if (now - t0 > DEADLINE_TICKS) {
        __hip_atomic_store(c.tmo, c.call, __ATOMIC_RELAXED, __HIP_MEMORY_SCOPE_AGENT);
        t2 = NPIECE; break;
      }
      __builtin_amdgcn_s_sleep(1);
    }
  }
  wait_vm<7>();
  c.t_prev = c.t_cur; c.t_cur = c.t_nxt; c.t_nxt = t2; c.n += 1u;
  return false;
}
template <bool LEADER> __device__ __forceinline__ void comm3_wave(C3& c) {
  while (true) {
    if (comm3_round<LEADER, 0>(c)) break;
    if (comm3_round<LEADER, 1>(c)) break;
  }
  wait_vm<0>();
  if constexpr (PF_U < 8 && !UF_END) group_done3(c);
  publish3(c, c.n, c.t_cur);
  if constexpr (PF_U < 8 && !UF_END) { wait_vm<0>(); group_done3(c); }
  if constexpr (PF_U < 8 && UF_END) {
    unsigned old = 0;
    if (c.lane == 0) old = __hip_atomic_fetch_add(c.cnt + 519, 1u, __ATOMIC_RELAXED, __HIP_MEMORY_SCOPE_WORKGROUP);
    if (__builtin_amdgcn_readfirstlane(old) == 3u) publish_groups(c.ucnt, c.uflagb, c.call, c.lane, NACT);
  }
}
__device__ __forceinline__ void comm_role3(const Params& p, unsigned char* smem_generic) {
  LAS unsigned char* smem = (LAS unsigned char*)smem_generic;
  const int j = blockIdx.x;
  const int tid = threadIdx.x, lane = tid & 63;
  const int w = __builtin_amdgcn_readfirstlane(tid >> 6);
  const int rank = p.rank;
  unsigned char* myws = p.ws[rank];
  STAMP(myws, 0);
  asm volatile("" ::: "v255");
  if (j >= NACT) return;
  C3 c;
  c.lane = lane;
  c.smem = smem;
  c.rs_x = mk_rsrc(p.x, SRC_BYTES);
  c.vo = (unsigned)((w >> 1) * 16 + (lane & 15)) * KDIM + (w & 1) * 64 + (lane >> 4) * 16;
  ld_chunk<0>(c.vo, c.rs_x, (unsigned)j);
  unsigned ticket = 0;
  if (lane == 0)
    asm volatile("global_atomic_add %0, %1, %2, off sc0" : "+v"(ticket) : "v"((unsigned*)(myws + CTRL_CALL + (j * 4 + w) * 32)), "v"(1u) : "memory");
  for (int i = tid; i < 2 * 520; i += 256) *(LAS unsigned*)(smem + 1024 + i * 4) = 0u;
  c.tk = (LAS unsigned*)(smem + 1024);
  c.cnt = (LAS unsigned*)(smem + 1024 + 520 * 4);
#pragma unroll
  for (int q = 0; q < 7; ++q) {
    const int pq = (rank + 1 + (C3_FIX ? q : (q + (C3_ROT ? j : j * 4 + w)) % 7)) & 7;
    c.rs[q] = mk_rsrc(p.ws[pq], NSTAGE * STAGE_BYTES);
  }
  c.flagb = p.ws[(rank + 1 + (lane < 7 ? lane : 0)) & 7] + CTRL_FLAG + rank * 2048;
  c.ticketp = (unsigned*)(myws + CTRL_TICKET);
  c.tmo = (GAS unsigned*)(myws + CTRL_TMO);
  c.vs = lane * 16;
  asm volatile("s_waitcnt lgkmcnt(0)\n\ts_barrier\n\ts_waitcnt vmcnt(0)" : "+v"(ticket) :: "memory");
  c.call = __builtin_amdgcn_readfirstlane(ticket) + 1u;
  c.tbase = (c.call - 1u) * (NPIECE - NACT) - 2u * NACT;
  c.pend = 0;
  c.uflagb = p.ws[(rank + 1 + (lane < 7 ? lane : 0)) & 7] + CTRL_UFLAG + rank * 64;
  c.ucnt = myws + CTRL_UCNT;
  c.dst0 = __builtin_amdgcn_readfirstlane(c.call % NSTAGE) * STAGE_BYTES + (unsigned)rank * SRC_BYTES + (unsigned)w * 1024u;
  c.t_prev = 0; c.t_cur = (unsigned)j; c.t_nxt = (unsigned)(NACT + j); c.n = 0;
  {
    const unsigned long long a = (unsigned long long)(p.ws[(rank + 1 + (lane < 7 ? lane : 0)) & 7] + (c.call % NSTAGE) * STAGE_BYTES + (unsigned)rank * SRC_BYTES + (unsigned)w * 1024u);
    c.blo = (unsigned)a; c.bhi = (unsigned)(a >> 32);
    c.alo = c.ahi = c.nlo = c.nhi = 0; c.w = (unsigned)w;
  }
  if (w == 0) comm3_wave<true>(c);
  else comm3_wave<false>(c);
}

constexpr unsigned NTK = NPIECE * 7u / 4u, NTK_U = NTK / 8u;
struct C4 {
  unsigned vo[4], vs, call, tbase, T_cur, T_nxt, t_prev, q_prev, n, pend, tk_out;
  unsigned blo, bhi;
  unsigned char *uflagb, *ucnt, *flagb;
  rsrc_t rs_x;
  unsigned* ticketp;
  GAS unsigned* tmo;
  LAS unsigned *tk, *cnt;
  LAS unsigned char* smem;
  unsigned long long* dbg;
  int lane, w;
};
__device__ __forceinline__ void group_done4(C4& c) {
  if (c.pend) {
    unsigned o;
    asm volatile("v_readfirstlane_b32 %0, v%c1" : "=s"(o) : "i"(FD) : "memory");
    if (o - (c.call - 1u) * NTK_U == NTK_U - 1u && c.lane < 7)
      asm volatile("global_store_dwordx2 %0, %1, off sc0 sc1" :: "v"(c.uflagb + (c.pend - 1u) * 8u), "v"((v2u){c.call, c.call}) : "memory");
    c.pend = 0;
  }
}
__device__ __forceinline__ void publish4(C4& c, unsigned n, unsigned t, unsigned q) {
  if (t < PF_U * 64u) {
    if (c.lane == (int)q) st_flag(c.flagb + ((t >> 1) & 3u) * 512u + (t >> 3) * 8u + (t & 1u) * 4u, c.call);
  } else {
    unsigned old = 0;
    if (c.lane == 0) old = __hip_atomic_fetch_add(c.cnt + n, 1u, __ATOMIC_RELAXED, __HIP_MEMORY_SCOPE_WORKGROUP);
    old = __builtin_amdgcn_readfirstlane(old);
    if (old == 3u) {
      if (c.lane == 0)
        asm volatile("global_atomic_add v%c2, %0, %1, off sc0 sc1" :: "v"(c.ucnt + (t >> 6) * 128u), "v"(1u), "i"(FD) : "memory");
      c.pend = (t >> 6) + 1u;
    }
  }
}
template <int PAR, int X> __device__ __forceinline__ void comm4_stores(C4& c, rsrc_t rs_d, unsigned dst) {
  constexpr int W = COMM_W;
  st_piece<PAR, 0>(c.vs, rs_d, dst);
  wait_vm<W - 1 + X>();
  st_piece<PAR, 1>(c.vs, rs_d, dst);
  wait_vm<W - 1 + X>();
  st_piece<PAR, 2>(c.vs, rs_d, dst);
  wait_vm<W - 1 + X>();
  group_done4(c);
  if (c.n > 0) publish4(c, c.n - 1, c.t_prev, c.q_prev);
  st_piece<PAR, 3>(c.vs, rs_d, dst);
}
template <bool LEADER, int PAR> __device__ __forceinline__ bool comm4_round(C4& c) {
  static_assert(COMM_MODE != 4 || COMM_W == 4, "window");
  wait_vm<3>();
  unsigned T2 = NTK;
  bool tk_now = false;
  if constexpr (LEADER) {
    if (c.n == 0) T2 = 2u * NACT + blockIdx.x;
    else if (c.tk_out) {
      unsigned t;
      asm volatile("v_readfirstlane_b32 %0, v%c1" : "=s"(t) : "i"(TK) : "memory");
      t -= c.tbase;
      T2 = t < NTK ? t : NTK;
    }
    if (c.lane == 0) c.tk[c.n + 2] = T2 + 1u;
    tk_now = T2 < NTK;
    if (tk_now && c.lane == 0) tk_issue(c.ticketp);
    c.tk_out = tk_now ? 1u : 0u;
  }
  const bool more = c.T_nxt < NTK;
  const unsigned tau = c.T_cur * 4u + (unsigned)c.w, t = (tau * 9363u) >> 16, q = tau - 7u * t;
  const unsigned lo = __builtin_amdgcn_readlane(c.blo, q), hi = __builtin_amdgcn_readlane(c.bhi, q);
  const rsrc_t rs_d = mk_rsrc((void*)(((unsigned long long)hi << 32) | lo), SRC_BYTES);
  if (more) {
    const unsigned taun = c.T_nxt * 4u + (unsigned)c.w;
    ld_piece<PAR ^ 1>(c.vo, c.rs_x, (taun * 9363u) >> 16);
    if (LEADER && tk_now) comm4_stores<PAR, 5>(c, rs_d, t * 4096u);
    else comm4_stores<PAR, 4>(c, rs_d, t * 4096u);
  } else {
    comm4_stores<PAR, 0>(c, rs_d, t * 4096u);
  }
  c.t_prev = t; c.q_prev = q;
  if (!more) return true;
  if constexpr (!LEADER) {
    unsigned long long t0 = 0;
    while (true) {
      unsigned v = __hip_atomic_load(c.tk + c.n + 2, __ATOMIC_RELAXED, __HIP_MEMORY_SCOPE_WORKGROUP);
      v = __builtin_amdgcn_readfirstlane(v);
      if (v) { T2 = v - 1u; break; }
      unsigned long long now = __builtin_amdgcn_s_memrealtime();
      if (!t0) t0 = now;
      if (now - t0 > DEADLINE_TICKS) {
        __hip_atomic_store(c.tmo, c.call, __ATOMIC_RELAXED, __HIP_MEMORY_SCOPE_AGENT);
        T2 = NTK; break;
      }
      __builtin_amdgcn_s_sleep(1);
    }
  }
  c.T_cur = c.T_nxt; c.T_nxt = T2; c.n += 1u;
  return false;
}
template <bool LEADER> __device__ __forceinline__ void comm4_wave(C4& c) {
  while (true) {
    if (comm4_round<LEADER, 0>(c)) break;
    if (comm4_round<LEADER, 1>(c)) break;
  }
  wait_vm<0>();
  group_done4(c);
  publish4(c, c.n, c.t_prev, c.q_prev);
  wait_vm<0>();
  group_done4(c);
}
__device__ __forceinline__ void comm_role4(const Params& p, unsigned char* smem_generic) {
  LAS unsigned char* smem = (LAS unsigned char*)smem_generic;
  const int j = blockIdx.x;
  const int tid = threadIdx.x, lane = tid & 63;
  const int w = __builtin_amdgcn_readfirstlane(tid >> 6);
  const int rank = p.rank;
  unsigned char* myws = p.ws[rank];
  STAMP(myws, 0);
  asm volatile("" ::: "v255");
  C4 c;
  c.lane = lane; c.w = w;
  c.smem = smem;
  c.rs_x = mk_rsrc(p.x, SRC_BYTES);
#pragma unroll
  for (int k = 0; k < 4; ++k) c.vo[k] = (unsigned)((k >> 1) * 16 + (lane & 15)) * KDIM + (k & 1) * 64 + (lane >> 4) * 16;
  ld_piece<0>(c.vo, c.rs_x, (((unsigned)j * 4u + (unsigned)w) * 9363u) >> 16);
  unsigned ticket = 0;
  if (lane == 0)
    asm volatile("global_atomic_add %0, %1, %2, off sc0" : "+v"(ticket) : "v"((unsigned*)(myws + CTRL_CALL + (j * 4 + w) * 32)), "v"(1u) : "memory");
  for (int i = tid; i < 2048; i += 256) *(LAS unsigned*)(smem + 32768 + i * 4) = 0u;
  c.dbg = nullptr;
  c.tk = (LAS unsigned*)(smem + 32768);
  c.cnt = (LAS unsigned*)(smem + 32768 + 4096);
  unsigned char* pw = p.ws[(rank + 1 + (lane < 7 ? lane : 0)) & 7];
  c.flagb = pw + CTRL_FLAG + rank * 2048;
  c.uflagb = pw + CTRL_UFLAG + rank * 64;
  c.ucnt = myws + CTRL_UCNT;
  c.ticketp = (unsigned*)(myws + CTRL_TICKET);
  c.tmo = (GAS unsigned*)(myws + CTRL_TMO);
  c.vs = lane * 16;
  asm volatile("s_waitcnt lgkmcnt(0)\n\ts_barrier\n\ts_waitcnt vmcnt(0)" : "+v"(ticket) :: "memory");
  c.call = __builtin_amdgcn_readfirstlane(ticket) + 1u;
  c.tbase = (c.call - 1u) * (NTK - 2u * NACT) - 3u * NACT;
  c.pend = 0; c.tk_out = 0;
  {
    const unsigned long long a = (unsigned long long)(pw + (c.call % NSTAGE) * STAGE_BYTES + (unsigned)rank * SRC_BYTES);
    c.blo = (unsigned)a; c.bhi = (unsigned)(a >> 32);
  }
  c.t_prev = 0; c.q_prev = 0; c.T_cur = (unsigned)j; c.T_nxt = (unsigned)(NACT + j); c.n = 0;
  if (w == 0) comm4_wave<true>(c);
  else comm4_wave<false>(c);
}

#define C6_W 5
#define C6_LD 7
constexpr unsigned NTASK = NPIECE * 7u, NTK6 = (NTASK + 2u) / 3u, C6_LEAD = C6_LD;
constexpr unsigned TK6_PER_CALL = NTK6 - C6_LEAD * NCOMM + 2u * NCOMM;
struct C6 {
  unsigned vo[4], vs, call, T0, T1, T2, n, sig;
  unsigned blo, bhi;
  rsrc_t rs_x;
  GAS unsigned* tmo;
  LAS unsigned *tk, *cnt;
  int lane, w;
};
template <int B, int K> __device__ __forceinline__ void ld_chunk6(unsigned vo, rsrc_t rs_x, unsigned so) {
  asm volatile("buffer_load_dwordx4 v[%c3:%c4], %0, %1, %2 offen" :: "v"(vo), "s"(rs_x), "s"(so), "i"(CB + 16 * B + 4 * K), "i"(CB + 16 * B + 4 * K + 3) : "memory");
}
template <int B> __device__ __forceinline__ void ld_piece6(const unsigned (&vo)[4], rsrc_t rs_x, unsigned t) {
  const unsigned so = (t >> 6) * UNIT_BYTES + (t & 1u) * (32u * KDIM) + ((t >> 1) & 31u) * 128u;
  ld_chunk6<B, 0>(vo[0], rs_x, so);
  ld_chunk6<B, 1>(vo[1], rs_x, so);
  ld_chunk6<B, 2>(vo[2], rs_x, so);
  ld_chunk6<B, 3>(vo[3], rs_x, so);
}
template <int B, int K> __device__ __forceinline__ void st_chunk6(unsigned vs, rsrc_t rs_d, unsigned dst) {
  asm volatile("buffer_store_dwordx4 v[%c3:%c4], %0, %1, %2 offen offset:%c5 sc0 sc1"
               :: "v"(vs), "s"(rs_d), "s"(dst), "i"(CB + 16 * B + 4 * K), "i"(CB + 16 * B + 4 * K + 3), "i"(K * 1024) : "memory");
}
constexpr int c6_cnt(int k) { return C6_W - 1 + 4 * (1 + (C6_W - k - 1) / 4 - (C6_W <= k ? 1 : 0)); }
__device__ __forceinline__ unsigned c6_task(unsigned T, int i) { const unsigned tau = T * 3u + (unsigned)i; return tau < NTASK ? tau : NTASK - 1u; }
template <int B> __device__ __forceinline__ bool comm6_round(C6& c) {
  constexpr int KS = (3 + C6_W) % 4, DR = (3 + C6_W) / 4;
  const unsigned tau = c6_task(c.T0, c.w - 1), t = (tau * 9363u) >> 16, q = tau - 7u * t;
  const unsigned lo = __builtin_amdgcn_readlane(c.blo, q), hi = __builtin_amdgcn_readlane(c.bhi, q);
  const rsrc_t rs_d = mk_rsrc((void*)(((unsigned long long)hi << 32) | lo), SRC_BYTES);
  ld_piece6<(B + 2) % 3>(c.vo, c.rs_x, (c6_task(c.T2 < NTK6 ? c.T2 : c.T0, c.w - 1) * 9363u) >> 16);
  sfor<4>([&](auto kc) {
    constexpr int k = decltype(kc)::value;
    wait_vm<c6_cnt(k)>();
    if constexpr (k == KS) {
      if (c.n >= (unsigned)DR) {
        if (c.lane == 0) __hip_atomic_fetch_add(c.cnt + c.sig, 1u, __ATOMIC_RELAXED, __HIP_MEMORY_SCOPE_WORKGROUP);
        c.sig += 1u;
      }
    }
    st_chunk6<B, k>(c.vs, rs_d, t * 4096u);
  });
  if (c.T1 >= NTK6) return true;
  unsigned T3 = NTK6;
  if (c.T2 < NTK6) {
    const unsigned v = lds_wait_nz(c.tk + c.n + 3, 1u, c.tmo, c.call);
    T3 = v ? v - 1u : NTK6;
  }
  c.T0 = c.T1; c.T1 = c.T2; c.T2 = T3; c.n += 1u;
  return false;
}
__device__ __forceinline__ void comm_role6(const Params& p, unsigned char* smem_generic) {
  LAS unsigned char* smem = (LAS unsigned char*)smem_generic;
  const int j = blockIdx.x;
  const int tid = threadIdx.x, lane = tid & 63;
  const int w = __builtin_amdgcn_readfirstlane(tid >> 6);
  const int rank = p.rank;
  unsigned char* myws = p.ws[rank];
  STAMP(myws, 0);
  asm volatile("" ::: "v255");
  C6 c;
  c.lane = lane; c.w = w;
  c.rs_x = mk_rsrc(p.x, SRC_BYTES);
#pragma unroll
  for (int k = 0; k < 4; ++k) c.vo[k] = (unsigned)((k >> 1) * 16 + (lane & 15)) * KDIM + (k & 1) * 64 + (lane >> 4) * 16;
  if (w > 0) {
    ld_piece6<0>(c.vo, c.rs_x, (c6_task((unsigned)j, w - 1) * 9363u) >> 16);
    ld_piece6<1>(c.vo, c.rs_x, (c6_task((unsigned)(NCOMM + j), w - 1) * 9363u) >> 16);
  }
  unsigned ticket = 0;
  if (lane == 0)
    asm volatile("global_atomic_add %0, %1, %2, off sc0" : "+v"(ticket) : "v"((unsigned*)(myws + CTRL_CALL + (j * 4 + w) * 32)), "v"(1u) : "memory");
  LAS unsigned* tk = (LAS unsigned*)(smem + 32768);
  LAS unsigned* cnt = (LAS unsigned*)(smem + 32768 + 8192);
  for (int i = tid; i < 4096; i += 256) tk[i] = i < (int)C6_LEAD ? (unsigned)(i * NCOMM + j + 1) : 0u;
  unsigned char* pw = p.ws[(rank + 1 + (lane < 7 ? lane : 0)) & 7];
  GAS unsigned* tmo = (GAS unsigned*)(myws + CTRL_TMO);
  asm volatile("s_waitcnt lgkmcnt(0)\n\ts_barrier\n\ts_waitcnt vmcnt(0)" : "+v"(ticket) :: "memory");
  const unsigned call = __builtin_amdgcn_readfirstlane(ticket) + 1u;
  if (w > 0) {
    c.call = call; c.tmo = tmo; c.tk = tk; c.cnt = cnt;
    c.vs = lane * 16;
    const unsigned long long a = (unsigned long long)(pw + (call % NSTAGE) * STAGE_BYTES + (unsigned)rank * SRC_BYTES);
    c.blo = (unsigned)a; c.bhi = (unsigned)(a >> 32);
    c.T0 = (unsigned)j; c.T1 = (unsigned)(NCOMM + j); c.T2 = (unsigned)(2 * NCOMM + j); c.n = 0; c.sig = 0;
    while (true) {
      if (comm6_round<0>(c)) break;
      if (comm6_round<1>(c)) break;
      if (comm6_round<2>(c)) break;
    }
    wait_vm<0>();
    for (; c.sig <= c.n; ++c.sig)
      if (lane == 0) __hip_atomic_fetch_add(cnt + c.sig, 1u, __ATOMIC_RELAXED, __HIP_MEMORY_SCOPE_WORKGROUP);
  } else {
    unsigned char* flagb = pw + CTRL_FLAG + rank * 2048;
    unsigned char* uflagb = pw + CTRL_UFLAG + rank * 64;
    unsigned char* ucnt = myws + CTRL_UCNT;
    unsigned* ticketp = (unsigned*)(myws + CTRL_TICKET);
    const unsigned tbase = (call - 1u) * TK6_PER_CALL - C6_LEAD * NCOMM;
    bool drawing = true;
    unsigned young = 0;
    for (unsigned i = 0;; ++i) {
      const unsigned T = __builtin_amdgcn_readfirstlane(__hip_atomic_load(tk + i, __ATOMIC_RELAXED, __HIP_MEMORY_SCOPE_WORKGROUP)) - 1u;
      if (T >= NTK6) break;
      if (lds_wait_nz(cnt + i, 3u, tmo, call) == 0u) break;
      unsigned nf = 0;
      sfor<3>([&](auto ic) {
        const unsigned tau = T * 3u + decltype(ic)::value;
        const unsigned t = (tau * 9363u) >> 16, q = tau - 7u * t;
        if (tau < NTASK && t < PF_U * 64u) {
          if (lane == (int)q) st_flag(flagb + ((t >> 1) & 3u) * 512u + (t >> 3) * 8u + (t & 1u) * 4u, call);
          nf += 1u;
        }
      });
      if (drawing) {
        if (i >= 2u) {
          wait_vm_dyn(young + nf);
          unsigned v;
          if (i & 1u) asm volatile("v_readfirstlane_b32 %0, v%c1" : "=s"(v) : "i"(HD) : "memory");
          else asm volatile("v_readfirstlane_b32 %0, v%c1" : "=s"(v) : "i"(TK) : "memory");
          v -= tbase;
          drawing = v < NTK6;
          if (lane == 0) tk[i - 2u + C6_LEAD] = (drawing ? v : NTK6) + 1u;
        }
        if (drawing && lane == 0) {
          if (i & 1u) asm volatile("global_atomic_add v%c2, %0, %1, off sc0 sc1" :: "v"(ticketp), "v"(1u), "i"(HD) : "memory");
          else asm volatile("global_atomic_add v%c2, %0, %1, off sc0 sc1" :: "v"(ticketp), "v"(1u), "i"(TK) : "memory");
        }
        young = nf + 1u;
      }
    }
    wait_vm<0>();
    if constexpr (PF_U < 8) publish_groups(ucnt, uflagb, call, lane, NCOMM);
  }
}

constexpr unsigned C7_STATIC = PF_U * 64u / NCOMM < 2u ? 2u : PF_U * 64u / NCOMM;
constexpr unsigned C7_TK = NPIECE - C7_STATIC * NCOMM + NCOMM;
struct C7 {
  unsigned vo, vs, call, tbase, dst0, t_prev, t_cur, t_nxt, n;
  rsrc_t rs_x, rs[7];
  unsigned char* flagb;
  unsigned* ticketp;
  LAS unsigned* cnt;
  LAS unsigned char* smem;
  unsigned long long* dbg;
  int lane, w;
};
__device__ __forceinline__ void publish7(C7& c, unsigned n, unsigned t) {
  if (t < PF_U * 64u) {
    unsigned old = 0;
    if (c.lane == 0) old = __hip_atomic_fetch_add(c.cnt + n, 1u, __ATOMIC_RELAXED, __HIP_MEMORY_SCOPE_WORKGROUP);
    old = __builtin_amdgcn_readfirstlane(old);
    if (old == 3u && c.lane < 7) st_flag(c.flagb + ((t >> 1) & 3u) * 512u + (t >> 3) * 8u + (t & 1u) * 4u, c.call);
  }
}
template <bool TKT, int PAR> __device__ __forceinline__ unsigned comm7_stores(C7& c) {
  constexpr int EX = TKT ? 2 : 1, W = COMM_W;
  unsigned t2 = NPIECE;
  auto ticket_done = [&]() {
    unsigned t;
    asm volatile("v_readfirstlane_b32 %0, v%c1" : "=s"(t) : "i"(TK) : "memory");
    t -= c.tbase;
    t2 = t < NPIECE ? t : NPIECE;
  };
  if constexpr (TKT) { if (c.lane == 0) tk_issue(c.ticketp); }
  ld_chunk<PAR ^ 1>(c.vo, c.rs_x, c.t_nxt < NPIECE ? c.t_nxt : 0u);
  const unsigned dst = c.dst0 + c.t_cur * 4096u;
  const unsigned nq = c.t_cur < NPIECE - 32u ? 7u : 7u - (((c.t_cur - (NPIECE - 32u)) * 7u) >> 5);
  if (nq < 7u) {
    sfor<6>([&](auto qc) {
      constexpr int q = decltype(qc)::value;
      if ((unsigned)q < nq) {
        wait_vm<(q < W ? W - 1 + EX : W - 1)>();
        st_chunk<PAR>(c.vs, c.rs[q], dst);
      }
    });
    wait_vm_dyn(nq);
    if (c.n > 0) publish7(c, c.n - 1, c.t_prev);
    if constexpr (TKT) ticket_done();
  } else {
    sfor<7>([&](auto qc) {
      constexpr int q = decltype(qc)::value;
      wait_vm<(q < W ? W - 1 + EX : W - 1)>();
      if constexpr (q == W - 1) { if (c.n > 0) publish7(c, c.n - 1, c.t_prev); }
      if constexpr (TKT && q == W) ticket_done();
      st_chunk<PAR>(c.vs, c.rs[q], dst);
    });
    wait_vm<7>();
  }
  return t2;
}
template <int PAR> __device__ __forceinline__ bool comm7_round(C7& c) {
  const bool more = c.t_nxt < NPIECE;
  unsigned t2;
  if (more && c.n + 2u >= C7_STATIC) t2 = comm7_stores<true, PAR>(c);
  else {
    comm7_stores<false, PAR>(c);
    t2 = more ? (c.n + 2u) * NCOMM + blockIdx.x : NPIECE;
  }
  if (!more) return true;
  c.t_prev = c.t_cur; c.t_cur = c.t_nxt; c.t_nxt = t2; c.n += 1u;
  return false;
}
__device__ __forceinline__ void comm_role7(const Params& p, unsigned char* smem_generic) {
  static_assert(COMM_MODE != 7 || (COMM_W <= 6 && C3_FIX), "window");
  LAS unsigned char* smem = (LAS unsigned char*)smem_generic;
  const int j = blockIdx.x;
  const int tid = threadIdx.x, lane = tid & 63;
  const int w = __builtin_amdgcn_readfirstlane(tid >> 6);
  const int rank = p.rank;
  unsigned char* myws = p.ws[rank];
  STAMP(myws, 0);
  asm volatile("" ::: "v255");
  C7 c;
  c.lane = lane; c.w = w;
  c.smem = smem;
  c.rs_x = mk_rsrc(p.x, SRC_BYTES);
  c.vo = (unsigned)((w >> 1) * 16 + (lane & 15)) * KDIM + (w & 1) * 64 + (lane >> 4) * 16;
  const unsigned ia = (unsigned)(j + 8 * w) & 31u, ta = NPIECE - 32u + ia, ca = (ia * 7u) >> 5;
  ld_chunk<2>(c.vo, c.rs_x, ta);
  ld_chunk<0>(c.vo, c.rs_x, (unsigned)j);
  unsigned ticket = 0;
  if (lane == 0)
    asm volatile("global_atomic_add %0, %1, %2, off sc0" : "+v"(ticket) : "v"((unsigned*)(myws + CTRL_CALL + (j * 4 + w) * 32)), "v"(1u) : "memory");
  if (tid < 64) *(LAS unsigned*)(smem + 1024 + tid * 4) = 0u;
  c.dbg = nullptr;
  c.cnt = (LAS unsigned*)(smem + 1024);
#pragma unroll
  for (int q = 0; q < 7; ++q) c.rs[q] = mk_rsrc(p.ws[(rank + 1 + q) & 7], NSTAGE * STAGE_BYTES);
  unsigned char* pw = p.ws[(rank + 1 + (lane < 7 ? lane : 0)) & 7];
  c.flagb = pw + CTRL_FLAG + rank * 2048;
  c.ticketp = (unsigned*)(myws + CTRL_TICKET + w * 128);
  c.vs = lane * 16;
  asm volatile("s_waitcnt lgkmcnt(0)\n\ts_barrier\n\ts_waitcnt vmcnt(0)" : "+v"(ticket) :: "memory");
  c.call = __builtin_amdgcn_readfirstlane(ticket) + 1u;
  c.tbase = (c.call - 1u) * C7_TK - C7_STATIC * NCOMM;
  c.dst0 = __builtin_amdgcn_readfirstlane(c.call % NSTAGE) * STAGE_BYTES + (unsigned)rank * SRC_BYTES + (unsigned)w * 1024u;
  c.t_prev = 0; c.t_cur = (unsigned)j; c.t_nxt = (unsigned)(NCOMM + j); c.n = 0;
  sfor<6>([&](auto qc) {
    constexpr int q = decltype(qc)::value + 1;
    if ((unsigned)q >= 7u - ca) {
      wait_vm<COMM_W - 1>();
      st_chunk<2>(c.vs, c.rs[q], c.dst0 + ta * 4096u);
    }
  });
  while (true) {
    if (comm7_round<0>(c)) break;
    if (comm7_round<1>(c)) break;
  }
  wait_vm<0>();
  publish7(c, c.n, c.t_cur);
  if constexpr (PF_U < 8) {
    unsigned old = 0;
    if (lane == 0) old = __hip_atomic_fetch_add(c.cnt + 63, 1u, __ATOMIC_RELAXED, __HIP_MEMORY_SCOPE_WORKGROUP);
    if (__builtin_amdgcn_readfirstlane(old) == 3u) publish_groups(myws + CTRL_UCNT, pw + CTRL_UFLAG + rank * 64, c.call, lane, NCOMM);
  }
}

__global__ void __launch_bounds__(256, 1) fused_ag_gemm(Params p) {
  extern __shared__ __attribute__((aligned(16))) unsigned char smem_dyn[];
  if (blockIdx.x < NCOMM) comm_role3(p, smem_dyn);
  else gemm_role(p, smem_dyn);
}

}

extern "C" void kernel_launch(int rank, int nranks, ncclComm_t comm, hipStream_t stream,
                              void* const* d_in, const int* in_sizes, int n_in,
                              void* d_out, int out_size,
                              const int* in_dtypes, int out_dtype,
                              void* const* d_out_all,
                              void* const* d_ws_all, size_t ws_bytes) {
  if (nranks != NRANKS || n_in < 4 || ws_bytes < WS_NEEDED || (unsigned)rank >= (unsigned)NRANKS) {
    fprintf(stderr, "unsupported configuration\n");
    return;
  }
  static bool attr_done = false;
  if (!attr_done) {
    HIP_CHECK(hipFuncSetAttribute((const void*)fused_ag_gemm, hipFuncAttributeMaxDynamicSharedMemorySize, (int)LDS_BYTES));
    attr_done = true;
  }
  Params p;
  p.x = (const unsigned char*)d_in[0];
  p.w = (const unsigned char*)d_in[1];
  p.sx = (const float*)d_in[2];
  p.sw = (const float*)d_in[3];
  p.out = (float*)d_out;
  for (int i = 0; i < NRANKS; ++i) p.ws[i] = (unsigned char*)d_ws_all[i];
  p.rank = rank;
  hipLaunchKernelGGL(fused_ag_gemm, dim3(256), dim3(256), LDS_BYTES, stream, p);
  HIP_CHECK(hipGetLastError());
}
```
